```python
import numpy as np

kernel_name = "gpu_gemm_rs_m4096_k4096_n16384_bf16_reluqe4m3"


ABSMAX_RTOL = 0.09

COMPUTE_DTYPE = "bfloat16"
EPILOGUE = "relu_quant_e4m3"
IN_QUANT = None
OUT_QUANT = "e4m3"
IN_DTYPES = {"x": "bfloat16", "w_mat": "bfloat16"}
WORLD_SIZE = 8

def _to_bf16(x):
    u = np.ascontiguousarray(x, dtype=np.float32).view(np.uint32)
    return ((u + 0x7FFF + ((u >> 16) & 1)) & 0xFFFF0000).view(np.float32)

def setup_inputs():

    x = _to_bf16(np.random.randn(4096, 4096).astype(np.float32))
    w_mat = _to_bf16((np.random.randn(4096, 16384) / np.sqrt(4096)).astype(np.float32))
    return {"x": x, "w_mat": w_mat}

def _snap_fp8(x, mant_bits, max_abs, min_normal):
    a = np.abs(x.astype(np.float64))
    _m, _e = np.frexp(a)
    q = np.where(a >= min_normal, np.ldexp(1.0, _e - 1 - mant_bits),
                 min_normal * 2.0 ** -mant_bits)
    snapped = np.minimum(np.round(a / q) * q, max_abs)
    return (np.sign(x) * snapped).astype(np.float32)
def _quant_dequant(y, amax):
    scale = amax / 448.0
    q = _snap_fp8(y / scale, 3, 448.0, 2.0 ** -6)
    return (q * scale).astype(np.float32)
def _epilogue(y):
    y = np.maximum(y, 0.0)
    return _quant_dequant(y, np.abs(y).max())

def reference(x, w_mat):
    y = (x @ w_mat).astype(np.float32)
    return _epilogue(y)
```

```cpp
#include <hip/hip_runtime.h>
#include <rccl/rccl.h>
#include <cstdio>
#include <cstdint>

#define HIP_CHECK(expr) do { hipError_t _e = (expr); if (_e != hipSuccess) { \
    fprintf(stderr, "HIP error %s at %s:%d\n", hipGetErrorString(_e), __FILE__, __LINE__); \
    return; } } while (0)

namespace {

typedef short bf16x8 __attribute__((ext_vector_type(8)));
typedef float f32x4 __attribute__((ext_vector_type(4)));
typedef unsigned u32x4 __attribute__((ext_vector_type(4)));
typedef unsigned u32x2 __attribute__((ext_vector_type(2)));
#define LAS __attribute__((address_space(3)))
#define GAS __attribute__((address_space(1)))

constexpr int NR = 8;
constexpr int MM = 4096, KSH = 512, NN = 16384, NCOL = NN / NR;
constexpr int NT = KSH / 8;
constexpr int NJOB = 12;
#define PUSH_WAVES 2
constexpr int NW = PUSH_WAVES;
constexpr int N_PUSH = NR * NJOB;
constexpr int N_GEMM = 128;
constexpr int GRID = N_PUSH + N_GEMM;
constexpr int BLOCK = 512;

constexpr size_t XT_UNIT = (size_t)MM * 16;
constexpr size_t WT_UNIT = (size_t)NCOL * 16;
constexpr size_t XT_OFF = 0;
constexpr size_t WT_OFF = XT_OFF + (size_t)NT * NR * XT_UNIT;
constexpr size_t CTL_OFF = WT_OFF + (size_t)NT * NR * WT_UNIT;
constexpr size_t PROG_OFF = CTL_OFF;
constexpr size_t AMAX_OFF = PROG_OFF + (size_t)NR * NJOB * 128;
constexpr size_t DONE_OFF = AMAX_OFF + (size_t)NR * N_GEMM * 8;
constexpr size_t CTR_OFF = DONE_OFF + 1024;
constexpr size_t TMO_OFF = CTR_OFF + (size_t)GRID * 128;
constexpr size_t DBG_OFF = TMO_OFF + 128;
constexpr size_t WS_NEED = DBG_OFF + 65536;

constexpr unsigned long long DEADLINE = 50000000ull;
constexpr int AUX_SYS = 17;

constexpr int LDS_BUF = 65536;
constexpr int LDS_RED = 2 * LDS_BUF;
constexpr int LDS_BYTES = LDS_RED + 256;

struct Params {
    const char* x; const char* w;
    char* ws[NR]; char* out[NR];
    int rank;
};

__device__ __forceinline__ __amdgpu_buffer_rsrc_t rsrc_of(const void* p, unsigned bytes) {
    return __builtin_amdgcn_make_buffer_rsrc(const_cast<void*>(p), 0, bytes, 0x00020000);
}
__device__ __forceinline__ unsigned ld_sys(const void* p) {
    return __hip_atomic_load((const GAS unsigned*)p, __ATOMIC_RELAXED, __HIP_MEMORY_SCOPE_SYSTEM);
}
__device__ __forceinline__ void st_sys(void* p, unsigned v) {
    __hip_atomic_store((GAS unsigned*)p, v, __ATOMIC_RELAXED, __HIP_MEMORY_SCOPE_SYSTEM);
}
__device__ __forceinline__ unsigned ld_agent(const void* p) {
    return __hip_atomic_load((const GAS unsigned*)p, __ATOMIC_RELAXED, __HIP_MEMORY_SCOPE_AGENT);
}
__device__ __forceinline__ bool give_up(char* ws, unsigned call, unsigned long long t0) {
    const unsigned t = __builtin_amdgcn_readfirstlane(ld_agent(ws + TMO_OFF));
    if (t == call || __builtin_amdgcn_s_memrealtime() - t0 > DEADLINE) {
        __hip_atomic_store((GAS unsigned*)(ws + TMO_OFF), call, __ATOMIC_RELAXED, __HIP_MEMORY_SCOPE_AGENT);
        return true;
    }
    __builtin_amdgcn_s_sleep(1);
    return false;
}
#define WAIT_VM0() asm volatile("s_waitcnt vmcnt(0)" ::: "memory")
#define STAMP(i) do {} while (0)

__global__ void __launch_bounds__(BLOCK, 2) fused_gemm_relu_q8(const Params p) {
    __shared__ __attribute__((aligned(1024))) char smem[LDS_BYTES];
    const int b = blockIdx.x;
    const int lane = threadIdx.x & 63;
    const int wid = __builtin_amdgcn_readfirstlane(threadIdx.x >> 6);
    const int rank = p.rank;
    char* const ws = p.ws[rank];

    if (b < N_PUSH) {
        if (wid >= NW) return;
        if (wid == 0) STAMP(0);
        unsigned ticket = 0;
        if (lane == 0) ticket = __hip_atomic_fetch_add((GAS unsigned*)(ws + CTR_OFF + (size_t)b * 128 + wid * 4), 1u,
                                                       __ATOMIC_RELAXED, __HIP_MEMORY_SCOPE_AGENT);
        const int r = b & 7, q = b >> 3;
        const bool is_x = q < 8;
        const int job = is_x ? r : q;
        const unsigned dst = (unsigned)(is_x ? (q + r + rank) : (r + 2 * q + rank)) & 7u;
        char* const dws = p.ws[dst];
        const __amdgpu_buffer_rsrc_t src = is_x ? rsrc_of(p.x, (unsigned)((size_t)MM * KSH * 2))
                                                : rsrc_of(p.w, (unsigned)((size_t)KSH * NN * 2));
        const __amdgpu_buffer_rsrc_t drs = rsrc_of(dws, (unsigned)CTL_OFF);
        const unsigned sbase = is_x ? (unsigned)((job * 512 + lane) * (KSH * 2))
                                    : (unsigned)((dst * NCOL + (job - 8) * 512 + lane * 8) * 2);
        const unsigned sstep_t = is_x ? 16u : (unsigned)(8 * NN * 2);
        const unsigned sstep_i = is_x ? (unsigned)(64 * KSH * 2) : (unsigned)(NN * 2);
        const unsigned dbase = is_x ? (unsigned)(XT_OFF + rank * XT_UNIT + (job * 512 + lane) * 16)
                                    : (unsigned)(WT_OFF + rank * WT_UNIT + ((job - 8) * 512 + (lane >> 4) * 128 + (lane & 15)) * 16);
        const unsigned dstep_t = is_x ? (unsigned)(NR * XT_UNIT) : (unsigned)(NR * WT_UNIT);
        const unsigned dstep_i = is_x ? 1024u : 256u;
        char* const flag = dws + PROG_OFF + (size_t)(rank * NJOB + job) * 128 + wid * 4;

        u32x4 cur[8], nxt[8];
#pragma unroll
        for (int i = 0; i < 8; ++i) cur[i] = __builtin_amdgcn_raw_buffer_load_b128(src, sbase + wid * sstep_t + i * sstep_i, 0, 0);
        const unsigned call = __builtin_amdgcn_readfirstlane(ticket) + 1u;
        const unsigned base = (call - 1u) * NT;
        for (int t = wid; t < NT; t += NW) {
            const int tn = t + NW < NT ? t + NW : t;
#pragma unroll
            for (int i = 0; i < 8; ++i) nxt[i] = __builtin_amdgcn_raw_buffer_load_b128(src, sbase + tn * sstep_t + i * sstep_i, 0, 0);
            if (!is_x) {
                u32x4 o[8];
#pragma unroll
                for (int i = 0; i < 8; ++i)
#pragma unroll
                    for (int d = 0; d < 4; ++d)
                        o[i][d] = __builtin_amdgcn_perm(cur[2 * d + 1][i >> 1], cur[2 * d][i >> 1], (i & 1) ? 0x07060302u : 0x05040100u);
#pragma unroll
                for (int i = 0; i < 8; ++i) cur[i] = o[i];
            }
#pragma unroll
            for (int i = 0; i < 8; ++i) __builtin_amdgcn_raw_buffer_store_b128(cur[i], drs, dbase + t * dstep_t + i * dstep_i, 0, AUX_SYS);
            WAIT_VM0();
            if (lane == 0) st_sys(flag, base + t + 1u);
#pragma unroll
            for (int i = 0; i < 8; ++i) cur[i] = nxt[i];
        }
        if (wid == 0) STAMP(1);
        if (b == 0 && wid == 0) {
            const char* dn = ws + DONE_OFF + lane * 8;
            const unsigned long long t0 = __builtin_amdgcn_s_memrealtime();
            for (;;) {
                const unsigned v0 = ld_sys(dn), v1 = ld_sys(dn + 4);
                if (__all((int)(v0 - call) >= 0 && (int)(v1 - call) >= 0)) break;
                if (give_up(ws, call, t0)) { if (lane == 0) p.out[rank][rank * NCOL] = (char)0x7f; break; }
            }
            STAMP(2);
        }
        return;
    }

    const int g = b - N_PUSH;
    const int pm = ((g & 7) >> 1) * 4 + ((g >> 3) >> 2);
    const int pn = (g & 1) * 4 + ((g >> 3) & 3);
    const int fr = lane & 15, fq = lane >> 4, wr = wid >> 1, wc = wid & 1;
    if (wid == 0) STAMP(0);
    unsigned ticket = 0;
    if (lane == 0) ticket = __hip_atomic_fetch_add((GAS unsigned*)(ws + CTR_OFF + (size_t)b * 128 + wid * 4), 1u,
                                                   __ATOMIC_RELAXED, __HIP_MEMORY_SCOPE_AGENT);
    if (wid == 0) __builtin_amdgcn_fence(__ATOMIC_ACQUIRE, "");
    const unsigned call = __builtin_amdgcn_readfirstlane(ticket) + 1u;
    const unsigned base = (call - 1u) * NT;
    WAIT_VM0();
    __builtin_amdgcn_s_barrier();

    const char* const pollp = ws + PROG_OFF + (size_t)(wid * NJOB + (lane < 32 ? (pm >> 1) : 8 + (pn >> 1))) * 128 + (lane & 31) * 4;
    const char* ga = ws + XT_OFF + (size_t)wid * XT_UNIT + (size_t)(pm * 256 + lane) * 16;
    const char* gb = ws + WT_OFF + (size_t)wid * WT_UNIT + (size_t)(pn * 256 + lane) * 16;
    LAS char* const lds = (LAS char*)smem;
    const int aoff = fq * 4096 + (wr * 64 + fr) * 16;
    const int boff = 32768 + fq * 4096 + (wc * 128 + fr) * 16;

    unsigned pv = ld_sys(pollp);
#define WAIT_TILE(t_) do { const unsigned want_ = base + (t_) + 1u; const int sl_ = (t_) % NW;                        \
        if ((int)(__builtin_amdgcn_readlane(pv, sl_) - want_) < 0 || (int)(__builtin_amdgcn_readlane(pv, 32 + sl_) - want_) < 0) { \
            const unsigned long long t0_ = __builtin_amdgcn_s_memrealtime();                                          \
            for (;;) { pv = ld_sys(pollp);                                                                            \
                if ((int)(__builtin_amdgcn_readlane(pv, sl_) - want_) >= 0 && (int)(__builtin_amdgcn_readlane(pv, 32 + sl_) - want_) >= 0) break; \
                if (give_up(ws, call, t0_)) break; } } } while (0)
#define STAGE(buf_) do { LAS char* la_ = lds + (buf_) * LDS_BUF + wid * 4096;                                         \
        __builtin_amdgcn_global_load_lds((const GAS unsigned*)ga, (LAS unsigned*)la_, 16, 0, 0);                      \
        __builtin_amdgcn_global_load_lds((const GAS unsigned*)ga, (LAS unsigned*)la_, 16, 1024, 0);                   \
        __builtin_amdgcn_global_load_lds((const GAS unsigned*)ga, (LAS unsigned*)la_, 16, 2048, 0);                   \
        __builtin_amdgcn_global_load_lds((const GAS unsigned*)ga, (LAS unsigned*)la_, 16, 3072, 0);                   \
        __builtin_amdgcn_global_load_lds((const GAS unsigned*)gb, (LAS unsigned*)(la_ + 32768), 16, 0, 0);            \
        __builtin_amdgcn_global_load_lds((const GAS unsigned*)gb, (LAS unsigned*)(la_ + 32768), 16, 1024, 0);         \
        __builtin_amdgcn_global_load_lds((const GAS unsigned*)gb, (LAS unsigned*)(la_ + 32768), 16, 2048, 0);         \
        __builtin_amdgcn_global_load_lds((const GAS unsigned*)gb, (LAS unsigned*)(la_ + 32768), 16, 3072, 0);         \
        ga += NR * XT_UNIT; gb += NR * WT_UNIT; } while (0)

    f32x4 acc[4][8];
#pragma unroll
    for (int m = 0; m < 4; ++m)
#pragma unroll
        for (int n = 0; n < 8; ++n) acc[m][n] = (f32x4){0.f, 0.f, 0.f, 0.f};

    WAIT_TILE(0);
    if (wid == 0) STAMP(1);
    STAGE(0);
    pv = ld_sys(pollp);
    WAIT_VM0();
    __builtin_amdgcn_s_barrier();
    for (int t = 0; t < NT; ++t) {
        const int cur = t & 1;
        if (t + 1 < NT) {
            WAIT_TILE(t + 1);
            STAGE(cur ^ 1);
            pv = ld_sys(pollp);
        }
        const LAS char* lb = lds + cur * LDS_BUF;
#pragma unroll
        for (int ks = 0; ks < 2; ++ks) {
            bf16x8 af[4], bfr[8];
#pragma unroll
            for (int m = 0; m < 4; ++m) af[m] = *(const LAS bf16x8*)(lb + aoff + ks * 16384 + m * 256);
#pragma unroll
            for (int n = 0; n < 8; ++n) bfr[n] = *(const LAS bf16x8*)(lb + boff + ks * 16384 + n * 256);
#pragma unroll
            for (int n = 0; n < 8; ++n)
#pragma unroll
                for (int m = 0; m < 4; ++m)
                    acc[m][n] = __builtin_amdgcn_mfma_f32_16x16x32_bf16(af[m], bfr[n], acc[m][n], 0, 0, 0);
        }
        asm volatile("s_waitcnt vmcnt(0) lgkmcnt(0)" ::: "memory");
        __builtin_amdgcn_s_barrier();
    }
    if (wid == 0) STAMP(2);

    float mx = 0.f;
#pragma unroll
    for (int m = 0; m < 4; ++m)
#pragma unroll
        for (int n = 0; n < 8; ++n)
#pragma unroll
            for (int j = 0; j < 4; ++j) { acc[m][n][j] = fmaxf(acc[m][n][j], 0.f); mx = fmaxf(mx, acc[m][n][j]); }
#pragma unroll
    for (int o = 32; o >= 1; o >>= 1) mx = fmaxf(mx, __shfl_xor(mx, o));
    LAS float* red = (LAS float*)(lds + LDS_RED);
    if (lane == 0) red[wid] = mx;
    __syncthreads();
    if (wid == 0 && lane < NR) {
        float t = red[0];
#pragma unroll
        for (int i = 1; i < 8; ++i) t = fmaxf(t, red[i]);
        __hip_atomic_store((GAS unsigned long long*)(p.ws[lane] + AMAX_OFF + (size_t)(rank * N_GEMM + g) * 8),
                           ((unsigned long long)call << 32) | __float_as_uint(t), __ATOMIC_RELAXED, __HIP_MEMORY_SCOPE_SYSTEM);
    }
    {
        const __amdgpu_buffer_rsrc_t ars = rsrc_of(ws + AMAX_OFF, NR * N_GEMM * 8);
        const unsigned long long t0 = __builtin_amdgcn_s_memrealtime();
        u32x4 v;
        for (;;) {
            v = __builtin_amdgcn_raw_buffer_load_b128(ars, threadIdx.x * 16, 0, AUX_SYS);
            if (__all((int)(v[1] - call) >= 0 && (int)(v[3] - call) >= 0)) break;
            if (give_up(ws, call, t0)) break;
        }
        mx = fmaxf(__uint_as_float(v[0]), __uint_as_float(v[2]));
#pragma unroll
        for (int o = 32; o >= 1; o >>= 1) mx = fmaxf(mx, __shfl_xor(mx, o));
        if (lane == 0) red[8 + wid] = mx;
        __syncthreads();
        mx = red[8];
#pragma unroll
        for (int i = 1; i < 8; ++i) mx = fmaxf(mx, red[8 + i]);
    }
    const bool bad = __builtin_amdgcn_readfirstlane(ld_agent(ws + TMO_OFF)) == call;
    if (wid == 0) STAMP(3);
    const float inv = 448.0f / mx;

    const unsigned owner = (unsigned)pm >> 1;
    const __amdgpu_buffer_rsrc_t ors = rsrc_of(p.out[owner], (unsigned)((size_t)(MM / NR) * NN));
    const unsigned obase = (unsigned)(((pm & 1) * 256 + wr * 64 + fq * 4) * NN + rank * NCOL + pn * 256 + wc * 128 + fr * 8);
#pragma unroll
    for (int m = 0; m < 4; ++m)
#pragma unroll
        for (int j = 0; j < 4; ++j) {
            u32x2 o;
            int lo = __builtin_amdgcn_cvt_pk_fp8_f32(fminf(acc[m][0][j] * inv, 448.f), fminf(acc[m][1][j] * inv, 448.f), 0, false);
            lo = __builtin_amdgcn_cvt_pk_fp8_f32(fminf(acc[m][2][j] * inv, 448.f), fminf(acc[m][3][j] * inv, 448.f), lo, true);
            int hi = __builtin_amdgcn_cvt_pk_fp8_f32(fminf(acc[m][4][j] * inv, 448.f), fminf(acc[m][5][j] * inv, 448.f), 0, false);
            hi = __builtin_amdgcn_cvt_pk_fp8_f32(fminf(acc[m][6][j] * inv, 448.f), fminf(acc[m][7][j] * inv, 448.f), hi, true);
            o[0] = bad ? 0x7f7f7f7fu : (unsigned)lo;
            o[1] = bad ? 0x7f7f7f7fu : (unsigned)hi;
            __builtin_amdgcn_raw_buffer_store_b64(o, ors, obase + (unsigned)((m * 16 + j) * NN), 0, AUX_SYS);
        }
    WAIT_VM0();
    __builtin_amdgcn_s_barrier();
    if (threadIdx.x == 0) st_sys(p.ws[owner] + DONE_OFF + (size_t)(rank * 16 + (pm & 1) * 8 + pn) * 4, call);
    if (wid == 0) STAMP(4);
}

}

extern "C" void kernel_launch(int rank, int nranks, ncclComm_t comm,
                              hipStream_t stream,
                              void* const* d_in, const int* in_sizes, int n_in,
                              void* d_out, int out_size,
                              const int* in_dtypes, int out_dtype,
                              void* const* d_out_all,
                              void* const* d_ws_all, size_t ws_bytes) {
    (void)comm; (void)in_dtypes; (void)out_dtype; (void)d_out;
    if (nranks != NR || n_in < 2 || (unsigned)rank >= (unsigned)NR || ws_bytes < WS_NEED ||
        in_sizes[0] != MM * KSH || in_sizes[1] != KSH * NN || out_size != (MM / NR) * NN) {
        fprintf(stderr, "kernel_launch: unexpected problem shape\n");
        return;
    }
    Params p;
    p.x = (const char*)d_in[0];
    p.w = (const char*)d_in[1];
    for (int i = 0; i < NR; ++i) { p.ws[i] = (char*)d_ws_all[i]; p.out[i] = (char*)d_out_all[i]; }
    p.rank = rank;
    hipLaunchKernelGGL(fused_gemm_relu_q8, dim3(GRID), dim3(BLOCK), 0, stream, p);
    HIP_CHECK(hipGetLastError());
}
```

```cpp
#include <hip/hip_runtime.h>
#include <rccl/rccl.h>
#include <cstdio>
#include <cstdint>

#define HIP_CHECK(expr) do { hipError_t _e = (expr); if (_e != hipSuccess) { \
    fprintf(stderr, "HIP error %s at %s:%d\n", hipGetErrorString(_e), __FILE__, __LINE__); \
    return; } } while (0)

namespace {

typedef short bf16x8 __attribute__((ext_vector_type(8)));
typedef float f32x4 __attribute__((ext_vector_type(4)));
typedef unsigned u32x4 __attribute__((ext_vector_type(4)));
typedef unsigned u32x2 __attribute__((ext_vector_type(2)));
typedef unsigned long long u64;
#define LAS __attribute__((address_space(3)))
#define GAS __attribute__((address_space(1)))

constexpr int NR = 8;
constexpr int MM = 4096, KSH = 512, NN = 16384, NCOL = NN / NR;
constexpr int NT = KSH / 8;
constexpr int NBLK = 24;
#define PULL_PER_LINK 22
#define PACK_DELAY 400
constexpr int Q_JOBS = NT * NBLK;
constexpr int Q_TICKETS = Q_JOBS + PULL_PER_LINK;
constexpr int N_COMM = 128;
constexpr int N_PULL = (NR - 1) * PULL_PER_LINK;
constexpr int N_GEMM = 128;
constexpr int GRID = N_COMM + N_GEMM;
constexpr int BLOCK = 512;
constexpr int N_PACKW = N_COMM * 6;
constexpr int NGRP = NT / 8;
static_assert(N_PULL <= 2 * N_COMM, "pull waves are waves 0-1 of the comm workgroups");

constexpr unsigned XT_T = NR * MM * 16, WT_T = NR * NCOL * 16;
constexpr unsigned XT_OFF = 0;
constexpr unsigned WT_OFF = XT_OFF + NT * XT_T;
constexpr unsigned WS_OFF = WT_OFF + NT * WT_T;
constexpr unsigned CTL_OFF = WS_OFF + NT * WT_T;
constexpr unsigned PK_OFF = CTL_OFF;
constexpr unsigned FL_OFF = PK_OFF + NR * NGRP * 256;
constexpr unsigned Q_OFF = FL_OFF + NBLK * NR * NT * 4;
constexpr unsigned AMAX_OFF = Q_OFF + NR * 128;
constexpr unsigned DONE_OFF = AMAX_OFF + NR * N_GEMM * 8;
constexpr unsigned CTR_OFF = DONE_OFF + 1024;
constexpr unsigned TMO_OFF = CTR_OFF + GRID * 128;
constexpr unsigned DBG_OFF = TMO_OFF + 128;
constexpr size_t WS_NEED = DBG_OFF + 131072;

constexpr u64 DEADLINE = 50000000ull;
#define AUX_LOC 17
#define LOC_SCOPE __HIP_MEMORY_SCOPE_SYSTEM
#define LOC_BITS "sc0 sc1"
constexpr int AUX_SYS = 17;

constexpr int NSTEP = 2 * NT;
constexpr int NSLOT = 5;
constexpr int LDS_SLOT = 32768;
constexpr int LDS_BYTES = NSLOT * LDS_SLOT;

struct Params {
    const char* x; const char* w;
    int rank, pad;
    char* ws[NR]; char* out[NR];
};

__device__ __forceinline__ __amdgpu_buffer_rsrc_t rsrc_of(const void* p, unsigned bytes) {
    return __builtin_amdgcn_make_buffer_rsrc(const_cast<void*>(p), 0, bytes, 0x00020000);
}
__device__ __forceinline__ unsigned ld_sys(const void* p) {
    return __hip_atomic_load((const GAS unsigned*)p, __ATOMIC_RELAXED, __HIP_MEMORY_SCOPE_SYSTEM);
}
__device__ __forceinline__ u64 ld_sys64(const void* p) {
    return __hip_atomic_load((const GAS u64*)p, __ATOMIC_RELAXED, __HIP_MEMORY_SCOPE_SYSTEM);
}
__device__ __forceinline__ void st_sys(void* p, unsigned v) {
    __hip_atomic_store((GAS unsigned*)p, v, __ATOMIC_RELAXED, __HIP_MEMORY_SCOPE_SYSTEM);
}
__device__ __forceinline__ unsigned ld_agent(const void* p) {
    return __hip_atomic_load((const GAS unsigned*)p, __ATOMIC_RELAXED, __HIP_MEMORY_SCOPE_AGENT);
}
__device__ __forceinline__ unsigned add_agent_lane0(void* p, int lane) {
    unsigned v = 0, z = 0;
    asm volatile("" : "+v"(z));
    if (lane == 0) v = __hip_atomic_fetch_add((GAS unsigned*)((char*)p + z), 1u, __ATOMIC_RELAXED, __HIP_MEMORY_SCOPE_AGENT);
    return v;
}
__device__ __forceinline__ u32x4 transposed(const u32x4 (&v)[8], int i) {
    u32x4 o;
#pragma unroll
    for (int d = 0; d < 4; ++d)
        o[d] = __builtin_amdgcn_perm(v[2 * d + 1][i >> 1], v[2 * d][i >> 1], (i & 1) ? 0x07060302u : 0x05040100u);
    return o;
}
__device__ __forceinline__ bool give_up(char* ws, unsigned call, u64 t0, unsigned& spins) {
    bool stop = __builtin_amdgcn_s_memrealtime() - t0 > DEADLINE;
    if ((++spins & 63u) == 0u) stop = stop || __builtin_amdgcn_readfirstlane(ld_agent(ws + TMO_OFF)) == call;
    if (stop) {
        __hip_atomic_store((GAS unsigned*)(ws + TMO_OFF), call, __ATOMIC_RELAXED, __HIP_MEMORY_SCOPE_AGENT);
        return true;
    }
    __builtin_amdgcn_s_sleep(1);
    return false;
}
__device__ __forceinline__ unsigned fl_word(int blk, int src, int t) { return FL_OFF + (unsigned)(((blk * 4 + (src & 3)) * NT + t) * 2 + (src >> 2)) * 4u; }
__device__ __forceinline__ int steps_known(unsigned f0, unsigned f1, unsigned call) {
    const u64 lo = __ballot((int)(f0 - call) >= 0), hi = __ballot((int)(f1 - call) >= 0);
    const u64 miss = ~(lo & hi);
    if (miss == 0) return NSTEP;
    const int c = __builtin_ctzll(miss);
    return 2 * c + (int)((lo >> c) & 1);
}
#define WAIT_VM(n) asm volatile("s_waitcnt vmcnt(" #n ")" ::: "memory")
#define STAMP(i) do {} while (0)

__global__ void __launch_bounds__(BLOCK, 2) fused_gemm_relu_q8(const Params p) {
    __shared__ __attribute__((aligned(1024))) char smem[LDS_BYTES];
    LAS char* const lds = (LAS char*)smem;
    const int hb = blockIdx.x;
    const int lane = threadIdx.x & 63;
    const int wid = __builtin_amdgcn_readfirstlane(threadIdx.x >> 6);
    const int rank = p.rank;
    char* const ws = p.ws[0];
    const u64 t_start = __builtin_amdgcn_s_memrealtime();
    const unsigned ticket = add_agent_lane0(ws + CTR_OFF + (size_t)hb * 128 + wid * 4, lane);
    const int b = hb & (N_COMM - 1);

    if (hb < N_COMM) {
        const __amdgpu_buffer_rsrc_t wsr = rsrc_of(ws, CTL_OFF);
        if (threadIdx.x == 256) *(LAS unsigned*)lds = 0u;
        __syncthreads();
        if (wid < 2) {
            const int cw = wid * N_COMM + b;
            const int dbg = cw;
            (void)dbg;
            if (cw >= N_PULL) return;
            STAMP(0);
            const unsigned s = (unsigned)(rank + 1 + cw % (NR - 1)) & 7u;
            const __amdgpu_buffer_rsrc_t srs = rsrc_of(p.ws[1 + cw % (NR - 1)], CTL_OFF);
            char* const queue = ws + Q_OFF + s * 128;
            unsigned tk = add_agent_lane0(queue, lane);
            u32x4 cur[4], nxt[4];
            const unsigned call = __builtin_amdgcn_readfirstlane(ticket) + 1u;
            const unsigned qbase = (call - 1u) * Q_TICKETS;
            const char* const pk = ws + PK_OFF + s * NGRP * 256 + lane * 4;
            unsigned pv = call - 1u;
            int pgrp = 0;
#define JOB_OF(n_, t_, j_, so_, do_) const int t_ = (int)(n_) / NBLK, j_ = (int)(n_) % NBLK;                           \
            const unsigned do_ = (j_ < 16 ? XT_OFF + t_ * XT_T + j_ * 32768 : WT_OFF + t_ * WT_T + (j_ - 16) * 32768) + s * 4096 + lane * 16; \
            const unsigned so_ = j_ < 16 ? do_ : WS_OFF + t_ * WT_T + rank * 32768 + (j_ - 16) * 4096 + lane * 16
#define WAIT_PACKED(t_, j_) do { const int sl_ = (j_) < 16 ? (j_) : 24 + ((t_) & 7) * 4 + ((j_) >> 1);                 \
            if (((t_) >> 3) != pgrp) { pgrp = (t_) >> 3; pv = call - 1u; }                                             \
            if ((int)(__builtin_amdgcn_readlane(pv, sl_) - call) < 0) {                                                \
                const u64 t0_ = __builtin_amdgcn_s_memrealtime(); unsigned sp_ = 0;                                    \
                for (;;) { pv = ld_sys(pk + pgrp * 256);                                                               \
                    if ((int)(__builtin_amdgcn_readlane(pv, sl_) - call) >= 0) break;                                  \
                    if (give_up(ws, call, t0_, sp_)) break; } } } while (0)
            unsigned n = __builtin_amdgcn_readfirstlane(tk) - qbase;
            unsigned dcur = 0, fcur = 0;
            if (n < (unsigned)Q_JOBS) {
                JOB_OF(n, t, j, so, dof);
                WAIT_PACKED(t, j);
                STAMP(2);
                tk = add_agent_lane0(queue, lane);
#pragma unroll
                for (int i = 0; i < 4; ++i) cur[i] = __builtin_amdgcn_raw_buffer_load_b128(srs, so + i * 1024, 0, AUX_SYS);
                dcur = dof; fcur = fl_word(j, s, t);
                for (;;) {
                    n = __builtin_amdgcn_readfirstlane(tk) - qbase;
                    const bool more = n < (unsigned)Q_JOBS;
                    unsigned dnxt = 0, fnxt = 0;
                    if (more) {
                        JOB_OF(n, t2, j2, so2, dof2);
                        WAIT_PACKED(t2, j2);
                        tk = add_agent_lane0(queue, lane);
#pragma unroll
                        for (int i = 0; i < 4; ++i) nxt[i] = __builtin_amdgcn_raw_buffer_load_b128(srs, so2 + i * 1024, 0, AUX_SYS);
                        dnxt = dof2; fnxt = fl_word(j2, s, t2);
                    }
#pragma unroll
                    for (int i = 0; i < 4; ++i) __builtin_amdgcn_raw_buffer_store_b128(cur[i], wsr, dcur + i * 1024, 0, AUX_LOC);
                    WAIT_VM(0);
                    if (lane == 0) __hip_atomic_store((GAS unsigned*)(ws + fcur), call, __ATOMIC_RELAXED, LOC_SCOPE);
                    if (!more) break;
                    dcur = dnxt; fcur = fnxt;
#pragma unroll
                    for (int i = 0; i < 4; ++i) cur[i] = nxt[i];
                }
            }
            STAMP(1);
            return;
        }
        const __amdgpu_buffer_rsrc_t xr = rsrc_of(p.x, MM * KSH * 2);
        const __amdgpu_buffer_rsrc_t wr_ = rsrc_of(p.w, KSH * NN * 2);
        const int dbg = 384;
        (void)dbg;
        bool x_todo = wid >= 4;
        for (int n = (wid - 2) * N_COMM + b; n < NT * 32 || x_todo;) {
            const bool do_x = x_todo && ((b >> 4) <= (n >> 8) || n >= NT * 32);
            const int g = do_x ? b >> 4 : n >> 8;
            while (__builtin_amdgcn_s_memrealtime() - t_start < (u64)(g * PACK_DELAY)) __builtin_amdgcn_s_sleep(2);
            u32x4 v[8];
            if (do_x) {
                const unsigned rb = (wid - 4) * 64 + (lane & 7), piece = lane >> 3;
#pragma unroll
                for (int i = 0; i < 8; ++i) v[i] = __builtin_amdgcn_raw_buffer_load_b128(xr, ((b & 15) * 256 + rb + i * 8) * (KSH * 2) + g * 128 + piece * 16, 0, 0);
                const unsigned d0 = XT_OFF + (g * 8 + piece) * XT_T + (b & 15) * 32768 + rank * 4096 + rb * 16;
#pragma unroll
                for (int i = 0; i < 8; ++i) __builtin_amdgcn_raw_buffer_store_b128(v[i], wsr, d0 + i * 128, 0, AUX_SYS);
                char* fl = lane < 8 ? p.ws[lane & 7] + PK_OFF + (rank * NGRP + g) * 256 + (b & 15) * 4 : ws + fl_word(b & 15, rank, g * 8 + (lane & 7));
                asm volatile("s_waitcnt vmcnt(0)" : "+v"(fl) :: "memory");
                unsigned old = 0;
                if (lane == 0) old = __hip_atomic_fetch_add((LAS unsigned*)lds, 1u, __ATOMIC_RELAXED, __HIP_MEMORY_SCOPE_WORKGROUP);
                const unsigned call = __builtin_amdgcn_readfirstlane(ticket) + 1u;
                if (__builtin_amdgcn_readfirstlane(old) == 3u && lane >= 1 && lane < 16) st_sys(fl, call);
                x_todo = false;
            } else {
                const int t = n >> 5, cb = n & 31;
                const unsigned dest = (unsigned)cb >> 2;
#pragma unroll
                for (int i = 0; i < 8; ++i) v[i] = __builtin_amdgcn_raw_buffer_load_b128(wr_, (unsigned)((t * 8 + i) * (NN * 2) + cb * 1024 + lane * 16), 0, 0);
                const unsigned d0 = (dest == (unsigned)rank ? WT_OFF + t * WT_T + ((cb & 3) * 2 + (lane >> 5)) * 32768 + rank * 4096
                                                            : WS_OFF + t * WT_T + dest * 32768 + ((cb & 3) * 2 + (lane >> 5)) * 4096)
                                  + (((lane >> 4) & 1) * 128 + (lane & 15)) * 16;
#pragma unroll
                for (int i = 0; i < 8; ++i) __builtin_amdgcn_raw_buffer_store_b128(transposed(v, i), wsr, d0 + i * 256, 0, AUX_SYS);
                char* const dws = p.ws[(dest - rank) & 7u];
                WAIT_VM(0);
                const unsigned call = __builtin_amdgcn_readfirstlane(ticket) + 1u;
                if (dest == (unsigned)rank) { if (lane < 2) st_sys(ws + fl_word(16 + (cb & 3) * 2 + lane, rank, t), call); }
                else if (lane == 0) st_sys(dws + PK_OFF + (rank * NGRP + g) * 256 + (32 + (t & 7) * 4 + (cb & 3)) * 4, call);
                n += N_PACKW;
            }
        }
        if (b == N_COMM - 1 && wid == 7) {
            const unsigned call = __builtin_amdgcn_readfirstlane(ticket) + 1u;
            const char* dn = ws + DONE_OFF + lane * 8;
            const u64 t0 = __builtin_amdgcn_s_memrealtime(); unsigned sp = 0;
            for (;;) {
                const u64 v = ld_sys64(dn);
                if (__all((int)((unsigned)v - call) >= 0 && (int)((unsigned)(v >> 32) - call) >= 0)) break;
                if (give_up(ws, call, t0, sp)) { if (lane == 0) p.out[0][rank * NCOL] = (char)0x7f; break; }
            }
            STAMP(5);
        }
        return;
    }

    const int g = b;
    const int dbg = 256 + g;
    (void)dbg;
    const int pm = ((g & 7) >> 1) * 4 + ((g >> 3) >> 2);
    const int pn = (g & 1) * 4 + ((g >> 3) & 3);
    const int fr = lane & 15, fq = lane >> 4, wr = wid >> 1, wc = wid & 1;
    if (wid == 0) STAMP(0);
    const unsigned owner = (unsigned)pm >> 1;
    char* amax_slot = p.ws[lane & 7] + AMAX_OFF + (size_t)(rank * N_GEMM + g) * 8;
    char* out_base = p.out[(owner - rank) & 7u];
    char* done_flag = p.ws[(owner - rank) & 7u] + DONE_OFF + (size_t)(rank * 16 + (pm & 1) * 8 + pn) * 4;
    asm volatile("" : "+v"(amax_slot), "+s"(out_base), "+s"(done_flag));
    if (wid == 0) __builtin_amdgcn_fence(__ATOMIC_ACQUIRE, "");
    const unsigned call = __builtin_amdgcn_readfirstlane(ticket) + 1u;
    WAIT_VM(0);
    __builtin_amdgcn_s_barrier();

    const int op = wid >> 2;
    const char* const pollp = ws + fl_word(op ? 16 + pn : pm, wid & 3, lane);
    const unsigned unit = op ? WT_T : XT_T;
    const char* const gbase = ws + (op ? WT_OFF + pn * 32768 : XT_OFF + pm * 32768) + (wid & 3) * 4096 + lane * 16;
    LAS char* const ldst = lds + wid * 4096;
    const int aoff = fq * 4096 + (wr * 64 + fr) * 16;
    const int boff = 16384 + fq * 4096 + (wc * 128 + fr) * 16;

    int known = 0;
    int next = 0, nslot = 0;
    u32x2 pvn = {0u, 0u};
#define OBS_STAGE() do {} while (0)
#define OBS_STEP(st_) do {} while (0)
#define LAT_PROBE(st_) do {} while (0)
#define STAGE_AUX 0
#define CTL_COL 6
#define POLL_AHEAD 6
#define STAGE_NEXT() do { const char* g_ = gbase + ((next & 1) * 16384u + (unsigned)(next >> 1) * unit);              \
        LAS char* l_ = ldst + nslot * LDS_SLOT;                                                                       \
        __builtin_amdgcn_global_load_lds((const GAS unsigned*)g_, (LAS unsigned*)l_, 16, 0, STAGE_AUX);                       \
        __builtin_amdgcn_global_load_lds((const GAS unsigned*)g_, (LAS unsigned*)l_, 16, 1024, STAGE_AUX);                    \
        __builtin_amdgcn_global_load_lds((const GAS unsigned*)g_, (LAS unsigned*)l_, 16, 2048, STAGE_AUX);                    \
        __builtin_amdgcn_global_load_lds((const GAS unsigned*)g_, (LAS unsigned*)l_, 16, 3072, STAGE_AUX);                    \
        OBS_STAGE();                                                                                                  \
        ++next; nslot = nslot == NSLOT - 1 ? 0 : nslot + 1; } while (0)
#define POLL_ISSUE() asm volatile("global_load_dwordx2 %0, %1, off " LOC_BITS : "+v"(pvn) : "v"(pollp) : "memory")
#define POLL_TAKE() do { u32x2 f_; asm volatile("s_waitcnt vmcnt(0)\n\tv_mov_b64 %0, %1" : "=v"(f_) : "v"(pvn) : "memory");    \
        known = steps_known(f_[0], f_[1], call); } while (0)
#define AWAIT_NEXT() do { if (next >= known) {                                                                        \
            const u64 t0_ = __builtin_amdgcn_s_memrealtime(); unsigned sp_ = 0;                                       \
            for (;;) { const u64 f_ = __hip_atomic_load((const GAS u64*)pollp, __ATOMIC_RELAXED, LOC_SCOPE); known = steps_known((unsigned)f_, (unsigned)(f_ >> 32), call); \
                if (next < known) break;                                                                              \
                if (give_up(ws, call, t0_, sp_)) { known = NSTEP; break; } } } } while (0)
#define LGKM0_BARRIER() do { __builtin_amdgcn_s_waitcnt(0xC07F); asm volatile("" ::: "memory");                       \
        __builtin_amdgcn_s_barrier(); asm volatile("" ::: "memory"); } while (0)
#define MFMA_COLS(A_, B_, n0_, n1_) do { __builtin_amdgcn_sched_barrier(0);                                           \
        _Pragma("unroll") for (int n = (n0_); n < (n1_); ++n)                                                         \
            _Pragma("unroll") for (int m = 0; m < 4; ++m)                                                             \
                acc[m][n] = __builtin_amdgcn_mfma_f32_16x16x32_bf16(A_[m], B_[n], acc[m][n], 0, 0, 0);                \
        __builtin_amdgcn_sched_barrier(0); } while (0)
#define RD_A(A_, m_) A_[m_] = *(const LAS bf16x8*)(lb_ + aoff + (m_) * 256)
#define RD_B(B_, n_) B_[n_] = *(const LAS bf16x8*)(lb_ + boff + (n_) * 256)
#define GSTAMP(st_) do {} while (0)
#define KNOWN_STAMP(st_) do {} while (0)
#define DIAG_T(v_) do {} while (0)
#define DIAG_ACC(st_, f_) do {} while (0)
#define KSTEP(st_, A_, B_, AN_, BN_) do {                                                                             \
        const LAS char* lb_ = lds + rslot * LDS_SLOT;               \
        MFMA_COLS(A_, B_, 0, 1);                                                                                      \
        RD_A(AN_, 0); RD_A(AN_, 1); RD_A(AN_, 2);                                                                     \
        const bool polled_ = known < min(next + POLL_AHEAD, NSTEP);                                                   \
        if (polled_) POLL_ISSUE();                                                                                    \
        MFMA_COLS(A_, B_, 1, 2);                                                                                      \
        RD_A(AN_, 3); RD_B(BN_, 0); RD_B(BN_, 1);                                                                     \
        MFMA_COLS(A_, B_, 2, 3);                                                                                      \
        RD_B(BN_, 2); RD_B(BN_, 3); RD_B(BN_, 4);                                                                     \
        MFMA_COLS(A_, B_, 3, 4);                                                                                      \
        RD_B(BN_, 5); RD_B(BN_, 6); RD_B(BN_, 7);                                                                     \
        rslot = rslot == NSLOT - 1 ? 0 : rslot + 1;                                                                   \
        MFMA_COLS(A_, B_, 4, CTL_COL);                                                                                \
        if (polled_) POLL_TAKE();                                         \
        const int had_ = next;                                                                                        \
        const int lim_ = min((st_) + NSLOT + 1, known);                                                               \
        LAT_PROBE(st_);                                                                                               \
        if (next < lim_) { STAGE_NEXT(); if (next < lim_) STAGE_NEXT(); }                                             \
        const bool full_ = (polled_ ? had_ + NSLOT - 2 : next) >= (st_) + NSLOT + 1;                                  \
        MFMA_COLS(A_, B_, CTL_COL, 8);                                                                                \
        DIAG_T(da);                                                                                                   \
        if (!full_) {                                                              \
            if (next == (st_) + 2 && next < NSTEP) {                                                                  \
                AWAIT_NEXT(); STAGE_NEXT();                                                                           \
                if (next < known) STAGE_NEXT();                                                                       \
            }                                                                                                         \
            const int k_ = next - (st_) - 3;                                                                          \
            if (k_ <= 0) WAIT_VM(0); else if (k_ == 1) WAIT_VM(4); else WAIT_VM(8);                                   \
        }                                                                                                             \
        WAIT_VM(12);                                                                                                  \
        DIAG_T(db);                                                                                                   \
        LGKM0_BARRIER();                                                                                              \
        DIAG_T(dc);                                                                                                   \
        OBS_STEP(st_);                                                                                                \
        KNOWN_STAMP(st_);                                                                                             \
        DIAG_ACC(st_, full_);                                                                                         \
        GSTAMP(st_); } while (0)

    f32x4 acc[4][8];
#pragma unroll
    for (int m = 0; m < 4; ++m)
#pragma unroll
        for (int n = 0; n < 8; ++n) acc[m][n] = (f32x4){0.f, 0.f, 0.f, 0.f};

    AWAIT_NEXT(); STAGE_NEXT();
    if (wid == 0) STAMP(1);
    AWAIT_NEXT(); STAGE_NEXT();
    WAIT_VM(0);
    LGKM0_BARRIER();
    int rslot = 1;
    bf16x8 a0[4], b0[8], a1[4], b1[8];
    {
        const LAS char* lb_ = lds;
#pragma unroll
        for (int m = 0; m < 4; ++m) RD_A(a0, m);
#pragma unroll
        for (int n = 0; n < 8; ++n) RD_B(b0, n);
    }
    LGKM0_BARRIER();
    for (int st = 0; st < NSTEP; st += 2) {
        KSTEP(st, a0, b0, a1, b1);
        KSTEP(st + 1, a1, b1, a0, b0);
    }
    if (wid == 0) STAMP(2);

    int mx = 0;
#pragma unroll
    for (int m = 0; m < 4; ++m)
#pragma unroll
        for (int n = 0; n < 8; ++n)
#pragma unroll
            for (int j = 0; j < 4; ++j) mx = max(mx, __float_as_int(acc[m][n][j]));
#pragma unroll
    for (int o = 32; o >= 1; o >>= 1) mx = max(mx, __shfl_xor(mx, o));
    LAS int* red = (LAS int*)lds;
    if (lane == 0) red[wid] = mx;
    __syncthreads();
    if (wid == 0 && lane < NR) {
        int t = red[0];
#pragma unroll
        for (int i = 1; i < 8; ++i) t = max(t, red[i]);
        __hip_atomic_store((GAS u64*)amax_slot, ((u64)call << 32) | (unsigned)t, __ATOMIC_RELAXED, __HIP_MEMORY_SCOPE_SYSTEM);
    }
#pragma unroll
    for (int m = 0; m < 4; ++m)
#pragma unroll
        for (int n = 0; n < 8; ++n)
#pragma unroll
            for (int j = 0; j < 4; ++j) acc[m][n][j] = __int_as_float(max(0, __float_as_int(acc[m][n][j])));
    {
        const __amdgpu_buffer_rsrc_t ars = rsrc_of(ws + AMAX_OFF, NR * N_GEMM * 8);
        const u64 t0 = __builtin_amdgcn_s_memrealtime(); unsigned sp = 0;
        u32x4 v;
        for (;;) {
            v = __builtin_amdgcn_raw_buffer_load_b128(ars, threadIdx.x * 16, 0, AUX_SYS);
            if (__all((int)(v[1] - call) >= 0 && (int)(v[3] - call) >= 0)) break;
            if (give_up(ws, call, t0, sp)) break;
        }
        mx = max((int)v[0], (int)v[2]);
#pragma unroll
        for (int o = 32; o >= 1; o >>= 1) mx = max(mx, __shfl_xor(mx, o));
        if (lane == 0) red[8 + wid] = mx;
        __syncthreads();
        mx = red[8];
#pragma unroll
        for (int i = 1; i < 8; ++i) mx = max(mx, red[8 + i]);
    }
    const bool bad = __builtin_amdgcn_readfirstlane(ld_agent(ws + TMO_OFF)) == call;
    if (wid == 0) STAMP(3);
    const float inv = 448.0f / __int_as_float(mx);

    const __amdgpu_buffer_rsrc_t ors = rsrc_of(out_base, (MM / NR) * NN);
    const unsigned obase = (unsigned)(((pm & 1) * 256 + wr * 64 + fq * 4) * NN + rank * NCOL + pn * 256 + wc * 128 + fr * 8);
#pragma unroll
    for (int m = 0; m < 4; ++m)
#pragma unroll
        for (int j = 0; j < 4; ++j) {
            u32x2 o;
            int lo = __builtin_amdgcn_cvt_pk_fp8_f32(acc[m][0][j] * inv, acc[m][1][j] * inv, 0, false);
            lo = __builtin_amdgcn_cvt_pk_fp8_f32(acc[m][2][j] * inv, acc[m][3][j] * inv, lo, true);
            int hi = __builtin_amdgcn_cvt_pk_fp8_f32(acc[m][4][j] * inv, acc[m][5][j] * inv, 0, false);
            hi = __builtin_amdgcn_cvt_pk_fp8_f32(acc[m][6][j] * inv, acc[m][7][j] * inv, hi, true);
            o[0] = bad ? 0x7f7f7f7fu : (unsigned)lo;
            o[1] = bad ? 0x7f7f7f7fu : (unsigned)hi;
            __builtin_amdgcn_raw_buffer_store_b64(o, ors, obase + (unsigned)((m * 16 + j) * NN), 0, AUX_SYS);
        }
    WAIT_VM(0);
    __builtin_amdgcn_s_barrier();
    if (threadIdx.x == 0) st_sys(done_flag, call);
    if (wid == 0) STAMP(4);
}

}

extern "C" void kernel_launch(int rank, int nranks, ncclComm_t comm,
                              hipStream_t stream,
                              void* const* d_in, const int* in_sizes, int n_in,
                              void* d_out, int out_size,
                              const int* in_dtypes, int out_dtype,
                              void* const* d_out_all,
                              void* const* d_ws_all, size_t ws_bytes) {
    (void)comm; (void)in_dtypes; (void)out_dtype; (void)d_out;
    if (nranks != NR || n_in < 2 || (unsigned)rank >= (unsigned)NR || ws_bytes < WS_NEED ||
        in_sizes[0] != MM * KSH || in_sizes[1] != KSH * NN || out_size != (MM / NR) * NN) {
        fprintf(stderr, "kernel_launch: unexpected problem shape\n");
        return;
    }
    Params p;
    p.x = (const char*)d_in[0];
    p.w = (const char*)d_in[1];
    for (int i = 0; i < NR; ++i) { p.ws[i] = (char*)d_ws_all[(rank + i) % NR]; p.out[i] = (char*)d_out_all[(rank + i) % NR]; }
    p.rank = rank; p.pad = 0;
    hipLaunchKernelGGL(fused_gemm_relu_q8, dim3(GRID), dim3(BLOCK), 0, stream, p);
    HIP_CHECK(hipGetLastError());
}
```

```cpp
#include <hip/hip_runtime.h>
#include <rccl/rccl.h>
#include <cstdio>
#include <cstdint>

#define HIP_CHECK(expr) do { hipError_t _e = (expr); if (_e != hipSuccess) { \
    fprintf(stderr, "HIP error %s at %s:%d\n", hipGetErrorString(_e), __FILE__, __LINE__); \
    return; } } while (0)

namespace {

typedef short bf16x8 __attribute__((ext_vector_type(8)));
typedef _Float16 f16x8 __attribute__((ext_vector_type(8)));
typedef float f32x4 __attribute__((ext_vector_type(4)));
typedef unsigned u32x4 __attribute__((ext_vector_type(4)));
typedef unsigned u32x2 __attribute__((ext_vector_type(2)));
typedef unsigned long long u64;
#define LAS __attribute__((address_space(3)))
#define GAS __attribute__((address_space(1)))

constexpr int NR = 8;
constexpr int MM = 4096, KSH = 512, NN = 16384, NCOL = NN / NR;
constexpr int NT = KSH / 8;
constexpr int NBLK = 24;
#define PULL_PER_LINK 22
#define MASK_AT 0
#define AMAX_REP 8
#define PACK_DELAY 100
constexpr int Q_JOBS = NT * NBLK;
constexpr int Q_TICKETS = Q_JOBS + PULL_PER_LINK;
constexpr int N_COMM = 128;
constexpr int N_PULL = (NR - 1) * PULL_PER_LINK;
constexpr int N_GEMM = 128;
constexpr int GRID = N_COMM + N_GEMM;
constexpr int BLOCK = 512;
constexpr int N_PACKW = N_COMM * 6;
constexpr int NGRP = NT / 8;
static_assert(N_PULL <= 2 * N_COMM, "pull waves are waves 0-1 of the comm workgroups");

constexpr unsigned XT_T = NR * MM * 16, WT_T = NR * NCOL * 16;
constexpr unsigned XT_OFF = 0;
constexpr unsigned WT_OFF = XT_OFF + NT * XT_T;
constexpr unsigned WS_OFF = WT_OFF + NT * WT_T;
constexpr unsigned OB_SLOT = 1024 + 8192;
constexpr unsigned OB_OFF = WS_OFF + NT * WT_T;
constexpr unsigned CTL_OFF = OB_OFF + NR * 128 * OB_SLOT;
constexpr unsigned PK_OFF = CTL_OFF;
constexpr unsigned FL_OFF = PK_OFF + NR * NGRP * 8192;
constexpr unsigned Q_OFF = FL_OFF + NBLK * NR * NT * 4;
constexpr unsigned TMAX_OFF = Q_OFF + NR * 128;
constexpr unsigned AMAX_OFF = TMAX_OFF + N_GEMM * 128;
constexpr unsigned OF_OFF = AMAX_OFF + 64 * 128;
constexpr unsigned CTR_OFF = OF_OFF + NR * 128 * 128;
constexpr unsigned TMO_OFF = CTR_OFF + GRID * 128 + 4;
constexpr unsigned DBG_OFF = CTR_OFF + GRID * 128 + 128;
constexpr size_t WS_NEED = DBG_OFF + 262144;

constexpr u64 DEADLINE = 50000000ull;
constexpr int AUX_SYS = 17;

constexpr int NSTEP = 2 * NT;
constexpr int NSLOT = 5;
constexpr int LDS_SLOT = 32768;
constexpr int LDS_BYTES = NSLOT * LDS_SLOT;
constexpr int LDS_WAVE = 16384;
constexpr int LDS_MISC = 8 * LDS_WAVE;

struct Params {
    const char* x; const char* w;
    int rank, pad;
    char* ws[NR];
    char* out;
};

__device__ __forceinline__ __amdgpu_buffer_rsrc_t rsrc_of(const void* p, unsigned bytes) {
    return __builtin_amdgcn_make_buffer_rsrc(const_cast<void*>(p), 0, bytes, 0x00020000);
}
__device__ __forceinline__ unsigned ld_sys(const void* p) {
    return __hip_atomic_load((const GAS unsigned*)p, __ATOMIC_RELAXED, __HIP_MEMORY_SCOPE_SYSTEM);
}
__device__ __forceinline__ u64 ld_sys64(const void* p) {
    return __hip_atomic_load((const GAS u64*)p, __ATOMIC_RELAXED, __HIP_MEMORY_SCOPE_SYSTEM);
}
__device__ __forceinline__ void st_sys(void* p, unsigned v) {
    __hip_atomic_store((GAS unsigned*)p, v, __ATOMIC_RELAXED, __HIP_MEMORY_SCOPE_SYSTEM);
}
__device__ __forceinline__ void st_sys64(void* p, unsigned lo, unsigned hi) {
    __hip_atomic_store((GAS u64*)p, ((u64)hi << 32) | lo, __ATOMIC_RELAXED, __HIP_MEMORY_SCOPE_SYSTEM);
}
__device__ __forceinline__ unsigned ld_agent(const void* p) {
    return __hip_atomic_load((const GAS unsigned*)p, __ATOMIC_RELAXED, __HIP_MEMORY_SCOPE_AGENT);
}
__device__ __forceinline__ unsigned add_agent_lane0(void* p, int lane) {
    unsigned v = 0, z = 0;
    asm volatile("" : "+v"(z));
    if (lane == 0) v = __hip_atomic_fetch_add((GAS unsigned*)((char*)p + z), 1u, __ATOMIC_RELAXED, __HIP_MEMORY_SCOPE_AGENT);
    return v;
}
__device__ __forceinline__ u32x4 transposed(const u32x4 (&v)[8], int i) {
    u32x4 o;
#pragma unroll
    for (int d = 0; d < 4; ++d)
        o[d] = __builtin_amdgcn_perm(v[2 * d + 1][i >> 1], v[2 * d][i >> 1], (i & 1) ? 0x07060302u : 0x05040100u);
    return o;
}
constexpr int NAN_BITS = 0x7fc00000;
#define ROW_SHR_MAX(v, n) v = max(v, __builtin_amdgcn_update_dpp(0, v, 0x110 + (n), 0xf, 0xf, true))
__device__ __forceinline__ int wave_max(int v) {
    ROW_SHR_MAX(v, 1); ROW_SHR_MAX(v, 2); ROW_SHR_MAX(v, 4); ROW_SHR_MAX(v, 8);
    return max(max(__builtin_amdgcn_readlane(v, 15), __builtin_amdgcn_readlane(v, 31)),
               max(__builtin_amdgcn_readlane(v, 47), __builtin_amdgcn_readlane(v, 63)));
}
__device__ __forceinline__ unsigned lanes_below(u64 m) {
    return __builtin_amdgcn_mbcnt_hi((unsigned)(m >> 32), __builtin_amdgcn_mbcnt_lo((unsigned)m, 0u));
}
template <int SLEEP = 1>
__device__ __forceinline__ bool give_up(char* ws, unsigned call, u64 t0, unsigned& spins) {
    bool stop = __builtin_amdgcn_s_memrealtime() - t0 > DEADLINE;
    if ((++spins & 63u) == 0u) stop = stop || __builtin_amdgcn_readfirstlane(ld_agent(ws + TMO_OFF)) == call;
    if (stop) {
        __hip_atomic_store((GAS unsigned*)(ws + TMO_OFF), call, __ATOMIC_RELAXED, __HIP_MEMORY_SCOPE_AGENT);
        return true;
    }
    __builtin_amdgcn_s_sleep(SLEEP);
    return false;
}
template <int SLEEP>
__device__ __forceinline__ bool await_word(char* ws, const char* p, unsigned call) {
    const u64 t0 = __builtin_amdgcn_s_memrealtime(); unsigned sp = 0;
    while ((int)(__builtin_amdgcn_readfirstlane(ld_sys(p)) - call) < 0)
        if (give_up<SLEEP>(ws, call, t0, sp)) return false;
    return true;
}
__device__ __forceinline__ unsigned fl_word(int blk, int src, int t) { return FL_OFF + (unsigned)(((blk * 4 + (src & 3)) * NT + t) * 2 + (src >> 2)) * 4u; }
__device__ __forceinline__ int steps_known(unsigned f0, unsigned f1, unsigned call) {
    const u64 lo = __ballot((int)(f0 - call) >= 0), hi = __ballot((int)(f1 - call) >= 0);
    const u64 miss = ~(lo & hi);
    if (miss == 0) return NSTEP;
    const int c = __builtin_ctzll(miss);
    return 2 * c + (int)((lo >> c) & 1);
}
__device__ __forceinline__ unsigned out_base(int src, int tile, int wave, int lane) {
    return (unsigned)(((tile >> 3) * 256 + (wave >> 1) * 64 + (lane >> 4) * 4) * NN + src * NCOL + (tile & 7) * 256 + (wave & 1) * 128 + (lane & 15) * 8);
}
#define WAIT_VM(n) asm volatile("s_waitcnt vmcnt(" #n ")" ::: "memory")
#define LGKM0_BARRIER() do { __builtin_amdgcn_s_waitcnt(0xC07F); asm volatile("" ::: "memory");                       \
        __builtin_amdgcn_s_barrier(); asm volatile("" ::: "memory"); } while (0)
#define STAMP(i) do {} while (0)

__global__ void __launch_bounds__(BLOCK, 2) fused_gemm_relu_q8(const Params p) {
    __shared__ __attribute__((aligned(1024))) char smem[LDS_BYTES];
    LAS char* const lds = (LAS char*)smem;
    const int hb = blockIdx.x;
    const int lane = threadIdx.x & 63;
    const int wid = __builtin_amdgcn_readfirstlane(threadIdx.x >> 6);
    const int rank = p.rank;
    char* const ws = p.ws[0];
    const u64 t_start = __builtin_amdgcn_s_memrealtime();
    const unsigned ticket = add_agent_lane0(ws + CTR_OFF + (size_t)hb * 128 + wid * 4, lane);
    const int b = hb & (N_COMM - 1);

    if (hb < N_COMM) {
        const __amdgpu_buffer_rsrc_t wsr = rsrc_of(ws, CTL_OFF);
        const int cw = wid * N_COMM + b;
        const int dbg = 128 + cw;
        (void)dbg;
        if (threadIdx.x == 256) *(LAS unsigned*)(lds + LDS_MISC) = 0u;
        __syncthreads();
        STAMP(0);
        if (wid < 2) {
            if (cw < N_PULL) {
                const unsigned s = (unsigned)(rank + 1 + cw % (NR - 1)) & 7u;
                const __amdgpu_buffer_rsrc_t srs = rsrc_of(p.ws[1 + cw % (NR - 1)], CTL_OFF);
                char* const queue = ws + Q_OFF + s * 128;
                unsigned tk = add_agent_lane0(queue, lane);
                u32x4 cur[4], nxt[4];
                const unsigned call = __builtin_amdgcn_readfirstlane(ticket) + 1u;
                const unsigned qbase = (call - 1u) * Q_TICKETS;
                const char* const pk = ws + PK_OFF + s * NGRP * 8192 + lane * 128;
                unsigned pv = call - 1u;
                int pgrp = 0;
#define JOB_OF(n_, t_, j_, so_, do_) const int t_ = (int)(n_) / NBLK, j_ = (int)(n_) % NBLK;                           \
                const unsigned do_ = (j_ < 16 ? XT_OFF + t_ * XT_T + j_ * 32768 : WT_OFF + t_ * WT_T + (j_ - 16) * 32768) + s * 4096 + lane * 16; \
                const unsigned so_ = j_ < 16 ? do_ : WS_OFF + t_ * WT_T + rank * 32768 + (j_ - 16) * 4096 + lane * 16
#define WAIT_PACKED(t_, j_) do { const int sl_ = (j_) < 16 ? (j_) : 24 + ((t_) & 7) * 4 + ((j_) >> 1);                 \
                if (((t_) >> 3) != pgrp) { pgrp = (t_) >> 3; pv = call - 1u; }                                         \
                if ((int)(__builtin_amdgcn_readlane(pv, sl_) - call) < 0) {                                            \
                    pv = ld_sys(pk + pgrp * 8192);                 \
                    const u64 t0_ = __builtin_amdgcn_s_memrealtime(); unsigned sp_ = 0;                                \
                    while ((int)(__builtin_amdgcn_readlane(pv, sl_) - call) < 0) {                                     \
                        if (give_up(ws, call, t0_, sp_)) break;                                                        \
                        if (lane == sl_) pv = ld_sys(pk + pgrp * 8192); } } } while (0)
                unsigned n = __builtin_amdgcn_readfirstlane(tk) - qbase;
                if (n < (unsigned)Q_JOBS) {
                    JOB_OF(n, t, j, so, dof);
                    WAIT_PACKED(t, j);
                    STAMP(2);
                    tk = add_agent_lane0(queue, lane);
#pragma unroll
                    for (int i = 0; i < 4; ++i) cur[i] = __builtin_amdgcn_raw_buffer_load_b128(srs, so + i * 1024, 0, AUX_SYS);
                    unsigned dcur = dof, fcur = fl_word(j, s, t);
                    for (;;) {
                        n = __builtin_amdgcn_readfirstlane(tk) - qbase;
                        const bool more = n < (unsigned)Q_JOBS;
                        unsigned dnxt = 0, fnxt = 0;
                        if (more) {
                            JOB_OF(n, t2, j2, so2, dof2);
                            WAIT_PACKED(t2, j2);
                            tk = add_agent_lane0(queue, lane);
#pragma unroll
                            for (int i = 0; i < 4; ++i) nxt[i] = __builtin_amdgcn_raw_buffer_load_b128(srs, so2 + i * 1024, 0, AUX_SYS);
                            dnxt = dof2; fnxt = fl_word(j2, s, t2);
                        }
#pragma unroll
                        for (int i = 0; i < 4; ++i) __builtin_amdgcn_raw_buffer_store_b128(cur[i], wsr, dcur + i * 1024, 0, AUX_SYS);
                        WAIT_VM(0);
                        if (lane == 0) st_sys(ws + fcur, call);
                        if (!more) break;
                        dcur = dnxt; fcur = fnxt;
#pragma unroll
                        for (int i = 0; i < 4; ++i) cur[i] = nxt[i];
                    }
                }
                STAMP(1);
            }
        } else {
            const __amdgpu_buffer_rsrc_t xr = rsrc_of(p.x, MM * KSH * 2);
            const __amdgpu_buffer_rsrc_t wr_ = rsrc_of(p.w, KSH * NN * 2);
            bool x_todo = wid >= 4;
            for (int n = (wid - 2) * N_COMM + b; n < NT * 32 || x_todo;) {
                const bool do_x = x_todo && ((b >> 4) <= (n >> 8) || n >= NT * 32);
                const int g = do_x ? b >> 4 : n >> 8;
                while (__builtin_amdgcn_s_memrealtime() - t_start < (u64)((do_x ? g * 8 : n >> 5) * PACK_DELAY)) __builtin_amdgcn_s_sleep(2);
                u32x4 v[8];
                if (do_x) {
                    const unsigned rb = (wid - 4) * 64 + (lane & 7), piece = lane >> 3;
#pragma unroll
                    for (int i = 0; i < 8; ++i) v[i] = __builtin_amdgcn_raw_buffer_load_b128(xr, ((b & 15) * 256 + rb + i * 8) * (KSH * 2) + g * 128 + piece * 16, 0, 0);
                    const unsigned d0 = XT_OFF + (g * 8 + piece) * XT_T + (b & 15) * 32768 + rank * 4096 + rb * 16;
#pragma unroll
                    for (int i = 0; i < 8; ++i) __builtin_amdgcn_raw_buffer_store_b128(v[i], wsr, d0 + i * 128, 0, AUX_SYS);
                    char* fl = lane < 8 ? p.ws[lane & 7] + PK_OFF + (rank * NGRP + g) * 8192 + (b & 15) * 128 : ws + fl_word(b & 15, rank, g * 8 + (lane & 7));
                    asm volatile("s_waitcnt vmcnt(0)" : "+v"(fl) :: "memory");
                    unsigned old = 0;
                    if (lane == 0) old = __hip_atomic_fetch_add((LAS unsigned*)(lds + LDS_MISC), 1u, __ATOMIC_RELAXED, __HIP_MEMORY_SCOPE_WORKGROUP);
                    const unsigned call = __builtin_amdgcn_readfirstlane(ticket) + 1u;
                    if (__builtin_amdgcn_readfirstlane(old) == 3u && lane >= 1 && lane < 16) st_sys(fl, call);
                    x_todo = false;
                } else {
                    const int t = n >> 5, cb = n & 31;
                    const unsigned dest = (unsigned)cb >> 2;
#pragma unroll
                    for (int i = 0; i < 8; ++i) v[i] = __builtin_amdgcn_raw_buffer_load_b128(wr_, (unsigned)((t * 8 + i) * (NN * 2) + cb * 1024 + lane * 16), 0, 0);
                    const unsigned d0 = (dest == (unsigned)rank ? WT_OFF + t * WT_T + ((cb & 3) * 2 + (lane >> 5)) * 32768 + rank * 4096
                                                                : WS_OFF + t * WT_T + dest * 32768 + ((cb & 3) * 2 + (lane >> 5)) * 4096)
                                      + (((lane >> 4) & 1) * 128 + (lane & 15)) * 16;
#pragma unroll
                    for (int i = 0; i < 8; ++i) __builtin_amdgcn_raw_buffer_store_b128(transposed(v, i), wsr, d0 + i * 256, 0, AUX_SYS);
                    char* const dws = p.ws[(dest - rank) & 7u];
                    WAIT_VM(0);
                    const unsigned call = __builtin_amdgcn_readfirstlane(ticket) + 1u;
                    if (dest == (unsigned)rank) { if (lane < 2) st_sys(ws + fl_word(16 + (cb & 3) * 2 + lane, rank, t), call); }
                    else if (lane == 0) st_sys(dws + PK_OFF + (rank * NGRP + g) * 8192 + (32 + (t & 7) * 4 + (cb & 3)) * 128, call);
                    n += N_PACKW;
                }
            }
        }
        const unsigned call = __builtin_amdgcn_readfirstlane(ticket) + 1u;
        if (wid == 7) {
            if (b == N_COMM - 1) {
                char* amax_out = p.ws[lane & 7] + AMAX_OFF + ((lane >> 3) * NR + rank) * 128;
                asm volatile("" : "+v"(amax_out));
                const char* tm = ws + TMAX_OFF + lane * 256;
                const u64 t0 = __builtin_amdgcn_s_memrealtime(); unsigned sp = 0;
                int mx;
                for (;;) {
                    const u64 v0 = ld_sys64(tm), v1 = ld_sys64(tm + 128);
                    mx = max((int)(unsigned)v0, (int)(unsigned)v1);
                    if (__all((int)((unsigned)(v0 >> 32) - call) >= 0 && (int)((unsigned)(v1 >> 32) - call) >= 0)) break;
                    if (give_up(ws, call, t0, sp)) { mx = NAN_BITS; break; }
                }
                if (lane < 8 * AMAX_REP) st_sys64(amax_out, (unsigned)wave_max(mx), call);
                STAMP(6);
            }
            return;
        }
        {
            const int src = (rank + 1 + wid) & 7;
            const unsigned slot = OB_OFF + (unsigned)(src * 128 + b) * OB_SLOT;
            const char* const of = ws + OF_OFF + (src * 128 + b) * 128;
            const unsigned lbase = wid * LDS_WAVE;
            if (lane == 0) *(LAS unsigned*)(lds + lbase + 8192) = 0u;
            char* outp = p.out;
            asm volatile("" : "+s"(outp));
            const unsigned ob = out_base(src, b >> 3, b & 7, lane);
            {
                bool ok = await_word<8>(ws, of, call);
                const u32x4 mk = __builtin_amdgcn_raw_buffer_load_b128(wsr, slot + lane * 16, 0, AUX_SYS);
                unsigned ad[128], pos = lbase;
#pragma unroll
                for (int r = 0; r < 128; ++r) {
                    const bool kept = (int)(mk[r >> 5] << (r & 31)) >= 0;
                    const u64 m = __ballot(kept);
                    unsigned a = pos + lanes_below(m);
                    asm("" : "+v"(a));
                    ad[r] = kept ? a : lbase + 8192;
                    pos += __builtin_popcountll(m);
                    asm("" : "+s"(pos));
                }
#pragma unroll
                for (int r = 0; r < 128; r += 16)
                    asm volatile("" : "+v"(ad[r]), "+v"(ad[r + 1]), "+v"(ad[r + 2]), "+v"(ad[r + 3]), "+v"(ad[r + 4]), "+v"(ad[r + 5]), "+v"(ad[r + 6]), "+v"(ad[r + 7]),
                                      "+v"(ad[r + 8]), "+v"(ad[r + 9]), "+v"(ad[r + 10]), "+v"(ad[r + 11]), "+v"(ad[r + 12]), "+v"(ad[r + 13]), "+v"(ad[r + 14]), "+v"(ad[r + 15]));
                STAMP(3);
                ok = await_word<1>(ws, of + 4, call) && ok;
                STAMP(4);
                const char* gp = ws + slot + 1024 + lane * 16;
                for (unsigned o = 0; o < pos - lbase; o += 1024)
                    __builtin_amdgcn_global_load_lds((const GAS unsigned*)(gp + o), (LAS unsigned*)(lds + lbase + o), 16, 0, AUX_SYS);
                const unsigned fill = ok && __builtin_amdgcn_readfirstlane(ld_agent(ws + TMO_OFF)) != call ? 0u : 0x7f7f7f7fu;
                WAIT_VM(0);
                const __amdgpu_buffer_rsrc_t ors = rsrc_of(outp, (MM / NR) * NN);
#pragma unroll
                for (int r = 0; r < 128; ++r) ad[r] = *(const LAS unsigned char*)(lds + ad[r]);
                __builtin_amdgcn_sched_barrier(0);
#pragma unroll
                for (int u = 0; u < 16; ++u) {
                    const unsigned* c = ad + u * 8;
                    u32x2 o;
                    o[0] = (c[0] | (c[1] << 8) | (c[2] << 16) | (c[3] << 24)) | fill;
                    o[1] = (c[4] | (c[5] << 8) | (c[6] << 16) | (c[7] << 24)) | fill;
                    __builtin_amdgcn_raw_buffer_store_b64(o, ors, ob + (unsigned)(((u >> 2) * 16 + (u & 3)) * NN), 0, 0);
                }
            }
        }
        return;
    }

    const int g = b;
    const int dbg = g;
    (void)dbg;
    const int pm = (g >> 4) * 2 + (g & 1);
    const int pn = ((g & 7) >> 1) * 2 + ((g >> 3) & 1);
    const int fr = lane & 15, fq = lane >> 4, wr = wid >> 1, wc = wid & 1;
    if (wid == 0) STAMP(0);
    const int owner = pm >> 1, tile = (pm & 1) * 8 + pn;
    char* ows = p.ws[(owner - rank) & 7];
    char* outp = p.out;
    asm volatile("" : "+s"(ows), "+s"(outp));
    if (wid == 0) __builtin_amdgcn_fence(__ATOMIC_ACQUIRE, "");
    const unsigned call = __builtin_amdgcn_readfirstlane(ticket) + 1u;
    WAIT_VM(0);
    __builtin_amdgcn_s_barrier();

    const int op = wid >> 2;
    const char* const pollp = ws + fl_word(op ? 16 + pn : pm, wid & 3, lane);
    const unsigned unit = op ? WT_T : XT_T;
    const char* const gbase = ws + (op ? WT_OFF + pn * 32768 : XT_OFF + pm * 32768) + (wid & 3) * 4096 + lane * 16;
    LAS char* const ldst = lds + wid * 4096;
    const int aoff = fq * 4096 + (wr * 64 + fr) * 16;
    const int boff = 16384 + fq * 4096 + (wc * 128 + fr) * 16;

    int known = 0;
    int next = 0, nslot = 0;
    u32x2 pvn = {0u, 0u};
#define STAGE_NEXT() do { const char* g_ = gbase + ((next & 1) * 16384u + (unsigned)(next >> 1) * unit);              \
        LAS char* l_ = ldst + nslot * LDS_SLOT;                                                                       \
        __builtin_amdgcn_global_load_lds((const GAS unsigned*)g_, (LAS unsigned*)l_, 16, 0, 0);                       \
        __builtin_amdgcn_global_load_lds((const GAS unsigned*)g_, (LAS unsigned*)l_, 16, 1024, 0);                    \
        __builtin_amdgcn_global_load_lds((const GAS unsigned*)g_, (LAS unsigned*)l_, 16, 2048, 0);                    \
        __builtin_amdgcn_global_load_lds((const GAS unsigned*)g_, (LAS unsigned*)l_, 16, 3072, 0);                    \
        ++next; nslot = nslot == NSLOT - 1 ? 0 : nslot + 1; } while (0)
#define POLL_ISSUE() asm volatile("global_load_dwordx2 %0, %1, off sc0 sc1" : "+v"(pvn) : "v"(pollp) : "memory")
#define POLL_TAKE() do { u32x2 f_; asm volatile("s_waitcnt vmcnt(0)\n\tv_mov_b64 %0, %1" : "=v"(f_) : "v"(pvn) : "memory");    \
        known = steps_known(f_[0], f_[1], call); } while (0)
#define AWAIT_NEXT() do { if (next >= known) {                                                                        \
            const u64 t0_ = __builtin_amdgcn_s_memrealtime(); unsigned sp_ = 0;                                       \
            for (;;) { const u64 f_ = ld_sys64(pollp); known = steps_known((unsigned)f_, (unsigned)(f_ >> 32), call); \
                if (next < known) break;                                                                              \
                if (give_up<2>(ws, call, t0_, sp_)) { known = NSTEP; break; } } } } while (0)
#define MFMA_COLS(A_, B_, n0_, n1_) do { __builtin_amdgcn_sched_barrier(0);                                           \
        _Pragma("unroll") for (int n = (n0_); n < (n1_); ++n)                                                         \
            _Pragma("unroll") for (int m = 0; m < 4; ++m)                                                             \
                acc[m][n] = __builtin_amdgcn_mfma_f32_16x16x32_bf16(A_[m], B_[n], acc[m][n], 0, 0, 0);                \
        __builtin_amdgcn_sched_barrier(0); } while (0)
#define RD_A(A_, m_) A_[m_] = *(const LAS bf16x8*)(lb_ + aoff + (m_) * 256)
#define RD_B(B_, n_) B_[n_] = *(const LAS bf16x8*)(lb_ + boff + (n_) * 256)
#define KSTEP(st_, A_, B_, AN_, BN_) do {                                                                             \
        const LAS char* lb_ = lds + rslot * LDS_SLOT;               \
        MFMA_COLS(A_, B_, 0, 1);                                                                                      \
        RD_A(AN_, 0); RD_A(AN_, 1); RD_A(AN_, 2);                                                                     \
        const bool polled_ = known < min(next + 6, NSTEP);                                                            \
        if (polled_) POLL_ISSUE();                                                                                    \
        MFMA_COLS(A_, B_, 1, 2);                                                                                      \
        RD_A(AN_, 3); RD_B(BN_, 0); RD_B(BN_, 1);                                                                     \
        MFMA_COLS(A_, B_, 2, 3);                                                                                      \
        RD_B(BN_, 2); RD_B(BN_, 3); RD_B(BN_, 4);                                                                     \
        MFMA_COLS(A_, B_, 3, 4);                                                                                      \
        RD_B(BN_, 5); RD_B(BN_, 6); RD_B(BN_, 7);                                                                     \
        rslot = rslot == NSLOT - 1 ? 0 : rslot + 1;                                                                   \
        MFMA_COLS(A_, B_, 4, 6);                                                                                      \
        if (polled_) POLL_TAKE();                                         \
        const int had_ = next;                                                                                        \
        const int lim_ = min((st_) + NSLOT + 1, known);                                                               \
        if (next < lim_) { STAGE_NEXT(); if (next < lim_) STAGE_NEXT(); }                                             \
        const bool full_ = (polled_ ? had_ + NSLOT - 2 : next) >= (st_) + NSLOT + 1;                                  \
        MFMA_COLS(A_, B_, 6, 8);                                                                                      \
        if (!full_) {                                                              \
            if (next == (st_) + 2 && next < NSTEP) {                                                                  \
                AWAIT_NEXT(); STAGE_NEXT();                                                                           \
                if (next < known) STAGE_NEXT();                                                                       \
            }                                                                                                         \
            const int k_ = next - (st_) - 3;                                                                          \
            if (k_ <= 0) WAIT_VM(0); else if (k_ == 1) WAIT_VM(4); else WAIT_VM(8);                                   \
        }                                                                                                             \
        WAIT_VM(12);                                                                                                  \
        LGKM0_BARRIER(); } while (0)

    f32x4 acc[4][8];
#pragma unroll
    for (int m = 0; m < 4; ++m)
#pragma unroll
        for (int n = 0; n < 8; ++n) acc[m][n] = (f32x4){0.f, 0.f, 0.f, 0.f};

    AWAIT_NEXT(); STAGE_NEXT();
    AWAIT_NEXT(); STAGE_NEXT();
    WAIT_VM(0);
    LGKM0_BARRIER();
    int rslot = 1;
    bf16x8 a0[4], b0[8], a1[4], b1[8];
    {
        const LAS char* lb_ = lds;
#pragma unroll
        for (int m = 0; m < 4; ++m) RD_A(a0, m);
#pragma unroll
        for (int n = 0; n < 8; ++n) RD_B(b0, n);
    }
    LGKM0_BARRIER();
    if (wid == 0) STAMP(1);
    for (int st = 0; st < NSTEP; st += 2) {
        KSTEP(st, a0, b0, a1, b1);
        KSTEP(st + 1, a1, b1, a0, b0);
    }
    if (wid == 0) STAMP(2);

    int mx = 0;
#pragma unroll
    for (int m = 0; m < 4; ++m)
#pragma unroll
        for (int n = 0; n < 8; ++n)
#pragma unroll
            for (int j = 0; j < 4; ++j) mx = max(mx, __float_as_int(acc[m][n][j]));
    mx = wave_max(mx);
    LAS int* red = (LAS int*)(lds + LDS_MISC);
    if (lane == 0) red[wid] = mx;
    LGKM0_BARRIER();
    if (wid == 0) {
        int t = red[lane & 7];
        ROW_SHR_MAX(t, 1); ROW_SHR_MAX(t, 2); ROW_SHR_MAX(t, 4);
        if (lane == 7) st_sys64(ws + TMAX_OFF + g * 128, (unsigned)t, call);
        STAMP(7);
    }

    const bool remote = owner != rank;
    const __amdgpu_buffer_rsrc_t sls = rsrc_of(ows + OB_OFF + (unsigned)(rank * 128 + tile * 8 + wid) * OB_SLOT, OB_SLOT);
    char* const of = ows + OF_OFF + (rank * 128 + tile * 8 + wid) * 128;
    const int efield = min(268 - (mx >> 23), 253);
    const float sc = __int_as_float(efield << 23);
    const unsigned pos0 = (unsigned)(size_t)(lds + wid * LDS_WAVE);
    const unsigned spill = (unsigned)(size_t)(lds + LDS_MISC + 64);
    unsigned pos = pos0;
#define VAL(r_) acc[(r_) >> 5][(r_) & 7][((r_) >> 3) & 3]
#define PACK2(r_) do { unsigned ta_, tb_, ha_, hb_, c_, p1_;                                                          \
        asm volatile("v_cmp_lt_i32 vcc, -1, %[a]\n\tv_cmp_lt_i32 s[88:89], -1, %[b]\n\t"                             \
                     "v_fma_mixlo_f16 %[ha], %[a], %[sc], 0\n\ts_bcnt1_i32_b64 %[c], vcc\n\t"                        \
                     "v_mbcnt_lo_u32_b32 %[ta], vcc_lo, 0\n\tv_fma_mixlo_f16 %[hb], %[b], %[sc], 0\n\t"              \
                     "s_lshl1_add_u32 %[p1], %[c], %[pos]\n\tv_mbcnt_lo_u32_b32 %[tb], s88, 0\n\t"                   \
                     "s_bcnt1_i32_b64 %[c], s[88:89]\n\tv_mbcnt_hi_u32_b32 %[ta], vcc_hi, %[ta]\n\t"                 \
                     "v_mbcnt_hi_u32_b32 %[tb], s89, %[tb]\n\tv_lshl_add_u32 %[ta], %[ta], 1, %[pos]\n\t"            \
                     "v_lshl_add_u32 %[tb], %[tb], 1, %[p1]\n\ts_lshl1_add_u32 %[pos], %[c], %[p1]\n\t"              \
                     "v_cndmask_b32 %[ta], %[sp], %[ta], vcc\n\tv_cndmask_b32 %[tb], %[sp], %[tb], s[88:89]\n\t"     \
                     "ds_write_b16 %[ta], %[ha]\n\tds_write_b16 %[tb], %[hb]"                                        \
                     : [ta] "=&v"(ta_), [tb] "=&v"(tb_), [ha] "=&v"(ha_), [hb] "=&v"(hb_), [c] "=&s"(c_),             \
                       [p1] "=&s"(p1_), [pos] "+s"(pos)                                                               \
                     : [a] "v"(VAL(r_)), [b] "v"(VAL((r_) + 1)), [sc] "s"(sc), [sp] "v"(spill)                        \
                     : "vcc", "scc", "s88", "s89", "memory"); } while (0)
#define PACK8(r_) PACK2(r_); PACK2((r_) + 2); PACK2((r_) + 4); PACK2((r_) + 6)
    if (remote) {
        u32x4 mk = {0u, 0u, 0u, 0u};
#pragma unroll
        for (int r = 0; r < 128; ++r) mk[r >> 5] = __builtin_amdgcn_alignbit(mk[r >> 5], __float_as_uint(VAL(r)), 31);
#define MASK_OUT(k_) do { if (MASK_AT == (k_)) __builtin_amdgcn_raw_buffer_store_b128(mk, sls, lane * 16, 0, AUX_SYS); } while (0)
        MASK_OUT(0); PACK8(0); MASK_OUT(1); PACK8(8); MASK_OUT(2); PACK8(16); MASK_OUT(3); PACK8(24); MASK_OUT(4); PACK8(32); MASK_OUT(5);
        if (MASK_AT < 3) { WAIT_VM(0); if (lane == 0) st_sys(of, call); }
    } else {
#pragma unroll
        for (int m = 0; m < 4; ++m) {
#pragma unroll
            for (int n = 0; n < 8; ++n)
#pragma unroll
                for (int j = 0; j < 4; ++j) acc[m][n][j] = __int_as_float(max(0, __float_as_int(acc[m][n][j])));
            asm volatile("" : "+v"(acc[m][0]), "+v"(acc[m][1]), "+v"(acc[m][2]), "+v"(acc[m][3]),
                              "+v"(acc[m][4]), "+v"(acc[m][5]), "+v"(acc[m][6]), "+v"(acc[m][7]));
        }
    }
    if (wid == 1) STAMP(6);
    if (wid == 0) {
        const char* ap = lane < NR ? ws + AMAX_OFF + ((g & (AMAX_REP - 1)) * NR + lane) * 128 : ws + TMO_OFF - 4;
        const u64 t0 = __builtin_amdgcn_s_memrealtime(); unsigned sp = 0;
        u64 v = 0;
        for (;;) {
            if (lane <= NR) v = ld_sys64(ap);
            if ((__ballot((int)((unsigned)(v >> 32) - call) >= 0) & 0xffu) == 0xffu) break;
            if (give_up(ws, call, t0, sp)) { v = (u64)call << 32; break; }
        }
        STAMP(8);
        int t = lane < NR ? (int)(unsigned)v : ((unsigned)(v >> 32) == call ? NAN_BITS : 0);
        ROW_SHR_MAX(t, 1); ROW_SHR_MAX(t, 2); ROW_SHR_MAX(t, 4); ROW_SHR_MAX(t, 8);
        if (lane == NR) red[8] = t;
    }
    LGKM0_BARRIER();
    const float inv = 448.0f / __int_as_float(red[8]);
    if (wid == 0) STAMP(3);

    if (remote) {
        const float f = inv * __int_as_float((254 - efield) << 23);
        unsigned sent = 0;
#define SEND(upto_) do { __builtin_amdgcn_s_waitcnt(0xC07F); asm volatile("" ::: "memory");                           \
            const unsigned cnt_ = (pos - pos0) >> 1;                                                                  \
            _Pragma("nounroll") for (; sent < (upto_); sent += 1024) {                                                                    \
                const unsigned o_ = sent + lane * 16;                                                                 \
                const f16x8 h0 = *(const LAS f16x8*)(lds + wid * LDS_WAVE + o_ * 2), h1 = *(const LAS f16x8*)(lds + wid * LDS_WAVE + o_ * 2 + 16); \
                u32x4 q;                                                                                              \
                _Pragma("unroll") for (int d = 0; d < 2; ++d) {                                                       \
                    int w0 = __builtin_amdgcn_cvt_pk_fp8_f32(__builtin_fmaf((float)h0[4 * d], f, 0.f), __builtin_fmaf((float)h0[4 * d + 1], f, 0.f), 0, false); \
                    q[d] = (unsigned)__builtin_amdgcn_cvt_pk_fp8_f32(__builtin_fmaf((float)h0[4 * d + 2], f, 0.f), __builtin_fmaf((float)h0[4 * d + 3], f, 0.f), w0, true); \
                    int w1 = __builtin_amdgcn_cvt_pk_fp8_f32(__builtin_fmaf((float)h1[4 * d], f, 0.f), __builtin_fmaf((float)h1[4 * d + 1], f, 0.f), 0, false); \
                    q[2 + d] = (unsigned)__builtin_amdgcn_cvt_pk_fp8_f32(__builtin_fmaf((float)h1[4 * d + 2], f, 0.f), __builtin_fmaf((float)h1[4 * d + 3], f, 0.f), w1, true); \
                }                                                                                                     \
                if (o_ < cnt_) __builtin_amdgcn_raw_buffer_store_b128(q, sls, 1024 + o_, 0, AUX_SYS);                 \
            } } while (0)
        if (MASK_AT >= 3) { WAIT_VM(0); if (lane == 0) st_sys(of, call); }
        SEND(cnt_ & ~1023u);
        PACK8(40); PACK8(48); PACK8(56); PACK8(64);
        SEND(cnt_ & ~1023u);
        PACK8(72); PACK8(80); PACK8(88); PACK8(96);
        SEND(cnt_ & ~1023u);
        PACK8(104); PACK8(112); PACK8(120);
        SEND(cnt_);
        WAIT_VM(0);
        if (lane == 0) st_sys(of + 4, call);
    } else {
        const __amdgpu_buffer_rsrc_t ors = rsrc_of(outp, (MM / NR) * NN);
        const unsigned ob = out_base(rank, tile, wid, lane);
#pragma unroll
        for (int m = 0; m < 4; ++m)
#pragma unroll
            for (int j = 0; j < 4; ++j) {
                u32x2 o;
                int lo = __builtin_amdgcn_cvt_pk_fp8_f32(acc[m][0][j] * inv, acc[m][1][j] * inv, 0, false);
                o[0] = (unsigned)__builtin_amdgcn_cvt_pk_fp8_f32(acc[m][2][j] * inv, acc[m][3][j] * inv, lo, true);
                int hi = __builtin_amdgcn_cvt_pk_fp8_f32(acc[m][4][j] * inv, acc[m][5][j] * inv, 0, false);
                o[1] = (unsigned)__builtin_amdgcn_cvt_pk_fp8_f32(acc[m][6][j] * inv, acc[m][7][j] * inv, hi, true);
                __builtin_amdgcn_raw_buffer_store_b64(o, ors, ob + (unsigned)((m * 16 + j) * NN), 0, 0);
            }
    }
    if (wid == 0) STAMP(4);
    if (wid == 7) STAMP(5);
}

}

extern "C" void kernel_launch(int rank, int nranks, ncclComm_t comm,
                              hipStream_t stream,
                              void* const* d_in, const int* in_sizes, int n_in,
                              void* d_out, int out_size,
                              const int* in_dtypes, int out_dtype,
                              void* const* d_out_all,
                              void* const* d_ws_all, size_t ws_bytes) {
    (void)comm; (void)in_dtypes; (void)out_dtype; (void)d_out_all;
    if (nranks != NR || n_in < 2 || (unsigned)rank >= (unsigned)NR || ws_bytes < WS_NEED ||
        in_sizes[0] != MM * KSH || in_sizes[1] != KSH * NN || out_size != (MM / NR) * NN) {
        fprintf(stderr, "kernel_launch: unexpected problem shape\n");
        return;
    }
    Params p;
    p.x = (const char*)d_in[0];
    p.w = (const char*)d_in[1];
    for (int i = 0; i < NR; ++i) p.ws[i] = (char*)d_ws_all[(rank + i) % NR];
    p.out = (char*)d_out;
    p.rank = rank; p.pad = 0;
    hipLaunchKernelGGL(fused_gemm_relu_q8, dim3(GRID), dim3(BLOCK), 0, stream, p);
    HIP_CHECK(hipGetLastError());
}
```

```cpp
#include <hip/hip_runtime.h>
#include <rccl/rccl.h>
#include <cstdio>
#include <cstdint>

#define HIP_CHECK(expr) do { hipError_t _e = (expr); if (_e != hipSuccess) { \
    fprintf(stderr, "HIP error %s at %s:%d\n", hipGetErrorString(_e), __FILE__, __LINE__); \
    return; } } while (0)

namespace {

typedef short bf16x8 __attribute__((ext_vector_type(8)));
typedef _Float16 f16x8 __attribute__((ext_vector_type(8)));
typedef float f32x4 __attribute__((ext_vector_type(4)));
typedef unsigned u32x4 __attribute__((ext_vector_type(4)));
typedef unsigned u32x2 __attribute__((ext_vector_type(2)));
typedef unsigned long long u64;
#define LAS __attribute__((address_space(3)))
#define GAS __attribute__((address_space(1)))

constexpr int NR = 8;
constexpr int MM = 4096, KSH = 512, NN = 16384, NCOL = NN / NR;
constexpr int NT = KSH / 8;
constexpr int NBLK = 24;
#define PULL_PER_LINK 17
#define MASK_AT 0
#define AMAX_REP 8
#define AUX_OUT 17
#define SEND_DEPTH 1
#define PACK_DELAY 100
constexpr int Q_JOBS = NT * NBLK;
constexpr int Q_TICKETS = Q_JOBS + PULL_PER_LINK;
constexpr int N_COMM = 128;
constexpr int N_PULL = (NR - 1) * PULL_PER_LINK;
constexpr int N_GEMM = 128;
constexpr int GRID = N_COMM + N_GEMM;
constexpr int BLOCK = 512;
constexpr int N_PACKW = N_COMM * 6;
constexpr int NGRP = NT / 8;
static_assert(N_PULL <= 2 * N_COMM, "pull waves are waves 0-1 of the comm workgroups");

constexpr unsigned XT_T = NR * MM * 16, WT_T = NR * NCOL * 16;
constexpr unsigned XT_OFF = 0;
constexpr unsigned WT_OFF = XT_OFF + NT * XT_T;
constexpr unsigned WS_OFF = WT_OFF + NT * WT_T;
constexpr unsigned OB_SLOT = 1024 + 8192;
constexpr unsigned OB_OFF = WS_OFF + NT * WT_T;
constexpr unsigned CTL_OFF = OB_OFF + NR * 128 * OB_SLOT;
constexpr unsigned PK_OFF = CTL_OFF;
constexpr unsigned FL_OFF = PK_OFF + NR * NGRP * 8192;
constexpr unsigned Q_OFF = FL_OFF + NBLK * NR * NT * 4;
constexpr unsigned TMAX_OFF = Q_OFF + NR * 128;
constexpr unsigned AMAX_OFF = TMAX_OFF + N_GEMM * 128;
constexpr unsigned OF_OFF = AMAX_OFF + 64 * 128;
constexpr unsigned CTR_OFF = OF_OFF + NR * 128 * 128;
constexpr unsigned TMO_OFF = CTR_OFF + GRID * 128 + 4;
constexpr unsigned DBG_OFF = CTR_OFF + GRID * 128 + 128;
constexpr size_t WS_NEED = DBG_OFF + 524288;

constexpr u64 DEADLINE = 50000000ull;
constexpr int AUX_SYS = 17;

constexpr int NSTEP = 2 * NT;
constexpr int NSLOT = 5;
constexpr int LDS_SLOT = 32768;
constexpr int LDS_BYTES = NSLOT * LDS_SLOT;
constexpr int LDS_WAVE = 16384;
constexpr int LDS_MISC = 8 * LDS_WAVE;

struct Params {
    const char* x; const char* w;
    int rank, pad;
    char* ws[NR];
    char* out;
};

__device__ __forceinline__ __amdgpu_buffer_rsrc_t rsrc_of(const void* p, unsigned bytes) {
    return __builtin_amdgcn_make_buffer_rsrc(const_cast<void*>(p), 0, bytes, 0x00020000);
}
__device__ __forceinline__ unsigned ld_sys(const void* p) {
    return __hip_atomic_load((const GAS unsigned*)p, __ATOMIC_RELAXED, __HIP_MEMORY_SCOPE_SYSTEM);
}
__device__ __forceinline__ u64 ld_sys64(const void* p) {
    return __hip_atomic_load((const GAS u64*)p, __ATOMIC_RELAXED, __HIP_MEMORY_SCOPE_SYSTEM);
}
__device__ __forceinline__ void st_sys(void* p, unsigned v) {
    __hip_atomic_store((GAS unsigned*)p, v, __ATOMIC_RELAXED, __HIP_MEMORY_SCOPE_SYSTEM);
}
__device__ __forceinline__ void st_sys64(void* p, unsigned lo, unsigned hi) {
    __hip_atomic_store((GAS u64*)p, ((u64)hi << 32) | lo, __ATOMIC_RELAXED, __HIP_MEMORY_SCOPE_SYSTEM);
}
__device__ __forceinline__ unsigned ld_agent(const void* p) {
    return __hip_atomic_load((const GAS unsigned*)p, __ATOMIC_RELAXED, __HIP_MEMORY_SCOPE_AGENT);
}
__device__ __forceinline__ unsigned add_agent_lane0(void* p, int lane) {
    unsigned v = 0, z = 0;
    asm volatile("" : "+v"(z));
    if (lane == 0) v = __hip_atomic_fetch_add((GAS unsigned*)((char*)p + z), 1u, __ATOMIC_RELAXED, __HIP_MEMORY_SCOPE_AGENT);
    return v;
}
__device__ __forceinline__ u32x4 transposed(const u32x4 (&v)[8], int i) {
    u32x4 o;
#pragma unroll
    for (int d = 0; d < 4; ++d)
        o[d] = __builtin_amdgcn_perm(v[2 * d + 1][i >> 1], v[2 * d][i >> 1], (i & 1) ? 0x07060302u : 0x05040100u);
    return o;
}
constexpr int NAN_BITS = 0x7fc00000;
#define ROW_SHR_MAX(v, n) v = max(v, __builtin_amdgcn_update_dpp(0, v, 0x110 + (n), 0xf, 0xf, true))
__device__ __forceinline__ int wave_max(int v) {
    ROW_SHR_MAX(v, 1); ROW_SHR_MAX(v, 2); ROW_SHR_MAX(v, 4); ROW_SHR_MAX(v, 8);
    return max(max(__builtin_amdgcn_readlane(v, 15), __builtin_amdgcn_readlane(v, 31)),
               max(__builtin_amdgcn_readlane(v, 47), __builtin_amdgcn_readlane(v, 63)));
}
__device__ __forceinline__ unsigned lanes_below(u64 m) {
    return __builtin_amdgcn_mbcnt_hi((unsigned)(m >> 32), __builtin_amdgcn_mbcnt_lo((unsigned)m, 0u));
}
template <int SLEEP = 1>
__device__ __forceinline__ bool give_up(char* ws, unsigned call, u64 t0, unsigned& spins) {
    bool stop = __builtin_amdgcn_s_memrealtime() - t0 > DEADLINE;
    if ((++spins & 63u) == 0u) stop = stop || __builtin_amdgcn_readfirstlane(ld_agent(ws + TMO_OFF)) == call;
    if (stop) {
        __hip_atomic_store((GAS unsigned*)(ws + TMO_OFF), call, __ATOMIC_RELAXED, __HIP_MEMORY_SCOPE_AGENT);
        return true;
    }
    __builtin_amdgcn_s_sleep(SLEEP);
    return false;
}
template <int SLEEP>
__device__ __forceinline__ bool await_word(char* ws, const char* p, unsigned call) {
    const u64 t0 = __builtin_amdgcn_s_memrealtime(); unsigned sp = 0;
    while ((int)(__builtin_amdgcn_readfirstlane(ld_sys(p)) - call) < 0)
        if (give_up<SLEEP>(ws, call, t0, sp)) return false;
    return true;
}
__device__ __forceinline__ unsigned fl_word(int blk, int src, int t) { return FL_OFF + (unsigned)(((blk * 4 + (src & 3)) * NT + t) * 2 + (src >> 2)) * 4u; }
__device__ __forceinline__ int steps_known(unsigned f0, unsigned f1, unsigned call) {
    const u64 lo = __ballot((int)(f0 - call) >= 0), hi = __ballot((int)(f1 - call) >= 0);
    const u64 miss = ~(lo & hi);
    if (miss == 0) return NSTEP;
    const int c = __builtin_ctzll(miss);
    return 2 * c + (int)((lo >> c) & 1);
}
__device__ __forceinline__ unsigned out_base(int src, int tile, int wave, int lane) {
    return (unsigned)(((tile >> 3) * 256 + (wave >> 1) * 64 + (lane >> 4) * 4) * NN + src * NCOL + (tile & 7) * 256 + (wave & 1) * 128 + (lane & 15) * 8);
}
#define WAIT_VM(n) asm volatile("s_waitcnt vmcnt(" #n ")" ::: "memory")
#define LGKM0_BARRIER() do { __builtin_amdgcn_s_waitcnt(0xC07F); asm volatile("" ::: "memory");                       \
        __builtin_amdgcn_s_barrier(); asm volatile("" ::: "memory"); } while (0)
#define STAMP(i) do {} while (0)

__global__ void __launch_bounds__(BLOCK, 2) fused_gemm_relu_q8(const Params p) {
    __shared__ __attribute__((aligned(1024))) char smem[LDS_BYTES];
    LAS char* const lds = (LAS char*)smem;
    const int hb = blockIdx.x;
    const int lane = threadIdx.x & 63;
    const int wid = __builtin_amdgcn_readfirstlane(threadIdx.x >> 6);
    int rank = p.rank;
    char* ws = p.ws[0];
    const char* px = p.x;
    const char* pw = p.w;
    const u64 t_start = __builtin_amdgcn_s_memrealtime();
    asm volatile("" : "+s"(rank), "+s"(ws), "+s"(px), "+s"(pw));
    const unsigned ticket = add_agent_lane0(ws + CTR_OFF + (size_t)hb * 128 + wid * 4, lane);
    const int b = hb & (N_COMM - 1);

    if (hb < N_COMM) {
        const __amdgpu_buffer_rsrc_t wsr = rsrc_of(ws, CTL_OFF);
        const int cw = wid * N_COMM + b;
        const int dbg = 128 + cw;
        (void)dbg;
        STAMP(0);
        if (wid < 2) {
            if (threadIdx.x == 0) *(LAS unsigned*)(lds + LDS_MISC) = 0u;
            LGKM0_BARRIER();
            if (cw < N_PULL) {
                const unsigned s = (unsigned)(rank + 1 + cw % (NR - 1)) & 7u;
                const __amdgpu_buffer_rsrc_t srs = rsrc_of(p.ws[1 + cw % (NR - 1)], CTL_OFF);
                char* const queue = ws + Q_OFF + s * 128;
                unsigned tk = add_agent_lane0(queue, lane);
                u32x4 cur[4];
                const unsigned call = __builtin_amdgcn_readfirstlane(ticket) + 1u;
                const unsigned qbase = (call - 1u) * Q_TICKETS;
                const char* const pk = ws + PK_OFF + s * NGRP * 8192 + lane * 128;
                unsigned pv = call - 1u;
                int pgrp = 0;
#define JOB_OF(n_, t_, j_, so_, do_) const int t_ = (int)(n_) / NBLK, j_ = (int)(n_) % NBLK;                           \
                const unsigned do_ = (j_ < 16 ? XT_OFF + t_ * XT_T + j_ * 32768 : WT_OFF + t_ * WT_T + (j_ - 16) * 32768) + s * 4096 + lane * 16; \
                const unsigned so_ = j_ < 16 ? do_ : WS_OFF + t_ * WT_T + rank * 32768 + (j_ - 16) * 4096 + lane * 16
#define WAIT_PACKED(t_, j_) do { const int sl_ = (j_) < 16 ? (j_) : 24 + ((t_) & 7) * 4 + ((j_) >> 1);                 \
                if (((t_) >> 3) != pgrp) { pgrp = (t_) >> 3; pv = call - 1u; }                                         \
                if ((int)(__builtin_amdgcn_readlane(pv, sl_) - call) < 0) {                                            \
                    pv = ld_sys(pk + pgrp * 8192);                 \
                    const u64 t0_ = __builtin_amdgcn_s_memrealtime(); unsigned sp_ = 0;                                \
                    while ((int)(__builtin_amdgcn_readlane(pv, sl_) - call) < 0) {                                     \
                        if (give_up(ws, call, t0_, sp_)) break;                                                        \
                        if (lane == sl_) pv = ld_sys(pk + pgrp * 8192); } } } while (0)
                unsigned n = __builtin_amdgcn_readfirstlane(tk) - qbase;
                if (n < (unsigned)Q_JOBS) {
                    JOB_OF(n, t, j, so, dof);
                    WAIT_PACKED(t, j);
                    STAMP(2);
                    tk = add_agent_lane0(queue, lane);
#pragma unroll
                    for (int i = 0; i < 4; ++i) cur[i] = __builtin_amdgcn_raw_buffer_load_b128(srs, so + i * 1024, 0, AUX_SYS);
                    unsigned dcur = dof, fcur = fl_word(j, s, t);
                    for (;;) {
#pragma unroll
                        for (int i = 0; i < 4; ++i) __builtin_amdgcn_raw_buffer_store_b128(cur[i], wsr, dcur + i * 1024, 0, AUX_SYS);
                        asm volatile("" ::: "memory");
                        n = __builtin_amdgcn_readfirstlane(tk) - qbase;
                        char* const flag = ws + fcur;
                        if (n >= (unsigned)Q_JOBS) {
                            WAIT_VM(0);
                            if (lane == 0) st_sys(flag, call);
                            break;
                        }
                        JOB_OF(n, t2, j2, so2, dof2);
                        WAIT_PACKED(t2, j2);
                        tk = add_agent_lane0(queue, lane);
#pragma unroll
                        for (int i = 0; i < 4; ++i) cur[i] = __builtin_amdgcn_raw_buffer_load_b128(srs, so2 + i * 1024, 0, AUX_SYS);
                        dcur = dof2; fcur = fl_word(j2, s, t2);
                        WAIT_VM(5);
                        if (lane == 0) st_sys(flag, call);
                    }
                }
                STAMP(1);
            }
        } else {
            const __amdgpu_buffer_rsrc_t xr = rsrc_of(px, MM * KSH * 2);
            const __amdgpu_buffer_rsrc_t wr_ = rsrc_of(pw, KSH * NN * 2);
            bool met = false;
            bool x_todo = wid >= 4;
            for (int n = (wid - 2) * N_COMM + b; n < NT * 32 || x_todo;) {
                const bool do_x = x_todo && ((b >> 4) <= (n >> 8) || n >= NT * 32);
                const int g = do_x ? b >> 4 : n >> 8;
                const int delay = (do_x ? g * 8 : n >> 5) * PACK_DELAY;
                if (!met && delay > 0) { __builtin_amdgcn_s_barrier(); met = true; }
                while (__builtin_amdgcn_s_memrealtime() - t_start < (u64)delay) __builtin_amdgcn_s_sleep(2);
                u32x4 v[8];
                if (do_x) {
                    const unsigned rb = (wid - 4) * 64 + (lane & 7), piece = lane >> 3;
#pragma unroll
                    for (int i = 0; i < 8; ++i) v[i] = __builtin_amdgcn_raw_buffer_load_b128(xr, ((b & 15) * 256 + rb + i * 8) * (KSH * 2) + g * 128 + piece * 16, 0, 0);
                    if (!met) { __builtin_amdgcn_s_barrier(); met = true; }
                    const unsigned d0 = XT_OFF + (g * 8 + piece) * XT_T + (b & 15) * 32768 + rank * 4096 + rb * 16;
#pragma unroll
                    for (int i = 0; i < 8; ++i) __builtin_amdgcn_raw_buffer_store_b128(v[i], wsr, d0 + i * 128, 0, AUX_SYS);
                    char* fl = lane < 8 ? p.ws[lane & 7] + PK_OFF + (rank * NGRP + g) * 8192 + (b & 15) * 128 : ws + fl_word(b & 15, rank, g * 8 + (lane & 7));
                    asm volatile("s_waitcnt vmcnt(0)" : "+v"(fl) :: "memory");
                    unsigned old = 0;
                    if (lane == 0) old = __hip_atomic_fetch_add((LAS unsigned*)(lds + LDS_MISC), 1u, __ATOMIC_RELAXED, __HIP_MEMORY_SCOPE_WORKGROUP);
                    const unsigned call = __builtin_amdgcn_readfirstlane(ticket) + 1u;
                    if (__builtin_amdgcn_readfirstlane(old) == 3u && lane >= 1 && lane < 16) st_sys(fl, call);
                    x_todo = false;
                } else {
                    const int t = n >> 5, cb = n & 31;
                    const unsigned dest = (unsigned)cb >> 2;
#pragma unroll
                    for (int i = 0; i < 8; ++i) v[i] = __builtin_amdgcn_raw_buffer_load_b128(wr_, (unsigned)((t * 8 + i) * (NN * 2) + cb * 1024 + lane * 16), 0, 0);
                    if (!met) { __builtin_amdgcn_s_barrier(); met = true; }
                    const unsigned d0 = (dest == (unsigned)rank ? WT_OFF + t * WT_T + ((cb & 3) * 2 + (lane >> 5)) * 32768 + rank * 4096
                                                                : WS_OFF + t * WT_T + dest * 32768 + ((cb & 3) * 2 + (lane >> 5)) * 4096)
                                      + (((lane >> 4) & 1) * 128 + (lane & 15)) * 16;
#pragma unroll
                    for (int i = 0; i < 8; ++i) __builtin_amdgcn_raw_buffer_store_b128(transposed(v, i), wsr, d0 + i * 256, 0, AUX_SYS);
                    char* const dws = p.ws[(dest - rank) & 7u];
                    WAIT_VM(0);
                    const unsigned call = __builtin_amdgcn_readfirstlane(ticket) + 1u;
                    if (dest == (unsigned)rank) { if (lane < 2) st_sys(ws + fl_word(16 + (cb & 3) * 2 + lane, rank, t), call); }
                    else if (lane == 0) st_sys(dws + PK_OFF + (rank * NGRP + g) * 8192 + (32 + (t & 7) * 4 + (cb & 3)) * 128, call);
                    n += N_PACKW;
                }
            }
        }
        const unsigned call = __builtin_amdgcn_readfirstlane(ticket) + 1u;
        if (wid == 7) {
            if (b == N_COMM - 1) {
                char* amax_out = p.ws[lane & 7] + AMAX_OFF + ((lane >> 3) * NR + rank) * 128;
                asm volatile("" : "+v"(amax_out));
                const char* tm = ws + TMAX_OFF + lane * 256;
                const u64 t0 = __builtin_amdgcn_s_memrealtime(); unsigned sp = 0;
                int mx;
                for (;;) {
                    const u64 v0 = ld_sys64(tm), v1 = ld_sys64(tm + 128);
                    mx = max((int)(unsigned)v0, (int)(unsigned)v1);
                    if (__all((int)((unsigned)(v0 >> 32) - call) >= 0 && (int)((unsigned)(v1 >> 32) - call) >= 0)) break;
                    if (give_up(ws, call, t0, sp)) { mx = NAN_BITS; break; }
                }
                if (lane < 8 * AMAX_REP) st_sys64(amax_out, (unsigned)wave_max(mx), call);
                STAMP(6);
            }
            return;
        }
        {
            const int src = (rank + 1 + wid) & 7;
            const unsigned slot = OB_OFF + (unsigned)(src * 128 + b) * OB_SLOT;
            const char* const of = ws + OF_OFF + (src * 128 + b) * 128;
            const unsigned lbase = wid * LDS_WAVE;
            if (lane == 0) *(LAS unsigned*)(lds + lbase + 8192) = 0u;
            char* outp = p.out;
            asm volatile("" : "+s"(outp));
            const unsigned ob = out_base(src, b >> 3, b & 7, lane);
            {
                bool ok = await_word<8>(ws, of, call);
                const u32x4 mk = __builtin_amdgcn_raw_buffer_load_b128(wsr, slot + lane * 16, 0, AUX_SYS);
                unsigned ad[128], pos = lbase;
#pragma unroll
                for (int r = 0; r < 128; ++r) {
                    const bool kept = (int)(mk[r >> 5] << (r & 31)) >= 0;
                    const u64 m = __ballot(kept);
                    unsigned a = pos + lanes_below(m);
                    asm("" : "+v"(a));
                    ad[r] = kept ? a : lbase + 8192;
                    pos += __builtin_popcountll(m);
                    asm("" : "+s"(pos));
                }
#pragma unroll
                for (int r = 0; r < 128; r += 16)
                    asm volatile("" : "+v"(ad[r]), "+v"(ad[r + 1]), "+v"(ad[r + 2]), "+v"(ad[r + 3]), "+v"(ad[r + 4]), "+v"(ad[r + 5]), "+v"(ad[r + 6]), "+v"(ad[r + 7]),
                                      "+v"(ad[r + 8]), "+v"(ad[r + 9]), "+v"(ad[r + 10]), "+v"(ad[r + 11]), "+v"(ad[r + 12]), "+v"(ad[r + 13]), "+v"(ad[r + 14]), "+v"(ad[r + 15]));
                STAMP(3);
                ok = await_word<1>(ws, of + 4, call) && ok;
                STAMP(4);
                const char* gp = ws + slot + 1024 + lane * 16;
                for (unsigned o = 0; o < pos - lbase; o += 1024)
                    __builtin_amdgcn_global_load_lds((const GAS unsigned*)(gp + o), (LAS unsigned*)(lds + lbase + o), 16, 0, AUX_SYS);
                const unsigned fill = ok && __builtin_amdgcn_readfirstlane(ld_agent(ws + TMO_OFF)) != call ? 0u : 0x7f7f7f7fu;
                WAIT_VM(0);
                const __amdgpu_buffer_rsrc_t ors = rsrc_of(outp, (MM / NR) * NN);
#pragma unroll
                for (int r = 0; r < 128; ++r) ad[r] = *(const LAS unsigned char*)(lds + ad[r]);
                __builtin_amdgcn_sched_barrier(0);
#pragma unroll
                for (int u = 0; u < 16; ++u) {
                    const unsigned* c = ad + u * 8;
                    u32x2 o;
                    o[0] = (c[0] | (c[1] << 8) | (c[2] << 16) | (c[3] << 24)) | fill;
                    o[1] = (c[4] | (c[5] << 8) | (c[6] << 16) | (c[7] << 24)) | fill;
                    __builtin_amdgcn_raw_buffer_store_b64(o, ors, ob + (unsigned)(((u >> 2) * 16 + (u & 3)) * NN), 0, AUX_OUT);
                }
            }
        }
        return;
    }

    const int g = b;
    const int dbg = g;
    (void)dbg;
    const int pm = (g >> 4) * 2 + (g & 1);
    const int pn = ((g & 7) >> 1) * 2 + ((g >> 3) & 1);
    const int fr = lane & 15, fq = lane >> 4, wr = wid >> 1, wc = wid & 1;
    if (wid == 0) STAMP(0);
    const int owner = pm >> 1, tile = (pm & 1) * 8 + pn;
    char* ows = p.ws[(owner - rank) & 7];
    char* outp = p.out;
    asm volatile("" : "+s"(ows), "+s"(outp));
    if (wid == 0) __builtin_amdgcn_fence(__ATOMIC_ACQUIRE, "");
    const unsigned call = __builtin_amdgcn_readfirstlane(ticket) + 1u;
    WAIT_VM(0);
    __builtin_amdgcn_s_barrier();

    const int op = wid >> 2;
    const char* const pollp = ws + fl_word(op ? 16 + pn : pm, wid & 3, lane);
    const unsigned unit = op ? WT_T : XT_T;
    const char* const gbase = ws + (op ? WT_OFF + pn * 32768 : XT_OFF + pm * 32768) + (wid & 3) * 4096 + lane * 16;
    LAS char* const ldst = lds + wid * 4096;
    const int aoff = fq * 4096 + (wr * 64 + fr) * 16;
    const int boff = 16384 + fq * 4096 + (wc * 128 + fr) * 16;

    int known = 0;
    int next = 0, nslot = 0;
    u32x2 pvn = {0u, 0u};
#define STAGE_NEXT() do { const char* g_ = gbase + ((next & 1) * 16384u + (unsigned)(next >> 1) * unit);              \
        LAS char* l_ = ldst + nslot * LDS_SLOT;                                                                       \
        __builtin_amdgcn_global_load_lds((const GAS unsigned*)g_, (LAS unsigned*)l_, 16, 0, 0);                       \
        __builtin_amdgcn_global_load_lds((const GAS unsigned*)g_, (LAS unsigned*)l_, 16, 1024, 0);                    \
        __builtin_amdgcn_global_load_lds((const GAS unsigned*)g_, (LAS unsigned*)l_, 16, 2048, 0);                    \
        __builtin_amdgcn_global_load_lds((const GAS unsigned*)g_, (LAS unsigned*)l_, 16, 3072, 0);                    \
        ++next; nslot = nslot == NSLOT - 1 ? 0 : nslot + 1; } while (0)
#define POLL_ISSUE() asm volatile("global_load_dwordx2 %0, %1, off sc0 sc1" : "+v"(pvn) : "v"(pollp) : "memory")
#define POLL_TAKE() do { u32x2 f_; asm volatile("s_waitcnt vmcnt(0)\n\tv_mov_b64 %0, %1" : "=v"(f_) : "v"(pvn) : "memory");    \
        known = steps_known(f_[0], f_[1], call); } while (0)
#define AWAIT_NEXT() do { if (next >= known) {                                                                        \
            const u64 t0_ = __builtin_amdgcn_s_memrealtime(); unsigned sp_ = 0;                                       \
            for (;;) { const u64 f_ = ld_sys64(pollp); known = steps_known((unsigned)f_, (unsigned)(f_ >> 32), call); \
                if (next < known) break;                                                                              \
                if (give_up<2>(ws, call, t0_, sp_)) { known = NSTEP; break; } } } } while (0)
#define MFMA_COLS(A_, B_, n0_, n1_) do { __builtin_amdgcn_sched_barrier(0);                                           \
        _Pragma("unroll") for (int n = (n0_); n < (n1_); ++n)                                                         \
            _Pragma("unroll") for (int m = 0; m < 4; ++m)                                                             \
                acc[m][n] = __builtin_amdgcn_mfma_f32_16x16x32_bf16(A_[m], B_[n], acc[m][n], 0, 0, 0);                \
        __builtin_amdgcn_sched_barrier(0); } while (0)
#define RD_A(A_, m_) A_[m_] = *(const LAS bf16x8*)(lb_ + aoff + (m_) * 256)
#define RD_B(B_, n_) B_[n_] = *(const LAS bf16x8*)(lb_ + boff + (n_) * 256)
#define TAIL_REC(st_) do {} while (0)
#define KSTEP(st_, A_, B_, AN_, BN_) do {                                                                             \
        MFMA_COLS(A_, B_, 0, 1);                                                                                      \
        if (polled) POLL_TAKE();                                          \
        const int had_ = next;                                                                                        \
        const int lim_ = min((st_) + NSLOT, known);                                   \
        if (next < lim_) { STAGE_NEXT(); if (next < lim_) STAGE_NEXT(); }                                             \
        const bool full_ = (polled ? had_ + NSLOT - 2 : next) >= (st_) + NSLOT;                                       \
        MFMA_COLS(A_, B_, 1, 4);                                                                                      \
        if (!full_) {                                                              \
            if (next == (st_) + 1 && next < NSTEP) {                                                                  \
                AWAIT_NEXT(); STAGE_NEXT();                                                                           \
                if (next < known) STAGE_NEXT();                                                                       \
            }                                                                                                         \
            const int k_ = next - (st_) - 2;                                                                          \
            if (k_ <= 0) WAIT_VM(0); else if (k_ == 1) WAIT_VM(4); else WAIT_VM(8);                                   \
        }                                                                                                             \
        WAIT_VM(12);                                                                                                  \
        LGKM0_BARRIER();                                                                                              \
        TAIL_REC(st_);                                                                                                \
        const LAS char* lb_ = lds + rslot * LDS_SLOT;               \
        RD_A(AN_, 0); RD_A(AN_, 1); RD_A(AN_, 2); RD_A(AN_, 3);                                                       \
        MFMA_COLS(A_, B_, 4, 5);                                                                                      \
        RD_B(BN_, 0); RD_B(BN_, 1); RD_B(BN_, 2); RD_B(BN_, 3);                                                       \
        MFMA_COLS(A_, B_, 5, 6);                                                                                      \
        RD_B(BN_, 4); RD_B(BN_, 5); RD_B(BN_, 6); RD_B(BN_, 7);                                                       \
        rslot = rslot == NSLOT - 1 ? 0 : rslot + 1;                                                                   \
        polled = known < min(next + 6, NSTEP);                                                                        \
        if (polled) POLL_ISSUE();                                                                                     \
        MFMA_COLS(A_, B_, 6, 8); } while (0)

    f32x4 acc[4][8];
#pragma unroll
    for (int m = 0; m < 4; ++m)
#pragma unroll
        for (int n = 0; n < 8; ++n) acc[m][n] = (f32x4){0.f, 0.f, 0.f, 0.f};

    AWAIT_NEXT(); STAGE_NEXT();
    WAIT_VM(0);
    LGKM0_BARRIER();
    int rslot = 1;
    bool polled = false;
    bf16x8 a0[4], b0[8], a1[4], b1[8];
    {
        const LAS char* lb_ = lds;
#pragma unroll
        for (int m = 0; m < 4; ++m) RD_A(a0, m);
#pragma unroll
        for (int n = 0; n < 8; ++n) RD_B(b0, n);
    }
    if (wid == 0) STAMP(1);
    for (int st = 0; st < NSTEP; st += 2) {
        KSTEP(st, a0, b0, a1, b1);
        KSTEP(st + 1, a1, b1, a0, b0);
    }
    if (wid == 0) STAMP(2);

    int mx = 0;
#pragma unroll
    for (int m = 0; m < 4; ++m)
#pragma unroll
        for (int n = 0; n < 8; ++n)
#pragma unroll
            for (int j = 0; j < 4; ++j) mx = max(mx, __float_as_int(acc[m][n][j]));
    mx = wave_max(mx);
    LAS int* red = (LAS int*)(lds + LDS_MISC);
    if (lane == 0) red[wid] = mx;
    LGKM0_BARRIER();
    if (wid == 0) {
        int t = red[lane & 7];
        ROW_SHR_MAX(t, 1); ROW_SHR_MAX(t, 2); ROW_SHR_MAX(t, 4);
        if (lane == 7) st_sys64(ws + TMAX_OFF + g * 128, (unsigned)t, call);
        STAMP(7);
    }

    const bool remote = owner != rank;
    const __amdgpu_buffer_rsrc_t sls = rsrc_of(ows + OB_OFF + (unsigned)(rank * 128 + tile * 8 + wid) * OB_SLOT, OB_SLOT);
    char* const of = ows + OF_OFF + (rank * 128 + tile * 8 + wid) * 128;
    const int efield = min(268 - (mx >> 23), 253);
    const float sc = __int_as_float(efield << 23);
    const unsigned pos0 = (unsigned)(size_t)(lds + wid * LDS_WAVE);
    const unsigned spill = (unsigned)(size_t)(lds + LDS_MISC + 64);
    unsigned pos = pos0;
#define VAL(r_) acc[(r_) >> 5][(r_) & 7][((r_) >> 3) & 3]
#define PACK2(r_) do { unsigned ta_, tb_, ha_, hb_, c_, p1_;                                                          \
        asm volatile("v_cmp_lt_i32 vcc, -1, %[a]\n\tv_cmp_lt_i32 s[88:89], -1, %[b]\n\t"                             \
                     "v_fma_mixlo_f16 %[ha], %[a], %[sc], 0\n\ts_bcnt1_i32_b64 %[c], vcc\n\t"                        \
                     "v_mbcnt_lo_u32_b32 %[ta], vcc_lo, 0\n\tv_fma_mixlo_f16 %[hb], %[b], %[sc], 0\n\t"              \
                     "s_lshl1_add_u32 %[p1], %[c], %[pos]\n\tv_mbcnt_lo_u32_b32 %[tb], s88, 0\n\t"                   \
                     "s_bcnt1_i32_b64 %[c], s[88:89]\n\tv_mbcnt_hi_u32_b32 %[ta], vcc_hi, %[ta]\n\t"                 \
                     "v_mbcnt_hi_u32_b32 %[tb], s89, %[tb]\n\tv_lshl_add_u32 %[ta], %[ta], 1, %[pos]\n\t"            \
                     "v_lshl_add_u32 %[tb], %[tb], 1, %[p1]\n\ts_lshl1_add_u32 %[pos], %[c], %[p1]\n\t"              \
                     "v_cndmask_b32 %[ta], %[sp], %[ta], vcc\n\tv_cndmask_b32 %[tb], %[sp], %[tb], s[88:89]\n\t"     \
                     "ds_write_b16 %[ta], %[ha]\n\tds_write_b16 %[tb], %[hb]"                                        \
                     : [ta] "=&v"(ta_), [tb] "=&v"(tb_), [ha] "=&v"(ha_), [hb] "=&v"(hb_), [c] "=&s"(c_),             \
                       [p1] "=&s"(p1_), [pos] "+s"(pos)                                                               \
                     : [a] "v"(VAL(r_)), [b] "v"(VAL((r_) + 1)), [sc] "s"(sc), [sp] "v"(spill)                        \
                     : "vcc", "scc", "s88", "s89", "memory"); } while (0)
#define PACK8(r_) PACK2(r_); PACK2((r_) + 2); PACK2((r_) + 4); PACK2((r_) + 6)
    if (remote) {
        u32x4 mk = {0u, 0u, 0u, 0u};
#pragma unroll
        for (int r = 0; r < 128; ++r) mk[r >> 5] = __builtin_amdgcn_alignbit(mk[r >> 5], __float_as_uint(VAL(r)), 31);
#define MASK_OUT(k_) do { if (MASK_AT == (k_)) __builtin_amdgcn_raw_buffer_store_b128(mk, sls, lane * 16, 0, AUX_SYS); } while (0)
        MASK_OUT(0); PACK8(0); MASK_OUT(1); PACK8(8); MASK_OUT(2); PACK8(16); MASK_OUT(3); PACK8(24); MASK_OUT(4); PACK8(32); MASK_OUT(5);
        if (MASK_AT < 3) { WAIT_VM(0); if (lane == 0) st_sys(of, call); }
    } else {
#pragma unroll
        for (int m = 0; m < 4; ++m) {
#pragma unroll
            for (int n = 0; n < 8; ++n)
#pragma unroll
                for (int j = 0; j < 4; ++j) acc[m][n][j] = __int_as_float(max(0, __float_as_int(acc[m][n][j])));
            asm volatile("" : "+v"(acc[m][0]), "+v"(acc[m][1]), "+v"(acc[m][2]), "+v"(acc[m][3]),
                              "+v"(acc[m][4]), "+v"(acc[m][5]), "+v"(acc[m][6]), "+v"(acc[m][7]));
        }
    }
    if (wid == 1) STAMP(6);
    if (wid == 0) {
        const char* ap = lane < NR ? ws + AMAX_OFF + ((g & (AMAX_REP - 1)) * NR + lane) * 128 : ws + TMO_OFF - 4;
        const u64 t0 = __builtin_amdgcn_s_memrealtime(); unsigned sp = 0;
        u64 v = 0;
        for (;;) {
            if (lane <= NR) v = ld_sys64(ap);
            if ((__ballot((int)((unsigned)(v >> 32) - call) >= 0) & 0xffu) == 0xffu) break;
            if (give_up(ws, call, t0, sp)) { v = (u64)call << 32; break; }
        }
        STAMP(8);
        int t = lane < NR ? (int)(unsigned)v : ((unsigned)(v >> 32) == call ? NAN_BITS : 0);
        ROW_SHR_MAX(t, 1); ROW_SHR_MAX(t, 2); ROW_SHR_MAX(t, 4); ROW_SHR_MAX(t, 8);
        if (lane == NR) red[8] = t;
    }
    LGKM0_BARRIER();
    const float inv = 448.0f / __int_as_float(red[8]);
    if (wid == 0) STAMP(3);

    if (remote) {
        const float f = inv * __int_as_float((254 - efield) << 23);
        unsigned sent = 0;
#define SEND_GATE() WAIT_VM(0)
#define SEND(upto_) do { __builtin_amdgcn_s_waitcnt(0xC07F); asm volatile("" ::: "memory");                           \
            const unsigned cnt_ = (pos - pos0) >> 1;                                                                  \
            _Pragma("nounroll") for (; sent < (upto_); sent += 1024) {                                                                    \
                const unsigned o_ = sent + lane * 16;                                                                 \
                const f16x8 h0 = *(const LAS f16x8*)(lds + wid * LDS_WAVE + o_ * 2), h1 = *(const LAS f16x8*)(lds + wid * LDS_WAVE + o_ * 2 + 16); \
                u32x4 q;                                                                                              \
                _Pragma("unroll") for (int d = 0; d < 2; ++d) {                                                       \
                    int w0 = __builtin_amdgcn_cvt_pk_fp8_f32(__builtin_fmaf((float)h0[4 * d], f, 0.f), __builtin_fmaf((float)h0[4 * d + 1], f, 0.f), 0, false); \
                    q[d] = (unsigned)__builtin_amdgcn_cvt_pk_fp8_f32(__builtin_fmaf((float)h0[4 * d + 2], f, 0.f), __builtin_fmaf((float)h0[4 * d + 3], f, 0.f), w0, true); \
                    int w1 = __builtin_amdgcn_cvt_pk_fp8_f32(__builtin_fmaf((float)h1[4 * d], f, 0.f), __builtin_fmaf((float)h1[4 * d + 1], f, 0.f), 0, false); \
                    q[2 + d] = (unsigned)__builtin_amdgcn_cvt_pk_fp8_f32(__builtin_fmaf((float)h1[4 * d + 2], f, 0.f), __builtin_fmaf((float)h1[4 * d + 3], f, 0.f), w1, true); \
                }                                                                                                     \
                SEND_GATE();                                                                                          \
                if (o_ < cnt_) __builtin_amdgcn_raw_buffer_store_b128(q, sls, 1024 + o_, 0, AUX_SYS);                 \
            } } while (0)
        if (MASK_AT >= 3) { WAIT_VM(0); if (lane == 0) st_sys(of, call); }
        SEND(cnt_ & ~1023u);
        PACK8(40); PACK8(48); PACK8(56); PACK8(64);
        SEND(cnt_ & ~1023u);
        PACK8(72); PACK8(80); PACK8(88); PACK8(96);
        SEND(cnt_ & ~1023u);
        PACK8(104); PACK8(112); PACK8(120);
        SEND(cnt_);
        WAIT_VM(0);
        if (lane == 0) st_sys(of + 4, call);
    } else {
        const __amdgpu_buffer_rsrc_t ors = rsrc_of(outp, (MM / NR) * NN);
        const unsigned ob = out_base(rank, tile, wid, lane);
#pragma unroll
        for (int m = 0; m < 4; ++m)
#pragma unroll
            for (int j = 0; j < 4; ++j) {
                u32x2 o;
                int lo = __builtin_amdgcn_cvt_pk_fp8_f32(acc[m][0][j] * inv, acc[m][1][j] * inv, 0, false);
                o[0] = (unsigned)__builtin_amdgcn_cvt_pk_fp8_f32(acc[m][2][j] * inv, acc[m][3][j] * inv, lo, true);
                int hi = __builtin_amdgcn_cvt_pk_fp8_f32(acc[m][4][j] * inv, acc[m][5][j] * inv, 0, false);
                o[1] = (unsigned)__builtin_amdgcn_cvt_pk_fp8_f32(acc[m][6][j] * inv, acc[m][7][j] * inv, hi, true);
                __builtin_amdgcn_raw_buffer_store_b64(o, ors, ob + (unsigned)((m * 16 + j) * NN), 0, AUX_OUT);
            }
    }
    if (wid == 0) STAMP(4);
    if (wid == 7) STAMP(5);
}

}

extern "C" void kernel_launch(int rank, int nranks, ncclComm_t comm,
                              hipStream_t stream,
                              void* const* d_in, const int* in_sizes, int n_in,
                              void* d_out, int out_size,
                              const int* in_dtypes, int out_dtype,
                              void* const* d_out_all,
                              void* const* d_ws_all, size_t ws_bytes) {
    (void)comm; (void)in_dtypes; (void)out_dtype; (void)d_out_all;
    if (nranks != NR || n_in < 2 || (unsigned)rank >= (unsigned)NR || ws_bytes < WS_NEED ||
        in_sizes[0] != MM * KSH || in_sizes[1] != KSH * NN || out_size != (MM / NR) * NN) {
        fprintf(stderr, "kernel_launch: unexpected problem shape\n");
        return;
    }
    Params p;
    p.x = (const char*)d_in[0];
    p.w = (const char*)d_in[1];
    for (int i = 0; i < NR; ++i) p.ws[i] = (char*)d_ws_all[(rank + i) % NR];
    p.out = (char*)d_out;
    p.rank = rank; p.pad = 0;
    hipLaunchKernelGGL(fused_gemm_relu_q8, dim3(GRID), dim3(BLOCK), 0, stream, p);
    HIP_CHECK(hipGetLastError());
}
```

```cpp
#include <hip/hip_runtime.h>
#include <rccl/rccl.h>
#include <cstdio>
#include <cstdint>

#define HIP_CHECK(expr) do { hipError_t _e = (expr); if (_e != hipSuccess) { \
    fprintf(stderr, "HIP error %s at %s:%d\n", hipGetErrorString(_e), __FILE__, __LINE__); \
    return; } } while (0)

namespace {

typedef short bf16x8 __attribute__((ext_vector_type(8)));
typedef _Float16 f16x8 __attribute__((ext_vector_type(8)));
typedef float f32x4 __attribute__((ext_vector_type(4)));
typedef unsigned u32x4 __attribute__((ext_vector_type(4)));
typedef unsigned u32x2 __attribute__((ext_vector_type(2)));
typedef unsigned long long u64;
#define LAS __attribute__((address_space(3)))
#define GAS __attribute__((address_space(1)))

constexpr int NR = 8;
constexpr int MM = 4096, KSH = 512, NN = 16384, NCOL = NN / NR;
constexpr int NT = KSH / 8;
constexpr int NBLK = 24;
#define PULL_PER_LINK 17
#define MASK_AT 0
#define AMAX_REP 8
#define AUX_OUT 17
#define SEND_DEPTH 1
#define PACK_DELAY 100
constexpr int Q_JOBS = NT * NBLK;
constexpr int Q_TICKETS = Q_JOBS + PULL_PER_LINK;
constexpr int N_COMM = 128;
constexpr int N_PULL = (NR - 1) * PULL_PER_LINK;
constexpr int N_GEMM = 128;
constexpr int GRID = N_COMM + N_GEMM;
constexpr int BLOCK = 512;
constexpr int N_PACKW = N_COMM * 6;
constexpr int NGRP = NT / 8;
static_assert(N_PULL <= 2 * N_COMM, "pull waves are waves 0-1 of the comm workgroups");

constexpr unsigned XT_T = NR * MM * 16, WT_T = NR * NCOL * 16;
constexpr unsigned XT_OFF = 0;
constexpr unsigned WT_OFF = XT_OFF + NT * XT_T;
constexpr unsigned WS_OFF = WT_OFF + NT * WT_T;
constexpr unsigned OB_SLOT = 1024 + 8192;
constexpr unsigned OB_OFF = WS_OFF + NT * WT_T;
constexpr unsigned CTL_OFF = OB_OFF + NR * 128 * OB_SLOT;
constexpr unsigned PK_OFF = CTL_OFF;
constexpr unsigned FL_OFF = PK_OFF + NR * NGRP * 8192;
constexpr unsigned Q_OFF = FL_OFF + NBLK * NR * NT * 4;
constexpr unsigned TMAX_OFF = Q_OFF + NR * 128;
constexpr unsigned AMAX_OFF = TMAX_OFF + N_GEMM * 128;
constexpr unsigned OF_OFF = AMAX_OFF + 64 * 128;
constexpr unsigned CTR_OFF = OF_OFF + NR * 128 * 128;
constexpr unsigned TMO_OFF = CTR_OFF + GRID * 128 + 4;
constexpr unsigned DBG_OFF = CTR_OFF + GRID * 128 + 128;
constexpr size_t WS_NEED = DBG_OFF + 524288;

constexpr u64 DEADLINE = 50000000ull;
constexpr int AUX_SYS = 17;

constexpr int NSTEP = 2 * NT;
constexpr int NSLOT = 5;
constexpr int LDS_SLOT = 32768;
constexpr int LDS_BYTES = NSLOT * LDS_SLOT;
constexpr int LDS_WAVE = 16384;
constexpr int LDS_MISC = 8 * LDS_WAVE;

struct Params {
    const char* x; const char* w;
    int rank, pad;
    char* ws[NR];
    char* out;
};

__device__ __forceinline__ __amdgpu_buffer_rsrc_t rsrc_of(const void* p, unsigned bytes) {
    return __builtin_amdgcn_make_buffer_rsrc(const_cast<void*>(p), 0, bytes, 0x00020000);
}
__device__ __forceinline__ unsigned ld_sys(const void* p) {
    return __hip_atomic_load((const GAS unsigned*)p, __ATOMIC_RELAXED, __HIP_MEMORY_SCOPE_SYSTEM);
}
__device__ __forceinline__ u64 ld_sys64(const void* p) {
    return __hip_atomic_load((const GAS u64*)p, __ATOMIC_RELAXED, __HIP_MEMORY_SCOPE_SYSTEM);
}
__device__ __forceinline__ void st_sys(void* p, unsigned v) {
    __hip_atomic_store((GAS unsigned*)p, v, __ATOMIC_RELAXED, __HIP_MEMORY_SCOPE_SYSTEM);
}
__device__ __forceinline__ void st_sys64(void* p, unsigned lo, unsigned hi) {
    __hip_atomic_store((GAS u64*)p, ((u64)hi << 32) | lo, __ATOMIC_RELAXED, __HIP_MEMORY_SCOPE_SYSTEM);
}
__device__ __forceinline__ unsigned ld_agent(const void* p) {
    return __hip_atomic_load((const GAS unsigned*)p, __ATOMIC_RELAXED, __HIP_MEMORY_SCOPE_AGENT);
}
__device__ __forceinline__ unsigned add_agent_lane0(void* p, int lane) {
    unsigned v = 0, z = 0;
    asm volatile("" : "+v"(z));
    if (lane == 0) v = __hip_atomic_fetch_add((GAS unsigned*)((char*)p + z), 1u, __ATOMIC_RELAXED, __HIP_MEMORY_SCOPE_AGENT);
    return v;
}
__device__ __forceinline__ u32x4 transposed(const u32x4 (&v)[8], int i) {
    u32x4 o;
#pragma unroll
    for (int d = 0; d < 4; ++d)
        o[d] = __builtin_amdgcn_perm(v[2 * d + 1][i >> 1], v[2 * d][i >> 1], (i & 1) ? 0x07060302u : 0x05040100u);
    return o;
}
constexpr int NAN_BITS = 0x7fc00000;
#define ROW_SHR_MAX(v, n) v = max(v, __builtin_amdgcn_update_dpp(0, v, 0x110 + (n), 0xf, 0xf, true))
__device__ __forceinline__ int wave_max(int v) {
    ROW_SHR_MAX(v, 1); ROW_SHR_MAX(v, 2); ROW_SHR_MAX(v, 4); ROW_SHR_MAX(v, 8);
    return max(max(__builtin_amdgcn_readlane(v, 15), __builtin_amdgcn_readlane(v, 31)),
               max(__builtin_amdgcn_readlane(v, 47), __builtin_amdgcn_readlane(v, 63)));
}
__device__ __forceinline__ unsigned lanes_below(u64 m) {
    return __builtin_amdgcn_mbcnt_hi((unsigned)(m >> 32), __builtin_amdgcn_mbcnt_lo((unsigned)m, 0u));
}
template <int SLEEP = 1>
__device__ __forceinline__ bool give_up(char* ws, unsigned call, u64 t0, unsigned& spins) {
    bool stop = __builtin_amdgcn_s_memrealtime() - t0 > DEADLINE;
    if ((++spins & 63u) == 0u) stop = stop || __builtin_amdgcn_readfirstlane(ld_agent(ws + TMO_OFF)) == call;
    if (stop) {
        __hip_atomic_store((GAS unsigned*)(ws + TMO_OFF), call, __ATOMIC_RELAXED, __HIP_MEMORY_SCOPE_AGENT);
        return true;
    }
    __builtin_amdgcn_s_sleep(SLEEP);
    return false;
}
template <int SLEEP>
__device__ __forceinline__ bool await_word(char* ws, const char* p, unsigned call) {
    const u64 t0 = __builtin_amdgcn_s_memrealtime(); unsigned sp = 0;
    while ((int)(__builtin_amdgcn_readfirstlane(ld_sys(p)) - call) < 0)
        if (give_up<SLEEP>(ws, call, t0, sp)) return false;
    return true;
}
__device__ __forceinline__ unsigned fl_word(int blk, int src, int t) { return FL_OFF + (unsigned)(((blk * 4 + (src & 3)) * NT + t) * 2 + (src >> 2)) * 4u; }
__device__ __forceinline__ int steps_known(unsigned f0, unsigned f1, unsigned call) {
    const u64 lo = __ballot((int)(f0 - call) >= 0), hi = __ballot((int)(f1 - call) >= 0);
    const u64 miss = ~(lo & hi);
    if (miss == 0) return NSTEP;
    const int c = __builtin_ctzll(miss);
    return 2 * c + (int)((lo >> c) & 1);
}
__device__ __forceinline__ unsigned out_base(int src, int tile, int wave, int lane) {
    return (unsigned)(((tile >> 3) * 256 + (wave >> 1) * 64 + (lane >> 4) * 4) * NN + src * NCOL + (tile & 7) * 256 + (wave & 1) * 128 + (lane & 15) * 8);
}
#define WAIT_VM(n) asm volatile("s_waitcnt vmcnt(" #n ")" ::: "memory")
#define LGKM0_BARRIER() do { __builtin_amdgcn_s_waitcnt(0xC07F); asm volatile("" ::: "memory");                       \
        __builtin_amdgcn_s_barrier(); asm volatile("" ::: "memory"); } while (0)
#define STAMP(i) do {} while (0)

__global__ void __launch_bounds__(BLOCK, 2) fused_gemm_relu_q8(const Params p) {
    __shared__ __attribute__((aligned(1024))) char smem[LDS_BYTES];
    LAS char* const lds = (LAS char*)smem;
    const int hb = blockIdx.x;
    const int lane = threadIdx.x & 63;
    const int wid = __builtin_amdgcn_readfirstlane(threadIdx.x >> 6);
    int rank = p.rank;
    char* ws = p.ws[0];
    const char* px = p.x;
    const char* pw = p.w;
    const u64 t_start = __builtin_amdgcn_s_memrealtime();
    asm volatile("" : "+s"(rank), "+s"(ws), "+s"(px), "+s"(pw));
    const unsigned ticket = add_agent_lane0(ws + CTR_OFF + (size_t)hb * 128 + wid * 4, lane);
    const int b = hb & (N_COMM - 1);

    if (hb < N_COMM) {
        const __amdgpu_buffer_rsrc_t wsr = rsrc_of(ws, CTL_OFF);
        const int cw = wid * N_COMM + b;
        const int dbg = 128 + cw;
        (void)dbg;
        STAMP(0);
        u64 skew_w = 0;
        if (wid < 2) {
            if (threadIdx.x == 0) *(LAS unsigned*)(lds + LDS_MISC) = 0u;
            LGKM0_BARRIER();
            if (cw == 255) {
                const unsigned call = __builtin_amdgcn_readfirstlane(ticket) + 1u;
                if (lane < NR) st_sys(p.ws[lane & 7] + DBG_OFF + rank * 128, call);
                const u64 t0 = __builtin_amdgcn_s_memrealtime(); unsigned sp = 0;
                for (;;) { const unsigned v = ld_sys(ws + DBG_OFF + (lane & 7) * 128); if (__all((int)(v - call) >= 0)) break; if (give_up(ws, call, t0, sp)) break; }
                skew_w = __builtin_amdgcn_s_memrealtime() - t_start;
            }
            if (cw < N_PULL) {
                const unsigned s = (unsigned)(rank + 1 + cw % (NR - 1)) & 7u;
                const __amdgpu_buffer_rsrc_t srs = rsrc_of(p.ws[1 + cw % (NR - 1)], CTL_OFF);
                char* const queue = ws + Q_OFF + s * 128;
                unsigned tk = add_agent_lane0(queue, lane);
                u32x4 cur[4];
                const unsigned call = __builtin_amdgcn_readfirstlane(ticket) + 1u;
                const unsigned qbase = (call - 1u) * Q_TICKETS;
                const char* const pk = ws + PK_OFF + s * NGRP * 8192 + lane * 128;
                unsigned pv = call - 1u;
                int pgrp = 0;
#define JOB_OF(n_, t_, j_, so_, do_) const int t_ = (int)(n_) / NBLK, j_ = (int)(n_) % NBLK;                           \
                const unsigned do_ = (j_ < 16 ? XT_OFF + t_ * XT_T + j_ * 32768 : WT_OFF + t_ * WT_T + (j_ - 16) * 32768) + s * 4096 + lane * 16; \
                const unsigned so_ = j_ < 16 ? do_ : WS_OFF + t_ * WT_T + rank * 32768 + (j_ - 16) * 4096 + lane * 16
#define WAIT_PACKED(t_, j_) do { const int sl_ = (j_) < 16 ? (j_) : 24 + ((t_) & 7) * 4 + ((j_) >> 1);                 \
                if (((t_) >> 3) != pgrp) { pgrp = (t_) >> 3; pv = call - 1u; }                                         \
                if ((int)(__builtin_amdgcn_readlane(pv, sl_) - call) < 0) {                                            \
                    pv = ld_sys(pk + pgrp * 8192);                 \
                    const u64 t0_ = __builtin_amdgcn_s_memrealtime(); unsigned sp_ = 0;                                \
                    while ((int)(__builtin_amdgcn_readlane(pv, sl_) - call) < 0) {                                     \
                        if (give_up(ws, call, t0_, sp_)) break;                                                        \
                        if (lane == sl_) pv = ld_sys(pk + pgrp * 8192); } } } while (0)
                unsigned n = __builtin_amdgcn_readfirstlane(tk) - qbase;
                if (n < (unsigned)Q_JOBS) {
                    JOB_OF(n, t, j, so, dof);
                    WAIT_PACKED(t, j);
                    STAMP(2);
                    tk = add_agent_lane0(queue, lane);
#pragma unroll
                    for (int i = 0; i < 4; ++i) cur[i] = __builtin_amdgcn_raw_buffer_load_b128(srs, so + i * 1024, 0, AUX_SYS);
                    unsigned dcur = dof, fcur = fl_word(j, s, t);
                    for (;;) {
#pragma unroll
                        for (int i = 0; i < 4; ++i) __builtin_amdgcn_raw_buffer_store_b128(cur[i], wsr, dcur + i * 1024, 0, AUX_SYS);
                        asm volatile("" ::: "memory");
                        n = __builtin_amdgcn_readfirstlane(tk) - qbase;
                        char* const flag = ws + fcur;
                        if (n >= (unsigned)Q_JOBS) {
                            WAIT_VM(0);
                            if (lane == 0) st_sys(flag, call);
                            break;
                        }
                        JOB_OF(n, t2, j2, so2, dof2);
                        WAIT_PACKED(t2, j2);
                        tk = add_agent_lane0(queue, lane);
#pragma unroll
                        for (int i = 0; i < 4; ++i) cur[i] = __builtin_amdgcn_raw_buffer_load_b128(srs, so2 + i * 1024, 0, AUX_SYS);
                        dcur = dof2; fcur = fl_word(j2, s, t2);
                        WAIT_VM(5);
                        if (lane == 0) st_sys(flag, call);
                    }
                }
                STAMP(1);
            }
        } else {
            const __amdgpu_buffer_rsrc_t xr = rsrc_of(px, MM * KSH * 2);
            const __amdgpu_buffer_rsrc_t wr_ = rsrc_of(pw, KSH * NN * 2);
            bool met = false;
            bool x_todo = wid >= 4;
            for (int n = (wid - 2) * N_COMM + b; n < NT * 32 || x_todo;) {
                const bool do_x = x_todo && ((b >> 4) <= (n >> 8) || n >= NT * 32);
                const int g = do_x ? b >> 4 : n >> 8;
                const int delay = (do_x ? g * 8 : n >> 5) * PACK_DELAY;
                if (!met && delay > 0) { __builtin_amdgcn_s_barrier(); met = true; }
                while (__builtin_amdgcn_s_memrealtime() - t_start < (u64)delay) __builtin_amdgcn_s_sleep(2);
                u32x4 v[8];
                if (do_x) {
                    const unsigned rb = (wid - 4) * 64 + (lane & 7), piece = lane >> 3;
#pragma unroll
                    for (int i = 0; i < 8; ++i) v[i] = __builtin_amdgcn_raw_buffer_load_b128(xr, ((b & 15) * 256 + rb + i * 8) * (KSH * 2) + g * 128 + piece * 16, 0, 0);
                    if (!met) { __builtin_amdgcn_s_barrier(); met = true; }
                    const unsigned d0 = XT_OFF + (g * 8 + piece) * XT_T + (b & 15) * 32768 + rank * 4096 + rb * 16;
#pragma unroll
                    for (int i = 0; i < 8; ++i) __builtin_amdgcn_raw_buffer_store_b128(v[i], wsr, d0 + i * 128, 0, AUX_SYS);
                    char* fl = lane < 8 ? p.ws[lane & 7] + PK_OFF + (rank * NGRP + g) * 8192 + (b & 15) * 128 : ws + fl_word(b & 15, rank, g * 8 + (lane & 7));
                    asm volatile("s_waitcnt vmcnt(0)" : "+v"(fl) :: "memory");
                    unsigned old = 0;
                    if (lane == 0) old = __hip_atomic_fetch_add((LAS unsigned*)(lds + LDS_MISC), 1u, __ATOMIC_RELAXED, __HIP_MEMORY_SCOPE_WORKGROUP);
                    const unsigned call = __builtin_amdgcn_readfirstlane(ticket) + 1u;
                    if (__builtin_amdgcn_readfirstlane(old) == 3u && lane >= 1 && lane < 16) st_sys(fl, call);
                    x_todo = false;
                } else {
                    const int t = n >> 5, cb = n & 31;
                    const unsigned dest = (unsigned)cb >> 2;
#pragma unroll
                    for (int i = 0; i < 8; ++i) v[i] = __builtin_amdgcn_raw_buffer_load_b128(wr_, (unsigned)((t * 8 + i) * (NN * 2) + cb * 1024 + lane * 16), 0, 0);
                    if (!met) { __builtin_amdgcn_s_barrier(); met = true; }
                    const unsigned d0 = (dest == (unsigned)rank ? WT_OFF + t * WT_T + ((cb & 3) * 2 + (lane >> 5)) * 32768 + rank * 4096
                                                                : WS_OFF + t * WT_T + dest * 32768 + ((cb & 3) * 2 + (lane >> 5)) * 4096)
                                      + (((lane >> 4) & 1) * 128 + (lane & 15)) * 16;
#pragma unroll
                    for (int i = 0; i < 8; ++i) __builtin_amdgcn_raw_buffer_store_b128(transposed(v, i), wsr, d0 + i * 256, 0, AUX_SYS);
                    char* const dws = p.ws[(dest - rank) & 7u];
                    WAIT_VM(0);
                    const unsigned call = __builtin_amdgcn_readfirstlane(ticket) + 1u;
                    if (dest == (unsigned)rank) { if (lane < 2) st_sys(ws + fl_word(16 + (cb & 3) * 2 + lane, rank, t), call); }
                    else if (lane == 0) st_sys(dws + PK_OFF + (rank * NGRP + g) * 8192 + (32 + (t & 7) * 4 + (cb & 3)) * 128, call);
                    n += N_PACKW;
                }
            }
        }
        const unsigned call = __builtin_amdgcn_readfirstlane(ticket) + 1u;
        if (wid == 7) {
            if (b == N_COMM - 1) {
                char* amax_out = p.ws[lane & 7] + AMAX_OFF + ((lane >> 3) * NR + rank) * 128;
                asm volatile("" : "+v"(amax_out));
                const char* tm = ws + TMAX_OFF + lane * 256;
                const u64 t0 = __builtin_amdgcn_s_memrealtime(); unsigned sp = 0;
                int mx;
                for (;;) {
                    const u64 v0 = ld_sys64(tm), v1 = ld_sys64(tm + 128);
                    mx = max((int)(unsigned)v0, (int)(unsigned)v1);
                    if (__all((int)((unsigned)(v0 >> 32) - call) >= 0 && (int)((unsigned)(v1 >> 32) - call) >= 0)) break;
                    if (give_up(ws, call, t0, sp)) { mx = NAN_BITS; break; }
                }
                if (lane < 8 * AMAX_REP) st_sys64(amax_out, (unsigned)wave_max(mx), call);
                STAMP(6);
            }
            return;
        }
        {
            const int src = (rank + 1 + wid) & 7;
            const unsigned slot = OB_OFF + (unsigned)(src * 128 + b) * OB_SLOT;
            const char* const of = ws + OF_OFF + (src * 128 + b) * 128;
            const unsigned lbase = wid * LDS_WAVE;
            if (lane == 0) *(LAS unsigned*)(lds + lbase + 8192) = 0u;
            char* outp = p.out;
            asm volatile("" : "+s"(outp));
            const unsigned ob = out_base(src, b >> 3, b & 7, lane);
            {
                bool ok = await_word<8>(ws, of, call);
                const u32x4 mk = __builtin_amdgcn_raw_buffer_load_b128(wsr, slot + lane * 16, 0, AUX_SYS);
                unsigned ad[128], pos = lbase;
#pragma unroll
                for (int r = 0; r < 128; ++r) {
                    const bool kept = (int)(mk[r >> 5] << (r & 31)) >= 0;
                    const u64 m = __ballot(kept);
                    unsigned a = pos + lanes_below(m);
                    asm("" : "+v"(a));
                    ad[r] = kept ? a : lbase + 8192;
                    pos += __builtin_popcountll(m);
                    asm("" : "+s"(pos));
                }
#pragma unroll
                for (int r = 0; r < 128; r += 16)
                    asm volatile("" : "+v"(ad[r]), "+v"(ad[r + 1]), "+v"(ad[r + 2]), "+v"(ad[r + 3]), "+v"(ad[r + 4]), "+v"(ad[r + 5]), "+v"(ad[r + 6]), "+v"(ad[r + 7]),
                                      "+v"(ad[r + 8]), "+v"(ad[r + 9]), "+v"(ad[r + 10]), "+v"(ad[r + 11]), "+v"(ad[r + 12]), "+v"(ad[r + 13]), "+v"(ad[r + 14]), "+v"(ad[r + 15]));
                STAMP(3);
                ok = await_word<1>(ws, of + 4, call) && ok;
                STAMP(4);
                const char* gp = ws + slot + 1024 + lane * 16;
                for (unsigned o = 0; o < pos - lbase; o += 1024)
                    __builtin_amdgcn_global_load_lds((const GAS unsigned*)(gp + o), (LAS unsigned*)(lds + lbase + o), 16, 0, AUX_SYS);
                const unsigned fill = ok && __builtin_amdgcn_readfirstlane(ld_agent(ws + TMO_OFF)) != call ? 0u : 0x7f7f7f7fu;
                WAIT_VM(0);
                const __amdgpu_buffer_rsrc_t ors = rsrc_of(outp, (MM / NR) * NN);
#pragma unroll
                for (int r = 0; r < 128; ++r) ad[r] = *(const LAS unsigned char*)(lds + ad[r]);
                __builtin_amdgcn_sched_barrier(0);
#pragma unroll
                for (int u = 0; u < 16; ++u) {
                    const unsigned* c = ad + u * 8;
                    u32x2 o;
                    o[0] = (c[0] | (c[1] << 8) | (c[2] << 16) | (c[3] << 24)) | fill;
                    o[1] = (c[4] | (c[5] << 8) | (c[6] << 16) | (c[7] << 24)) | fill;
                    __builtin_amdgcn_raw_buffer_store_b64(o, ors, ob + (unsigned)(((u >> 2) * 16 + (u & 3)) * NN), 0, AUX_OUT);
                }
            }
            if (cw == 255) { if (skew_w > 20000) skew_w = 20000; while (__builtin_amdgcn_s_memrealtime() - t_start < 14000 + 10 * skew_w) __builtin_amdgcn_s_sleep(8); }
        }
        return;
    }

    const int g = b;
    const int dbg = g;
    (void)dbg;
    const int pm = (g >> 4) * 2 + (g & 1);
    const int pn = ((g & 7) >> 1) * 2 + ((g >> 3) & 1);
    const int fr = lane & 15, fq = lane >> 4, wr = wid >> 1, wc = wid & 1;
    if (wid == 0) STAMP(0);
    const int owner = pm >> 1, tile = (pm & 1) * 8 + pn;
    char* ows = p.ws[(owner - rank) & 7];
    char* outp = p.out;
    asm volatile("" : "+s"(ows), "+s"(outp));
    if (wid == 0) __builtin_amdgcn_fence(__ATOMIC_ACQUIRE, "");
    const unsigned call = __builtin_amdgcn_readfirstlane(ticket) + 1u;
    WAIT_VM(0);
    __builtin_amdgcn_s_barrier();

    const int op = wid >> 2;
    const char* const pollp = ws + fl_word(op ? 16 + pn : pm, wid & 3, lane);
    const unsigned unit = op ? WT_T : XT_T;
    const char* const gbase = ws + (op ? WT_OFF + pn * 32768 : XT_OFF + pm * 32768) + (wid & 3) * 4096 + lane * 16;
    LAS char* const ldst = lds + wid * 4096;
    const int aoff = fq * 4096 + (wr * 64 + fr) * 16;
    const int boff = 16384 + fq * 4096 + (wc * 128 + fr) * 16;

    int known = 0;
    int next = 0, nslot = 0;
    u32x2 pvn = {0u, 0u};
#define STAGE_NEXT() do { const char* g_ = gbase + ((next & 1) * 16384u + (unsigned)(next >> 1) * unit);              \
        LAS char* l_ = ldst + nslot * LDS_SLOT;                                                                       \
        __builtin_amdgcn_global_load_lds((const GAS unsigned*)g_, (LAS unsigned*)l_, 16, 0, 0);                       \
        __builtin_amdgcn_global_load_lds((const GAS unsigned*)g_, (LAS unsigned*)l_, 16, 1024, 0);                    \
        __builtin_amdgcn_global_load_lds((const GAS unsigned*)g_, (LAS unsigned*)l_, 16, 2048, 0);                    \
        __builtin_amdgcn_global_load_lds((const GAS unsigned*)g_, (LAS unsigned*)l_, 16, 3072, 0);                    \
        ++next; nslot = nslot == NSLOT - 1 ? 0 : nslot + 1; } while (0)
#define POLL_ISSUE() asm volatile("global_load_dwordx2 %0, %1, off sc0 sc1" : "+v"(pvn) : "v"(pollp) : "memory")
#define POLL_TAKE() do { u32x2 f_; asm volatile("s_waitcnt vmcnt(0)\n\tv_mov_b64 %0, %1" : "=v"(f_) : "v"(pvn) : "memory");    \
        known = steps_known(f_[0], f_[1], call); } while (0)
#define AWAIT_NEXT() do { if (next >= known) {                                                                        \
            const u64 t0_ = __builtin_amdgcn_s_memrealtime(); unsigned sp_ = 0;                                       \
            for (;;) { const u64 f_ = ld_sys64(pollp); known = steps_known((unsigned)f_, (unsigned)(f_ >> 32), call); \
                if (next < known) break;                                                                              \
                if (give_up<2>(ws, call, t0_, sp_)) { known = NSTEP; break; } } } } while (0)
#define MFMA_COLS(A_, B_, n0_, n1_) do { __builtin_amdgcn_sched_barrier(0);                                           \
        _Pragma("unroll") for (int n = (n0_); n < (n1_); ++n)                                                         \
            _Pragma("unroll") for (int m = 0; m < 4; ++m)                                                             \
                acc[m][n] = __builtin_amdgcn_mfma_f32_16x16x32_bf16(A_[m], B_[n], acc[m][n], 0, 0, 0);                \
        __builtin_amdgcn_sched_barrier(0); } while (0)
#define RD_A(A_, m_) A_[m_] = *(const LAS bf16x8*)(lb_ + aoff + (m_) * 256)
#define RD_B(B_, n_) B_[n_] = *(const LAS bf16x8*)(lb_ + boff + (n_) * 256)
#define TAIL_REC(st_) do {} while (0)
#define KSTEP(st_, A_, B_, AN_, BN_) do {                                                                             \
        MFMA_COLS(A_, B_, 0, 1);                                                                                      \
        if (polled) POLL_TAKE();                                          \
        const int had_ = next;                                                                                        \
        const int lim_ = min((st_) + NSLOT, known);                                   \
        if (next < lim_) { STAGE_NEXT(); if (next < lim_) STAGE_NEXT(); }                                             \
        const bool full_ = (polled ? had_ + NSLOT - 2 : next) >= (st_) + NSLOT;                                       \
        MFMA_COLS(A_, B_, 1, 4);                                                                                      \
        if (!full_) {                                                              \
            if (next == (st_) + 1 && next < NSTEP) {                                                                  \
                AWAIT_NEXT(); STAGE_NEXT();                                                                           \
                if (next < known) STAGE_NEXT();                                                                       \
            }                                                                                                         \
            const int k_ = next - (st_) - 2;                                                                          \
            if (k_ <= 0) WAIT_VM(0); else if (k_ == 1) WAIT_VM(4); else WAIT_VM(8);                                   \
        }                                                                                                             \
        WAIT_VM(12);                                                                                                  \
        LGKM0_BARRIER();                                                                                              \
        TAIL_REC(st_);                                                                                                \
        const LAS char* lb_ = lds + rslot * LDS_SLOT;               \
        RD_A(AN_, 0); RD_A(AN_, 1); RD_A(AN_, 2); RD_A(AN_, 3);                                                       \
        MFMA_COLS(A_, B_, 4, 5);                                                                                      \
        RD_B(BN_, 0); RD_B(BN_, 1); RD_B(BN_, 2); RD_B(BN_, 3);                                                       \
        MFMA_COLS(A_, B_, 5, 6);                                                                                      \
        RD_B(BN_, 4); RD_B(BN_, 5); RD_B(BN_, 6); RD_B(BN_, 7);                                                       \
        rslot = rslot == NSLOT - 1 ? 0 : rslot + 1;                                                                   \
        polled = known < min(next + 6, NSTEP);                                                                        \
        if (polled) POLL_ISSUE();                                                                                     \
        MFMA_COLS(A_, B_, 6, 8); } while (0)

    f32x4 acc[4][8];
#pragma unroll
    for (int m = 0; m < 4; ++m)
#pragma unroll
        for (int n = 0; n < 8; ++n) acc[m][n] = (f32x4){0.f, 0.f, 0.f, 0.f};

    AWAIT_NEXT(); STAGE_NEXT();
    WAIT_VM(0);
    LGKM0_BARRIER();
    int rslot = 1;
    bool polled = false;
    bf16x8 a0[4], b0[8], a1[4], b1[8];
    {
        const LAS char* lb_ = lds;
#pragma unroll
        for (int m = 0; m < 4; ++m) RD_A(a0, m);
#pragma unroll
        for (int n = 0; n < 8; ++n) RD_B(b0, n);
    }
    if (wid == 0) STAMP(1);
    for (int st = 0; st < NSTEP; st += 2) {
        KSTEP(st, a0, b0, a1, b1);
        KSTEP(st + 1, a1, b1, a0, b0);
    }
    if (wid == 0) STAMP(2);

    int mx = 0;
#pragma unroll
    for (int m = 0; m < 4; ++m)
#pragma unroll
        for (int n = 0; n < 8; ++n)
#pragma unroll
            for (int j = 0; j < 4; ++j) mx = max(mx, __float_as_int(acc[m][n][j]));
    mx = wave_max(mx);
    LAS int* red = (LAS int*)(lds + LDS_MISC);
    if (lane == 0) red[wid] = mx;
    LGKM0_BARRIER();
    if (wid == 0) {
        int t = red[lane & 7];
        ROW_SHR_MAX(t, 1); ROW_SHR_MAX(t, 2); ROW_SHR_MAX(t, 4);
        if (lane == 7) st_sys64(ws + TMAX_OFF + g * 128, (unsigned)t, call);
        STAMP(7);
    }

    const bool remote = owner != rank;
    const __amdgpu_buffer_rsrc_t sls = rsrc_of(ows + OB_OFF + (unsigned)(rank * 128 + tile * 8 + wid) * OB_SLOT, OB_SLOT);
    char* const of = ows + OF_OFF + (rank * 128 + tile * 8 + wid) * 128;
    const int efield = min(268 - (mx >> 23), 253);
    const float sc = __int_as_float(efield << 23);
    const unsigned pos0 = (unsigned)(size_t)(lds + wid * LDS_WAVE);
    const unsigned spill = (unsigned)(size_t)(lds + LDS_MISC + 64);
    unsigned pos = pos0;
#define VAL(r_) acc[(r_) >> 5][(r_) & 7][((r_) >> 3) & 3]
#define PACK2(r_) do { unsigned ta_, tb_, ha_, hb_, c_, p1_;                                                          \
        asm volatile("v_cmp_lt_i32 vcc, -1, %[a]\n\tv_cmp_lt_i32 s[88:89], -1, %[b]\n\t"                             \
                     "v_fma_mixlo_f16 %[ha], %[a], %[sc], 0\n\ts_bcnt1_i32_b64 %[c], vcc\n\t"                        \
                     "v_mbcnt_lo_u32_b32 %[ta], vcc_lo, 0\n\tv_fma_mixlo_f16 %[hb], %[b], %[sc], 0\n\t"              \
                     "s_lshl1_add_u32 %[p1], %[c], %[pos]\n\tv_mbcnt_lo_u32_b32 %[tb], s88, 0\n\t"                   \
                     "s_bcnt1_i32_b64 %[c], s[88:89]\n\tv_mbcnt_hi_u32_b32 %[ta], vcc_hi, %[ta]\n\t"                 \
                     "v_mbcnt_hi_u32_b32 %[tb], s89, %[tb]\n\tv_lshl_add_u32 %[ta], %[ta], 1, %[pos]\n\t"            \
                     "v_lshl_add_u32 %[tb], %[tb], 1, %[p1]\n\ts_lshl1_add_u32 %[pos], %[c], %[p1]\n\t"              \
                     "v_cndmask_b32 %[ta], %[sp], %[ta], vcc\n\tv_cndmask_b32 %[tb], %[sp], %[tb], s[88:89]\n\t"     \
                     "ds_write_b16 %[ta], %[ha]\n\tds_write_b16 %[tb], %[hb]"                                        \
                     : [ta] "=&v"(ta_), [tb] "=&v"(tb_), [ha] "=&v"(ha_), [hb] "=&v"(hb_), [c] "=&s"(c_),             \
                       [p1] "=&s"(p1_), [pos] "+s"(pos)                                                               \
                     : [a] "v"(VAL(r_)), [b] "v"(VAL((r_) + 1)), [sc] "s"(sc), [sp] "v"(spill)                        \
                     : "vcc", "scc", "s88", "s89", "memory"); } while (0)
#define PACK8(r_) PACK2(r_); PACK2((r_) + 2); PACK2((r_) + 4); PACK2((r_) + 6)
    if (remote) {
        u32x4 mk = {0u, 0u, 0u, 0u};
#pragma unroll
        for (int r = 0; r < 128; ++r) mk[r >> 5] = __builtin_amdgcn_alignbit(mk[r >> 5], __float_as_uint(VAL(r)), 31);
#define MASK_OUT(k_) do { if (MASK_AT == (k_)) __builtin_amdgcn_raw_buffer_store_b128(mk, sls, lane * 16, 0, AUX_SYS); } while (0)
        MASK_OUT(0); PACK8(0); MASK_OUT(1); PACK8(8); MASK_OUT(2); PACK8(16); MASK_OUT(3); PACK8(24); MASK_OUT(4); PACK8(32); MASK_OUT(5);
        if (MASK_AT < 3) { WAIT_VM(0); if (lane == 0) st_sys(of, call); }
    } else {
#pragma unroll
        for (int m = 0; m < 4; ++m) {
#pragma unroll
            for (int n = 0; n < 8; ++n)
#pragma unroll
                for (int j = 0; j < 4; ++j) acc[m][n][j] = __int_as_float(max(0, __float_as_int(acc[m][n][j])));
            asm volatile("" : "+v"(acc[m][0]), "+v"(acc[m][1]), "+v"(acc[m][2]), "+v"(acc[m][3]),
                              "+v"(acc[m][4]), "+v"(acc[m][5]), "+v"(acc[m][6]), "+v"(acc[m][7]));
        }
    }
    if (wid == 1) STAMP(6);
    if (wid == 0) {
        const char* ap = lane < NR ? ws + AMAX_OFF + ((g & (AMAX_REP - 1)) * NR + lane) * 128 : ws + TMO_OFF - 4;
        const u64 t0 = __builtin_amdgcn_s_memrealtime(); unsigned sp = 0;
        u64 v = 0;
        for (;;) {
            if (lane <= NR) v = ld_sys64(ap);
            if ((__ballot((int)((unsigned)(v >> 32) - call) >= 0) & 0xffu) == 0xffu) break;
            if (give_up(ws, call, t0, sp)) { v = (u64)call << 32; break; }
        }
        STAMP(8);
        int t = lane < NR ? (int)(unsigned)v : ((unsigned)(v >> 32) == call ? NAN_BITS : 0);
        ROW_SHR_MAX(t, 1); ROW_SHR_MAX(t, 2); ROW_SHR_MAX(t, 4); ROW_SHR_MAX(t, 8);
        if (lane == NR) red[8] = t;
    }
    LGKM0_BARRIER();
    const float inv = 448.0f / __int_as_float(red[8]);
    if (wid == 0) STAMP(3);

    if (remote) {
        const float f = inv * __int_as_float((254 - efield) << 23);
        unsigned sent = 0;
#define SEND_GATE() WAIT_VM(0)
#define SEND(upto_) do { __builtin_amdgcn_s_waitcnt(0xC07F); asm volatile("" ::: "memory");                           \
            const unsigned cnt_ = (pos - pos0) >> 1;                                                                  \
            _Pragma("nounroll") for (; sent < (upto_); sent += 1024) {                                                                    \
                const unsigned o_ = sent + lane * 16;                                                                 \
                const f16x8 h0 = *(const LAS f16x8*)(lds + wid * LDS_WAVE + o_ * 2), h1 = *(const LAS f16x8*)(lds + wid * LDS_WAVE + o_ * 2 + 16); \
                u32x4 q;                                                                                              \
                _Pragma("unroll") for (int d = 0; d < 2; ++d) {                                                       \
                    int w0 = __builtin_amdgcn_cvt_pk_fp8_f32(__builtin_fmaf((float)h0[4 * d], f, 0.f), __builtin_fmaf((float)h0[4 * d + 1], f, 0.f), 0, false); \
                    q[d] = (unsigned)__builtin_amdgcn_cvt_pk_fp8_f32(__builtin_fmaf((float)h0[4 * d + 2], f, 0.f), __builtin_fmaf((float)h0[4 * d + 3], f, 0.f), w0, true); \
                    int w1 = __builtin_amdgcn_cvt_pk_fp8_f32(__builtin_fmaf((float)h1[4 * d], f, 0.f), __builtin_fmaf((float)h1[4 * d + 1], f, 0.f), 0, false); \
                    q[2 + d] = (unsigned)__builtin_amdgcn_cvt_pk_fp8_f32(__builtin_fmaf((float)h1[4 * d + 2], f, 0.f), __builtin_fmaf((float)h1[4 * d + 3], f, 0.f), w1, true); \
                }                                                                                                     \
                SEND_GATE();                                                                                          \
                if (o_ < cnt_) __builtin_amdgcn_raw_buffer_store_b128(q, sls, 1024 + o_, 0, AUX_SYS);                 \
            } } while (0)
        if (MASK_AT >= 3) { WAIT_VM(0); if (lane == 0) st_sys(of, call); }
        SEND(cnt_ & ~1023u);
        PACK8(40); PACK8(48); PACK8(56); PACK8(64);
        SEND(cnt_ & ~1023u);
        PACK8(72); PACK8(80); PACK8(88); PACK8(96);
        SEND(cnt_ & ~1023u);
        PACK8(104); PACK8(112); PACK8(120);
        SEND(cnt_);
        WAIT_VM(0);
        if (lane == 0) st_sys(of + 4, call);
    } else {
        const __amdgpu_buffer_rsrc_t ors = rsrc_of(outp, (MM / NR) * NN);
        const unsigned ob = out_base(rank, tile, wid, lane);
#pragma unroll
        for (int m = 0; m < 4; ++m)
#pragma unroll
            for (int j = 0; j < 4; ++j) {
                u32x2 o;
                int lo = __builtin_amdgcn_cvt_pk_fp8_f32(acc[m][0][j] * inv, acc[m][1][j] * inv, 0, false);
                o[0] = (unsigned)__builtin_amdgcn_cvt_pk_fp8_f32(acc[m][2][j] * inv, acc[m][3][j] * inv, lo, true);
                int hi = __builtin_amdgcn_cvt_pk_fp8_f32(acc[m][4][j] * inv, acc[m][5][j] * inv, 0, false);
                o[1] = (unsigned)__builtin_amdgcn_cvt_pk_fp8_f32(acc[m][6][j] * inv, acc[m][7][j] * inv, hi, true);
                __builtin_amdgcn_raw_buffer_store_b64(o, ors, ob + (unsigned)((m * 16 + j) * NN), 0, AUX_OUT);
            }
    }
    if (wid == 0) STAMP(4);
    if (wid == 7) STAMP(5);
}

}

extern "C" void kernel_launch(int rank, int nranks, ncclComm_t comm,
                              hipStream_t stream,
                              void* const* d_in, const int* in_sizes, int n_in,
                              void* d_out, int out_size,
                              const int* in_dtypes, int out_dtype,
                              void* const* d_out_all,
                              void* const* d_ws_all, size_t ws_bytes) {
    (void)comm; (void)in_dtypes; (void)out_dtype; (void)d_out_all;
    if (nranks != NR || n_in < 2 || (unsigned)rank >= (unsigned)NR || ws_bytes < WS_NEED ||
        in_sizes[0] != MM * KSH || in_sizes[1] != KSH * NN || out_size != (MM / NR) * NN) {
        fprintf(stderr, "kernel_launch: unexpected problem shape\n");
        return;
    }
    Params p;
    p.x = (const char*)d_in[0];
    p.w = (const char*)d_in[1];
    for (int i = 0; i < NR; ++i) p.ws[i] = (char*)d_ws_all[(rank + i) % NR];
    p.out = (char*)d_out;
    p.rank = rank; p.pad = 0;
    hipLaunchKernelGGL(fused_gemm_relu_q8, dim3(GRID), dim3(BLOCK), 0, stream, p);
    HIP_CHECK(hipGetLastError());
}
```

```cpp
#include <hip/hip_runtime.h>
#include <rccl/rccl.h>
#include <cstdio>
#include <cstdint>

#define HIP_CHECK(expr) do { hipError_t _e = (expr); if (_e != hipSuccess) { \
    fprintf(stderr, "HIP error %s at %s:%d\n", hipGetErrorString(_e), __FILE__, __LINE__); \
    return; } } while (0)

namespace {

typedef short bf16x8 __attribute__((ext_vector_type(8)));
typedef _Float16 f16x8 __attribute__((ext_vector_type(8)));
typedef float f32x4 __attribute__((ext_vector_type(4)));
typedef unsigned u32x4 __attribute__((ext_vector_type(4)));
typedef unsigned u32x2 __attribute__((ext_vector_type(2)));
typedef unsigned long long u64;
#define LAS __attribute__((address_space(3)))
#define GAS __attribute__((address_space(1)))

constexpr int NR = 8;
constexpr int MM = 4096, KSH = 512, NN = 16384, NCOL = NN / NR;
constexpr int NT = KSH / 8;
constexpr int NBLK = 24;
#define PULL_PER_LINK 18
#define MASK_AT 0
#define AMAX_REP 8
#define AUX_OUT 17
#define SEND_DEPTH 1
#define JMODE 2
#define JORDER(k) ((k) % 3 == 2 ? 16 + (k) / 3 : (k) / 3 * 2 + (k) % 3)
#define SLEEP_K 2
#define ROT 1
#define SLOT(src, blk) ((((src) + ROT * (blk)) & 7) * 4096)
#define PACK_DELAY 100
constexpr int Q_JOBS = NT * NBLK;
constexpr int Q_TICKETS = Q_JOBS + PULL_PER_LINK;
constexpr int N_COMM = 128;
constexpr int N_PULL = (NR - 1) * PULL_PER_LINK;
constexpr int N_GEMM = 128;
constexpr int GRID = N_COMM + N_GEMM;
constexpr int BLOCK = 512;
constexpr int N_PACKW = N_COMM * 6;
constexpr int NGRP = NT / 8;
static_assert(N_PULL <= 2 * N_COMM, "pull waves are waves 0-1 of the comm workgroups");

constexpr unsigned XT_T = NR * MM * 16, WT_T = NR * NCOL * 16;
constexpr unsigned XT_OFF = 0;
constexpr unsigned WT_OFF = XT_OFF + NT * XT_T;
constexpr unsigned WS_OFF = WT_OFF + NT * WT_T;
constexpr unsigned OB_SLOT = 1024 + 8192;
constexpr unsigned OB_OFF = WS_OFF + NT * WT_T;
constexpr unsigned CTL_OFF = OB_OFF + NR * 128 * OB_SLOT;
constexpr unsigned PK_OFF = CTL_OFF;
constexpr unsigned FL_OFF = PK_OFF + NR * NGRP * 8192;
constexpr unsigned Q_OFF = FL_OFF + NBLK * NR * NT * 4;
constexpr unsigned TMAX_OFF = Q_OFF + NR * 128;
constexpr unsigned AMAX_OFF = TMAX_OFF + N_GEMM * 128;
constexpr unsigned OF_OFF = AMAX_OFF + 64 * 128;
constexpr unsigned CTR_OFF = OF_OFF + NR * 128 * 128;
constexpr unsigned TMO_OFF = CTR_OFF + GRID * 128 + 4;
constexpr unsigned DBG_OFF = CTR_OFF + GRID * 128 + 128;
constexpr size_t WS_NEED = DBG_OFF + 524288;

constexpr u64 DEADLINE = 50000000ull;
constexpr int AUX_SYS = 17;

constexpr int NSTEP = 2 * NT;
constexpr int NSLOT = 5;
constexpr int LDS_SLOT = 32768;
constexpr int LDS_BYTES = NSLOT * LDS_SLOT;
constexpr int LDS_WAVE = 16384;
constexpr int LDS_MISC = 8 * LDS_WAVE;

struct Params {
    const char* x; const char* w;
    int rank, pad;
    char* ws[NR];
    char* out;
};

__device__ __forceinline__ __amdgpu_buffer_rsrc_t rsrc_of(const void* p, unsigned bytes) {
    return __builtin_amdgcn_make_buffer_rsrc(const_cast<void*>(p), 0, bytes, 0x00020000);
}
__device__ __forceinline__ unsigned ld_sys(const void* p) {
    return __hip_atomic_load((const GAS unsigned*)p, __ATOMIC_RELAXED, __HIP_MEMORY_SCOPE_SYSTEM);
}
__device__ __forceinline__ u64 ld_sys64(const void* p) {
    return __hip_atomic_load((const GAS u64*)p, __ATOMIC_RELAXED, __HIP_MEMORY_SCOPE_SYSTEM);
}
__device__ __forceinline__ void st_sys(void* p, unsigned v) {
    __hip_atomic_store((GAS unsigned*)p, v, __ATOMIC_RELAXED, __HIP_MEMORY_SCOPE_SYSTEM);
}
__device__ __forceinline__ void st_sys64(void* p, unsigned lo, unsigned hi) {
    __hip_atomic_store((GAS u64*)p, ((u64)hi << 32) | lo, __ATOMIC_RELAXED, __HIP_MEMORY_SCOPE_SYSTEM);
}
__device__ __forceinline__ unsigned ld_agent(const void* p) {
    return __hip_atomic_load((const GAS unsigned*)p, __ATOMIC_RELAXED, __HIP_MEMORY_SCOPE_AGENT);
}
__device__ __forceinline__ unsigned add_agent_lane0(void* p, int lane) {
    unsigned v = 0, z = 0;
    asm volatile("" : "+v"(z));
    if (lane == 0) v = __hip_atomic_fetch_add((GAS unsigned*)((char*)p + z), 1u, __ATOMIC_RELAXED, __HIP_MEMORY_SCOPE_AGENT);
    return v;
}
__device__ __forceinline__ u32x4 transposed(const u32x4 (&v)[8], int i) {
    u32x4 o;
#pragma unroll
    for (int d = 0; d < 4; ++d)
        o[d] = __builtin_amdgcn_perm(v[2 * d + 1][i >> 1], v[2 * d][i >> 1], (i & 1) ? 0x07060302u : 0x05040100u);
    return o;
}
constexpr int NAN_BITS = 0x7fc00000;
#define ROW_SHR_MAX(v, n) v = max(v, __builtin_amdgcn_update_dpp(0, v, 0x110 + (n), 0xf, 0xf, true))
__device__ __forceinline__ int wave_max(int v) {
    ROW_SHR_MAX(v, 1); ROW_SHR_MAX(v, 2); ROW_SHR_MAX(v, 4); ROW_SHR_MAX(v, 8);
    return max(max(__builtin_amdgcn_readlane(v, 15), __builtin_amdgcn_readlane(v, 31)),
               max(__builtin_amdgcn_readlane(v, 47), __builtin_amdgcn_readlane(v, 63)));
}
__device__ __forceinline__ unsigned lanes_below(u64 m) {
    return __builtin_amdgcn_mbcnt_hi((unsigned)(m >> 32), __builtin_amdgcn_mbcnt_lo((unsigned)m, 0u));
}
template <int SLEEP = 1>
__device__ __forceinline__ bool give_up(char* ws, unsigned call, u64 t0, unsigned& spins) {
    bool stop = __builtin_amdgcn_s_memrealtime() - t0 > DEADLINE;
    if ((++spins & 63u) == 0u) stop = stop || __builtin_amdgcn_readfirstlane(ld_agent(ws + TMO_OFF)) == call;
    if (stop) {
        __hip_atomic_store((GAS unsigned*)(ws + TMO_OFF), call, __ATOMIC_RELAXED, __HIP_MEMORY_SCOPE_AGENT);
        return true;
    }
    __builtin_amdgcn_s_sleep(SLEEP);
    return false;
}
template <int SLEEP>
__device__ __forceinline__ bool await_word(char* ws, const char* p, unsigned call) {
    const u64 t0 = __builtin_amdgcn_s_memrealtime(); unsigned sp = 0;
    while ((int)(__builtin_amdgcn_readfirstlane(ld_sys(p)) - call) < 0)
        if (give_up<SLEEP>(ws, call, t0, sp)) return false;
    return true;
}
__device__ __forceinline__ unsigned fl_word(int blk, int src, int t) { return FL_OFF + (unsigned)(((blk * 4 + (src & 3)) * NT + t) * 2 + (src >> 2)) * 4u; }
__device__ __forceinline__ int steps_known(unsigned f0, unsigned f1, unsigned call) {
    const u64 lo = __ballot((int)(f0 - call) >= 0), hi = __ballot((int)(f1 - call) >= 0);
    const u64 miss = ~(lo & hi);
    if (miss == 0) return NSTEP;
    const int c = __builtin_ctzll(miss);
    return 2 * c + (int)((lo >> c) & 1);
}
__device__ __forceinline__ unsigned out_base(int src, int tile, int wave, int lane) {
    return (unsigned)(((tile >> 3) * 256 + (wave >> 1) * 64 + (lane >> 4) * 4) * NN + src * NCOL + (tile & 7) * 256 + (wave & 1) * 128 + (lane & 15) * 8);
}
#define WAIT_VM(n) asm volatile("s_waitcnt vmcnt(" #n ")" ::: "memory")
#define LGKM0_BARRIER() do { __builtin_amdgcn_s_waitcnt(0xC07F); asm volatile("" ::: "memory");                       \
        __builtin_amdgcn_s_barrier(); asm volatile("" ::: "memory"); } while (0)
#define STAMP(i) do {} while (0)

__global__ void __launch_bounds__(BLOCK, 2) fused_gemm_relu_q8(const Params p) {
    __shared__ __attribute__((aligned(1024))) char smem[LDS_BYTES];
    LAS char* const lds = (LAS char*)smem;
    const int hb = blockIdx.x;
    const int lane = threadIdx.x & 63;
    const int wid = __builtin_amdgcn_readfirstlane(threadIdx.x >> 6);
    int rank = p.rank;
    char* ws = p.ws[0];
    const char* px = p.x;
    const char* pw = p.w;
    const u64 t_start = __builtin_amdgcn_s_memrealtime();
    asm volatile("" : "+s"(rank), "+s"(ws), "+s"(px), "+s"(pw));
    const unsigned ticket = add_agent_lane0(ws + CTR_OFF + (size_t)hb * 128 + wid * 4, lane);
    const int b = hb & (N_COMM - 1);

    if (hb < N_COMM) {
        const __amdgpu_buffer_rsrc_t wsr = rsrc_of(ws, CTL_OFF);
        const int cw = wid * N_COMM + b;
        const int dbg = 128 + cw;
        (void)dbg;
        STAMP(0);
        if (wid < 2) {
            if (threadIdx.x == 0) *(LAS unsigned*)(lds + LDS_MISC) = 0u;
            LGKM0_BARRIER();
            if (cw < N_PULL) {
                const unsigned s = (unsigned)(rank + 1 + cw % (NR - 1)) & 7u;
                const __amdgpu_buffer_rsrc_t srs = rsrc_of(p.ws[1 + cw % (NR - 1)], CTL_OFF);
                char* const queue = ws + Q_OFF + s * 128;
                unsigned tk = add_agent_lane0(queue, lane);
                u32x4 cur[4];
                const unsigned call = __builtin_amdgcn_readfirstlane(ticket) + 1u;
                const unsigned qbase = (call - 1u) * Q_TICKETS;
                const char* const pk = ws + PK_OFF + s * NGRP * 8192 + lane * 128;
                unsigned pv = call - 1u;
                int pgrp = 0;
#define JOB_OF(n_, t_, j_, so_, do_) const int t_ = (int)(n_) / NBLK, j_ = JORDER((int)(n_) % NBLK);                           \
                const unsigned do_ = (j_ < 16 ? XT_OFF + t_ * XT_T + j_ * 32768 : WT_OFF + t_ * WT_T + (j_ - 16) * 32768) + SLOT(s, j_) + lane * 16; \
                const unsigned so_ = j_ < 16 ? do_ : WS_OFF + t_ * WT_T + rank * 32768 + (j_ - 16) * 4096 + lane * 16
#define WAIT_PACKED(t_, j_) do { const int sl_ = (j_) < 16 ? (j_) : 24 + ((t_) & 7) * 4 + ((j_) >> 1);                 \
                if (((t_) >> 3) != pgrp) { pgrp = (t_) >> 3; pv = call - 1u; }                                         \
                if ((int)(__builtin_amdgcn_readlane(pv, sl_) - call) < 0) {                                            \
                    pv = ld_sys(pk + pgrp * 8192);                 \
                    const u64 t0_ = __builtin_amdgcn_s_memrealtime(); unsigned sp_ = 0;                                \
                    while ((int)(__builtin_amdgcn_readlane(pv, sl_) - call) < 0) {                                     \
                        if (give_up(ws, call, t0_, sp_)) break;                                                        \
                        if (lane == sl_) pv = ld_sys(pk + pgrp * 8192); } } } while (0)
                unsigned n = __builtin_amdgcn_readfirstlane(tk) - qbase;
                if (n < (unsigned)Q_JOBS) {
                    JOB_OF(n, t, j, so, dof);
                    WAIT_PACKED(t, j);
                    STAMP(2);
                    tk = add_agent_lane0(queue, lane);
#pragma unroll
                    for (int i = 0; i < 4; ++i) cur[i] = __builtin_amdgcn_raw_buffer_load_b128(srs, so + i * 1024, 0, AUX_SYS);
                    unsigned dcur = dof, fcur = fl_word(j, s, t);
                    for (;;) {
#pragma unroll
                        for (int i = 0; i < 4; ++i) __builtin_amdgcn_raw_buffer_store_b128(cur[i], wsr, dcur + i * 1024, 0, AUX_SYS);
                        asm volatile("" ::: "memory");
                        n = __builtin_amdgcn_readfirstlane(tk) - qbase;
                        char* const flag = ws + fcur;
                        if (n >= (unsigned)Q_JOBS) {
                            WAIT_VM(0);
                            if (lane == 0) st_sys(flag, call);
                            break;
                        }
                        JOB_OF(n, t2, j2, so2, dof2);
                        WAIT_PACKED(t2, j2);
                        tk = add_agent_lane0(queue, lane);
#pragma unroll
                        for (int i = 0; i < 4; ++i) cur[i] = __builtin_amdgcn_raw_buffer_load_b128(srs, so2 + i * 1024, 0, AUX_SYS);
                        dcur = dof2; fcur = fl_word(j2, s, t2);
                        WAIT_VM(5);
                        if (lane == 0) st_sys(flag, call);
                    }
                }
                STAMP(1);
            }
        } else {
            const __amdgpu_buffer_rsrc_t xr = rsrc_of(px, MM * KSH * 2);
            const __amdgpu_buffer_rsrc_t wr_ = rsrc_of(pw, KSH * NN * 2);
            bool met = false;
            bool x_todo = wid >= 4;
            for (int n = (wid - 2) * N_COMM + b; n < NT * 32 || x_todo;) {
                const bool do_x = x_todo && ((b >> 4) <= (n >> 8) || n >= NT * 32);
                const int g = do_x ? b >> 4 : n >> 8;
                const int delay = (do_x ? g * 8 : n >> 5) * PACK_DELAY;
                if (!met && delay > 0) { __builtin_amdgcn_s_barrier(); met = true; }
                while (__builtin_amdgcn_s_memrealtime() - t_start < (u64)delay) __builtin_amdgcn_s_sleep(2);
                u32x4 v[8];
                if (do_x) {
                    const unsigned rb = (wid - 4) * 64 + (lane & 7), piece = lane >> 3;
#pragma unroll
                    for (int i = 0; i < 8; ++i) v[i] = __builtin_amdgcn_raw_buffer_load_b128(xr, ((b & 15) * 256 + rb + i * 8) * (KSH * 2) + g * 128 + piece * 16, 0, 0);
                    if (!met) { __builtin_amdgcn_s_barrier(); met = true; }
                    const unsigned d0 = XT_OFF + (g * 8 + piece) * XT_T + (b & 15) * 32768 + SLOT(rank, b & 15) + rb * 16;
#pragma unroll
                    for (int i = 0; i < 8; ++i) __builtin_amdgcn_raw_buffer_store_b128(v[i], wsr, d0 + i * 128, 0, AUX_SYS);
                    char* fl = lane < 8 ? p.ws[lane & 7] + PK_OFF + (rank * NGRP + g) * 8192 + (b & 15) * 128 : ws + fl_word(b & 15, rank, g * 8 + (lane & 7));
                    asm volatile("s_waitcnt vmcnt(0)" : "+v"(fl) :: "memory");
                    unsigned old = 0;
                    if (lane == 0) old = __hip_atomic_fetch_add((LAS unsigned*)(lds + LDS_MISC), 1u, __ATOMIC_RELAXED, __HIP_MEMORY_SCOPE_WORKGROUP);
                    const unsigned call = __builtin_amdgcn_readfirstlane(ticket) + 1u;
                    if (__builtin_amdgcn_readfirstlane(old) == 3u && lane >= 1 && lane < 16) st_sys(fl, call);
                    x_todo = false;
                } else {
                    const int t = n >> 5, cb = n & 31;
                    const unsigned dest = (unsigned)cb >> 2;
#pragma unroll
                    for (int i = 0; i < 8; ++i) v[i] = __builtin_amdgcn_raw_buffer_load_b128(wr_, (unsigned)((t * 8 + i) * (NN * 2) + cb * 1024 + lane * 16), 0, 0);
                    if (!met) { __builtin_amdgcn_s_barrier(); met = true; }
                    const unsigned d0 = (dest == (unsigned)rank ? WT_OFF + t * WT_T + ((cb & 3) * 2 + (lane >> 5)) * 32768 + SLOT(rank, (cb & 3) * 2 + (lane >> 5))
                                                                : WS_OFF + t * WT_T + dest * 32768 + ((cb & 3) * 2 + (lane >> 5)) * 4096)
                                      + (((lane >> 4) & 1) * 128 + (lane & 15)) * 16;
#pragma unroll
                    for (int i = 0; i < 8; ++i) __builtin_amdgcn_raw_buffer_store_b128(transposed(v, i), wsr, d0 + i * 256, 0, AUX_SYS);
                    char* const dws = p.ws[(dest - rank) & 7u];
                    WAIT_VM(0);
                    const unsigned call = __builtin_amdgcn_readfirstlane(ticket) + 1u;
                    if (dest == (unsigned)rank) { if (lane < 2) st_sys(ws + fl_word(16 + (cb & 3) * 2 + lane, rank, t), call); }
                    else if (lane == 0) st_sys(dws + PK_OFF + (rank * NGRP + g) * 8192 + (32 + (t & 7) * 4 + (cb & 3)) * 128, call);
                    n += N_PACKW;
                }
            }
        }
        const unsigned call = __builtin_amdgcn_readfirstlane(ticket) + 1u;
        if (wid == 7) {
            if (b == N_COMM - 1) {
                char* amax_out = p.ws[lane & 7] + AMAX_OFF + ((lane >> 3) * NR + rank) * 128;
                asm volatile("" : "+v"(amax_out));
                const char* tm = ws + TMAX_OFF + lane * 256;
                const u64 t0 = __builtin_amdgcn_s_memrealtime(); unsigned sp = 0;
                int mx;
                for (;;) {
                    const u64 v0 = ld_sys64(tm), v1 = ld_sys64(tm + 128);
                    mx = max((int)(unsigned)v0, (int)(unsigned)v1);
                    if (__all((int)((unsigned)(v0 >> 32) - call) >= 0 && (int)((unsigned)(v1 >> 32) - call) >= 0)) break;
                    if (give_up(ws, call, t0, sp)) { mx = NAN_BITS; break; }
                }
                if (lane < 8 * AMAX_REP) st_sys64(amax_out, (unsigned)wave_max(mx), call);
                STAMP(6);
            }
            return;
        }
        {
            const int src = (rank + 1 + wid) & 7;
            const unsigned slot = OB_OFF + (unsigned)(src * 128 + b) * OB_SLOT;
            const char* const of = ws + OF_OFF + (src * 128 + b) * 128;
            const unsigned lbase = wid * LDS_WAVE;
            if (lane == 0) *(LAS unsigned*)(lds + lbase + 8192) = 0u;
            char* outp = p.out;
            asm volatile("" : "+s"(outp));
            const unsigned ob = out_base(src, b >> 3, b & 7, lane);
            {
                bool ok = await_word<8>(ws, of, call);
                const u32x4 mk = __builtin_amdgcn_raw_buffer_load_b128(wsr, slot + lane * 16, 0, AUX_SYS);
                unsigned ad[128], pos = lbase;
#pragma unroll
                for (int r = 0; r < 128; ++r) {
                    const bool kept = (int)(mk[r >> 5] << (r & 31)) >= 0;
                    const u64 m = __ballot(kept);
                    unsigned a = pos + lanes_below(m);
                    asm("" : "+v"(a));
                    ad[r] = kept ? a : lbase + 8192;
                    pos += __builtin_popcountll(m);
                    asm("" : "+s"(pos));
                }
#pragma unroll
                for (int r = 0; r < 128; r += 16)
                    asm volatile("" : "+v"(ad[r]), "+v"(ad[r + 1]), "+v"(ad[r + 2]), "+v"(ad[r + 3]), "+v"(ad[r + 4]), "+v"(ad[r + 5]), "+v"(ad[r + 6]), "+v"(ad[r + 7]),
                                      "+v"(ad[r + 8]), "+v"(ad[r + 9]), "+v"(ad[r + 10]), "+v"(ad[r + 11]), "+v"(ad[r + 12]), "+v"(ad[r + 13]), "+v"(ad[r + 14]), "+v"(ad[r + 15]));
                STAMP(3);
                ok = await_word<1>(ws, of + 4, call) && ok;
                STAMP(4);
                const char* gp = ws + slot + 1024 + lane * 16;
                for (unsigned o = 0; o < pos - lbase; o += 1024)
                    __builtin_amdgcn_global_load_lds((const GAS unsigned*)(gp + o), (LAS unsigned*)(lds + lbase + o), 16, 0, AUX_SYS);
                const unsigned fill = ok && __builtin_amdgcn_readfirstlane(ld_agent(ws + TMO_OFF)) != call ? 0u : 0x7f7f7f7fu;
                WAIT_VM(0);
                const __amdgpu_buffer_rsrc_t ors = rsrc_of(outp, (MM / NR) * NN);
#pragma unroll
                for (int r = 0; r < 128; ++r) ad[r] = *(const LAS unsigned char*)(lds + ad[r]);
                __builtin_amdgcn_sched_barrier(0);
#pragma unroll
                for (int u = 0; u < 16; ++u) {
                    const unsigned* c = ad + u * 8;
                    u32x2 o;
                    o[0] = (c[0] | (c[1] << 8) | (c[2] << 16) | (c[3] << 24)) | fill;
                    o[1] = (c[4] | (c[5] << 8) | (c[6] << 16) | (c[7] << 24)) | fill;
                    __builtin_amdgcn_raw_buffer_store_b64(o, ors, ob + (unsigned)(((u >> 2) * 16 + (u & 3)) * NN), 0, AUX_OUT);
                }
            }
        }
        return;
    }

    const int g = b;
    const int dbg = g;
    (void)dbg;
    const int pm = (g >> 4) * 2 + (g & 1);
    const int pn = ((g & 7) >> 1) * 2 + ((g >> 3) & 1);
    const int fr = lane & 15, fq = lane >> 4, wr = wid >> 1, wc = wid & 1;
    if (wid == 0) STAMP(0);
    const int owner = pm >> 1, tile = (pm & 1) * 8 + pn;
    char* ows = p.ws[(owner - rank) & 7];
    char* outp = p.out;
    asm volatile("" : "+s"(ows), "+s"(outp));
    if (wid == 0) __builtin_amdgcn_fence(__ATOMIC_ACQUIRE, "");
    const unsigned call = __builtin_amdgcn_readfirstlane(ticket) + 1u;
    WAIT_VM(0);
    __builtin_amdgcn_s_barrier();

    const int op = wid >> 2;
    const char* const pollp = ws + fl_word(op ? 16 + pn : pm, wid & 3, lane);
    const unsigned unit = op ? WT_T : XT_T;
    const char* const gbase = ws + (op ? WT_OFF + pn * 32768 : XT_OFF + pm * 32768) + lane * 16;
    const unsigned slot0 = SLOT(wid & 3, op ? pn : pm);
    LAS char* const ldst = lds + wid * 4096;
    const int aoff = fq * 4096 + (wr * 64 + fr) * 16;
    const int boff = 16384 + fq * 4096 + (wc * 128 + fr) * 16;

    int known = 0;
    int next = 0, nslot = 0;
    u32x2 pvn = {0u, 0u};
#define STAGE_NEXT() do { const char* g_ = gbase + ((slot0 ^ ((next & 1) << 14)) + (unsigned)(next >> 1) * unit);              \
        LAS char* l_ = ldst + nslot * LDS_SLOT;                                                                       \
        __builtin_amdgcn_global_load_lds((const GAS unsigned*)g_, (LAS unsigned*)l_, 16, 0, 0);                       \
        __builtin_amdgcn_global_load_lds((const GAS unsigned*)g_, (LAS unsigned*)l_, 16, 1024, 0);                    \
        __builtin_amdgcn_global_load_lds((const GAS unsigned*)g_, (LAS unsigned*)l_, 16, 2048, 0);                    \
        __builtin_amdgcn_global_load_lds((const GAS unsigned*)g_, (LAS unsigned*)l_, 16, 3072, 0);                    \
        ++next; nslot = nslot == NSLOT - 1 ? 0 : nslot + 1; } while (0)
#define POLL_ISSUE() asm volatile("global_load_dwordx2 %0, %1, off sc0 sc1" : "+v"(pvn) : "v"(pollp) : "memory")
#define POLL_TAKE() do { u32x2 f_; asm volatile("s_waitcnt vmcnt(0)\n\tv_mov_b64 %0, %1" : "=v"(f_) : "v"(pvn) : "memory");    \
        known = steps_known(f_[0], f_[1], call); } while (0)
#define AWAIT_NEXT() do { if (next >= known) {                                                                        \
            const u64 t0_ = __builtin_amdgcn_s_memrealtime(); unsigned sp_ = 0;                                       \
            for (;;) { const u64 f_ = ld_sys64(pollp); known = steps_known((unsigned)f_, (unsigned)(f_ >> 32), call); \
                if (next < known) break;                                                                              \
                if (give_up<SLEEP_K>(ws, call, t0_, sp_)) { known = NSTEP; break; } } } } while (0)
#define MFMA_COLS(A_, B_, n0_, n1_) do { __builtin_amdgcn_sched_barrier(0);                                           \
        _Pragma("unroll") for (int n = (n0_); n < (n1_); ++n)                                                         \
            _Pragma("unroll") for (int m = 0; m < 4; ++m)                                                             \
                acc[m][n] = __builtin_amdgcn_mfma_f32_16x16x32_bf16(A_[m], B_[n], acc[m][n], 0, 0, 0);                \
        __builtin_amdgcn_sched_barrier(0); } while (0)
#define RD_A(A_, m_) A_[m_] = *(const LAS bf16x8*)(lb_ + aoff + (m_) * 256)
#define RD_B(B_, n_) B_[n_] = *(const LAS bf16x8*)(lb_ + boff + (n_) * 256)
#define TAIL_REC(st_) do {} while (0)
#define KSTEP(st_, A_, B_, AN_, BN_) do {                                                                             \
        MFMA_COLS(A_, B_, 0, 1);                                                                                      \
        if (polled) POLL_TAKE();                                          \
        const int had_ = next;                                                                                        \
        const int lim_ = min((st_) + NSLOT, known);                                   \
        if (next < lim_) { STAGE_NEXT(); if (next < lim_) STAGE_NEXT(); }                                             \
        const bool full_ = (polled ? had_ + NSLOT - 2 : next) >= (st_) + NSLOT;                                       \
        MFMA_COLS(A_, B_, 1, 4);                                                                                      \
        if (!full_) {                                                              \
            if (next == (st_) + 1 && next < NSTEP) {                                                                  \
                AWAIT_NEXT(); STAGE_NEXT();                                                                           \
                if (next < known) STAGE_NEXT();                                                                       \
            }                                                                                                         \
            const int k_ = next - (st_) - 2;                                                                          \
            if (k_ <= 0) WAIT_VM(0); else if (k_ == 1) WAIT_VM(4); else WAIT_VM(8);                                   \
        }                                                                                                             \
        WAIT_VM(12);                                                                                                  \
        LGKM0_BARRIER();                                                                                              \
        TAIL_REC(st_);                                                                                                \
        const LAS char* lb_ = lds + rslot * LDS_SLOT;               \
        RD_A(AN_, 0); RD_A(AN_, 1); RD_A(AN_, 2); RD_A(AN_, 3);                                                       \
        MFMA_COLS(A_, B_, 4, 5);                                                                                      \
        RD_B(BN_, 0); RD_B(BN_, 1); RD_B(BN_, 2); RD_B(BN_, 3);                                                       \
        MFMA_COLS(A_, B_, 5, 6);                                                                                      \
        RD_B(BN_, 4); RD_B(BN_, 5); RD_B(BN_, 6); RD_B(BN_, 7);                                                       \
        rslot = rslot == NSLOT - 1 ? 0 : rslot + 1;                                                                   \
        polled = known < min(next + 6, NSTEP);                                                                        \
        if (polled) POLL_ISSUE();                                                                                     \
        MFMA_COLS(A_, B_, 6, 8); } while (0)

    f32x4 acc[4][8];
#pragma unroll
    for (int m = 0; m < 4; ++m)
#pragma unroll
        for (int n = 0; n < 8; ++n) acc[m][n] = (f32x4){0.f, 0.f, 0.f, 0.f};

    AWAIT_NEXT(); STAGE_NEXT();
    WAIT_VM(0);
    LGKM0_BARRIER();
    int rslot = 1;
    bool polled = false;
    bf16x8 a0[4], b0[8], a1[4], b1[8];
    {
        const LAS char* lb_ = lds;
#pragma unroll
        for (int m = 0; m < 4; ++m) RD_A(a0, m);
#pragma unroll
        for (int n = 0; n < 8; ++n) RD_B(b0, n);
    }
    if (wid == 0) STAMP(1);
    for (int st = 0; st < NSTEP; st += 2) {
        KSTEP(st, a0, b0, a1, b1);
        KSTEP(st + 1, a1, b1, a0, b0);
    }
    if (wid == 0) STAMP(2);

    int mx = 0;
#pragma unroll
    for (int m = 0; m < 4; ++m)
#pragma unroll
        for (int n = 0; n < 8; ++n)
#pragma unroll
            for (int j = 0; j < 4; ++j) mx = max(mx, __float_as_int(acc[m][n][j]));
    mx = wave_max(mx);
    LAS int* red = (LAS int*)(lds + LDS_MISC);
    if (lane == 0) red[wid] = mx;
    LGKM0_BARRIER();
    if (wid == 0) {
        int t = red[lane & 7];
        ROW_SHR_MAX(t, 1); ROW_SHR_MAX(t, 2); ROW_SHR_MAX(t, 4);
        if (lane == 7) st_sys64(ws + TMAX_OFF + g * 128, (unsigned)t, call);
        STAMP(7);
    }

    const bool remote = owner != rank;
    const __amdgpu_buffer_rsrc_t sls = rsrc_of(ows + OB_OFF + (unsigned)(rank * 128 + tile * 8 + wid) * OB_SLOT, OB_SLOT);
    char* const of = ows + OF_OFF + (rank * 128 + tile * 8 + wid) * 128;
    const int efield = min(268 - (mx >> 23), 253);
    const float sc = __int_as_float(efield << 23);
    const unsigned pos0 = (unsigned)(size_t)(lds + wid * LDS_WAVE);
    const unsigned spill = (unsigned)(size_t)(lds + LDS_MISC + 64);
    unsigned pos = pos0;
#define VAL(r_) acc[(r_) >> 5][(r_) & 7][((r_) >> 3) & 3]
#define PACK2(r_) do { unsigned ta_, tb_, ha_, hb_, c_, p1_;                                                          \
        asm volatile("v_cmp_lt_i32 vcc, -1, %[a]\n\tv_cmp_lt_i32 s[88:89], -1, %[b]\n\t"                             \
                     "v_fma_mixlo_f16 %[ha], %[a], %[sc], 0\n\ts_bcnt1_i32_b64 %[c], vcc\n\t"                        \
                     "v_mbcnt_lo_u32_b32 %[ta], vcc_lo, 0\n\tv_fma_mixlo_f16 %[hb], %[b], %[sc], 0\n\t"              \
                     "s_lshl1_add_u32 %[p1], %[c], %[pos]\n\tv_mbcnt_lo_u32_b32 %[tb], s88, 0\n\t"                   \
                     "s_bcnt1_i32_b64 %[c], s[88:89]\n\tv_mbcnt_hi_u32_b32 %[ta], vcc_hi, %[ta]\n\t"                 \
                     "v_mbcnt_hi_u32_b32 %[tb], s89, %[tb]\n\tv_lshl_add_u32 %[ta], %[ta], 1, %[pos]\n\t"            \
                     "v_lshl_add_u32 %[tb], %[tb], 1, %[p1]\n\ts_lshl1_add_u32 %[pos], %[c], %[p1]\n\t"              \
                     "v_cndmask_b32 %[ta], %[sp], %[ta], vcc\n\tv_cndmask_b32 %[tb], %[sp], %[tb], s[88:89]\n\t"     \
                     "ds_write_b16 %[ta], %[ha]\n\tds_write_b16 %[tb], %[hb]"                                        \
                     : [ta] "=&v"(ta_), [tb] "=&v"(tb_), [ha] "=&v"(ha_), [hb] "=&v"(hb_), [c] "=&s"(c_),             \
                       [p1] "=&s"(p1_), [pos] "+s"(pos)                                                               \
                     : [a] "v"(VAL(r_)), [b] "v"(VAL((r_) + 1)), [sc] "s"(sc), [sp] "v"(spill)                        \
                     : "vcc", "scc", "s88", "s89", "memory"); } while (0)
#define PACK8(r_) PACK2(r_); PACK2((r_) + 2); PACK2((r_) + 4); PACK2((r_) + 6)
    if (remote) {
        u32x4 mk = {0u, 0u, 0u, 0u};
#pragma unroll
        for (int r = 0; r < 128; ++r) mk[r >> 5] = __builtin_amdgcn_alignbit(mk[r >> 5], __float_as_uint(VAL(r)), 31);
#define MASK_OUT(k_) do { if (MASK_AT == (k_)) __builtin_amdgcn_raw_buffer_store_b128(mk, sls, lane * 16, 0, AUX_SYS); } while (0)
        MASK_OUT(0); PACK8(0); MASK_OUT(1); PACK8(8); MASK_OUT(2); PACK8(16); MASK_OUT(3); PACK8(24); MASK_OUT(4); PACK8(32); MASK_OUT(5);
        if (MASK_AT < 3) { WAIT_VM(0); if (lane == 0) st_sys(of, call); }
    } else {
#pragma unroll
        for (int m = 0; m < 4; ++m) {
#pragma unroll
            for (int n = 0; n < 8; ++n)
#pragma unroll
                for (int j = 0; j < 4; ++j) acc[m][n][j] = __int_as_float(max(0, __float_as_int(acc[m][n][j])));
            asm volatile("" : "+v"(acc[m][0]), "+v"(acc[m][1]), "+v"(acc[m][2]), "+v"(acc[m][3]),
                              "+v"(acc[m][4]), "+v"(acc[m][5]), "+v"(acc[m][6]), "+v"(acc[m][7]));
        }
    }
    if (wid == 1) STAMP(6);
    if (wid == 0) {
        const char* ap = lane < NR ? ws + AMAX_OFF + ((g & (AMAX_REP - 1)) * NR + lane) * 128 : ws + TMO_OFF - 4;
        const u64 t0 = __builtin_amdgcn_s_memrealtime(); unsigned sp = 0;
        u64 v = 0;
        for (;;) {
            if (lane <= NR) v = ld_sys64(ap);
            if ((__ballot((int)((unsigned)(v >> 32) - call) >= 0) & 0xffu) == 0xffu) break;
            if (give_up(ws, call, t0, sp)) { v = (u64)call << 32; break; }
        }
        STAMP(8);
        int t = lane < NR ? (int)(unsigned)v : ((unsigned)(v >> 32) == call ? NAN_BITS : 0);
        ROW_SHR_MAX(t, 1); ROW_SHR_MAX(t, 2); ROW_SHR_MAX(t, 4); ROW_SHR_MAX(t, 8);
        if (lane == NR) red[8] = t;
    }
    LGKM0_BARRIER();
    const float inv = 448.0f / __int_as_float(red[8]);
    if (wid == 0) STAMP(3);

    if (remote) {
        const float f = inv * __int_as_float((254 - efield) << 23);
        unsigned sent = 0;
#define SEND_GATE() WAIT_VM(0)
#define SEND(upto_) do { __builtin_amdgcn_s_waitcnt(0xC07F); asm volatile("" ::: "memory");                           \
            const unsigned cnt_ = (pos - pos0) >> 1;                                                                  \
            _Pragma("nounroll") for (; sent < (upto_); sent += 1024) {                                                                    \
                const unsigned o_ = sent + lane * 16;                                                                 \
                const f16x8 h0 = *(const LAS f16x8*)(lds + wid * LDS_WAVE + o_ * 2), h1 = *(const LAS f16x8*)(lds + wid * LDS_WAVE + o_ * 2 + 16); \
                u32x4 q;                                                                                              \
                _Pragma("unroll") for (int d = 0; d < 2; ++d) {                                                       \
                    int w0 = __builtin_amdgcn_cvt_pk_fp8_f32(__builtin_fmaf((float)h0[4 * d], f, 0.f), __builtin_fmaf((float)h0[4 * d + 1], f, 0.f), 0, false); \
                    q[d] = (unsigned)__builtin_amdgcn_cvt_pk_fp8_f32(__builtin_fmaf((float)h0[4 * d + 2], f, 0.f), __builtin_fmaf((float)h0[4 * d + 3], f, 0.f), w0, true); \
                    int w1 = __builtin_amdgcn_cvt_pk_fp8_f32(__builtin_fmaf((float)h1[4 * d], f, 0.f), __builtin_fmaf((float)h1[4 * d + 1], f, 0.f), 0, false); \
                    q[2 + d] = (unsigned)__builtin_amdgcn_cvt_pk_fp8_f32(__builtin_fmaf((float)h1[4 * d + 2], f, 0.f), __builtin_fmaf((float)h1[4 * d + 3], f, 0.f), w1, true); \
                }                                                                                                     \
                SEND_GATE();                                                                                          \
                if (o_ < cnt_) __builtin_amdgcn_raw_buffer_store_b128(q, sls, 1024 + o_, 0, AUX_SYS);                 \
            } } while (0)
        if (MASK_AT >= 3) { WAIT_VM(0); if (lane == 0) st_sys(of, call); }
        SEND(cnt_ & ~1023u);
        PACK8(40); PACK8(48); PACK8(56); PACK8(64);
        SEND(cnt_ & ~1023u);
        PACK8(72); PACK8(80); PACK8(88); PACK8(96);
        SEND(cnt_ & ~1023u);
        PACK8(104); PACK8(112); PACK8(120);
        SEND(cnt_);
        WAIT_VM(0);
        if (lane == 0) st_sys(of + 4, call);
    } else {
        const __amdgpu_buffer_rsrc_t ors = rsrc_of(outp, (MM / NR) * NN);
        const unsigned ob = out_base(rank, tile, wid, lane);
#pragma unroll
        for (int m = 0; m < 4; ++m)
#pragma unroll
            for (int j = 0; j < 4; ++j) {
                u32x2 o;
                int lo = __builtin_amdgcn_cvt_pk_fp8_f32(acc[m][0][j] * inv, acc[m][1][j] * inv, 0, false);
                o[0] = (unsigned)__builtin_amdgcn_cvt_pk_fp8_f32(acc[m][2][j] * inv, acc[m][3][j] * inv, lo, true);
                int hi = __builtin_amdgcn_cvt_pk_fp8_f32(acc[m][4][j] * inv, acc[m][5][j] * inv, 0, false);
                o[1] = (unsigned)__builtin_amdgcn_cvt_pk_fp8_f32(acc[m][6][j] * inv, acc[m][7][j] * inv, hi, true);
                __builtin_amdgcn_raw_buffer_store_b64(o, ors, ob + (unsigned)((m * 16 + j) * NN), 0, AUX_OUT);
            }
    }
    if (wid == 0) STAMP(4);
    if (wid == 7) STAMP(5);
}

}

extern "C" void kernel_launch(int rank, int nranks, ncclComm_t comm,
                              hipStream_t stream,
                              void* const* d_in, const int* in_sizes, int n_in,
                              void* d_out, int out_size,
                              const int* in_dtypes, int out_dtype,
                              void* const* d_out_all,
                              void* const* d_ws_all, size_t ws_bytes) {
    (void)comm; (void)in_dtypes; (void)out_dtype; (void)d_out_all;
    if (nranks != NR || n_in < 2 || (unsigned)rank >= (unsigned)NR || ws_bytes < WS_NEED ||
        in_sizes[0] != MM * KSH || in_sizes[1] != KSH * NN || out_size != (MM / NR) * NN) {
        fprintf(stderr, "kernel_launch: unexpected problem shape\n");
        return;
    }
    Params p;
    p.x = (const char*)d_in[0];
    p.w = (const char*)d_in[1];
    for (int i = 0; i < NR; ++i) p.ws[i] = (char*)d_ws_all[(rank + i) % NR];
    p.out = (char*)d_out;
    p.rank = rank; p.pad = 0;
    hipLaunchKernelGGL(fused_gemm_relu_q8, dim3(GRID), dim3(BLOCK), 0, stream, p);
    HIP_CHECK(hipGetLastError());
}
```

```cpp
#include <hip/hip_runtime.h>
#include <rccl/rccl.h>
#include <cstdio>
#include <cstdint>

#define HIP_CHECK(expr) do { hipError_t _e = (expr); if (_e != hipSuccess) { \
    fprintf(stderr, "HIP error %s at %s:%d\n", hipGetErrorString(_e), __FILE__, __LINE__); \
    return; } } while (0)

namespace {

typedef short bf16x8 __attribute__((ext_vector_type(8)));
typedef _Float16 f16x8 __attribute__((ext_vector_type(8)));
typedef float f32x4 __attribute__((ext_vector_type(4)));
typedef unsigned u32x4 __attribute__((ext_vector_type(4)));
typedef unsigned u32x3 __attribute__((ext_vector_type(3)));
typedef unsigned u32x2 __attribute__((ext_vector_type(2)));
typedef unsigned long long u64;
#define LAS __attribute__((address_space(3)))
#define GAS __attribute__((address_space(1)))

constexpr int NR = 8;
constexpr int MM = 4096, KSH = 512, NN = 16384, NCOL = NN / NR;
constexpr int NT = KSH / 8;
constexpr int NBLK = 24;
#define PULL_PER_LINK 18
#define AMAX_REP 8
#define AUX_OUT 17
#define SEND_DEPTH 2
#define JMODE 2
#define JORDER(k) ((k) % 3 == 2 ? 16 + (k) / 3 : (k) / 3 * 2 + (k) % 3)
#define SLEEP_K 2
#define ROT 1
#define SLOT(src, blk) ((((src) + ROT * (blk)) & 7) * 4096)
#define LATE_FLAG 1
#define PACK_DELAY 100
constexpr int Q_JOBS = NT * NBLK;
constexpr int Q_TICKETS = Q_JOBS + PULL_PER_LINK;
constexpr int N_COMM = 128;
constexpr int N_PULL = (NR - 1) * PULL_PER_LINK;
constexpr int N_GEMM = 128;
constexpr int GRID = N_COMM + N_GEMM;
constexpr int BLOCK = 512;
constexpr int N_PACKW = N_COMM * 6;
constexpr int NGRP = NT / 8;
static_assert(N_PULL <= 2 * N_COMM, "pull waves are waves 0-1 of the comm workgroups");

constexpr unsigned XT_T = NR * MM * 16, WT_T = NR * NCOL * 16;
constexpr unsigned XT_OFF = 0;
constexpr unsigned WT_OFF = XT_OFF + NT * XT_T;
constexpr unsigned WS_OFF = WT_OFF + NT * WT_T;
constexpr unsigned OB_SLOT = 1024 + 8192;
constexpr unsigned OB_OFF = WS_OFF + NT * WT_T;
constexpr unsigned CTL_OFF = OB_OFF + NR * 128 * OB_SLOT;
constexpr unsigned PK_OFF = CTL_OFF;
constexpr unsigned FL_OFF = PK_OFF + NR * NGRP * 8192;
constexpr unsigned Q_OFF = FL_OFF + NBLK * NR * NT * 4;
constexpr unsigned TMAX_OFF = Q_OFF + NR * 128;
constexpr unsigned AMAX_OFF = TMAX_OFF + N_GEMM * 128;
constexpr unsigned OF_OFF = AMAX_OFF + 64 * 128;
constexpr unsigned CTR_OFF = OF_OFF + NR * 128 * 128;
constexpr unsigned TMO_OFF = CTR_OFF + GRID * 128 + 4;
constexpr unsigned DBG_OFF = CTR_OFF + GRID * 128 + 128;
constexpr size_t WS_NEED = DBG_OFF + 524288;

constexpr u64 DEADLINE = 50000000ull;
constexpr int AUX_SYS = 17;

constexpr int NSTEP = 2 * NT;
constexpr int NSLOT = 5;
constexpr int LDS_SLOT = 32768;
constexpr int LDS_BYTES = NSLOT * LDS_SLOT;
constexpr int LDS_WAVE = 16384;
constexpr int LDS_MISC = 8 * LDS_WAVE;

struct Params {
    const char* x; const char* w;
    int rank, pad;
    char* ws[NR];
    char* out;
};

__device__ __forceinline__ __amdgpu_buffer_rsrc_t rsrc_of(const void* p, unsigned bytes) {
    return __builtin_amdgcn_make_buffer_rsrc(const_cast<void*>(p), 0, bytes, 0x00020000);
}
__device__ __forceinline__ unsigned ld_sys(const void* p) {
    return __hip_atomic_load((const GAS unsigned*)p, __ATOMIC_RELAXED, __HIP_MEMORY_SCOPE_SYSTEM);
}
__device__ __forceinline__ u64 ld_sys64(const void* p) {
    return __hip_atomic_load((const GAS u64*)p, __ATOMIC_RELAXED, __HIP_MEMORY_SCOPE_SYSTEM);
}
__device__ __forceinline__ void st_sys(void* p, unsigned v) {
    __hip_atomic_store((GAS unsigned*)p, v, __ATOMIC_RELAXED, __HIP_MEMORY_SCOPE_SYSTEM);
}
__device__ __forceinline__ void st_sys64(void* p, unsigned lo, unsigned hi) {
    __hip_atomic_store((GAS u64*)p, ((u64)hi << 32) | lo, __ATOMIC_RELAXED, __HIP_MEMORY_SCOPE_SYSTEM);
}
__device__ __forceinline__ unsigned ld_agent(const void* p) {
    return __hip_atomic_load((const GAS unsigned*)p, __ATOMIC_RELAXED, __HIP_MEMORY_SCOPE_AGENT);
}
__device__ __forceinline__ unsigned add_agent_lane0(void* p, int lane) {
    unsigned v = 0, z = 0;
    asm volatile("" : "+v"(z));
    if (lane == 0) v = __hip_atomic_fetch_add((GAS unsigned*)((char*)p + z), 1u, __ATOMIC_RELAXED, __HIP_MEMORY_SCOPE_AGENT);
    return v;
}
__device__ __forceinline__ u32x4 transposed(const u32x4 (&v)[8], int i) {
    u32x4 o;
#pragma unroll
    for (int d = 0; d < 4; ++d)
        o[d] = __builtin_amdgcn_perm(v[2 * d + 1][i >> 1], v[2 * d][i >> 1], (i & 1) ? 0x07060302u : 0x05040100u);
    return o;
}
constexpr int NAN_BITS = 0x7fc00000;
#define ROW_SHR_MAX(v, n) v = max(v, __builtin_amdgcn_update_dpp(0, v, 0x110 + (n), 0xf, 0xf, true))
__device__ __forceinline__ int wave_max(int v) {
    ROW_SHR_MAX(v, 1); ROW_SHR_MAX(v, 2); ROW_SHR_MAX(v, 4); ROW_SHR_MAX(v, 8);
    return max(max(__builtin_amdgcn_readlane(v, 15), __builtin_amdgcn_readlane(v, 31)),
               max(__builtin_amdgcn_readlane(v, 47), __builtin_amdgcn_readlane(v, 63)));
}
__device__ __forceinline__ unsigned sign_byte(const u32x4& mk, int u) { return (mk[u >> 2] >> (24 - 8 * (u & 3))) & 0xffu; }
__device__ __forceinline__ void stream_layout(const u32x4& mk, unsigned (&first)[16], unsigned (&last)[16]) {
    unsigned base = 0;
#pragma unroll
    for (int u = 0; u < 16; u += 2) {
        const int c = 0x00080008 - (int)(__builtin_popcount(sign_byte(mk, u)) | (__builtin_popcount(sign_byte(mk, u + 1)) << 16));
        int v = c;
        v += __builtin_amdgcn_update_dpp(0, v, 0x111, 0xf, 0xf, true);
        v += __builtin_amdgcn_update_dpp(0, v, 0x112, 0xf, 0xf, true);
        v += __builtin_amdgcn_update_dpp(0, v, 0x114, 0xf, 0xf, true);
        v += __builtin_amdgcn_update_dpp(0, v, 0x118, 0xf, 0xf, true);
        v += __builtin_amdgcn_update_dpp(0, v, 0x142, 0xa, 0xf, false);
        v += __builtin_amdgcn_update_dpp(0, v, 0x143, 0xc, 0xf, false);
        const unsigned tot = (unsigned)__builtin_amdgcn_readlane(v, 63), exc = (unsigned)(v - c);
        first[u] = base + (exc & 0xffffu); base += tot & 0xffffu; last[u] = base;
        first[u + 1] = base + (exc >> 16); base += tot >> 16; last[u + 1] = base;
    }
}
template <int SLEEP = 1>
__device__ __forceinline__ bool give_up(char* ws, unsigned call, u64 t0, unsigned& spins) {
    bool stop = __builtin_amdgcn_s_memrealtime() - t0 > DEADLINE;
    if ((++spins & 63u) == 0u) stop = stop || __builtin_amdgcn_readfirstlane(ld_agent(ws + TMO_OFF)) == call;
    if (stop) {
        __hip_atomic_store((GAS unsigned*)(ws + TMO_OFF), call, __ATOMIC_RELAXED, __HIP_MEMORY_SCOPE_AGENT);
        return true;
    }
    __builtin_amdgcn_s_sleep(SLEEP);
    return false;
}
template <int SLEEP>
__device__ __forceinline__ bool await_word(char* ws, const char* p, unsigned call) {
    const u64 t0 = __builtin_amdgcn_s_memrealtime(); unsigned sp = 0;
    while ((int)(__builtin_amdgcn_readfirstlane(ld_sys(p)) - call) < 0)
        if (give_up<SLEEP>(ws, call, t0, sp)) return false;
    return true;
}
__device__ __forceinline__ unsigned fl_word(int blk, int src, int t) { return FL_OFF + (unsigned)(((blk * 4 + (src & 3)) * NT + t) * 2 + (src >> 2)) * 4u; }
__device__ __forceinline__ int steps_known(unsigned f0, unsigned f1, unsigned call) {
    const u64 lo = __ballot((int)(f0 - call) >= 0), hi = __ballot((int)(f1 - call) >= 0);
    const u64 miss = ~(lo & hi);
    if (miss == 0) return NSTEP;
    const int c = __builtin_ctzll(miss);
    return 2 * c + (int)((lo >> c) & 1);
}
__device__ __forceinline__ unsigned out_base(int src, int tile, int wave, int lane) {
    return (unsigned)(((tile >> 3) * 256 + (wave >> 1) * 64 + (lane >> 4) * 4) * NN + src * NCOL + (tile & 7) * 256 + (wave & 1) * 128 + (lane & 15) * 8);
}
constexpr unsigned PIECE_BYTES = 896;
__device__ __forceinline__ unsigned piece_off(int lane) { return lane & 1 ? 512 + (lane >> 1) * 12 : (lane >> 1) * 16; }
#define WAIT_VM(n) asm volatile("s_waitcnt vmcnt(" #n ")" ::: "memory")
#define LGKM0_BARRIER() do { __builtin_amdgcn_s_waitcnt(0xC07F); asm volatile("" ::: "memory");                       \
        __builtin_amdgcn_s_barrier(); asm volatile("" ::: "memory"); } while (0)
#define STAMP(i) do {} while (0)

__global__ void __launch_bounds__(BLOCK, 2) fused_gemm_relu_q8(const Params p) {
    __shared__ __attribute__((aligned(1024))) char smem[LDS_BYTES];
    LAS char* const lds = (LAS char*)smem;
    const int hb = blockIdx.x;
    const int lane = threadIdx.x & 63;
    const int wid = __builtin_amdgcn_readfirstlane(threadIdx.x >> 6);
    int rank = p.rank;
    char* ws = p.ws[0];
    const char* px = p.x;
    const char* pw = p.w;
    const u64 t_start = __builtin_amdgcn_s_memrealtime();
    asm volatile("" : "+s"(rank), "+s"(ws), "+s"(px), "+s"(pw));
    const unsigned ticket = add_agent_lane0(ws + CTR_OFF + (size_t)hb * 128 + wid * 4, lane);
    const int b = hb & (N_COMM - 1);

    if (hb < N_COMM) {
        const __amdgpu_buffer_rsrc_t wsr = rsrc_of(ws, CTL_OFF);
        const int cw = wid * N_COMM + b;
        const int dbg = 128 + cw;
        (void)dbg;
        STAMP(0);
        if (wid < 2) {
            if (threadIdx.x == 0) *(LAS unsigned*)(lds + LDS_MISC) = 0u;
            LGKM0_BARRIER();
            if (cw < N_PULL) {
                const unsigned s = (unsigned)(rank + 1 + cw % (NR - 1)) & 7u;
                const __amdgpu_buffer_rsrc_t srs = rsrc_of(p.ws[1 + cw % (NR - 1)], CTL_OFF);
                char* const queue = ws + Q_OFF + s * 128;
                unsigned tk = add_agent_lane0(queue, lane);
                u32x4 cur[4];
                const unsigned call = __builtin_amdgcn_readfirstlane(ticket) + 1u;
                const unsigned qbase = (call - 1u) * Q_TICKETS;
                const char* const pk = ws + PK_OFF + s * NGRP * 8192 + lane * 128;
                unsigned pv = call - 1u;
                int pgrp = 0;
#define JOB_OF(n_, t_, j_, so_, do_) const int t_ = (int)(n_) / NBLK, j_ = JORDER((int)(n_) % NBLK);                           \
                const unsigned do_ = (j_ < 16 ? XT_OFF + t_ * XT_T + j_ * 32768 : WT_OFF + t_ * WT_T + (j_ - 16) * 32768) + SLOT(s, j_) + lane * 16; \
                const unsigned so_ = j_ < 16 ? do_ : WS_OFF + t_ * WT_T + rank * 32768 + (j_ - 16) * 4096 + lane * 16
#define WAIT_PACKED(t_, j_) do { const int sl_ = (j_) < 16 ? (j_) : 24 + ((t_) & 7) * 4 + ((j_) >> 1);                 \
                if (((t_) >> 3) != pgrp) { pgrp = (t_) >> 3; pv = call - 1u; }                                         \
                if ((int)(__builtin_amdgcn_readlane(pv, sl_) - call) < 0) {                                            \
                    pv = ld_sys(pk + pgrp * 8192);                 \
                    const u64 t0_ = __builtin_amdgcn_s_memrealtime(); unsigned sp_ = 0;                                \
                    while ((int)(__builtin_amdgcn_readlane(pv, sl_) - call) < 0) {                                     \
                        if (give_up(ws, call, t0_, sp_)) break;                                                        \
                        if (lane == sl_) pv = ld_sys(pk + pgrp * 8192); } } } while (0)
                unsigned n = __builtin_amdgcn_readfirstlane(tk) - qbase;
                if (n < (unsigned)Q_JOBS) {
                    JOB_OF(n, t, j, so, dof);
                    WAIT_PACKED(t, j);
                    STAMP(2);
                    tk = add_agent_lane0(queue, lane);
#pragma unroll
                    for (int i = 0; i < 4; ++i) cur[i] = __builtin_amdgcn_raw_buffer_load_b128(srs, so + i * 1024, 0, AUX_SYS);
                    unsigned dcur = dof, fcur = fl_word(j, s, t);
                    for (;;) {
#pragma unroll
                        for (int i = 0; i < 4; ++i) __builtin_amdgcn_raw_buffer_store_b128(cur[i], wsr, dcur + i * 1024, 0, AUX_SYS);
                        asm volatile("" ::: "memory");
                        n = __builtin_amdgcn_readfirstlane(tk) - qbase;
                        char* const flag = ws + fcur;
                        if (n >= (unsigned)Q_JOBS) {
                            WAIT_VM(0);
                            if (lane == 0) st_sys(flag, call);
                            break;
                        }
                        JOB_OF(n, t2, j2, so2, dof2);
                        WAIT_PACKED(t2, j2);
                        tk = add_agent_lane0(queue, lane);
#pragma unroll
                        for (int i = 0; i < 4; ++i) cur[i] = __builtin_amdgcn_raw_buffer_load_b128(srs, so2 + i * 1024, 0, AUX_SYS);
                        dcur = dof2; fcur = fl_word(j2, s, t2);
                        WAIT_VM(5);
                        if (lane == 0) st_sys(flag, call);
                    }
                }
                STAMP(1);
            }
        } else {
            const __amdgpu_buffer_rsrc_t xr = rsrc_of(px, MM * KSH * 2);
            const __amdgpu_buffer_rsrc_t wr_ = rsrc_of(pw, KSH * NN * 2);
            bool met = false;
            bool x_todo = wid >= 4;
            for (int n = (wid - 2) * N_COMM + b; n < NT * 32 || x_todo;) {
                const bool do_x = x_todo && ((b >> 4) <= (n >> 8) || n >= NT * 32);
                const int g = do_x ? b >> 4 : n >> 8;
                const int delay = (do_x ? g * 8 : n >> 5) * PACK_DELAY;
                if (!met && delay > 0) { __builtin_amdgcn_s_barrier(); met = true; }
                while (__builtin_amdgcn_s_memrealtime() - t_start < (u64)delay) __builtin_amdgcn_s_sleep(2);
                u32x4 v[8];
                if (do_x) {
                    const unsigned rb = (wid - 4) * 64 + (lane & 7), piece = lane >> 3;
#pragma unroll
                    for (int i = 0; i < 8; ++i) v[i] = __builtin_amdgcn_raw_buffer_load_b128(xr, ((b & 15) * 256 + rb + i * 8) * (KSH * 2) + g * 128 + piece * 16, 0, 0);
                    if (!met) { __builtin_amdgcn_s_barrier(); met = true; }
                    const unsigned d0 = XT_OFF + (g * 8 + piece) * XT_T + (b & 15) * 32768 + SLOT(rank, b & 15) + rb * 16;
#pragma unroll
                    for (int i = 0; i < 8; ++i) __builtin_amdgcn_raw_buffer_store_b128(v[i], wsr, d0 + i * 128, 0, AUX_SYS);
                    char* fl = lane < 8 ? p.ws[lane & 7] + PK_OFF + (rank * NGRP + g) * 8192 + (b & 15) * 128 : ws + fl_word(b & 15, rank, g * 8 + (lane & 7));
                    asm volatile("s_waitcnt vmcnt(0)" : "+v"(fl) :: "memory");
                    unsigned old = 0;
                    if (lane == 0) old = __hip_atomic_fetch_add((LAS unsigned*)(lds + LDS_MISC), 1u, __ATOMIC_RELAXED, __HIP_MEMORY_SCOPE_WORKGROUP);
                    const unsigned call = __builtin_amdgcn_readfirstlane(ticket) + 1u;
                    if (__builtin_amdgcn_readfirstlane(old) == 3u && lane >= 1 && lane < 16) st_sys(fl, call);
                    x_todo = false;
                } else {
                    const int t = n >> 5, cb = n & 31;
                    const unsigned dest = (unsigned)cb >> 2;
#pragma unroll
                    for (int i = 0; i < 8; ++i) v[i] = __builtin_amdgcn_raw_buffer_load_b128(wr_, (unsigned)((t * 8 + i) * (NN * 2) + cb * 1024 + lane * 16), 0, 0);
                    if (!met) { __builtin_amdgcn_s_barrier(); met = true; }
                    const unsigned d0 = (dest == (unsigned)rank ? WT_OFF + t * WT_T + ((cb & 3) * 2 + (lane >> 5)) * 32768 + SLOT(rank, (cb & 3) * 2 + (lane >> 5))
                                                                : WS_OFF + t * WT_T + dest * 32768 + ((cb & 3) * 2 + (lane >> 5)) * 4096)
                                      + (((lane >> 4) & 1) * 128 + (lane & 15)) * 16;
#pragma unroll
                    for (int i = 0; i < 8; ++i) __builtin_amdgcn_raw_buffer_store_b128(transposed(v, i), wsr, d0 + i * 256, 0, AUX_SYS);
                    char* const dws = p.ws[(dest - rank) & 7u];
                    WAIT_VM(0);
                    const unsigned call = __builtin_amdgcn_readfirstlane(ticket) + 1u;
                    if (dest == (unsigned)rank) { if (lane < 2) st_sys(ws + fl_word(16 + (cb & 3) * 2 + lane, rank, t), call); }
                    else if (lane == 0) st_sys(dws + PK_OFF + (rank * NGRP + g) * 8192 + (32 + (t & 7) * 4 + (cb & 3)) * 128, call);
                    n += N_PACKW;
                }
            }
        }
        const unsigned call = __builtin_amdgcn_readfirstlane(ticket) + 1u;
        if (wid == 7) {
            if (b == N_COMM - 1) {
                char* amax_out = p.ws[lane & 7] + AMAX_OFF + ((lane >> 3) * NR + rank) * 128;
                asm volatile("" : "+v"(amax_out));
                const char* tm = ws + TMAX_OFF + lane * 256;
                const u64 t0 = __builtin_amdgcn_s_memrealtime(); unsigned sp = 0;
                int mx;
                for (;;) {
                    const u64 v0 = ld_sys64(tm), v1 = ld_sys64(tm + 128);
                    mx = max((int)(unsigned)v0, (int)(unsigned)v1);
                    if (__all((int)((unsigned)(v0 >> 32) - call) >= 0 && (int)((unsigned)(v1 >> 32) - call) >= 0)) break;
                    if (give_up(ws, call, t0, sp)) { mx = NAN_BITS; break; }
                }
                if (lane < 8 * AMAX_REP) st_sys64(amax_out, (unsigned)wave_max(mx), call);
                STAMP(6);
            }
            return;
        }
        {
            const int src = (rank + 1 + wid) & 7;
            const unsigned slot = OB_OFF + (unsigned)(src * 128 + b) * OB_SLOT;
            const char* const of = ws + OF_OFF + (src * 128 + b) * 128;
            const unsigned lbase = wid * LDS_WAVE;
#pragma unroll
            for (int e = 0; e < 4; ++e) {
                const unsigned idx = lane * 4 + e;
                u32x2 sel = {0u, 0u};
                unsigned rk = 0;
#pragma unroll
                for (int n = 0; n < 8; ++n) {
                    const bool neg = (idx >> (7 - n)) & 1u;
                    sel[n >> 2] |= (neg ? 0x0cu : rk) << (8 * (n & 3));
                    rk += neg ? 0u : 1u;
                }
                *(LAS u32x2*)(lds + lbase + 12288 + idx * 8) = sel;
            }
            char* outp = p.out;
            asm volatile("" : "+s"(outp));
            const unsigned ob = out_base(src, b >> 3, b & 7, lane);
            {
                bool ok = await_word<8>(ws, of, call);
                const u32x4 mk = __builtin_amdgcn_raw_buffer_load_b128(wsr, slot + lane * 16, 0, AUX_SYS);
                unsigned first[16], last[16];
                stream_layout(mk, first, last);
                u32x2 sel[16];
#pragma unroll
                for (int u = 0; u < 16; ++u) {
                    sel[u] = *(const LAS u32x2*)(lds + lbase + 12288 + sign_byte(mk, u) * 8);
                    first[u] += lbase;
                    asm volatile("" : "+v"(first[u]), "+v"(sel[u]));
                }
                STAMP(3);
                const unsigned cnt = last[15];
                const __amdgpu_buffer_rsrc_t ors = rsrc_of(outp, (MM / NR) * NN);
#define TAKE_PIECES(i0_, i1_) do { u32x4 pc[(i1_) - (i0_)];                                                            \
                    _Pragma("unroll") for (int i = (i0_); i < (i1_); ++i)                                              \
                        if (i * 1024u < cnt) pc[i - (i0_)] = __builtin_amdgcn_raw_buffer_load_b128(wsr, slot + 1024 + i * PIECE_BYTES + piece_off(lane), 0, AUX_SYS); \
                    fill = ok && __builtin_amdgcn_readfirstlane(ld_agent(ws + TMO_OFF)) != call ? 0u : 0x7f7f7f7fu;    \
                    _Pragma("unroll") for (int i = (i0_); i < (i1_); ++i)                                              \
                        if (i * 1024u < cnt) {                                                                         \
                            u32x4 q = pc[i - (i0_)];                                                                   \
                            unsigned h = ((q[0] & 0x80808080u) >> 7) | ((q[1] & 0x80808080u) >> 6) | ((q[2] & 0x80808080u) >> 5); \
                            const unsigned he = h | ((q[3] & 0x80808080u) >> 4);     \
                            h = (h << 4) | (unsigned)__builtin_amdgcn_update_dpp(0, (int)he, 0xa0, 0xf, 0xf, true);    \
                            q[0] &= 0x7f7f7f7fu; q[1] &= 0x7f7f7f7fu; q[2] &= 0x7f7f7f7fu;                             \
                            q[3] = lane & 1 ? h : q[3] & 0x7f7f7f7fu;                                                  \
                            *(LAS u32x4*)(lds + lbase + i * 1024 + lane * 16) = q;                                     \
                        } } while (0)
                unsigned fill;
                ok = await_word<1>(ws, of + 4, call) && ok;
                STAMP(4);
                TAKE_PIECES(0, 8);
                unsigned d[16][3];
#pragma unroll
                for (int u = 0; u < 16; ++u)
#pragma unroll
                    for (int i = 0; i < 3; ++i) d[u][i] = *(const LAS unsigned*)(lds + (first[u] & ~3u) + i * 4);
                __builtin_amdgcn_sched_barrier(0);
#pragma unroll
                for (int u = 0; u < 16; ++u) {
                    const unsigned lo = __builtin_amdgcn_alignbyte(d[u][1], d[u][0], first[u]), hi = __builtin_amdgcn_alignbyte(d[u][2], d[u][1], first[u]);
                    u32x2 o;
                    o[0] = __builtin_amdgcn_perm(hi, lo, sel[u][0]) | fill;
                    o[1] = __builtin_amdgcn_perm(hi, lo, sel[u][1]) | fill;
                    __builtin_amdgcn_raw_buffer_store_b64(o, ors, ob + (unsigned)(((u >> 2) * 16 + (u & 3)) * NN), 0, AUX_OUT);
                }
            }
        }
        return;
    }

    const int g = b;
    const int dbg = g;
    (void)dbg;
    const int pm = (g >> 4) * 2 + (g & 1);
    const int pn = ((g & 7) >> 1) * 2 + ((g >> 3) & 1);
    const int fr = lane & 15, fq = lane >> 4, wr = wid >> 1, wc = wid & 1;
    if (wid == 0) STAMP(0);
    const int owner = pm >> 1, tile = (pm & 1) * 8 + pn;
    char* ows = p.ws[(owner - rank) & 7];
    char* outp = p.out;
    asm volatile("" : "+s"(ows), "+s"(outp));
    if (wid == 0) __builtin_amdgcn_fence(__ATOMIC_ACQUIRE, "");
    const unsigned call = __builtin_amdgcn_readfirstlane(ticket) + 1u;
    WAIT_VM(0);
    __builtin_amdgcn_s_barrier();

    const int op = wid >> 2;
    const char* const pollp = ws + fl_word(op ? 16 + pn : pm, wid & 3, lane);
    const unsigned unit = op ? WT_T : XT_T;
    const char* const gbase = ws + (op ? WT_OFF + pn * 32768 : XT_OFF + pm * 32768) + lane * 16;
    const unsigned slot0 = SLOT(wid & 3, op ? pn : pm);
    LAS char* const ldst = lds + wid * 4096;
    const int aoff = fq * 4096 + (wr * 64 + fr) * 16;
    const int boff = 16384 + fq * 4096 + (wc * 128 + fr) * 16;

    int known = 0;
    int next = 0, nslot = 0;
    u32x2 pvn = {0u, 0u};
#define STAGE_NEXT() do { const char* g_ = gbase + ((slot0 ^ ((next & 1) << 14)) + (unsigned)(next >> 1) * unit);              \
        LAS char* l_ = ldst + nslot * LDS_SLOT;                                                                       \
        __builtin_amdgcn_global_load_lds((const GAS unsigned*)g_, (LAS unsigned*)l_, 16, 0, 0);                       \
        __builtin_amdgcn_global_load_lds((const GAS unsigned*)g_, (LAS unsigned*)l_, 16, 1024, 0);                    \
        __builtin_amdgcn_global_load_lds((const GAS unsigned*)g_, (LAS unsigned*)l_, 16, 2048, 0);                    \
        __builtin_amdgcn_global_load_lds((const GAS unsigned*)g_, (LAS unsigned*)l_, 16, 3072, 0);                    \
        ++next; nslot = nslot == NSLOT - 1 ? 0 : nslot + 1; } while (0)
#define POLL_ISSUE() asm volatile("global_load_dwordx2 %0, %1, off sc0 sc1" : "+v"(pvn) : "v"(pollp) : "memory")
#define POLL_TAKE() do { u32x2 f_; asm volatile("s_waitcnt vmcnt(0)\n\tv_mov_b64 %0, %1" : "=v"(f_) : "v"(pvn) : "memory");    \
        known = steps_known(f_[0], f_[1], call); } while (0)
#define AWAIT_NEXT() do { if (next >= known) {                                                                        \
            const u64 t0_ = __builtin_amdgcn_s_memrealtime(); unsigned sp_ = 0;                                       \
            for (;;) { const u64 f_ = ld_sys64(pollp); known = steps_known((unsigned)f_, (unsigned)(f_ >> 32), call); \
                if (next < known) break;                                                                              \
                if (give_up<SLEEP_K>(ws, call, t0_, sp_)) { known = NSTEP; break; } } } } while (0)
#define MFMA_COLS(A_, B_, n0_, n1_) do { __builtin_amdgcn_sched_barrier(0);                                           \
        _Pragma("unroll") for (int n = (n0_); n < (n1_); ++n)                                                         \
            _Pragma("unroll") for (int m = 0; m < 4; ++m)                                                             \
                acc[m][n] = __builtin_amdgcn_mfma_f32_16x16x32_bf16(A_[m], B_[n], acc[m][n], 0, 0, 0);                \
        __builtin_amdgcn_sched_barrier(0); } while (0)
#define RD_A(A_, m_) A_[m_] = *(const LAS bf16x8*)(lb_ + aoff + (m_) * 256)
#define RD_B(B_, n_) B_[n_] = *(const LAS bf16x8*)(lb_ + boff + (n_) * 256)
#define TAIL_REC(st_) do {} while (0)
#define KSTEP(st_, A_, B_, AN_, BN_) do {                                                                             \
        MFMA_COLS(A_, B_, 0, 1);                                                                                      \
        if (polled) POLL_TAKE();                                          \
        const int had_ = next;                                                                                        \
        const int lim_ = min((st_) + NSLOT, known);                                   \
        if (next < lim_) { STAGE_NEXT(); if (next < lim_) STAGE_NEXT(); }                                             \
        const bool full_ = (polled ? had_ + NSLOT - 2 : next) >= (st_) + NSLOT;                                       \
        MFMA_COLS(A_, B_, 1, 4);                                                                                      \
        if (!full_) {                                                              \
            if (next == (st_) + 1 && next < NSTEP) {                                                                  \
                AWAIT_NEXT(); STAGE_NEXT();                                                                           \
                if (next < known) STAGE_NEXT();                                                                       \
            }                                                                                                         \
            const int k_ = next - (st_) - 2;                                                                          \
            if (k_ <= 0) WAIT_VM(0); else if (k_ == 1) WAIT_VM(4); else WAIT_VM(8);                                   \
        }                                                                                                             \
        WAIT_VM(12);                                                                                                  \
        LGKM0_BARRIER();                                                                                              \
        TAIL_REC(st_);                                                                                                \
        const LAS char* lb_ = lds + rslot * LDS_SLOT;               \
        RD_A(AN_, 0); RD_A(AN_, 1); RD_A(AN_, 2); RD_A(AN_, 3);                                                       \
        MFMA_COLS(A_, B_, 4, 5);                                                                                      \
        RD_B(BN_, 0); RD_B(BN_, 1); RD_B(BN_, 2); RD_B(BN_, 3);                                                       \
        MFMA_COLS(A_, B_, 5, 6);                                                                                      \
        RD_B(BN_, 4); RD_B(BN_, 5); RD_B(BN_, 6); RD_B(BN_, 7);                                                       \
        rslot = rslot == NSLOT - 1 ? 0 : rslot + 1;                                                                   \
        polled = known < min(next + 6, NSTEP);                                                                        \
        if (polled) POLL_ISSUE();                                                                                     \
        MFMA_COLS(A_, B_, 6, 8); } while (0)

    f32x4 acc[4][8];
#pragma unroll
    for (int m = 0; m < 4; ++m)
#pragma unroll
        for (int n = 0; n < 8; ++n) acc[m][n] = (f32x4){0.f, 0.f, 0.f, 0.f};

    AWAIT_NEXT(); STAGE_NEXT();
    WAIT_VM(0);
    LGKM0_BARRIER();
    int rslot = 1;
    bool polled = false;
    bf16x8 a0[4], b0[8], a1[4], b1[8];
    {
        const LAS char* lb_ = lds;
#pragma unroll
        for (int m = 0; m < 4; ++m) RD_A(a0, m);
#pragma unroll
        for (int n = 0; n < 8; ++n) RD_B(b0, n);
    }
    if (wid == 0) STAMP(1);
    for (int st = 0; st < NSTEP; st += 2) {
        KSTEP(st, a0, b0, a1, b1);
        KSTEP(st + 1, a1, b1, a0, b0);
    }
    if (wid == 0) STAMP(2);

    int mx = 0;
#pragma unroll
    for (int m = 0; m < 4; ++m)
#pragma unroll
        for (int n = 0; n < 8; ++n)
#pragma unroll
            for (int j = 0; j < 4; ++j) mx = max(mx, __float_as_int(acc[m][n][j]));
    mx = wave_max(mx);
    LAS int* red = (LAS int*)(lds + LDS_MISC);
    if (lane == 0) red[wid] = mx;
    LGKM0_BARRIER();
    if (wid == 0) {
        int t = red[lane & 7];
        ROW_SHR_MAX(t, 1); ROW_SHR_MAX(t, 2); ROW_SHR_MAX(t, 4);
        if (lane == 7) st_sys64(ws + TMAX_OFF + g * 128, (unsigned)t, call);
        STAMP(7);
    }

    const bool remote = owner != rank;
    const __amdgpu_buffer_rsrc_t sls = rsrc_of(ows + OB_OFF + (unsigned)(rank * 128 + tile * 8 + wid) * OB_SLOT, OB_SLOT);
    char* const of = ows + OF_OFF + (rank * 128 + tile * 8 + wid) * 128;
    const int efield = min(268 - (mx >> 23), 253);
    const float sc = __int_as_float(efield << 23);
    const unsigned pos0 = (unsigned)(size_t)(lds + wid * LDS_WAVE);
    const unsigned spill = (unsigned)(size_t)(lds + LDS_MISC + 64);
    unsigned first[16], last[16];
#define VAL(u_, n_) acc[(u_) >> 2][n_][(u_) & 3]
#define PACK1(n_) "v_ashrrev_i32 %[s], 31, %[v" #n_ "]\n\tv_fma_mixlo_f16 %[h], %[v" #n_ "], %[sc], 0\n\t"              \
                  "v_bfi_b32 %[a], %[s], %[sp], %[p]\n\tv_lshl_add_u32 %[p], %[s], 1, %[p]\n\t"                      \
                  "ds_write_b16 %[a], %[h] offset:2*" #n_ "\n\t"
#define PACK8(u_) do { unsigned p_ = pos0 + 2 * first[u_], s_, h_, a_;                                                \
        asm volatile(PACK1(0) PACK1(1) PACK1(2) PACK1(3) PACK1(4) PACK1(5) PACK1(6) PACK1(7)                          \
                     : [p] "+v"(p_), [s] "=&v"(s_), [h] "=&v"(h_), [a] "=&v"(a_)                                      \
                     : [v0] "v"(VAL(u_, 0)), [v1] "v"(VAL(u_, 1)), [v2] "v"(VAL(u_, 2)), [v3] "v"(VAL(u_, 3)),        \
                       [v4] "v"(VAL(u_, 4)), [v5] "v"(VAL(u_, 5)), [v6] "v"(VAL(u_, 6)), [v7] "v"(VAL(u_, 7)),        \
                       [sc] "s"(sc), [sp] "v"(spill) : "memory"); } while (0)
    u32x4 mk = {0u, 0u, 0u, 0u};
    if (remote) {
#pragma unroll
        for (int r = 0; r < 128; ++r) mk[r >> 5] = __builtin_amdgcn_alignbit(mk[r >> 5], __float_as_uint(VAL(r >> 3, r & 7)), 31);
        if (!LATE_FLAG || wid != 0) __builtin_amdgcn_raw_buffer_store_b128(mk, sls, lane * 16, 0, AUX_SYS);
        stream_layout(mk, first, last);
        PACK8(0); PACK8(1); PACK8(2); PACK8(3); PACK8(4);
        if (!LATE_FLAG) { WAIT_VM(0); if (lane == 0) st_sys(of, call); }
    } else {
#pragma unroll
        for (int m = 0; m < 4; ++m) {
#pragma unroll
            for (int n = 0; n < 8; ++n)
#pragma unroll
                for (int j = 0; j < 4; ++j) acc[m][n][j] = __int_as_float(max(0, __float_as_int(acc[m][n][j])));
            asm volatile("" : "+v"(acc[m][0]), "+v"(acc[m][1]), "+v"(acc[m][2]), "+v"(acc[m][3]),
                              "+v"(acc[m][4]), "+v"(acc[m][5]), "+v"(acc[m][6]), "+v"(acc[m][7]));
        }
    }
    if (wid == 1) STAMP(6);
    if (wid == 0) {
        const char* ap = lane < NR ? ws + AMAX_OFF + ((g & (AMAX_REP - 1)) * NR + lane) * 128 : ws + TMO_OFF - 4;
        const u64 t0 = __builtin_amdgcn_s_memrealtime(); unsigned sp = 0;
        u64 v = 0;
        for (;;) {
            if (lane <= NR) v = ld_sys64(ap);
            if ((__ballot((int)((unsigned)(v >> 32) - call) >= 0) & 0xffu) == 0xffu) break;
            if (give_up(ws, call, t0, sp)) { v = (u64)call << 32; break; }
        }
        STAMP(8);
        int t = lane < NR ? (int)(unsigned)v : ((unsigned)(v >> 32) == call ? NAN_BITS : 0);
        ROW_SHR_MAX(t, 1); ROW_SHR_MAX(t, 2); ROW_SHR_MAX(t, 4); ROW_SHR_MAX(t, 8);
        if (lane == NR) red[8] = t;
    }
    LGKM0_BARRIER();
    const float inv = 448.0f / __int_as_float(red[8]);
    if (wid == 0) STAMP(3);

    if (remote) {
        const float f = inv * __int_as_float((254 - efield) << 23);
        unsigned sent = 0;
        const unsigned hshift = (lane & 1) * 4, woff = piece_off(lane);
#define SEND_GATE() WAIT_VM(2)
#define SEND(cnt_, upto_) do { __builtin_amdgcn_s_waitcnt(0xC07F); asm volatile("" ::: "memory");                           \
            _Pragma("nounroll") for (; sent < (upto_); sent += 1024) {                                                                    \
                const unsigned o_ = sent + lane * 16;                                                                 \
                const f16x8 h0 = *(const LAS f16x8*)(lds + wid * LDS_WAVE + o_ * 2), h1 = *(const LAS f16x8*)(lds + wid * LDS_WAVE + o_ * 2 + 16); \
                u32x4 q;                                                                                              \
                _Pragma("unroll") for (int d = 0; d < 2; ++d) {                                                       \
                    int w0 = __builtin_amdgcn_cvt_pk_fp8_f32(__builtin_fmaf((float)h0[4 * d], f, 0.f), __builtin_fmaf((float)h0[4 * d + 1], f, 0.f), 0, false); \
                    q[d] = (unsigned)__builtin_amdgcn_cvt_pk_fp8_f32(__builtin_fmaf((float)h0[4 * d + 2], f, 0.f), __builtin_fmaf((float)h0[4 * d + 3], f, 0.f), w0, true); \
                    int w1 = __builtin_amdgcn_cvt_pk_fp8_f32(__builtin_fmaf((float)h1[4 * d], f, 0.f), __builtin_fmaf((float)h1[4 * d + 1], f, 0.f), 0, false); \
                    q[2 + d] = (unsigned)__builtin_amdgcn_cvt_pk_fp8_f32(__builtin_fmaf((float)h1[4 * d + 2], f, 0.f), __builtin_fmaf((float)h1[4 * d + 3], f, 0.f), w1, true); \
                }                                                                                                     \
                  \
                const unsigned h_ = (unsigned)__builtin_amdgcn_update_dpp(0, (int)q[3], 0xf5, 0xf, 0xf, true) >> hshift; \
                _Pragma("unroll") for (int d = 0; d < 4; ++d) q[d] |= (h_ << (7 - d)) & 0x80808080u;                  \
                SEND_GATE();                                                                                          \
                if ((o_ & ~31u) < (cnt_)) {                                                                             \
                    const unsigned w_ = 1024 + (sent >> 10) * PIECE_BYTES + woff;                                     \
                    if (lane & 1) __builtin_amdgcn_raw_buffer_store_b96((u32x3){q[0], q[1], q[2]}, sls, w_, 0, AUX_SYS); \
                    else __builtin_amdgcn_raw_buffer_store_b128(q, sls, w_, 0, AUX_SYS);                              \
                }                                                                                                     \
            } } while (0)
        if (LATE_FLAG && wid == 0) __builtin_amdgcn_raw_buffer_store_b128(mk, sls, lane * 16, 0, AUX_SYS);
        SEND(last[4], last[4] & ~1023u);
        PACK8(5); PACK8(6); PACK8(7); PACK8(8);
        if (LATE_FLAG) { if (sent == 1024u) WAIT_VM(2); else WAIT_VM(0); if (lane == 0) st_sys(of, call); }
        SEND(last[8], last[8] & ~1023u);
        PACK8(9); PACK8(10); PACK8(11); PACK8(12);
        SEND(last[12], last[12] & ~1023u);
        PACK8(13); PACK8(14); PACK8(15);
        SEND(last[15], last[15]);
        WAIT_VM(0);
        if (lane == 0) st_sys(of + 4, call);
    } else {
        const __amdgpu_buffer_rsrc_t ors = rsrc_of(outp, (MM / NR) * NN);
        const unsigned ob = out_base(rank, tile, wid, lane);
#pragma unroll
        for (int m = 0; m < 4; ++m)
#pragma unroll
            for (int j = 0; j < 4; ++j) {
                u32x2 o;
                int lo = __builtin_amdgcn_cvt_pk_fp8_f32(acc[m][0][j] * inv, acc[m][1][j] * inv, 0, false);
                o[0] = (unsigned)__builtin_amdgcn_cvt_pk_fp8_f32(acc[m][2][j] * inv, acc[m][3][j] * inv, lo, true);
                int hi = __builtin_amdgcn_cvt_pk_fp8_f32(acc[m][4][j] * inv, acc[m][5][j] * inv, 0, false);
                o[1] = (unsigned)__builtin_amdgcn_cvt_pk_fp8_f32(acc[m][6][j] * inv, acc[m][7][j] * inv, hi, true);
                __builtin_amdgcn_raw_buffer_store_b64(o, ors, ob + (unsigned)((m * 16 + j) * NN), 0, AUX_OUT);
            }
    }
    if (wid == 0) STAMP(4);
    if (wid == 7) STAMP(5);
}

}

extern "C" void kernel_launch(int rank, int nranks, ncclComm_t comm,
                              hipStream_t stream,
                              void* const* d_in, const int* in_sizes, int n_in,
                              void* d_out, int out_size,
                              const int* in_dtypes, int out_dtype,
                              void* const* d_out_all,
                              void* const* d_ws_all, size_t ws_bytes) {
    (void)comm; (void)in_dtypes; (void)out_dtype; (void)d_out_all;
    if (nranks != NR || n_in < 2 || (unsigned)rank >= (unsigned)NR || ws_bytes < WS_NEED ||
        in_sizes[0] != MM * KSH || in_sizes[1] != KSH * NN || out_size != (MM / NR) * NN) {
        fprintf(stderr, "kernel_launch: unexpected problem shape\n");
        return;
    }
    Params p;
    p.x = (const char*)d_in[0];
    p.w = (const char*)d_in[1];
    for (int i = 0; i < NR; ++i) p.ws[i] = (char*)d_ws_all[(rank + i) % NR];
    p.out = (char*)d_out;
    p.rank = rank; p.pad = 0;
    hipLaunchKernelGGL(fused_gemm_relu_q8, dim3(GRID), dim3(BLOCK), 0, stream, p);
    HIP_CHECK(hipGetLastError());
}
```

```cpp
#include <hip/hip_runtime.h>
#include <rccl/rccl.h>
#include <cstdio>
#include <cstdint>

#define HIP_CHECK(expr) do { hipError_t _e = (expr); if (_e != hipSuccess) { \
    fprintf(stderr, "HIP error %s at %s:%d\n", hipGetErrorString(_e), __FILE__, __LINE__); \
    return; } } while (0)

namespace {

typedef short bf16x8 __attribute__((ext_vector_type(8)));
typedef _Float16 f16x8 __attribute__((ext_vector_type(8)));
typedef float f32x4 __attribute__((ext_vector_type(4)));
typedef unsigned u32x4 __attribute__((ext_vector_type(4)));
typedef unsigned u32x3 __attribute__((ext_vector_type(3)));
typedef unsigned u32x2 __attribute__((ext_vector_type(2)));
typedef unsigned long long u64;
#define LAS __attribute__((address_space(3)))
#define GAS __attribute__((address_space(1)))

constexpr int NR = 8;
constexpr int MM = 4096, KSH = 512, NN = 16384, NCOL = NN / NR;
constexpr int NT = KSH / 8;
constexpr int NBLK = 24;
#define PULL_PER_LINK 18
#define AMAX_REP 8
#define AUX_OUT 17
#define SEND_DEPTH 2
#define JMODE 2
#define JORDER(k) ((k) % 3 == 2 ? 16 + (k) / 3 : (k) / 3 * 2 + (k) % 3)
#define SLEEP_K 2
#define ROT 1
#define SLOT(src, blk) ((((src) + ROT * (blk)) & 7) * 4096)
#define LATE_FLAG 1
#define STAGE_AUX 17
#define POLL_POS 0
#define TAKE_COL 1
#define POLL_AHEAD 6
#define PACK_DELAY 100
constexpr int Q_JOBS = NT * NBLK;
constexpr int Q_TICKETS = Q_JOBS + PULL_PER_LINK;
constexpr int N_COMM = 128;
constexpr int N_PULL = (NR - 1) * PULL_PER_LINK;
constexpr int N_GEMM = 128;
constexpr int GRID = N_COMM + N_GEMM;
constexpr int BLOCK = 512;
constexpr int N_PACKW = N_COMM * 6;
constexpr int NGRP = NT / 8;
static_assert(N_PULL <= 2 * N_COMM, "pull waves are waves 0-1 of the comm workgroups");

constexpr unsigned XT_T = NR * MM * 16, WT_T = NR * NCOL * 16;
constexpr unsigned XT_OFF = 0;
constexpr unsigned WT_OFF = XT_OFF + NT * XT_T;
constexpr unsigned WS_OFF = WT_OFF + NT * WT_T;
constexpr unsigned OB_SLOT = 1024 + 8192;
constexpr unsigned OB_OFF = WS_OFF + NT * WT_T;
constexpr unsigned CTL_OFF = OB_OFF + NR * 128 * OB_SLOT;
constexpr unsigned PK_OFF = CTL_OFF;
constexpr unsigned FL_OFF = PK_OFF + NR * NGRP * 8192;
constexpr unsigned Q_OFF = FL_OFF + NBLK * NR * NT * 4;
constexpr unsigned TMAX_OFF = Q_OFF + NR * 128;
constexpr unsigned AMAX_OFF = TMAX_OFF + N_GEMM * 128;
constexpr unsigned OF_OFF = AMAX_OFF + 64 * 128;
constexpr unsigned CTR_OFF = OF_OFF + NR * 128 * 128;
constexpr unsigned TMO_OFF = CTR_OFF + GRID * 128 + 4;
constexpr unsigned DBG_OFF = CTR_OFF + GRID * 128 + 128;
constexpr size_t WS_NEED = DBG_OFF + 524288;

constexpr u64 DEADLINE = 50000000ull;
constexpr int AUX_SYS = 17;

constexpr int NSTEP = 2 * NT;
constexpr int NSLOT = 5;
constexpr int LDS_SLOT = 32768;
constexpr int LDS_BYTES = NSLOT * LDS_SLOT;
constexpr int LDS_WAVE = 16384;
constexpr int LDS_MISC = 8 * LDS_WAVE;

struct Params {
    const char* x; const char* w;
    int rank, pad;
    char* ws[NR];
    char* out;
};

__device__ __forceinline__ __amdgpu_buffer_rsrc_t rsrc_of(const void* p, unsigned bytes) {
    return __builtin_amdgcn_make_buffer_rsrc(const_cast<void*>(p), 0, bytes, 0x00020000);
}
__device__ __forceinline__ unsigned ld_sys(const void* p) {
    return __hip_atomic_load((const GAS unsigned*)p, __ATOMIC_RELAXED, __HIP_MEMORY_SCOPE_SYSTEM);
}
__device__ __forceinline__ u64 ld_sys64(const void* p) {
    return __hip_atomic_load((const GAS u64*)p, __ATOMIC_RELAXED, __HIP_MEMORY_SCOPE_SYSTEM);
}
__device__ __forceinline__ void st_sys(void* p, unsigned v) {
    __hip_atomic_store((GAS unsigned*)p, v, __ATOMIC_RELAXED, __HIP_MEMORY_SCOPE_SYSTEM);
}
__device__ __forceinline__ void st_sys64(void* p, unsigned lo, unsigned hi) {
    __hip_atomic_store((GAS u64*)p, ((u64)hi << 32) | lo, __ATOMIC_RELAXED, __HIP_MEMORY_SCOPE_SYSTEM);
}
__device__ __forceinline__ unsigned ld_agent(const void* p) {
    return __hip_atomic_load((const GAS unsigned*)p, __ATOMIC_RELAXED, __HIP_MEMORY_SCOPE_AGENT);
}
__device__ __forceinline__ unsigned add_agent_lane0(void* p, int lane) {
    unsigned v = 0, z = 0;
    asm volatile("" : "+v"(z));
    if (lane == 0) v = __hip_atomic_fetch_add((GAS unsigned*)((char*)p + z), 1u, __ATOMIC_RELAXED, __HIP_MEMORY_SCOPE_AGENT);
    return v;
}
__device__ __forceinline__ u32x4 transposed(const u32x4 (&v)[8], int i) {
    u32x4 o;
#pragma unroll
    for (int d = 0; d < 4; ++d)
        o[d] = __builtin_amdgcn_perm(v[2 * d + 1][i >> 1], v[2 * d][i >> 1], (i & 1) ? 0x07060302u : 0x05040100u);
    return o;
}
constexpr int NAN_BITS = 0x7fc00000;
#define ROW_SHR_MAX(v, n) v = max(v, __builtin_amdgcn_update_dpp(0, v, 0x110 + (n), 0xf, 0xf, true))
__device__ __forceinline__ int wave_max(int v) {
    ROW_SHR_MAX(v, 1); ROW_SHR_MAX(v, 2); ROW_SHR_MAX(v, 4); ROW_SHR_MAX(v, 8);
    return max(max(__builtin_amdgcn_readlane(v, 15), __builtin_amdgcn_readlane(v, 31)),
               max(__builtin_amdgcn_readlane(v, 47), __builtin_amdgcn_readlane(v, 63)));
}
__device__ __forceinline__ unsigned sign_byte(const u32x4& mk, int u) { return (mk[u >> 2] >> (24 - 8 * (u & 3))) & 0xffu; }
__device__ __forceinline__ void stream_layout(const u32x4& mk, unsigned (&first)[16], unsigned (&last)[16]) {
    unsigned base = 0;
#pragma unroll
    for (int u = 0; u < 16; u += 2) {
        const int c = 0x00080008 - (int)(__builtin_popcount(sign_byte(mk, u)) | (__builtin_popcount(sign_byte(mk, u + 1)) << 16));
        int v = c;
        v += __builtin_amdgcn_update_dpp(0, v, 0x111, 0xf, 0xf, true);
        v += __builtin_amdgcn_update_dpp(0, v, 0x112, 0xf, 0xf, true);
        v += __builtin_amdgcn_update_dpp(0, v, 0x114, 0xf, 0xf, true);
        v += __builtin_amdgcn_update_dpp(0, v, 0x118, 0xf, 0xf, true);
        v += __builtin_amdgcn_update_dpp(0, v, 0x142, 0xa, 0xf, false);
        v += __builtin_amdgcn_update_dpp(0, v, 0x143, 0xc, 0xf, false);
        const unsigned tot = (unsigned)__builtin_amdgcn_readlane(v, 63), exc = (unsigned)(v - c);
        first[u] = base + (exc & 0xffffu); base += tot & 0xffffu; last[u] = base;
        first[u + 1] = base + (exc >> 16); base += tot >> 16; last[u + 1] = base;
    }
}
template <int SLEEP = 1>
__device__ __forceinline__ bool give_up(char* ws, unsigned call, u64 t0, unsigned& spins) {
    bool stop = __builtin_amdgcn_s_memrealtime() - t0 > DEADLINE;
    if ((++spins & 63u) == 0u) stop = stop || __builtin_amdgcn_readfirstlane(ld_agent(ws + TMO_OFF)) == call;
    if (stop) {
        __hip_atomic_store((GAS unsigned*)(ws + TMO_OFF), call, __ATOMIC_RELAXED, __HIP_MEMORY_SCOPE_AGENT);
        return true;
    }
    __builtin_amdgcn_s_sleep(SLEEP);
    return false;
}
template <int SLEEP>
__device__ __forceinline__ bool await_word(char* ws, const char* p, unsigned call) {
    const u64 t0 = __builtin_amdgcn_s_memrealtime(); unsigned sp = 0;
    while ((int)(__builtin_amdgcn_readfirstlane(ld_sys(p)) - call) < 0)
        if (give_up<SLEEP>(ws, call, t0, sp)) return false;
    return true;
}
__device__ __forceinline__ unsigned fl_word(int blk, int src, int t) { return FL_OFF + (unsigned)(((blk * 4 + (src & 3)) * NT + t) * 2 + (src >> 2)) * 4u; }
__device__ __forceinline__ int steps_known(unsigned f0, unsigned f1, unsigned call) {
    const u64 lo = __ballot((int)(f0 - call) >= 0), hi = __ballot((int)(f1 - call) >= 0);
    const u64 miss = ~(lo & hi);
    if (miss == 0) return NSTEP;
    const int c = __builtin_ctzll(miss);
    return 2 * c + (int)((lo >> c) & 1);
}
__device__ __forceinline__ unsigned out_base(int src, int tile, int wave, int lane) {
    return (unsigned)(((tile >> 3) * 256 + (wave >> 1) * 64 + (lane >> 4) * 4) * NN + src * NCOL + (tile & 7) * 256 + (wave & 1) * 128 + (lane & 15) * 8);
}
constexpr unsigned PIECE_BYTES = 896;
__device__ __forceinline__ unsigned piece_off(int lane) { return lane & 1 ? 512 + (lane >> 1) * 12 : (lane >> 1) * 16; }
#define WAIT_VM(n) asm volatile("s_waitcnt vmcnt(" #n ")" ::: "memory")
#define LGKM0_BARRIER() do { __builtin_amdgcn_s_waitcnt(0xC07F); asm volatile("" ::: "memory");                       \
        __builtin_amdgcn_s_barrier(); asm volatile("" ::: "memory"); } while (0)
#define STAMP(i) do {} while (0)

__global__ void __launch_bounds__(BLOCK, 2) fused_gemm_relu_q8(const Params p) {
    __shared__ __attribute__((aligned(1024))) char smem[LDS_BYTES];
    LAS char* const lds = (LAS char*)smem;
    const int hb = blockIdx.x;
    const int lane = threadIdx.x & 63;
    const int wid = __builtin_amdgcn_readfirstlane(threadIdx.x >> 6);
    int rank = p.rank;
    char* ws = p.ws[0];
    const char* px = p.x;
    const char* pw = p.w;
    const u64 t_start = __builtin_amdgcn_s_memrealtime();
    asm volatile("" : "+s"(rank), "+s"(ws), "+s"(px), "+s"(pw));
    const unsigned ticket = add_agent_lane0(ws + CTR_OFF + (size_t)hb * 128 + wid * 4, lane);
    const int b = hb & (N_COMM - 1);

    if (hb < N_COMM) {
        const __amdgpu_buffer_rsrc_t wsr = rsrc_of(ws, CTL_OFF);
        const int cw = wid * N_COMM + b;
        const int dbg = 128 + cw;
        (void)dbg;
        STAMP(0);
        if (wid < 2) {
            if (threadIdx.x == 0) *(LAS unsigned*)(lds + LDS_MISC) = 0u;
            LGKM0_BARRIER();
            if (cw < N_PULL) {
                const unsigned s = (unsigned)(rank + 1 + cw % (NR - 1)) & 7u;
                const __amdgpu_buffer_rsrc_t srs = rsrc_of(p.ws[1 + cw % (NR - 1)], CTL_OFF);
                char* const queue = ws + Q_OFF + s * 128;
                unsigned tk = add_agent_lane0(queue, lane);
                u32x4 cur[4];
                const unsigned call = __builtin_amdgcn_readfirstlane(ticket) + 1u;
                const unsigned qbase = (call - 1u) * Q_TICKETS;
                const char* const pk = ws + PK_OFF + s * NGRP * 8192 + lane * 128;
                unsigned pv = call - 1u;
                int pgrp = 0;
#define JOB_OF(n_, t_, j_, so_, do_) const int t_ = (int)(n_) / NBLK, j_ = JORDER((int)(n_) % NBLK);                           \
                const unsigned do_ = (j_ < 16 ? XT_OFF + t_ * XT_T + j_ * 32768 : WT_OFF + t_ * WT_T + (j_ - 16) * 32768) + SLOT(s, j_) + lane * 16; \
                const unsigned so_ = j_ < 16 ? do_ : WS_OFF + t_ * WT_T + rank * 32768 + (j_ - 16) * 4096 + lane * 16
#define WAIT_PACKED(t_, j_) do { const int sl_ = (j_) < 16 ? (j_) : 24 + ((t_) & 7) * 4 + ((j_) >> 1);                 \
                if (((t_) >> 3) != pgrp) { pgrp = (t_) >> 3; pv = call - 1u; }                                         \
                if ((int)(__builtin_amdgcn_readlane(pv, sl_) - call) < 0) {                                            \
                    pv = ld_sys(pk + pgrp * 8192);                 \
                    const u64 t0_ = __builtin_amdgcn_s_memrealtime(); unsigned sp_ = 0;                                \
                    while ((int)(__builtin_amdgcn_readlane(pv, sl_) - call) < 0) {                                     \
                        if (give_up(ws, call, t0_, sp_)) break;                                                        \
                        if (lane == sl_) pv = ld_sys(pk + pgrp * 8192); } } } while (0)
#define PTAIL() do {} while (0)
                unsigned n = __builtin_amdgcn_readfirstlane(tk) - qbase;
                if (n < (unsigned)Q_JOBS) {
                    JOB_OF(n, t, j, so, dof);
                    WAIT_PACKED(t, j);
                    STAMP(2);
                    tk = add_agent_lane0(queue, lane);
#pragma unroll
                    for (int i = 0; i < 4; ++i) cur[i] = __builtin_amdgcn_raw_buffer_load_b128(srs, so + i * 1024, 0, AUX_SYS);
                    unsigned dcur = dof, fcur = fl_word(j, s, t);
                    for (;;) {
#pragma unroll
                        for (int i = 0; i < 4; ++i) __builtin_amdgcn_raw_buffer_store_b128(cur[i], wsr, dcur + i * 1024, 0, AUX_SYS);
                        asm volatile("" ::: "memory");
                        n = __builtin_amdgcn_readfirstlane(tk) - qbase;
                        char* const flag = ws + fcur;
                        if (n >= (unsigned)Q_JOBS) {
                            WAIT_VM(0);
                            if (lane == 0) st_sys(flag, call);
                            PTAIL();
                            break;
                        }
                        JOB_OF(n, t2, j2, so2, dof2);
                        WAIT_PACKED(t2, j2);
                        tk = add_agent_lane0(queue, lane);
#pragma unroll
                        for (int i = 0; i < 4; ++i) cur[i] = __builtin_amdgcn_raw_buffer_load_b128(srs, so2 + i * 1024, 0, AUX_SYS);
                        dcur = dof2; fcur = fl_word(j2, s, t2);
                        WAIT_VM(5);
                        if (lane == 0) st_sys(flag, call);
                        PTAIL();
                    }
                }
                STAMP(1);
            }
        } else {
            const __amdgpu_buffer_rsrc_t xr = rsrc_of(px, MM * KSH * 2);
            const __amdgpu_buffer_rsrc_t wr_ = rsrc_of(pw, KSH * NN * 2);
            bool met = false;
            bool x_todo = wid >= 4;
            for (int n = (wid - 2) * N_COMM + b; n < NT * 32 || x_todo;) {
                const bool do_x = x_todo && ((b >> 4) <= (n >> 8) || n >= NT * 32);
                const int g = do_x ? b >> 4 : n >> 8;
                const int delay = (do_x ? g * 8 : n >> 5) * PACK_DELAY;
                if (!met && delay > 0) { __builtin_amdgcn_s_barrier(); met = true; }
                while (__builtin_amdgcn_s_memrealtime() - t_start < (u64)delay) __builtin_amdgcn_s_sleep(2);
                u32x4 v[8];
                if (do_x) {
                    const unsigned rb = (wid - 4) * 64 + (lane & 7), piece = lane >> 3;
#pragma unroll
                    for (int i = 0; i < 8; ++i) v[i] = __builtin_amdgcn_raw_buffer_load_b128(xr, ((b & 15) * 256 + rb + i * 8) * (KSH * 2) + g * 128 + piece * 16, 0, 0);
                    if (!met) { __builtin_amdgcn_s_barrier(); met = true; }
                    const unsigned d0 = XT_OFF + (g * 8 + piece) * XT_T + (b & 15) * 32768 + SLOT(rank, b & 15) + rb * 16;
#pragma unroll
                    for (int i = 0; i < 8; ++i) __builtin_amdgcn_raw_buffer_store_b128(v[i], wsr, d0 + i * 128, 0, AUX_SYS);
                    char* fl = lane < 8 ? p.ws[lane & 7] + PK_OFF + (rank * NGRP + g) * 8192 + (b & 15) * 128 : ws + fl_word(b & 15, rank, g * 8 + (lane & 7));
                    asm volatile("s_waitcnt vmcnt(0)" : "+v"(fl) :: "memory");
                    unsigned old = 0;
                    if (lane == 0) old = __hip_atomic_fetch_add((LAS unsigned*)(lds + LDS_MISC), 1u, __ATOMIC_RELAXED, __HIP_MEMORY_SCOPE_WORKGROUP);
                    const unsigned call = __builtin_amdgcn_readfirstlane(ticket) + 1u;
                    if (__builtin_amdgcn_readfirstlane(old) == 3u && lane >= 1 && lane < 16) st_sys(fl, call);
                    x_todo = false;
                } else {
                    const int t = n >> 5, cb = n & 31;
                    const unsigned dest = (unsigned)cb >> 2;
#pragma unroll
                    for (int i = 0; i < 8; ++i) v[i] = __builtin_amdgcn_raw_buffer_load_b128(wr_, (unsigned)((t * 8 + i) * (NN * 2) + cb * 1024 + lane * 16), 0, 0);
                    if (!met) { __builtin_amdgcn_s_barrier(); met = true; }
                    const unsigned d0 = (dest == (unsigned)rank ? WT_OFF + t * WT_T + ((cb & 3) * 2 + (lane >> 5)) * 32768 + SLOT(rank, (cb & 3) * 2 + (lane >> 5))
                                                                : WS_OFF + t * WT_T + dest * 32768 + ((cb & 3) * 2 + (lane >> 5)) * 4096)
                                      + (((lane >> 4) & 1) * 128 + (lane & 15)) * 16;
#pragma unroll
                    for (int i = 0; i < 8; ++i) __builtin_amdgcn_raw_buffer_store_b128(transposed(v, i), wsr, d0 + i * 256, 0, AUX_SYS);
                    char* const dws = p.ws[(dest - rank) & 7u];
                    WAIT_VM(0);
                    const unsigned call = __builtin_amdgcn_readfirstlane(ticket) + 1u;
                    if (dest == (unsigned)rank) { if (lane < 2) st_sys(ws + fl_word(16 + (cb & 3) * 2 + lane, rank, t), call); }
                    else if (lane == 0) st_sys(dws + PK_OFF + (rank * NGRP + g) * 8192 + (32 + (t & 7) * 4 + (cb & 3)) * 128, call);
                    n += N_PACKW;
                }
            }
        }
        const unsigned call = __builtin_amdgcn_readfirstlane(ticket) + 1u;
        if (wid == 7) {
            if (b == N_COMM - 1) {
                char* amax_out = p.ws[lane & 7] + AMAX_OFF + ((lane >> 3) * NR + rank) * 128;
                asm volatile("" : "+v"(amax_out));
                const char* tm = ws + TMAX_OFF + lane * 256;
                const u64 t0 = __builtin_amdgcn_s_memrealtime(); unsigned sp = 0;
                int mx;
                for (;;) {
                    const u64 v0 = ld_sys64(tm), v1 = ld_sys64(tm + 128);
                    mx = max((int)(unsigned)v0, (int)(unsigned)v1);
                    if (__all((int)((unsigned)(v0 >> 32) - call) >= 0 && (int)((unsigned)(v1 >> 32) - call) >= 0)) break;
                    if (give_up(ws, call, t0, sp)) { mx = NAN_BITS; break; }
                }
                if (lane < 8 * AMAX_REP) st_sys64(amax_out, (unsigned)wave_max(mx), call);
                STAMP(6);
            }
            return;
        }
        {
            const int src = (rank + 1 + wid) & 7;
            const unsigned slot = OB_OFF + (unsigned)(src * 128 + b) * OB_SLOT;
            const char* const of = ws + OF_OFF + (src * 128 + b) * 128;
            const unsigned lbase = wid * LDS_WAVE;
#pragma unroll
            for (int e = 0; e < 4; ++e) {
                const unsigned idx = lane * 4 + e;
                u32x2 sel = {0u, 0u};
                unsigned rk = 0;
#pragma unroll
                for (int n = 0; n < 8; ++n) {
                    const bool neg = (idx >> (7 - n)) & 1u;
                    sel[n >> 2] |= (neg ? 0x0cu : rk) << (8 * (n & 3));
                    rk += neg ? 0u : 1u;
                }
                *(LAS u32x2*)(lds + lbase + 12288 + idx * 8) = sel;
            }
            char* outp = p.out;
            asm volatile("" : "+s"(outp));
            const unsigned ob = out_base(src, b >> 3, b & 7, lane);
            {
                bool ok = await_word<8>(ws, of, call);
                const u32x4 mk = __builtin_amdgcn_raw_buffer_load_b128(wsr, slot + lane * 16, 0, AUX_SYS);
                unsigned first[16], last[16];
                stream_layout(mk, first, last);
                u32x2 sel[16];
#pragma unroll
                for (int u = 0; u < 16; ++u) {
                    sel[u] = *(const LAS u32x2*)(lds + lbase + 12288 + sign_byte(mk, u) * 8);
                    first[u] += lbase;
                    asm volatile("" : "+v"(first[u]), "+v"(sel[u]));
                }
                STAMP(3);
                const unsigned cnt = last[15];
                const __amdgpu_buffer_rsrc_t ors = rsrc_of(outp, (MM / NR) * NN);
#define TAKE_PIECES(i0_, i1_) do { u32x4 pc[(i1_) - (i0_)];                                                            \
                    _Pragma("unroll") for (int i = (i0_); i < (i1_); ++i)                                              \
                        if (i * 1024u < cnt) pc[i - (i0_)] = __builtin_amdgcn_raw_buffer_load_b128(wsr, slot + 1024 + i * PIECE_BYTES + piece_off(lane), 0, AUX_SYS); \
                    fill = ok && !__any(ld_sys(lane & 1 ? ws + TMO_OFF : of + 12) == call) ? 0u : 0x7f7f7f7fu;          \
                    _Pragma("unroll") for (int i = (i0_); i < (i1_); ++i)                                              \
                        if (i * 1024u < cnt) {                                                                         \
                            u32x4 q = pc[i - (i0_)];                                                                   \
                            unsigned h = ((q[0] & 0x80808080u) >> 7) | ((q[1] & 0x80808080u) >> 6) | ((q[2] & 0x80808080u) >> 5); \
                            const unsigned he = h | ((q[3] & 0x80808080u) >> 4);     \
                            h = (h << 4) | (unsigned)__builtin_amdgcn_update_dpp(0, (int)he, 0xa0, 0xf, 0xf, true);    \
                            q[0] &= 0x7f7f7f7fu; q[1] &= 0x7f7f7f7fu; q[2] &= 0x7f7f7f7fu;                             \
                            q[3] = lane & 1 ? h : q[3] & 0x7f7f7f7fu;                                                  \
                            *(LAS u32x4*)(lds + lbase + i * 1024 + lane * 16) = q;                                     \
                        } } while (0)
                unsigned fill;
                ok = await_word<1>(ws, of + 8, call) && ok;
                STAMP(4);
                TAKE_PIECES(0, 8);
                unsigned d[16][3];
#pragma unroll
                for (int u = 0; u < 16; ++u)
#pragma unroll
                    for (int i = 0; i < 3; ++i) d[u][i] = *(const LAS unsigned*)(lds + (first[u] & ~3u) + i * 4);
                __builtin_amdgcn_sched_barrier(0);
#pragma unroll
                for (int u = 0; u < 16; ++u) {
                    const unsigned lo = __builtin_amdgcn_alignbyte(d[u][1], d[u][0], first[u]), hi = __builtin_amdgcn_alignbyte(d[u][2], d[u][1], first[u]);
                    u32x2 o;
                    o[0] = __builtin_amdgcn_perm(hi, lo, sel[u][0]) | fill;
                    o[1] = __builtin_amdgcn_perm(hi, lo, sel[u][1]) | fill;
                    __builtin_amdgcn_raw_buffer_store_b64(o, ors, ob + (unsigned)(((u >> 2) * 16 + (u & 3)) * NN), 0, AUX_OUT);
                }
            }
        }
        return;
    }

    const int g = b;
    const int dbg = g;
    (void)dbg;
    const int pm = (g >> 4) * 2 + (g & 1);
    const int pn = ((g & 7) >> 1) * 2 + ((g >> 3) & 1);
    const int fr = lane & 15, fq = lane >> 4, wr = wid >> 1, wc = wid & 1;
    if (wid == 0) STAMP(0);
    const int owner = pm >> 1, tile = (pm & 1) * 8 + pn;
    char* ows = p.ws[(owner - rank) & 7];
    char* outp = p.out;
    asm volatile("" : "+s"(ows), "+s"(outp));
    const unsigned call = __builtin_amdgcn_readfirstlane(ticket) + 1u;

    const int op = wid >> 2;
    const char* const pollp = ws + fl_word(op ? 16 + pn : pm, wid & 3, lane);
    const unsigned unit = op ? WT_T : XT_T;
    const char* const gbase = ws + (op ? WT_OFF + pn * 32768 : XT_OFF + pm * 32768) + lane * 16;
    const unsigned slot0 = SLOT(wid & 3, op ? pn : pm);
    LAS char* const ldst = lds + wid * 4096;
    const int aoff = fq * 4096 + (wr * 64 + fr) * 16;
    const int boff = 16384 + fq * 4096 + (wc * 128 + fr) * 16;

    int known = 0;
    int next = 0, nslot = 0;
    u32x2 pvn = {0u, 0u};
#define STAGE_NEXT() do { const char* g_ = gbase + ((slot0 ^ ((next & 1) << 14)) + (unsigned)(next >> 1) * unit);              \
        LAS char* l_ = ldst + nslot * LDS_SLOT;                                                                       \
        __builtin_amdgcn_global_load_lds((const GAS unsigned*)g_, (LAS unsigned*)l_, 16, 0, STAGE_AUX);                       \
        __builtin_amdgcn_global_load_lds((const GAS unsigned*)g_, (LAS unsigned*)l_, 16, 1024, STAGE_AUX);                    \
        __builtin_amdgcn_global_load_lds((const GAS unsigned*)g_, (LAS unsigned*)l_, 16, 2048, STAGE_AUX);                    \
        __builtin_amdgcn_global_load_lds((const GAS unsigned*)g_, (LAS unsigned*)l_, 16, 3072, STAGE_AUX);                    \
        ++next; nslot = nslot == NSLOT - 1 ? 0 : nslot + 1; } while (0)
#define POLL_ISSUE() asm volatile("global_load_dwordx2 %0, %1, off sc0 sc1" : "+v"(pvn) : "v"(pollp) : "memory")
#define POLL_TAKE() do { u32x2 f_; asm volatile("s_waitcnt vmcnt(0)\n\tv_mov_b64 %0, %1" : "=v"(f_) : "v"(pvn) : "memory");    \
        known = steps_known(f_[0], f_[1], call); } while (0)
#define AWAIT_NEXT() do { if (next >= known) {                                                                        \
            const u64 t0_ = __builtin_amdgcn_s_memrealtime(); unsigned sp_ = 0;                                       \
            for (;;) { const u64 f_ = ld_sys64(pollp); known = steps_known((unsigned)f_, (unsigned)(f_ >> 32), call); \
                if (next < known) break;                                                                              \
                if (give_up<SLEEP_K>(ws, call, t0_, sp_)) { known = NSTEP; break; } } } } while (0)
#define MFMA_COLS(A_, B_, n0_, n1_) do { __builtin_amdgcn_sched_barrier(0);                                           \
        _Pragma("unroll") for (int n = (n0_); n < (n1_); ++n)                                                         \
            _Pragma("unroll") for (int m = 0; m < 4; ++m)                                                             \
                acc[m][n] = __builtin_amdgcn_mfma_f32_16x16x32_bf16(A_[m], B_[n], acc[m][n], 0, 0, 0);                \
        __builtin_amdgcn_sched_barrier(0); } while (0)
#define RD_A(A_, m_) A_[m_] = *(const LAS bf16x8*)(lb_ + aoff + (m_) * 256)
#define RD_B(B_, n_) B_[n_] = *(const LAS bf16x8*)(lb_ + boff + (n_) * 256)
#define TAIL_REC(st_) do {} while (0)
#define KSTEP(st_, A_, B_, AN_, BN_) do {                                                                             \
        MFMA_COLS(A_, B_, 0, TAKE_COL);                                                                               \
        if (polled) POLL_TAKE();                                          \
        const int had_ = next;                                                                                        \
        const int lim_ = min((st_) + NSLOT, known);                                   \
        if (next < lim_) { STAGE_NEXT(); if (next < lim_) STAGE_NEXT(); }                                             \
        const bool full_ = (polled ? had_ + NSLOT - 2 : next) >= (st_) + NSLOT;                                       \
        MFMA_COLS(A_, B_, TAKE_COL, 4);                                                                                      \
        if (!full_) {                                                              \
            if (next == (st_) + 1 && next < NSTEP) {                                                                  \
                AWAIT_NEXT(); STAGE_NEXT();                                                                           \
                if (next < known) STAGE_NEXT();                                                                       \
            }                                                                                                         \
            const int k_ = next - (st_) - 2;                                                                          \
            if (k_ <= 0) WAIT_VM(0); else if (k_ == 1) WAIT_VM(4); else WAIT_VM(8);                                   \
        }                                                                                                             \
        WAIT_VM(12);                                                                                                  \
        LGKM0_BARRIER();                                                                                              \
        TAIL_REC(st_);                                                                                                \
        const LAS char* lb_ = lds + rslot * LDS_SLOT;               \
        RD_A(AN_, 0); RD_A(AN_, 1); RD_A(AN_, 2); RD_A(AN_, 3);                                                       \
        if (POLL_POS == 0) { polled = known < min(next + POLL_AHEAD, NSTEP); if (polled) POLL_ISSUE(); }              \
        MFMA_COLS(A_, B_, 4, 5);                                                                                      \
        RD_B(BN_, 0); RD_B(BN_, 1); RD_B(BN_, 2); RD_B(BN_, 3);                                                       \
        MFMA_COLS(A_, B_, 5, 6);                                                                                      \
        RD_B(BN_, 4); RD_B(BN_, 5); RD_B(BN_, 6); RD_B(BN_, 7);                                                       \
        rslot = rslot == NSLOT - 1 ? 0 : rslot + 1;                                                                   \
        if (POLL_POS == 2) { polled = known < min(next + POLL_AHEAD, NSTEP); if (polled) POLL_ISSUE(); }              \
        MFMA_COLS(A_, B_, 6, 8); } while (0)

    f32x4 acc[4][8];
#pragma unroll
    for (int m = 0; m < 4; ++m)
#pragma unroll
        for (int n = 0; n < 8; ++n) acc[m][n] = (f32x4){0.f, 0.f, 0.f, 0.f};

    AWAIT_NEXT(); STAGE_NEXT();
    WAIT_VM(0);
    LGKM0_BARRIER();
    int rslot = 1;
    bool polled = false;
    bf16x8 a0[4], b0[8], a1[4], b1[8];
    {
        const LAS char* lb_ = lds;
#pragma unroll
        for (int m = 0; m < 4; ++m) RD_A(a0, m);
#pragma unroll
        for (int n = 0; n < 8; ++n) RD_B(b0, n);
    }
    if (wid == 0) STAMP(1);
    for (int st = 0; st < NSTEP; st += 2) {
        KSTEP(st, a0, b0, a1, b1);
        KSTEP(st + 1, a1, b1, a0, b0);
    }
    if (wid == 0) STAMP(2);

    int mx = 0;
#pragma unroll
    for (int m = 0; m < 4; ++m)
#pragma unroll
        for (int n = 0; n < 8; ++n)
#pragma unroll
            for (int j = 0; j < 4; ++j) mx = max(mx, __float_as_int(acc[m][n][j]));
    mx = wave_max(mx);
    LAS int* red = (LAS int*)(lds + LDS_MISC);
    if (lane == 0) red[wid] = mx;
    LGKM0_BARRIER();
    if (wid == 0) {
        int t = red[lane & 7];
        ROW_SHR_MAX(t, 1); ROW_SHR_MAX(t, 2); ROW_SHR_MAX(t, 4);
        if (lane == 7) st_sys64(ws + TMAX_OFF + g * 128, (unsigned)t, call);
        STAMP(7);
    }

    const bool remote = owner != rank;
    const __amdgpu_buffer_rsrc_t sls = rsrc_of(ows + OB_OFF + (unsigned)(rank * 128 + tile * 8 + wid) * OB_SLOT, OB_SLOT);
    char* const of = ows + OF_OFF + (rank * 128 + tile * 8 + wid) * 128;
    const int efield = min(268 - (mx >> 23), 253);
    const float sc = __int_as_float(efield << 23);
    const unsigned pos0 = (unsigned)(size_t)(lds + wid * LDS_WAVE);
    const unsigned spill = (unsigned)(size_t)(lds + LDS_MISC + 64);
    unsigned first[16], last[16];
#define VAL(u_, n_) acc[(u_) >> 2][n_][(u_) & 3]
#define PACK1(n_) "v_ashrrev_i32 %[s], 31, %[v" #n_ "]\n\tv_fma_mixlo_f16 %[h], %[v" #n_ "], %[sc], 0\n\t"              \
                  "v_bfi_b32 %[a], %[s], %[sp], %[p]\n\tv_lshl_add_u32 %[p], %[s], 1, %[p]\n\t"                      \
                  "ds_write_b16 %[a], %[h] offset:2*" #n_ "\n\t"
#define PACK8(u_) do { unsigned p_ = pos0 + 2 * first[u_], s_, h_, a_;                                                \
        asm volatile(PACK1(0) PACK1(1) PACK1(2) PACK1(3) PACK1(4) PACK1(5) PACK1(6) PACK1(7)                          \
                     : [p] "+v"(p_), [s] "=&v"(s_), [h] "=&v"(h_), [a] "=&v"(a_)                                      \
                     : [v0] "v"(VAL(u_, 0)), [v1] "v"(VAL(u_, 1)), [v2] "v"(VAL(u_, 2)), [v3] "v"(VAL(u_, 3)),        \
                       [v4] "v"(VAL(u_, 4)), [v5] "v"(VAL(u_, 5)), [v6] "v"(VAL(u_, 6)), [v7] "v"(VAL(u_, 7)),        \
                       [sc] "s"(sc), [sp] "v"(spill) : "memory"); } while (0)
    u32x4 mk = {0u, 0u, 0u, 0u};
    if (remote) {
#pragma unroll
        for (int r = 0; r < 128; ++r) mk[r >> 5] = __builtin_amdgcn_alignbit(mk[r >> 5], __float_as_uint(VAL(r >> 3, r & 7)), 31);
        if (!LATE_FLAG || wid != 0) __builtin_amdgcn_raw_buffer_store_b128(mk, sls, lane * 16, 0, AUX_SYS);
        stream_layout(mk, first, last);
        PACK8(0); PACK8(1); PACK8(2); PACK8(3); PACK8(4);
        if (!LATE_FLAG) { WAIT_VM(0); if (lane == 0) st_sys(of, call); }
    } else {
#pragma unroll
        for (int m = 0; m < 4; ++m) {
#pragma unroll
            for (int n = 0; n < 8; ++n)
#pragma unroll
                for (int j = 0; j < 4; ++j) acc[m][n][j] = __int_as_float(max(0, __float_as_int(acc[m][n][j])));
            asm volatile("" : "+v"(acc[m][0]), "+v"(acc[m][1]), "+v"(acc[m][2]), "+v"(acc[m][3]),
                              "+v"(acc[m][4]), "+v"(acc[m][5]), "+v"(acc[m][6]), "+v"(acc[m][7]));
        }
    }
    if (wid == 1) STAMP(6);
    if (wid == 0) {
        const char* ap = lane < NR ? ws + AMAX_OFF + ((g & (AMAX_REP - 1)) * NR + lane) * 128 : ws + TMO_OFF - 4;
        const u64 t0 = __builtin_amdgcn_s_memrealtime(); unsigned sp = 0;
        u64 v = 0;
        for (;;) {
            if (lane <= NR) v = ld_sys64(ap);
            if ((__ballot((int)((unsigned)(v >> 32) - call) >= 0) & 0xffu) == 0xffu) break;
            if (give_up(ws, call, t0, sp)) { v = (u64)call << 32; break; }
        }
        STAMP(8);
        int t = lane < NR ? (int)(unsigned)v : ((unsigned)(v >> 32) == call ? NAN_BITS : 0);
        ROW_SHR_MAX(t, 1); ROW_SHR_MAX(t, 2); ROW_SHR_MAX(t, 4); ROW_SHR_MAX(t, 8);
        if (lane == NR) red[8] = t;
    }
    LGKM0_BARRIER();
    const float inv = 448.0f / __int_as_float(red[8]);
    if (wid == 0) STAMP(3);

    if (remote) {
        const float f = inv * __int_as_float((254 - efield) << 23);
        unsigned sent = 0;
        const unsigned hshift = (lane & 1) * 4, woff = piece_off(lane);
#define SEND_GATE() WAIT_VM(2)
#define SEND(cnt_, upto_) do { __builtin_amdgcn_s_waitcnt(0xC07F); asm volatile("" ::: "memory");                           \
            _Pragma("nounroll") for (; sent < (upto_); sent += 1024) {                                                                    \
                const unsigned o_ = sent + lane * 16;                                                                 \
                const f16x8 h0 = *(const LAS f16x8*)(lds + wid * LDS_WAVE + o_ * 2), h1 = *(const LAS f16x8*)(lds + wid * LDS_WAVE + o_ * 2 + 16); \
                u32x4 q;                                                                                              \
                _Pragma("unroll") for (int d = 0; d < 2; ++d) {                                                       \
                    int w0 = __builtin_amdgcn_cvt_pk_fp8_f32(__builtin_fmaf((float)h0[4 * d], f, 0.f), __builtin_fmaf((float)h0[4 * d + 1], f, 0.f), 0, false); \
                    q[d] = (unsigned)__builtin_amdgcn_cvt_pk_fp8_f32(__builtin_fmaf((float)h0[4 * d + 2], f, 0.f), __builtin_fmaf((float)h0[4 * d + 3], f, 0.f), w0, true); \
                    int w1 = __builtin_amdgcn_cvt_pk_fp8_f32(__builtin_fmaf((float)h1[4 * d], f, 0.f), __builtin_fmaf((float)h1[4 * d + 1], f, 0.f), 0, false); \
                    q[2 + d] = (unsigned)__builtin_amdgcn_cvt_pk_fp8_f32(__builtin_fmaf((float)h1[4 * d + 2], f, 0.f), __builtin_fmaf((float)h1[4 * d + 3], f, 0.f), w1, true); \
                }                                                                                                     \
                  \
                const unsigned h_ = (unsigned)__builtin_amdgcn_update_dpp(0, (int)q[3], 0xf5, 0xf, 0xf, true) >> hshift; \
                _Pragma("unroll") for (int d = 0; d < 4; ++d) q[d] |= (h_ << (7 - d)) & 0x80808080u;                  \
                SEND_GATE();                                                                                          \
                if ((o_ & ~31u) < (cnt_)) {                                                                             \
                    const unsigned w_ = 1024 + (sent >> 10) * PIECE_BYTES + woff;                                     \
                    if (lane & 1) __builtin_amdgcn_raw_buffer_store_b96((u32x3){q[0], q[1], q[2]}, sls, w_, 0, AUX_SYS); \
                    else __builtin_amdgcn_raw_buffer_store_b128(q, sls, w_, 0, AUX_SYS);                              \
                }                                                                                                     \
            } } while (0)
        if (LATE_FLAG && wid == 0) __builtin_amdgcn_raw_buffer_store_b128(mk, sls, lane * 16, 0, AUX_SYS);
        SEND(last[4], last[4] & ~1023u);
        PACK8(5); PACK8(6); PACK8(7); PACK8(8);
        if (LATE_FLAG) { if (sent == 1024u) WAIT_VM(2); else WAIT_VM(0); if (lane == 0) st_sys(of, call); }
        SEND(last[8], last[8] & ~1023u);
        PACK8(9); PACK8(10); PACK8(11); PACK8(12);
        SEND(last[12], last[12] & ~1023u);
        PACK8(13); PACK8(14); PACK8(15);
        SEND(last[15], last[15]);
        WAIT_VM(0);
        if (lane == 0) st_sys64(of + 8, call, red[8] == NAN_BITS ? call : 0u);
    } else {
        const __amdgpu_buffer_rsrc_t ors = rsrc_of(outp, (MM / NR) * NN);
        const unsigned ob = out_base(rank, tile, wid, lane);
#pragma unroll
        for (int m = 0; m < 4; ++m)
#pragma unroll
            for (int j = 0; j < 4; ++j) {
                u32x2 o;
                int lo = __builtin_amdgcn_cvt_pk_fp8_f32(acc[m][0][j] * inv, acc[m][1][j] * inv, 0, false);
                o[0] = (unsigned)__builtin_amdgcn_cvt_pk_fp8_f32(acc[m][2][j] * inv, acc[m][3][j] * inv, lo, true);
                int hi = __builtin_amdgcn_cvt_pk_fp8_f32(acc[m][4][j] * inv, acc[m][5][j] * inv, 0, false);
                o[1] = (unsigned)__builtin_amdgcn_cvt_pk_fp8_f32(acc[m][6][j] * inv, acc[m][7][j] * inv, hi, true);
                __builtin_amdgcn_raw_buffer_store_b64(o, ors, ob + (unsigned)((m * 16 + j) * NN), 0, AUX_OUT);
            }
    }
    if (wid == 0) STAMP(4);
    if (wid == 7) STAMP(5);
}

}

extern "C" void kernel_launch(int rank, int nranks, ncclComm_t comm,
                              hipStream_t stream,
                              void* const* d_in, const int* in_sizes, int n_in,
                              void* d_out, int out_size,
                              const int* in_dtypes, int out_dtype,
                              void* const* d_out_all,
                              void* const* d_ws_all, size_t ws_bytes) {
    (void)comm; (void)in_dtypes; (void)out_dtype; (void)d_out_all;
    if (nranks != NR || n_in < 2 || (unsigned)rank >= (unsigned)NR || ws_bytes < WS_NEED ||
        in_sizes[0] != MM * KSH || in_sizes[1] != KSH * NN || out_size != (MM / NR) * NN) {
        fprintf(stderr, "kernel_launch: unexpected problem shape\n");
        return;
    }
    Params p;
    p.x = (const char*)d_in[0];
    p.w = (const char*)d_in[1];
    for (int i = 0; i < NR; ++i) p.ws[i] = (char*)d_ws_all[(rank + i) % NR];
    p.out = (char*)d_out;
    p.rank = rank; p.pad = 0;
    hipLaunchKernelGGL(fused_gemm_relu_q8, dim3(GRID), dim3(BLOCK), 0, stream, p);
    HIP_CHECK(hipGetLastError());
}
```

```cpp
#include <hip/hip_runtime.h>
#include <rccl/rccl.h>
#include <cstdio>
#include <cstdint>

#define HIP_CHECK(expr) do { hipError_t _e = (expr); if (_e != hipSuccess) { \
    fprintf(stderr, "HIP error %s at %s:%d\n", hipGetErrorString(_e), __FILE__, __LINE__); \
    return; } } while (0)

namespace {

typedef short bf16x8 __attribute__((ext_vector_type(8)));
typedef _Float16 f16x8 __attribute__((ext_vector_type(8)));
typedef float f32x4 __attribute__((ext_vector_type(4)));
typedef unsigned u32x4 __attribute__((ext_vector_type(4)));
typedef unsigned u32x3 __attribute__((ext_vector_type(3)));
typedef unsigned u32x2 __attribute__((ext_vector_type(2)));
typedef unsigned long long u64;
#define LAS __attribute__((address_space(3)))
#define GAS __attribute__((address_space(1)))

constexpr int NR = 8;
constexpr int MM = 4096, KSH = 512, NN = 16384, NCOL = NN / NR;
constexpr int NT = KSH / 8;
constexpr int NBLK = 24;
#define PULL_PER_LINK 18
#define AMAX_REP 8
#define AUX_OUT 17
#define SEND_DEPTH 2
#define JMODE 2
#define JORDER(k) ((k) % 3 == 2 ? 16 + (k) / 3 : (k) / 3 * 2 + (k) % 3)
#define SLEEP_K 2
#define ROT 1
#define SLOT(src, blk) ((((src) + ROT * (blk)) & 7) * 4096)
#define LATE_FLAG 1
#define STAGE_AUX 17
#define POLL_POS 0
#define TAKE_COL 1
#define POLL_AHEAD 6
#define PACK_DELAY 100
constexpr int Q_JOBS = NT * NBLK;
constexpr int Q_TICKETS = Q_JOBS + PULL_PER_LINK;
constexpr int N_COMM = 128;
constexpr int N_PULL = (NR - 1) * PULL_PER_LINK;
constexpr int N_GEMM = 128;
constexpr int GRID = N_COMM + N_GEMM;
constexpr int BLOCK = 512;
constexpr int N_PACKW = N_COMM * 6;
constexpr int NGRP = NT / 8;
static_assert(N_PULL <= 2 * N_COMM, "pull waves are waves 0-1 of the comm workgroups");

constexpr unsigned XT_T = NR * MM * 16, WT_T = NR * NCOL * 16;
constexpr unsigned XT_OFF = 0;
constexpr unsigned WT_OFF = XT_OFF + NT * XT_T;
constexpr unsigned WS_OFF = WT_OFF + NT * WT_T;
constexpr unsigned OB_SLOT = 1024 + 8192;
constexpr unsigned OB_OFF = WS_OFF + NT * WT_T;
constexpr unsigned CTL_OFF = OB_OFF + NR * 128 * OB_SLOT;
constexpr unsigned PK_OFF = CTL_OFF;
constexpr unsigned FL_OFF = PK_OFF + NR * NGRP * 8192;
constexpr unsigned Q_OFF = FL_OFF + NBLK * NR * NT * 4;
constexpr unsigned TMAX_OFF = Q_OFF + NR * 128;
constexpr unsigned AMAX_OFF = TMAX_OFF + N_GEMM * 128;
constexpr unsigned OF_OFF = AMAX_OFF + 64 * 128;
constexpr unsigned CTR_OFF = OF_OFF + NR * 128 * 128;
constexpr unsigned TMO_OFF = CTR_OFF + GRID * 128 + 4;
constexpr unsigned DBG_OFF = CTR_OFF + GRID * 128 + 128;
constexpr size_t WS_NEED = DBG_OFF + 524288;

constexpr u64 DEADLINE = 50000000ull;
constexpr int AUX_SYS = 17;

constexpr int NSTEP = 2 * NT;
constexpr int NSLOT = 5;
constexpr int LDS_SLOT = 32768;
constexpr int LDS_BYTES = NSLOT * LDS_SLOT;
constexpr int LDS_WAVE = 16384;
constexpr int LDS_MISC = 8 * LDS_WAVE;

struct Params {
    const char* x; const char* w;
    int rank, pad;
    char* ws[NR];
    char* out;
};

__device__ __forceinline__ __amdgpu_buffer_rsrc_t rsrc_of(const void* p, unsigned bytes) {
    return __builtin_amdgcn_make_buffer_rsrc(const_cast<void*>(p), 0, bytes, 0x00020000);
}
__device__ __forceinline__ unsigned ld_sys(const void* p) {
    return __hip_atomic_load((const GAS unsigned*)p, __ATOMIC_RELAXED, __HIP_MEMORY_SCOPE_SYSTEM);
}
__device__ __forceinline__ u64 ld_sys64(const void* p) {
    return __hip_atomic_load((const GAS u64*)p, __ATOMIC_RELAXED, __HIP_MEMORY_SCOPE_SYSTEM);
}
__device__ __forceinline__ void st_sys(void* p, unsigned v) {
    __hip_atomic_store((GAS unsigned*)p, v, __ATOMIC_RELAXED, __HIP_MEMORY_SCOPE_SYSTEM);
}
__device__ __forceinline__ void st_sys64(void* p, unsigned lo, unsigned hi) {
    __hip_atomic_store((GAS u64*)p, ((u64)hi << 32) | lo, __ATOMIC_RELAXED, __HIP_MEMORY_SCOPE_SYSTEM);
}
__device__ __forceinline__ unsigned ld_agent(const void* p) {
    return __hip_atomic_load((const GAS unsigned*)p, __ATOMIC_RELAXED, __HIP_MEMORY_SCOPE_AGENT);
}
__device__ __forceinline__ unsigned add_agent_lane0(void* p, int lane) {
    unsigned v = 0, z = 0;
    asm volatile("" : "+v"(z));
    if (lane == 0) v = __hip_atomic_fetch_add((GAS unsigned*)((char*)p + z), 1u, __ATOMIC_RELAXED, __HIP_MEMORY_SCOPE_AGENT);
    return v;
}
__device__ __forceinline__ u32x4 transposed(const u32x4 (&v)[8], int i) {
    u32x4 o;
#pragma unroll
    for (int d = 0; d < 4; ++d)
        o[d] = __builtin_amdgcn_perm(v[2 * d + 1][i >> 1], v[2 * d][i >> 1], (i & 1) ? 0x07060302u : 0x05040100u);
    return o;
}
constexpr int NAN_BITS = 0x7fc00000;
#define ROW_SHR_MAX(v, n) v = max(v, __builtin_amdgcn_update_dpp(0, v, 0x110 + (n), 0xf, 0xf, true))
__device__ __forceinline__ int wave_max(int v) {
    ROW_SHR_MAX(v, 1); ROW_SHR_MAX(v, 2); ROW_SHR_MAX(v, 4); ROW_SHR_MAX(v, 8);
    return max(max(__builtin_amdgcn_readlane(v, 15), __builtin_amdgcn_readlane(v, 31)),
               max(__builtin_amdgcn_readlane(v, 47), __builtin_amdgcn_readlane(v, 63)));
}
__device__ __forceinline__ unsigned sign_byte(const u32x4& mk, int u) { return (mk[u >> 2] >> (24 - 8 * (u & 3))) & 0xffu; }
__device__ __forceinline__ void stream_layout(const u32x4& mk, unsigned (&first)[16], unsigned (&last)[16]) {
    unsigned base = 0;
#pragma unroll
    for (int u = 0; u < 16; u += 2) {
        const int c = 0x00080008 - (int)(__builtin_popcount(sign_byte(mk, u)) | (__builtin_popcount(sign_byte(mk, u + 1)) << 16));
        int v = c;
        v += __builtin_amdgcn_update_dpp(0, v, 0x111, 0xf, 0xf, true);
        v += __builtin_amdgcn_update_dpp(0, v, 0x112, 0xf, 0xf, true);
        v += __builtin_amdgcn_update_dpp(0, v, 0x114, 0xf, 0xf, true);
        v += __builtin_amdgcn_update_dpp(0, v, 0x118, 0xf, 0xf, true);
        v += __builtin_amdgcn_update_dpp(0, v, 0x142, 0xa, 0xf, false);
        v += __builtin_amdgcn_update_dpp(0, v, 0x143, 0xc, 0xf, false);
        const unsigned tot = (unsigned)__builtin_amdgcn_readlane(v, 63), exc = (unsigned)(v - c);
        first[u] = base + (exc & 0xffffu); base += tot & 0xffffu; last[u] = base;
        first[u + 1] = base + (exc >> 16); base += tot >> 16; last[u + 1] = base;
    }
}
template <int SLEEP = 1>
__device__ __forceinline__ bool give_up(char* ws, unsigned call, u64 t0, unsigned& spins) {
    bool stop = __builtin_amdgcn_s_memrealtime() - t0 > DEADLINE;
    if ((++spins & 63u) == 0u) stop = stop || __builtin_amdgcn_readfirstlane(ld_agent(ws + TMO_OFF)) == call;
    if (stop) {
        __hip_atomic_store((GAS unsigned*)(ws + TMO_OFF), call, __ATOMIC_RELAXED, __HIP_MEMORY_SCOPE_AGENT);
        return true;
    }
    __builtin_amdgcn_s_sleep(SLEEP);
    return false;
}
template <int SLEEP>
__device__ __forceinline__ bool await_word(char* ws, const char* p, unsigned call) {
    const u64 t0 = __builtin_amdgcn_s_memrealtime(); unsigned sp = 0;
    while ((int)(__builtin_amdgcn_readfirstlane(ld_sys(p)) - call) < 0)
        if (give_up<SLEEP>(ws, call, t0, sp)) return false;
    return true;
}
__device__ __forceinline__ unsigned fl_word(int blk, int src, int t) { return FL_OFF + (unsigned)(((blk * 4 + (src & 3)) * NT + t) * 2 + (src >> 2)) * 4u; }
__device__ __forceinline__ int steps_known(unsigned f0, unsigned f1, unsigned call) {
    const u64 lo = __ballot((int)(f0 - call) >= 0), hi = __ballot((int)(f1 - call) >= 0);
    const u64 miss = ~(lo & hi);
    if (miss == 0) return NSTEP;
    const int c = __builtin_ctzll(miss);
    return 2 * c + (int)((lo >> c) & 1);
}
__device__ __forceinline__ unsigned out_base(int src, int tile, int wave, int lane) {
    return (unsigned)(((tile >> 3) * 256 + (wave >> 1) * 64 + (lane >> 4) * 4) * NN + src * NCOL + (tile & 7) * 256 + (wave & 1) * 128 + (lane & 15) * 8);
}
constexpr unsigned PIECE_BYTES = 896;
__device__ __forceinline__ unsigned piece_off(int lane) { return lane & 1 ? 512 + (lane >> 1) * 12 : (lane >> 1) * 16; }
#define WAIT_VM(n) asm volatile("s_waitcnt vmcnt(" #n ")" ::: "memory")
#define LGKM0_BARRIER() do { __builtin_amdgcn_s_waitcnt(0xC07F); asm volatile("" ::: "memory");                       \
        __builtin_amdgcn_s_barrier(); asm volatile("" ::: "memory"); } while (0)
#define STAMP(i) do {} while (0)

__global__ void __launch_bounds__(BLOCK, 2) fused_gemm_relu_q8(const Params p) {
    __shared__ __attribute__((aligned(1024))) char smem[LDS_BYTES];
    LAS char* const lds = (LAS char*)smem;
    const int hb = blockIdx.x;
    const int lane = threadIdx.x & 63;
    const int wid = __builtin_amdgcn_readfirstlane(threadIdx.x >> 6);
    int rank = p.rank;
    char* ws = p.ws[0];
    const char* px = p.x;
    const char* pw = p.w;
    const u64 t_start = __builtin_amdgcn_s_memrealtime();
    asm volatile("" : "+s"(rank), "+s"(ws), "+s"(px), "+s"(pw));
    const unsigned ticket = add_agent_lane0(ws + CTR_OFF + (size_t)hb * 128 + wid * 4, lane);
    const int b = hb & (N_COMM - 1);

    if (hb < N_COMM) {
        const __amdgpu_buffer_rsrc_t wsr = rsrc_of(ws, CTL_OFF);
        const int cw = wid * N_COMM + b;
        const int dbg = 128 + cw;
        (void)dbg;
        STAMP(0);
        u64 all_t = 0;
        if (wid < 2) {
            if (threadIdx.x == 0) *(LAS unsigned*)(lds + LDS_MISC) = 0u;
            LGKM0_BARRIER();
            if (cw == 255) {
                const unsigned call = __builtin_amdgcn_readfirstlane(ticket) + 1u;
                if (lane < NR) st_sys(p.ws[lane & 7] + DBG_OFF + rank * 128, call);
                const u64 t0 = __builtin_amdgcn_s_memrealtime(); unsigned sp = 0;
                for (;;) { const unsigned v = ld_sys(ws + DBG_OFF + (lane & 7) * 128); if (__all((int)(v - call) >= 0)) break; if (give_up(ws, call, t0, sp)) break; }
                all_t = __builtin_amdgcn_s_memrealtime();
            }
            if (cw < N_PULL) {
                const unsigned s = (unsigned)(rank + 1 + cw % (NR - 1)) & 7u;
                const __amdgpu_buffer_rsrc_t srs = rsrc_of(p.ws[1 + cw % (NR - 1)], CTL_OFF);
                char* const queue = ws + Q_OFF + s * 128;
                unsigned tk = add_agent_lane0(queue, lane);
                u32x4 cur[4];
                const unsigned call = __builtin_amdgcn_readfirstlane(ticket) + 1u;
                const unsigned qbase = (call - 1u) * Q_TICKETS;
                const char* const pk = ws + PK_OFF + s * NGRP * 8192 + lane * 128;
                unsigned pv = call - 1u;
                int pgrp = 0;
#define JOB_OF(n_, t_, j_, so_, do_) const int t_ = (int)(n_) / NBLK, j_ = JORDER((int)(n_) % NBLK);                           \
                const unsigned do_ = (j_ < 16 ? XT_OFF + t_ * XT_T + j_ * 32768 : WT_OFF + t_ * WT_T + (j_ - 16) * 32768) + SLOT(s, j_) + lane * 16; \
                const unsigned so_ = j_ < 16 ? do_ : WS_OFF + t_ * WT_T + rank * 32768 + (j_ - 16) * 4096 + lane * 16
#define WAIT_PACKED(t_, j_) do { const int sl_ = (j_) < 16 ? (j_) : 24 + ((t_) & 7) * 4 + ((j_) >> 1);                 \
                if (((t_) >> 3) != pgrp) { pgrp = (t_) >> 3; pv = call - 1u; }                                         \
                if ((int)(__builtin_amdgcn_readlane(pv, sl_) - call) < 0) {                                            \
                    pv = ld_sys(pk + pgrp * 8192);                 \
                    const u64 t0_ = __builtin_amdgcn_s_memrealtime(); unsigned sp_ = 0;                                \
                    while ((int)(__builtin_amdgcn_readlane(pv, sl_) - call) < 0) {                                     \
                        if (give_up(ws, call, t0_, sp_)) break;                                                        \
                        if (lane == sl_) pv = ld_sys(pk + pgrp * 8192); } } } while (0)
#define PTAIL() do {} while (0)
                unsigned n = __builtin_amdgcn_readfirstlane(tk) - qbase;
                if (n < (unsigned)Q_JOBS) {
                    JOB_OF(n, t, j, so, dof);
                    WAIT_PACKED(t, j);
                    STAMP(2);
                    tk = add_agent_lane0(queue, lane);
#pragma unroll
                    for (int i = 0; i < 4; ++i) cur[i] = __builtin_amdgcn_raw_buffer_load_b128(srs, so + i * 1024, 0, AUX_SYS);
                    unsigned dcur = dof, fcur = fl_word(j, s, t);
                    for (;;) {
#pragma unroll
                        for (int i = 0; i < 4; ++i) __builtin_amdgcn_raw_buffer_store_b128(cur[i], wsr, dcur + i * 1024, 0, AUX_SYS);
                        asm volatile("" ::: "memory");
                        n = __builtin_amdgcn_readfirstlane(tk) - qbase;
                        char* const flag = ws + fcur;
                        if (n >= (unsigned)Q_JOBS) {
                            WAIT_VM(0);
                            if (lane == 0) st_sys(flag, call);
                            PTAIL();
                            break;
                        }
                        JOB_OF(n, t2, j2, so2, dof2);
                        WAIT_PACKED(t2, j2);
                        tk = add_agent_lane0(queue, lane);
#pragma unroll
                        for (int i = 0; i < 4; ++i) cur[i] = __builtin_amdgcn_raw_buffer_load_b128(srs, so2 + i * 1024, 0, AUX_SYS);
                        dcur = dof2; fcur = fl_word(j2, s, t2);
                        WAIT_VM(5);
                        if (lane == 0) st_sys(flag, call);
                        PTAIL();
                    }
                }
                STAMP(1);
            }
        } else {
            const __amdgpu_buffer_rsrc_t xr = rsrc_of(px, MM * KSH * 2);
            const __amdgpu_buffer_rsrc_t wr_ = rsrc_of(pw, KSH * NN * 2);
            bool met = false;
            bool x_todo = wid >= 4;
            for (int n = (wid - 2) * N_COMM + b; n < NT * 32 || x_todo;) {
                const bool do_x = x_todo && ((b >> 4) <= (n >> 8) || n >= NT * 32);
                const int g = do_x ? b >> 4 : n >> 8;
                const int delay = (do_x ? g * 8 : n >> 5) * PACK_DELAY;
                if (!met && delay > 0) { __builtin_amdgcn_s_barrier(); met = true; }
                while (__builtin_amdgcn_s_memrealtime() - t_start < (u64)delay) __builtin_amdgcn_s_sleep(2);
                u32x4 v[8];
                if (do_x) {
                    const unsigned rb = (wid - 4) * 64 + (lane & 7), piece = lane >> 3;
#pragma unroll
                    for (int i = 0; i < 8; ++i) v[i] = __builtin_amdgcn_raw_buffer_load_b128(xr, ((b & 15) * 256 + rb + i * 8) * (KSH * 2) + g * 128 + piece * 16, 0, 0);
                    if (!met) { __builtin_amdgcn_s_barrier(); met = true; }
                    const unsigned d0 = XT_OFF + (g * 8 + piece) * XT_T + (b & 15) * 32768 + SLOT(rank, b & 15) + rb * 16;
#pragma unroll
                    for (int i = 0; i < 8; ++i) __builtin_amdgcn_raw_buffer_store_b128(v[i], wsr, d0 + i * 128, 0, AUX_SYS);
                    char* fl = lane < 8 ? p.ws[lane & 7] + PK_OFF + (rank * NGRP + g) * 8192 + (b & 15) * 128 : ws + fl_word(b & 15, rank, g * 8 + (lane & 7));
                    asm volatile("s_waitcnt vmcnt(0)" : "+v"(fl) :: "memory");
                    unsigned old = 0;
                    if (lane == 0) old = __hip_atomic_fetch_add((LAS unsigned*)(lds + LDS_MISC), 1u, __ATOMIC_RELAXED, __HIP_MEMORY_SCOPE_WORKGROUP);
                    const unsigned call = __builtin_amdgcn_readfirstlane(ticket) + 1u;
                    if (__builtin_amdgcn_readfirstlane(old) == 3u && lane >= 1 && lane < 16) st_sys(fl, call);
                    x_todo = false;
                } else {
                    const int t = n >> 5, cb = n & 31;
                    const unsigned dest = (unsigned)cb >> 2;
#pragma unroll
                    for (int i = 0; i < 8; ++i) v[i] = __builtin_amdgcn_raw_buffer_load_b128(wr_, (unsigned)((t * 8 + i) * (NN * 2) + cb * 1024 + lane * 16), 0, 0);
                    if (!met) { __builtin_amdgcn_s_barrier(); met = true; }
                    const unsigned d0 = (dest == (unsigned)rank ? WT_OFF + t * WT_T + ((cb & 3) * 2 + (lane >> 5)) * 32768 + SLOT(rank, (cb & 3) * 2 + (lane >> 5))
                                                                : WS_OFF + t * WT_T + dest * 32768 + ((cb & 3) * 2 + (lane >> 5)) * 4096)
                                      + (((lane >> 4) & 1) * 128 + (lane & 15)) * 16;
#pragma unroll
                    for (int i = 0; i < 8; ++i) __builtin_amdgcn_raw_buffer_store_b128(transposed(v, i), wsr, d0 + i * 256, 0, AUX_SYS);
                    char* const dws = p.ws[(dest - rank) & 7u];
                    WAIT_VM(0);
                    const unsigned call = __builtin_amdgcn_readfirstlane(ticket) + 1u;
                    if (dest == (unsigned)rank) { if (lane < 2) st_sys(ws + fl_word(16 + (cb & 3) * 2 + lane, rank, t), call); }
                    else if (lane == 0) st_sys(dws + PK_OFF + (rank * NGRP + g) * 8192 + (32 + (t & 7) * 4 + (cb & 3)) * 128, call);
                    n += N_PACKW;
                }
            }
        }
        const unsigned call = __builtin_amdgcn_readfirstlane(ticket) + 1u;
        if (wid == 7) {
            if (b == N_COMM - 1) {
                char* amax_out = p.ws[lane & 7] + AMAX_OFF + ((lane >> 3) * NR + rank) * 128;
                asm volatile("" : "+v"(amax_out));
                const char* tm = ws + TMAX_OFF + lane * 256;
                const u64 t0 = __builtin_amdgcn_s_memrealtime(); unsigned sp = 0;
                int mx;
                for (;;) {
                    const u64 v0 = ld_sys64(tm), v1 = ld_sys64(tm + 128);
                    mx = max((int)(unsigned)v0, (int)(unsigned)v1);
                    if (__all((int)((unsigned)(v0 >> 32) - call) >= 0 && (int)((unsigned)(v1 >> 32) - call) >= 0)) break;
                    if (give_up(ws, call, t0, sp)) { mx = NAN_BITS; break; }
                }
                if (lane < 8 * AMAX_REP) st_sys64(amax_out, (unsigned)wave_max(mx), call);
                STAMP(6);
            }
            return;
        }
        {
            const int src = (rank + 1 + wid) & 7;
            const unsigned slot = OB_OFF + (unsigned)(src * 128 + b) * OB_SLOT;
            const char* const of = ws + OF_OFF + (src * 128 + b) * 128;
            const unsigned lbase = wid * LDS_WAVE;
#pragma unroll
            for (int e = 0; e < 4; ++e) {
                const unsigned idx = lane * 4 + e;
                u32x2 sel = {0u, 0u};
                unsigned rk = 0;
#pragma unroll
                for (int n = 0; n < 8; ++n) {
                    const bool neg = (idx >> (7 - n)) & 1u;
                    sel[n >> 2] |= (neg ? 0x0cu : rk) << (8 * (n & 3));
                    rk += neg ? 0u : 1u;
                }
                *(LAS u32x2*)(lds + lbase + 12288 + idx * 8) = sel;
            }
            char* outp = p.out;
            asm volatile("" : "+s"(outp));
            const unsigned ob = out_base(src, b >> 3, b & 7, lane);
            {
                bool ok = await_word<8>(ws, of, call);
                const u32x4 mk = __builtin_amdgcn_raw_buffer_load_b128(wsr, slot + lane * 16, 0, AUX_SYS);
                unsigned first[16], last[16];
                stream_layout(mk, first, last);
                u32x2 sel[16];
#pragma unroll
                for (int u = 0; u < 16; ++u) {
                    sel[u] = *(const LAS u32x2*)(lds + lbase + 12288 + sign_byte(mk, u) * 8);
                    first[u] += lbase;
                    asm volatile("" : "+v"(first[u]), "+v"(sel[u]));
                }
                STAMP(3);
                const unsigned cnt = last[15];
                const __amdgpu_buffer_rsrc_t ors = rsrc_of(outp, (MM / NR) * NN);
#define TAKE_PIECES(i0_, i1_) do { u32x4 pc[(i1_) - (i0_)];                                                            \
                    _Pragma("unroll") for (int i = (i0_); i < (i1_); ++i)                                              \
                        if (i * 1024u < cnt) pc[i - (i0_)] = __builtin_amdgcn_raw_buffer_load_b128(wsr, slot + 1024 + i * PIECE_BYTES + piece_off(lane), 0, AUX_SYS); \
                    fill = ok && !__any(ld_sys(lane & 1 ? ws + TMO_OFF : of + 12) == call) ? 0u : 0x7f7f7f7fu;          \
                    _Pragma("unroll") for (int i = (i0_); i < (i1_); ++i)                                              \
                        if (i * 1024u < cnt) {                                                                         \
                            u32x4 q = pc[i - (i0_)];                                                                   \
                            unsigned h = ((q[0] & 0x80808080u) >> 7) | ((q[1] & 0x80808080u) >> 6) | ((q[2] & 0x80808080u) >> 5); \
                            const unsigned he = h | ((q[3] & 0x80808080u) >> 4);     \
                            h = (h << 4) | (unsigned)__builtin_amdgcn_update_dpp(0, (int)he, 0xa0, 0xf, 0xf, true);    \
                            q[0] &= 0x7f7f7f7fu; q[1] &= 0x7f7f7f7fu; q[2] &= 0x7f7f7f7fu;                             \
                            q[3] = lane & 1 ? h : q[3] & 0x7f7f7f7fu;                                                  \
                            *(LAS u32x4*)(lds + lbase + i * 1024 + lane * 16) = q;                                     \
                        } } while (0)
                unsigned fill;
                ok = await_word<1>(ws, of + 8, call) && ok;
                STAMP(4);
                TAKE_PIECES(0, 8);
                unsigned d[16][3];
#pragma unroll
                for (int u = 0; u < 16; ++u)
#pragma unroll
                    for (int i = 0; i < 3; ++i) d[u][i] = *(const LAS unsigned*)(lds + (first[u] & ~3u) + i * 4);
                __builtin_amdgcn_sched_barrier(0);
#pragma unroll
                for (int u = 0; u < 16; ++u) {
                    const unsigned lo = __builtin_amdgcn_alignbyte(d[u][1], d[u][0], first[u]), hi = __builtin_amdgcn_alignbyte(d[u][2], d[u][1], first[u]);
                    u32x2 o;
                    o[0] = __builtin_amdgcn_perm(hi, lo, sel[u][0]) | fill;
                    o[1] = __builtin_amdgcn_perm(hi, lo, sel[u][1]) | fill;
                    __builtin_amdgcn_raw_buffer_store_b64(o, ors, ob + (unsigned)(((u >> 2) * 16 + (u & 3)) * NN), 0, AUX_OUT);
                }
            }
            if (cw == 255) { WAIT_VM(0); const u64 now = __builtin_amdgcn_s_memrealtime(); long long c = (long long)(now - all_t) - 12000; if (c < 0) c = 0; if (c > 5000) c = 5000;
                while (__builtin_amdgcn_s_memrealtime() - all_t < (u64)(13500 + 10 * c)) __builtin_amdgcn_s_sleep(8); }
        }
        return;
    }

    const int g = b;
    const int dbg = g;
    (void)dbg;
    const int pm = (g >> 4) * 2 + (g & 1);
    const int pn = ((g & 7) >> 1) * 2 + ((g >> 3) & 1);
    const int fr = lane & 15, fq = lane >> 4, wr = wid >> 1, wc = wid & 1;
    if (wid == 0) STAMP(0);
    const int owner = pm >> 1, tile = (pm & 1) * 8 + pn;
    char* ows = p.ws[(owner - rank) & 7];
    char* outp = p.out;
    asm volatile("" : "+s"(ows), "+s"(outp));
    const unsigned call = __builtin_amdgcn_readfirstlane(ticket) + 1u;

    const int op = wid >> 2;
    const char* const pollp = ws + fl_word(op ? 16 + pn : pm, wid & 3, lane);
    const unsigned unit = op ? WT_T : XT_T;
    const char* const gbase = ws + (op ? WT_OFF + pn * 32768 : XT_OFF + pm * 32768) + lane * 16;
    const unsigned slot0 = SLOT(wid & 3, op ? pn : pm);
    LAS char* const ldst = lds + wid * 4096;
    const int aoff = fq * 4096 + (wr * 64 + fr) * 16;
    const int boff = 16384 + fq * 4096 + (wc * 128 + fr) * 16;

    int known = 0;
    int next = 0, nslot = 0;
    u32x2 pvn = {0u, 0u};
#define STAGE_NEXT() do { const char* g_ = gbase + ((slot0 ^ ((next & 1) << 14)) + (unsigned)(next >> 1) * unit);              \
        LAS char* l_ = ldst + nslot * LDS_SLOT;                                                                       \
        __builtin_amdgcn_global_load_lds((const GAS unsigned*)g_, (LAS unsigned*)l_, 16, 0, STAGE_AUX);                       \
        __builtin_amdgcn_global_load_lds((const GAS unsigned*)g_, (LAS unsigned*)l_, 16, 1024, STAGE_AUX);                    \
        __builtin_amdgcn_global_load_lds((const GAS unsigned*)g_, (LAS unsigned*)l_, 16, 2048, STAGE_AUX);                    \
        __builtin_amdgcn_global_load_lds((const GAS unsigned*)g_, (LAS unsigned*)l_, 16, 3072, STAGE_AUX);                    \
        ++next; nslot = nslot == NSLOT - 1 ? 0 : nslot + 1; } while (0)
#define POLL_ISSUE() asm volatile("global_load_dwordx2 %0, %1, off sc0 sc1" : "+v"(pvn) : "v"(pollp) : "memory")
#define POLL_TAKE() do { u32x2 f_; asm volatile("s_waitcnt vmcnt(0)\n\tv_mov_b64 %0, %1" : "=v"(f_) : "v"(pvn) : "memory");    \
        known = steps_known(f_[0], f_[1], call); } while (0)
#define AWAIT_NEXT() do { if (next >= known) {                                                                        \
            const u64 t0_ = __builtin_amdgcn_s_memrealtime(); unsigned sp_ = 0;                                       \
            for (;;) { const u64 f_ = ld_sys64(pollp); known = steps_known((unsigned)f_, (unsigned)(f_ >> 32), call); \
                if (next < known) break;                                                                              \
                if (give_up<SLEEP_K>(ws, call, t0_, sp_)) { known = NSTEP; break; } } } } while (0)
#define MFMA_COLS(A_, B_, n0_, n1_) do { __builtin_amdgcn_sched_barrier(0);                                           \
        _Pragma("unroll") for (int n = (n0_); n < (n1_); ++n)                                                         \
            _Pragma("unroll") for (int m = 0; m < 4; ++m)                                                             \
                acc[m][n] = __builtin_amdgcn_mfma_f32_16x16x32_bf16(A_[m], B_[n], acc[m][n], 0, 0, 0);                \
        __builtin_amdgcn_sched_barrier(0); } while (0)
#define RD_A(A_, m_) A_[m_] = *(const LAS bf16x8*)(lb_ + aoff + (m_) * 256)
#define RD_B(B_, n_) B_[n_] = *(const LAS bf16x8*)(lb_ + boff + (n_) * 256)
#define TAIL_REC(st_) do {} while (0)
#define KSTEP(st_, A_, B_, AN_, BN_) do {                                                                             \
        MFMA_COLS(A_, B_, 0, TAKE_COL);                                                                               \
        if (polled) POLL_TAKE();                                          \
        const int had_ = next;                                                                                        \
        const int lim_ = min((st_) + NSLOT, known);                                   \
        if (next < lim_) { STAGE_NEXT(); if (next < lim_) STAGE_NEXT(); }                                             \
        const bool full_ = (polled ? had_ + NSLOT - 2 : next) >= (st_) + NSLOT;                                       \
        MFMA_COLS(A_, B_, TAKE_COL, 4);                                                                                      \
        if (!full_) {                                                              \
            if (next == (st_) + 1 && next < NSTEP) {                                                                  \
                AWAIT_NEXT(); STAGE_NEXT();                                                                           \
                if (next < known) STAGE_NEXT();                                                                       \
            }                                                                                                         \
            const int k_ = next - (st_) - 2;                                                                          \
            if (k_ <= 0) WAIT_VM(0); else if (k_ == 1) WAIT_VM(4); else WAIT_VM(8);                                   \
        }                                                                                                             \
        WAIT_VM(12);                                                                                                  \
        LGKM0_BARRIER();                                                                                              \
        TAIL_REC(st_);                                                                                                \
        const LAS char* lb_ = lds + rslot * LDS_SLOT;               \
        RD_A(AN_, 0); RD_A(AN_, 1); RD_A(AN_, 2); RD_A(AN_, 3);                                                       \
        if (POLL_POS == 0) { polled = known < min(next + POLL_AHEAD, NSTEP); if (polled) POLL_ISSUE(); }              \
        MFMA_COLS(A_, B_, 4, 5);                                                                                      \
        RD_B(BN_, 0); RD_B(BN_, 1); RD_B(BN_, 2); RD_B(BN_, 3);                                                       \
        MFMA_COLS(A_, B_, 5, 6);                                                                                      \
        RD_B(BN_, 4); RD_B(BN_, 5); RD_B(BN_, 6); RD_B(BN_, 7);                                                       \
        rslot = rslot == NSLOT - 1 ? 0 : rslot + 1;                                                                   \
        if (POLL_POS == 2) { polled = known < min(next + POLL_AHEAD, NSTEP); if (polled) POLL_ISSUE(); }              \
        MFMA_COLS(A_, B_, 6, 8); } while (0)

    f32x4 acc[4][8];
#pragma unroll
    for (int m = 0; m < 4; ++m)
#pragma unroll
        for (int n = 0; n < 8; ++n) acc[m][n] = (f32x4){0.f, 0.f, 0.f, 0.f};

    AWAIT_NEXT(); STAGE_NEXT();
    WAIT_VM(0);
    LGKM0_BARRIER();
    int rslot = 1;
    bool polled = false;
    bf16x8 a0[4], b0[8], a1[4], b1[8];
    {
        const LAS char* lb_ = lds;
#pragma unroll
        for (int m = 0; m < 4; ++m) RD_A(a0, m);
#pragma unroll
        for (int n = 0; n < 8; ++n) RD_B(b0, n);
    }
    if (wid == 0) STAMP(1);
    for (int st = 0; st < NSTEP; st += 2) {
        KSTEP(st, a0, b0, a1, b1);
        KSTEP(st + 1, a1, b1, a0, b0);
    }
    if (wid == 0) STAMP(2);

    int mx = 0;
#pragma unroll
    for (int m = 0; m < 4; ++m)
#pragma unroll
        for (int n = 0; n < 8; ++n)
#pragma unroll
            for (int j = 0; j < 4; ++j) mx = max(mx, __float_as_int(acc[m][n][j]));
    mx = wave_max(mx);
    LAS int* red = (LAS int*)(lds + LDS_MISC);
    if (lane == 0) red[wid] = mx;
    LGKM0_BARRIER();
    if (wid == 0) {
        int t = red[lane & 7];
        ROW_SHR_MAX(t, 1); ROW_SHR_MAX(t, 2); ROW_SHR_MAX(t, 4);
        if (lane == 7) st_sys64(ws + TMAX_OFF + g * 128, (unsigned)t, call);
        STAMP(7);
    }

    const bool remote = owner != rank;
    const __amdgpu_buffer_rsrc_t sls = rsrc_of(ows + OB_OFF + (unsigned)(rank * 128 + tile * 8 + wid) * OB_SLOT, OB_SLOT);
    char* const of = ows + OF_OFF + (rank * 128 + tile * 8 + wid) * 128;
    const int efield = min(268 - (mx >> 23), 253);
    const float sc = __int_as_float(efield << 23);
    const unsigned pos0 = (unsigned)(size_t)(lds + wid * LDS_WAVE);
    const unsigned spill = (unsigned)(size_t)(lds + LDS_MISC + 64);
    unsigned first[16], last[16];
#define VAL(u_, n_) acc[(u_) >> 2][n_][(u_) & 3]
#define PACK1(n_) "v_ashrrev_i32 %[s], 31, %[v" #n_ "]\n\tv_fma_mixlo_f16 %[h], %[v" #n_ "], %[sc], 0\n\t"              \
                  "v_bfi_b32 %[a], %[s], %[sp], %[p]\n\tv_lshl_add_u32 %[p], %[s], 1, %[p]\n\t"                      \
                  "ds_write_b16 %[a], %[h] offset:2*" #n_ "\n\t"
#define PACK8(u_) do { unsigned p_ = pos0 + 2 * first[u_], s_, h_, a_;                                                \
        asm volatile(PACK1(0) PACK1(1) PACK1(2) PACK1(3) PACK1(4) PACK1(5) PACK1(6) PACK1(7)                          \
                     : [p] "+v"(p_), [s] "=&v"(s_), [h] "=&v"(h_), [a] "=&v"(a_)                                      \
                     : [v0] "v"(VAL(u_, 0)), [v1] "v"(VAL(u_, 1)), [v2] "v"(VAL(u_, 2)), [v3] "v"(VAL(u_, 3)),        \
                       [v4] "v"(VAL(u_, 4)), [v5] "v"(VAL(u_, 5)), [v6] "v"(VAL(u_, 6)), [v7] "v"(VAL(u_, 7)),        \
                       [sc] "s"(sc), [sp] "v"(spill) : "memory"); } while (0)
    u32x4 mk = {0u, 0u, 0u, 0u};
    if (remote) {
#pragma unroll
        for (int r = 0; r < 128; ++r) mk[r >> 5] = __builtin_amdgcn_alignbit(mk[r >> 5], __float_as_uint(VAL(r >> 3, r & 7)), 31);
        if (!LATE_FLAG || wid != 0) __builtin_amdgcn_raw_buffer_store_b128(mk, sls, lane * 16, 0, AUX_SYS);
        stream_layout(mk, first, last);
        PACK8(0); PACK8(1); PACK8(2); PACK8(3); PACK8(4);
        if (!LATE_FLAG) { WAIT_VM(0); if (lane == 0) st_sys(of, call); }
    } else {
#pragma unroll
        for (int m = 0; m < 4; ++m) {
#pragma unroll
            for (int n = 0; n < 8; ++n)
#pragma unroll
                for (int j = 0; j < 4; ++j) acc[m][n][j] = __int_as_float(max(0, __float_as_int(acc[m][n][j])));
            asm volatile("" : "+v"(acc[m][0]), "+v"(acc[m][1]), "+v"(acc[m][2]), "+v"(acc[m][3]),
                              "+v"(acc[m][4]), "+v"(acc[m][5]), "+v"(acc[m][6]), "+v"(acc[m][7]));
        }
    }
    if (wid == 1) STAMP(6);
    if (wid == 0) {
        const char* ap = lane < NR ? ws + AMAX_OFF + ((g & (AMAX_REP - 1)) * NR + lane) * 128 : ws + TMO_OFF - 4;
        const u64 t0 = __builtin_amdgcn_s_memrealtime(); unsigned sp = 0;
        u64 v = 0;
        for (;;) {
            if (lane <= NR) v = ld_sys64(ap);
            if ((__ballot((int)((unsigned)(v >> 32) - call) >= 0) & 0xffu) == 0xffu) break;
            if (give_up(ws, call, t0, sp)) { v = (u64)call << 32; break; }
        }
        STAMP(8);
        int t = lane < NR ? (int)(unsigned)v : ((unsigned)(v >> 32) == call ? NAN_BITS : 0);
        ROW_SHR_MAX(t, 1); ROW_SHR_MAX(t, 2); ROW_SHR_MAX(t, 4); ROW_SHR_MAX(t, 8);
        if (lane == NR) red[8] = t;
    }
    LGKM0_BARRIER();
    const float inv = 448.0f / __int_as_float(red[8]);
    if (wid == 0) STAMP(3);

    if (remote) {
        const float f = inv * __int_as_float((254 - efield) << 23);
        unsigned sent = 0;
        const unsigned hshift = (lane & 1) * 4, woff = piece_off(lane);
#define SEND_GATE() WAIT_VM(2)
#define SEND(cnt_, upto_) do { __builtin_amdgcn_s_waitcnt(0xC07F); asm volatile("" ::: "memory");                           \
            _Pragma("nounroll") for (; sent < (upto_); sent += 1024) {                                                                    \
                const unsigned o_ = sent + lane * 16;                                                                 \
                const f16x8 h0 = *(const LAS f16x8*)(lds + wid * LDS_WAVE + o_ * 2), h1 = *(const LAS f16x8*)(lds + wid * LDS_WAVE + o_ * 2 + 16); \
                u32x4 q;                                                                                              \
                _Pragma("unroll") for (int d = 0; d < 2; ++d) {                                                       \
                    int w0 = __builtin_amdgcn_cvt_pk_fp8_f32(__builtin_fmaf((float)h0[4 * d], f, 0.f), __builtin_fmaf((float)h0[4 * d + 1], f, 0.f), 0, false); \
                    q[d] = (unsigned)__builtin_amdgcn_cvt_pk_fp8_f32(__builtin_fmaf((float)h0[4 * d + 2], f, 0.f), __builtin_fmaf((float)h0[4 * d + 3], f, 0.f), w0, true); \
                    int w1 = __builtin_amdgcn_cvt_pk_fp8_f32(__builtin_fmaf((float)h1[4 * d], f, 0.f), __builtin_fmaf((float)h1[4 * d + 1], f, 0.f), 0, false); \
                    q[2 + d] = (unsigned)__builtin_amdgcn_cvt_pk_fp8_f32(__builtin_fmaf((float)h1[4 * d + 2], f, 0.f), __builtin_fmaf((float)h1[4 * d + 3], f, 0.f), w1, true); \
                }                                                                                                     \
                  \
                const unsigned h_ = (unsigned)__builtin_amdgcn_update_dpp(0, (int)q[3], 0xf5, 0xf, 0xf, true) >> hshift; \
                _Pragma("unroll") for (int d = 0; d < 4; ++d) q[d] |= (h_ << (7 - d)) & 0x80808080u;                  \
                SEND_GATE();                                                                                          \
                if ((o_ & ~31u) < (cnt_)) {                                                                             \
                    const unsigned w_ = 1024 + (sent >> 10) * PIECE_BYTES + woff;                                     \
                    if (lane & 1) __builtin_amdgcn_raw_buffer_store_b96((u32x3){q[0], q[1], q[2]}, sls, w_, 0, AUX_SYS); \
                    else __builtin_amdgcn_raw_buffer_store_b128(q, sls, w_, 0, AUX_SYS);                              \
                }                                                                                                     \
            } } while (0)
        if (LATE_FLAG && wid == 0) __builtin_amdgcn_raw_buffer_store_b128(mk, sls, lane * 16, 0, AUX_SYS);
        SEND(last[4], last[4] & ~1023u);
        PACK8(5); PACK8(6); PACK8(7); PACK8(8);
        if (LATE_FLAG) { if (sent == 1024u) WAIT_VM(2); else WAIT_VM(0); if (lane == 0) st_sys(of, call); }
        SEND(last[8], last[8] & ~1023u);
        PACK8(9); PACK8(10); PACK8(11); PACK8(12);
        SEND(last[12], last[12] & ~1023u);
        PACK8(13); PACK8(14); PACK8(15);
        SEND(last[15], last[15]);
        WAIT_VM(0);
        if (lane == 0) st_sys64(of + 8, call, red[8] == NAN_BITS ? call : 0u);
    } else {
        const __amdgpu_buffer_rsrc_t ors = rsrc_of(outp, (MM / NR) * NN);
        const unsigned ob = out_base(rank, tile, wid, lane);
#pragma unroll
        for (int m = 0; m < 4; ++m)
#pragma unroll
            for (int j = 0; j < 4; ++j) {
                u32x2 o;
                int lo = __builtin_amdgcn_cvt_pk_fp8_f32(acc[m][0][j] * inv, acc[m][1][j] * inv, 0, false);
                o[0] = (unsigned)__builtin_amdgcn_cvt_pk_fp8_f32(acc[m][2][j] * inv, acc[m][3][j] * inv, lo, true);
                int hi = __builtin_amdgcn_cvt_pk_fp8_f32(acc[m][4][j] * inv, acc[m][5][j] * inv, 0, false);
                o[1] = (unsigned)__builtin_amdgcn_cvt_pk_fp8_f32(acc[m][6][j] * inv, acc[m][7][j] * inv, hi, true);
                __builtin_amdgcn_raw_buffer_store_b64(o, ors, ob + (unsigned)((m * 16 + j) * NN), 0, AUX_OUT);
            }
    }
    if (wid == 0) STAMP(4);
    if (wid == 7) STAMP(5);
}

}

extern "C" void kernel_launch(int rank, int nranks, ncclComm_t comm,
                              hipStream_t stream,
                              void* const* d_in, const int* in_sizes, int n_in,
                              void* d_out, int out_size,
                              const int* in_dtypes, int out_dtype,
                              void* const* d_out_all,
                              void* const* d_ws_all, size_t ws_bytes) {
    (void)comm; (void)in_dtypes; (void)out_dtype; (void)d_out_all;
    if (nranks != NR || n_in < 2 || (unsigned)rank >= (unsigned)NR || ws_bytes < WS_NEED ||
        in_sizes[0] != MM * KSH || in_sizes[1] != KSH * NN || out_size != (MM / NR) * NN) {
        fprintf(stderr, "kernel_launch: unexpected problem shape\n");
        return;
    }
    Params p;
    p.x = (const char*)d_in[0];
    p.w = (const char*)d_in[1];
    for (int i = 0; i < NR; ++i) p.ws[i] = (char*)d_ws_all[(rank + i) % NR];
    p.out = (char*)d_out;
    p.rank = rank; p.pad = 0;
    hipLaunchKernelGGL(fused_gemm_relu_q8, dim3(GRID), dim3(BLOCK), 0, stream, p);
    HIP_CHECK(hipGetLastError());
}
```

```cpp
#include <hip/hip_runtime.h>
#include <rccl/rccl.h>
#include <cstdio>
#include <cstdint>

#define HIP_CHECK(expr) do { hipError_t _e = (expr); if (_e != hipSuccess) { \
    fprintf(stderr, "HIP error %s at %s:%d\n", hipGetErrorString(_e), __FILE__, __LINE__); \
    return; } } while (0)

namespace {

typedef short bf16x8 __attribute__((ext_vector_type(8)));
typedef _Float16 f16x8 __attribute__((ext_vector_type(8)));
typedef float f32x4 __attribute__((ext_vector_type(4)));
typedef unsigned u32x4 __attribute__((ext_vector_type(4)));
typedef unsigned u32x3 __attribute__((ext_vector_type(3)));
typedef unsigned u32x2 __attribute__((ext_vector_type(2)));
typedef unsigned long long u64;
#define LAS __attribute__((address_space(3)))
#define GAS __attribute__((address_space(1)))

constexpr int NR = 8;
constexpr int MM = 4096, KSH = 512, NN = 16384, NCOL = NN / NR;
constexpr int NT = KSH / 8;
constexpr int NBLK = 24;
constexpr int PULL_PER_LINK = 18;
constexpr int PACK_DELAY = 100;
constexpr int Q_JOBS = NT * NBLK;
constexpr int Q_TICKETS = Q_JOBS + PULL_PER_LINK;
constexpr int N_COMM = 128;
constexpr int N_PULL = (NR - 1) * PULL_PER_LINK;
constexpr int N_GEMM = 128;
constexpr int GRID = N_COMM + N_GEMM;
constexpr int BLOCK = 512;
constexpr int N_PACKW = N_COMM * 6;
constexpr int NGRP = NT / 8;
static_assert(N_PULL <= 2 * N_COMM, "pull waves are waves 0-1 of the comm workgroups");

__host__ __device__ constexpr int job_block(int k) { return k % 3 == 2 ? 16 + k / 3 : k / 3 * 2 + k % 3; }
__host__ __device__ constexpr unsigned chunk_pos(unsigned src, unsigned blk) { return ((src + blk) & 7u) * 4096u; }

constexpr unsigned XT_T = NR * MM * 16, WT_T = NR * NCOL * 16;
constexpr unsigned XT_OFF = 0;
constexpr unsigned WT_OFF = XT_OFF + NT * XT_T;
constexpr unsigned WS_OFF = WT_OFF + NT * WT_T;
constexpr unsigned OB_SLOT = 1024 + 8192;
constexpr unsigned OB_OFF = WS_OFF + NT * WT_T;
constexpr unsigned CTL_OFF = OB_OFF + NR * 128 * OB_SLOT;
constexpr unsigned PK_OFF = CTL_OFF;
constexpr unsigned FL_OFF = PK_OFF + NR * NGRP * 8192;
constexpr unsigned Q_OFF = FL_OFF + NBLK * NR * NT * 4;
constexpr unsigned TMAX_OFF = Q_OFF + NR * 128;
constexpr unsigned AMAX_OFF = TMAX_OFF + N_GEMM * 128;
constexpr unsigned OF_OFF = AMAX_OFF + 64 * 128;
constexpr unsigned CTR_OFF = OF_OFF + NR * 128 * 128;
constexpr unsigned TMO_OFF = CTR_OFF + GRID * 128 + 4;
constexpr size_t WS_NEED = CTR_OFF + GRID * 128 + 128;

constexpr u64 DEADLINE = 50000000ull;
constexpr int AUX_SYS = 17;

constexpr int NSTEP = 2 * NT;
constexpr int NSLOT = 5;
constexpr int LDS_SLOT = 32768;
constexpr int LDS_BYTES = NSLOT * LDS_SLOT;
constexpr int LDS_WAVE = 16384;
constexpr int LDS_MISC = 8 * LDS_WAVE;

struct Params {
    const char* x; const char* w;
    int rank, pad;
    char* ws[NR];
    char* out;
};

__device__ __forceinline__ __amdgpu_buffer_rsrc_t rsrc_of(const void* p, unsigned bytes) {
    return __builtin_amdgcn_make_buffer_rsrc(const_cast<void*>(p), 0, bytes, 0x00020000);
}
__device__ __forceinline__ unsigned ld_sys(const void* p) {
    return __hip_atomic_load((const GAS unsigned*)p, __ATOMIC_RELAXED, __HIP_MEMORY_SCOPE_SYSTEM);
}
__device__ __forceinline__ u64 ld_sys64(const void* p) {
    return __hip_atomic_load((const GAS u64*)p, __ATOMIC_RELAXED, __HIP_MEMORY_SCOPE_SYSTEM);
}
__device__ __forceinline__ void st_sys(void* p, unsigned v) {
    __hip_atomic_store((GAS unsigned*)p, v, __ATOMIC_RELAXED, __HIP_MEMORY_SCOPE_SYSTEM);
}
__device__ __forceinline__ void st_sys64(void* p, unsigned lo, unsigned hi) {
    __hip_atomic_store((GAS u64*)p, ((u64)hi << 32) | lo, __ATOMIC_RELAXED, __HIP_MEMORY_SCOPE_SYSTEM);
}
__device__ __forceinline__ unsigned ld_agent(const void* p) {
    return __hip_atomic_load((const GAS unsigned*)p, __ATOMIC_RELAXED, __HIP_MEMORY_SCOPE_AGENT);
}
__device__ __forceinline__ unsigned add_agent_lane0(void* p, int lane) {
    unsigned v = 0, z = 0;
    asm volatile("" : "+v"(z));
    if (lane == 0) v = __hip_atomic_fetch_add((GAS unsigned*)((char*)p + z), 1u, __ATOMIC_RELAXED, __HIP_MEMORY_SCOPE_AGENT);
    return v;
}
__device__ __forceinline__ u32x4 transposed(const u32x4 (&v)[8], int i) {
    u32x4 o;
#pragma unroll
    for (int d = 0; d < 4; ++d)
        o[d] = __builtin_amdgcn_perm(v[2 * d + 1][i >> 1], v[2 * d][i >> 1], (i & 1) ? 0x07060302u : 0x05040100u);
    return o;
}
constexpr int NAN_BITS = 0x7fc00000;
#define ROW_SHR_MAX(v, n) v = max(v, __builtin_amdgcn_update_dpp(0, v, 0x110 + (n), 0xf, 0xf, true))
__device__ __forceinline__ int wave_max(int v) {
    ROW_SHR_MAX(v, 1); ROW_SHR_MAX(v, 2); ROW_SHR_MAX(v, 4); ROW_SHR_MAX(v, 8);
    return max(max(__builtin_amdgcn_readlane(v, 15), __builtin_amdgcn_readlane(v, 31)),
               max(__builtin_amdgcn_readlane(v, 47), __builtin_amdgcn_readlane(v, 63)));
}
__device__ __forceinline__ unsigned sign_byte(const u32x4& mk, int u) { return (mk[u >> 2] >> (24 - 8 * (u & 3))) & 0xffu; }
__device__ __forceinline__ void stream_layout(const u32x4& mk, unsigned (&first)[16], unsigned (&last)[16]) {
    unsigned base = 0;
#pragma unroll
    for (int u = 0; u < 16; u += 2) {
        const int c = 0x00080008 - (int)(__builtin_popcount(sign_byte(mk, u)) | (__builtin_popcount(sign_byte(mk, u + 1)) << 16));
        int v = c;
        v += __builtin_amdgcn_update_dpp(0, v, 0x111, 0xf, 0xf, true);
        v += __builtin_amdgcn_update_dpp(0, v, 0x112, 0xf, 0xf, true);
        v += __builtin_amdgcn_update_dpp(0, v, 0x114, 0xf, 0xf, true);
        v += __builtin_amdgcn_update_dpp(0, v, 0x118, 0xf, 0xf, true);
        v += __builtin_amdgcn_update_dpp(0, v, 0x142, 0xa, 0xf, false);
        v += __builtin_amdgcn_update_dpp(0, v, 0x143, 0xc, 0xf, false);
        const unsigned tot = (unsigned)__builtin_amdgcn_readlane(v, 63), exc = (unsigned)(v - c);
        first[u] = base + (exc & 0xffffu); base += tot & 0xffffu; last[u] = base;
        first[u + 1] = base + (exc >> 16); base += tot >> 16; last[u + 1] = base;
    }
}
template <int SLEEP = 1>
__device__ __forceinline__ bool give_up(char* ws, unsigned call, u64 t0, unsigned& spins) {
    bool stop = __builtin_amdgcn_s_memrealtime() - t0 > DEADLINE;
    if ((++spins & 63u) == 0u) stop = stop || __builtin_amdgcn_readfirstlane(ld_agent(ws + TMO_OFF)) == call;
    if (stop) {
        __hip_atomic_store((GAS unsigned*)(ws + TMO_OFF), call, __ATOMIC_RELAXED, __HIP_MEMORY_SCOPE_AGENT);
        return true;
    }
    __builtin_amdgcn_s_sleep(SLEEP);
    return false;
}
template <int SLEEP>
__device__ __forceinline__ bool await_word(char* ws, const char* p, unsigned call) {
    const u64 t0 = __builtin_amdgcn_s_memrealtime(); unsigned sp = 0;
    while ((int)(__builtin_amdgcn_readfirstlane(ld_sys(p)) - call) < 0)
        if (give_up<SLEEP>(ws, call, t0, sp)) return false;
    return true;
}
__device__ __forceinline__ unsigned fl_word(int blk, int src, int t) { return FL_OFF + (unsigned)(((blk * 4 + (src & 3)) * NT + t) * 2 + (src >> 2)) * 4u; }
__device__ __forceinline__ int steps_known(unsigned f0, unsigned f1, unsigned call) {
    const u64 lo = __ballot((int)(f0 - call) >= 0), hi = __ballot((int)(f1 - call) >= 0);
    const u64 miss = ~(lo & hi);
    if (miss == 0) return NSTEP;
    const int c = __builtin_ctzll(miss);
    return 2 * c + (int)((lo >> c) & 1);
}
__device__ __forceinline__ unsigned out_base(int src, int tile, int wave, int lane) {
    return (unsigned)(((tile >> 3) * 256 + (wave >> 1) * 64 + (lane >> 4) * 4) * NN + src * NCOL + (tile & 7) * 256 + (wave & 1) * 128 + (lane & 15) * 8);
}
constexpr unsigned PIECE_BYTES = 896;
__device__ __forceinline__ unsigned piece_off(int lane) { return lane & 1 ? 512 + (lane >> 1) * 12 : (lane >> 1) * 16; }
#define WAIT_VM(n) asm volatile("s_waitcnt vmcnt(" #n ")" ::: "memory")
#define LGKM0_BARRIER() do { __builtin_amdgcn_s_waitcnt(0xC07F); asm volatile("" ::: "memory");                       \
        __builtin_amdgcn_s_barrier(); asm volatile("" ::: "memory"); } while (0)

__global__ void __launch_bounds__(BLOCK, 2) fused_gemm_relu_q8(const Params p) {
    __shared__ __attribute__((aligned(1024))) char smem[LDS_BYTES];
    LAS char* const lds = (LAS char*)smem;
    const int hb = blockIdx.x;
    const int lane = threadIdx.x & 63;
    const int wid = __builtin_amdgcn_readfirstlane(threadIdx.x >> 6);
    int rank = p.rank;
    char* ws = p.ws[0];
    const char* px = p.x;
    const char* pw = p.w;
    const u64 t_start = __builtin_amdgcn_s_memrealtime();
    asm volatile("" : "+s"(rank), "+s"(ws), "+s"(px), "+s"(pw));
    const unsigned ticket = add_agent_lane0(ws + CTR_OFF + (size_t)hb * 128 + wid * 4, lane);
    const int b = hb & (N_COMM - 1);

    if (hb < N_COMM) {
        const __amdgpu_buffer_rsrc_t wsr = rsrc_of(ws, CTL_OFF);
        const int cw = wid * N_COMM + b;
        if (wid < 2) {
            if (threadIdx.x == 0) *(LAS unsigned*)(lds + LDS_MISC) = 0u;
            LGKM0_BARRIER();
            if (cw < N_PULL) {
                const unsigned s = (unsigned)(rank + 1 + cw % (NR - 1)) & 7u;
                const __amdgpu_buffer_rsrc_t srs = rsrc_of(p.ws[1 + cw % (NR - 1)], CTL_OFF);
                char* const queue = ws + Q_OFF + s * 128;
                unsigned tk = add_agent_lane0(queue, lane);
                u32x4 cur[4];
                const unsigned call = __builtin_amdgcn_readfirstlane(ticket) + 1u;
                const unsigned qbase = (call - 1u) * Q_TICKETS;
                const char* const pk = ws + PK_OFF + s * NGRP * 8192 + lane * 128;
                unsigned pv = call - 1u;
                int pgrp = 0;
#define JOB_OF(n_, t_, j_, so_, do_) const int t_ = (int)(n_) / NBLK, j_ = job_block((int)(n_) % NBLK);                           \
                const unsigned do_ = (j_ < 16 ? XT_OFF + t_ * XT_T + j_ * 32768 : WT_OFF + t_ * WT_T + (j_ - 16) * 32768) + chunk_pos(s, j_) + lane * 16; \
                const unsigned so_ = j_ < 16 ? do_ : WS_OFF + t_ * WT_T + rank * 32768 + (j_ - 16) * 4096 + lane * 16
#define WAIT_PACKED(t_, j_) do { const int sl_ = (j_) < 16 ? (j_) : 24 + ((t_) & 7) * 4 + ((j_) >> 1);                 \
                if (((t_) >> 3) != pgrp) { pgrp = (t_) >> 3; pv = call - 1u; }                                         \
                if ((int)(__builtin_amdgcn_readlane(pv, sl_) - call) < 0) {                                            \
                    pv = ld_sys(pk + pgrp * 8192);                 \
                    const u64 t0_ = __builtin_amdgcn_s_memrealtime(); unsigned sp_ = 0;                                \
                    while ((int)(__builtin_amdgcn_readlane(pv, sl_) - call) < 0) {                                     \
                        if (give_up(ws, call, t0_, sp_)) break;                                                        \
                        if (lane == sl_) pv = ld_sys(pk + pgrp * 8192); } } } while (0)
                unsigned n = __builtin_amdgcn_readfirstlane(tk) - qbase;
                if (n < (unsigned)Q_JOBS) {
                    JOB_OF(n, t, j, so, dof);
                    WAIT_PACKED(t, j);
                    tk = add_agent_lane0(queue, lane);
#pragma unroll
                    for (int i = 0; i < 4; ++i) cur[i] = __builtin_amdgcn_raw_buffer_load_b128(srs, so + i * 1024, 0, AUX_SYS);
                    unsigned dcur = dof, fcur = fl_word(j, s, t);
                    for (;;) {
#pragma unroll
                        for (int i = 0; i < 4; ++i) __builtin_amdgcn_raw_buffer_store_b128(cur[i], wsr, dcur + i * 1024, 0, AUX_SYS);
                        asm volatile("" ::: "memory");
                        n = __builtin_amdgcn_readfirstlane(tk) - qbase;
                        char* const flag = ws + fcur;
                        if (n >= (unsigned)Q_JOBS) {
                            WAIT_VM(0);
                            if (lane == 0) st_sys(flag, call);
                            break;
                        }
                        JOB_OF(n, t2, j2, so2, dof2);
                        WAIT_PACKED(t2, j2);
                        tk = add_agent_lane0(queue, lane);
#pragma unroll
                        for (int i = 0; i < 4; ++i) cur[i] = __builtin_amdgcn_raw_buffer_load_b128(srs, so2 + i * 1024, 0, AUX_SYS);
                        dcur = dof2; fcur = fl_word(j2, s, t2);
                        WAIT_VM(5);
                        if (lane == 0) st_sys(flag, call);
                    }
                }
            }
        } else {
            const __amdgpu_buffer_rsrc_t xr = rsrc_of(px, MM * KSH * 2);
            const __amdgpu_buffer_rsrc_t wr_ = rsrc_of(pw, KSH * NN * 2);
            bool met = false;
            bool x_todo = wid >= 4;
            for (int n = (wid - 2) * N_COMM + b; n < NT * 32 || x_todo;) {
                const bool do_x = x_todo && ((b >> 4) <= (n >> 8) || n >= NT * 32);
                const int g = do_x ? b >> 4 : n >> 8;
                const int delay = (do_x ? g * 8 : n >> 5) * PACK_DELAY;
                if (!met && delay > 0) { __builtin_amdgcn_s_barrier(); met = true; }
                while (__builtin_amdgcn_s_memrealtime() - t_start < (u64)delay) __builtin_amdgcn_s_sleep(2);
                u32x4 v[8];
                if (do_x) {
                    const unsigned rb = (wid - 4) * 64 + (lane & 7), piece = lane >> 3;
#pragma unroll
                    for (int i = 0; i < 8; ++i) v[i] = __builtin_amdgcn_raw_buffer_load_b128(xr, ((b & 15) * 256 + rb + i * 8) * (KSH * 2) + g * 128 + piece * 16, 0, 0);
                    if (!met) { __builtin_amdgcn_s_barrier(); met = true; }
                    const unsigned d0 = XT_OFF + (g * 8 + piece) * XT_T + (b & 15) * 32768 + chunk_pos(rank, b & 15) + rb * 16;
#pragma unroll
                    for (int i = 0; i < 8; ++i) __builtin_amdgcn_raw_buffer_store_b128(v[i], wsr, d0 + i * 128, 0, AUX_SYS);
                    char* fl = lane < 8 ? p.ws[lane & 7] + PK_OFF + (rank * NGRP + g) * 8192 + (b & 15) * 128 : ws + fl_word(b & 15, rank, g * 8 + (lane & 7));
                    asm volatile("s_waitcnt vmcnt(0)" : "+v"(fl) :: "memory");
                    unsigned old = 0;
                    if (lane == 0) old = __hip_atomic_fetch_add((LAS unsigned*)(lds + LDS_MISC), 1u, __ATOMIC_RELAXED, __HIP_MEMORY_SCOPE_WORKGROUP);
                    const unsigned call = __builtin_amdgcn_readfirstlane(ticket) + 1u;
                    if (__builtin_amdgcn_readfirstlane(old) == 3u && lane >= 1 && lane < 16) st_sys(fl, call);
                    x_todo = false;
                } else {
                    const int t = n >> 5, cb = n & 31;
                    const unsigned dest = (unsigned)cb >> 2;
#pragma unroll
                    for (int i = 0; i < 8; ++i) v[i] = __builtin_amdgcn_raw_buffer_load_b128(wr_, (unsigned)((t * 8 + i) * (NN * 2) + cb * 1024 + lane * 16), 0, 0);
                    if (!met) { __builtin_amdgcn_s_barrier(); met = true; }
                    const unsigned d0 = (dest == (unsigned)rank ? WT_OFF + t * WT_T + ((cb & 3) * 2 + (lane >> 5)) * 32768 + chunk_pos(rank, (cb & 3) * 2 + (lane >> 5))
                                                                : WS_OFF + t * WT_T + dest * 32768 + ((cb & 3) * 2 + (lane >> 5)) * 4096)
                                      + (((lane >> 4) & 1) * 128 + (lane & 15)) * 16;
#pragma unroll
                    for (int i = 0; i < 8; ++i) __builtin_amdgcn_raw_buffer_store_b128(transposed(v, i), wsr, d0 + i * 256, 0, AUX_SYS);
                    char* const dws = p.ws[(dest - rank) & 7u];
                    WAIT_VM(0);
                    const unsigned call = __builtin_amdgcn_readfirstlane(ticket) + 1u;
                    if (dest == (unsigned)rank) { if (lane < 2) st_sys(ws + fl_word(16 + (cb & 3) * 2 + lane, rank, t), call); }
                    else if (lane == 0) st_sys(dws + PK_OFF + (rank * NGRP + g) * 8192 + (32 + (t & 7) * 4 + (cb & 3)) * 128, call);
                    n += N_PACKW;
                }
            }
        }
        const unsigned call = __builtin_amdgcn_readfirstlane(ticket) + 1u;
        if (wid == 7) {
            if (b == N_COMM - 1) {
                char* amax_out = p.ws[lane & 7] + AMAX_OFF + ((lane >> 3) * NR + rank) * 128;
                asm volatile("" : "+v"(amax_out));
                const char* tm = ws + TMAX_OFF + lane * 256;
                const u64 t0 = __builtin_amdgcn_s_memrealtime(); unsigned sp = 0;
                int mx;
                for (;;) {
                    const u64 v0 = ld_sys64(tm), v1 = ld_sys64(tm + 128);
                    mx = max((int)(unsigned)v0, (int)(unsigned)v1);
                    if (__all((int)((unsigned)(v0 >> 32) - call) >= 0 && (int)((unsigned)(v1 >> 32) - call) >= 0)) break;
                    if (give_up(ws, call, t0, sp)) { mx = NAN_BITS; break; }
                }
                st_sys64(amax_out, (unsigned)wave_max(mx), call);
            }
            return;
        }
        {
            const int src = (rank + 1 + wid) & 7;
            const unsigned slot = OB_OFF + (unsigned)(src * 128 + b) * OB_SLOT;
            const char* const of = ws + OF_OFF + (src * 128 + b) * 128;
            const unsigned lbase = wid * LDS_WAVE;
#pragma unroll
            for (int e = 0; e < 4; ++e) {
                const unsigned idx = lane * 4 + e;
                u32x2 sel = {0u, 0u};
                unsigned rk = 0;
#pragma unroll
                for (int n = 0; n < 8; ++n) {
                    const bool neg = (idx >> (7 - n)) & 1u;
                    sel[n >> 2] |= (neg ? 0x0cu : rk) << (8 * (n & 3));
                    rk += neg ? 0u : 1u;
                }
                *(LAS u32x2*)(lds + lbase + 12288 + idx * 8) = sel;
            }
            char* outp = p.out;
            asm volatile("" : "+s"(outp));
            const unsigned ob = out_base(src, b >> 3, b & 7, lane);
            {
                bool ok = await_word<8>(ws, of, call);
                const u32x4 mk = __builtin_amdgcn_raw_buffer_load_b128(wsr, slot + lane * 16, 0, AUX_SYS);
                unsigned first[16], last[16];
                stream_layout(mk, first, last);
                u32x2 sel[16];
#pragma unroll
                for (int u = 0; u < 16; ++u) {
                    sel[u] = *(const LAS u32x2*)(lds + lbase + 12288 + sign_byte(mk, u) * 8);
                    first[u] += lbase;
                    asm volatile("" : "+v"(first[u]), "+v"(sel[u]));
                }
                const unsigned cnt = last[15];
                const __amdgpu_buffer_rsrc_t ors = rsrc_of(outp, (MM / NR) * NN);
#define TAKE_PIECES(i0_, i1_) do { u32x4 pc[(i1_) - (i0_)];                                                            \
                    _Pragma("unroll") for (int i = (i0_); i < (i1_); ++i)                                              \
                        if (i * 1024u < cnt) pc[i - (i0_)] = __builtin_amdgcn_raw_buffer_load_b128(wsr, slot + 1024 + i * PIECE_BYTES + piece_off(lane), 0, AUX_SYS); \
                    fill = ok && !__any(ld_sys(lane & 1 ? ws + TMO_OFF : of + 12) == call) ? 0u : 0x7f7f7f7fu;          \
                    _Pragma("unroll") for (int i = (i0_); i < (i1_); ++i)                                              \
                        if (i * 1024u < cnt) {                                                                         \
                            u32x4 q = pc[i - (i0_)];                                                                   \
                            unsigned h = ((q[0] & 0x80808080u) >> 7) | ((q[1] & 0x80808080u) >> 6) | ((q[2] & 0x80808080u) >> 5); \
                            const unsigned he = h | ((q[3] & 0x80808080u) >> 4);     \
                            h = (h << 4) | (unsigned)__builtin_amdgcn_update_dpp(0, (int)he, 0xa0, 0xf, 0xf, true);    \
                            q[0] &= 0x7f7f7f7fu; q[1] &= 0x7f7f7f7fu; q[2] &= 0x7f7f7f7fu;                             \
                            q[3] = lane & 1 ? h : q[3] & 0x7f7f7f7fu;                                                  \
                            *(LAS u32x4*)(lds + lbase + i * 1024 + lane * 16) = q;                                     \
                        } } while (0)
                unsigned fill;
                ok = await_word<1>(ws, of + 8, call) && ok;
                TAKE_PIECES(0, 8);
                unsigned d[16][3];
#pragma unroll
                for (int u = 0; u < 16; ++u)
#pragma unroll
                    for (int i = 0; i < 3; ++i) d[u][i] = *(const LAS unsigned*)(lds + (first[u] & ~3u) + i * 4);
                __builtin_amdgcn_sched_barrier(0);
#pragma unroll
                for (int u = 0; u < 16; ++u) {
                    const unsigned lo = __builtin_amdgcn_alignbyte(d[u][1], d[u][0], first[u]), hi = __builtin_amdgcn_alignbyte(d[u][2], d[u][1], first[u]);
                    u32x2 o;
                    o[0] = __builtin_amdgcn_perm(hi, lo, sel[u][0]) | fill;
                    o[1] = __builtin_amdgcn_perm(hi, lo, sel[u][1]) | fill;
                    __builtin_amdgcn_raw_buffer_store_b64(o, ors, ob + (unsigned)(((u >> 2) * 16 + (u & 3)) * NN), 0, AUX_SYS);
                }
            }
        }
        return;
    }

    const int g = b;
    const int pm = (g >> 4) * 2 + (g & 1);
    const int pn = ((g & 7) >> 1) * 2 + ((g >> 3) & 1);
    const int fr = lane & 15, fq = lane >> 4, wr = wid >> 1, wc = wid & 1;
    const int owner = pm >> 1, tile = (pm & 1) * 8 + pn;
    char* ows = p.ws[(owner - rank) & 7];
    char* outp = p.out;
    asm volatile("" : "+s"(ows), "+s"(outp));
    const unsigned call = __builtin_amdgcn_readfirstlane(ticket) + 1u;

    const int op = wid >> 2;
    const char* const pollp = ws + fl_word(op ? 16 + pn : pm, wid & 3, lane);
    const unsigned unit = op ? WT_T : XT_T;
    const char* const gbase = ws + (op ? WT_OFF + pn * 32768 : XT_OFF + pm * 32768) + lane * 16;
    const unsigned slot0 = chunk_pos(wid & 3, op ? pn : pm);
    LAS char* const ldst = lds + wid * 4096;
    const int aoff = fq * 4096 + (wr * 64 + fr) * 16;
    const int boff = 16384 + fq * 4096 + (wc * 128 + fr) * 16;

    int known = 0;
    int next = 0, nslot = 0;
    u32x2 pvn = {0u, 0u};
#define STAGE_NEXT() do { const char* g_ = gbase + ((slot0 ^ ((next & 1) << 14)) + (unsigned)(next >> 1) * unit);              \
        LAS char* l_ = ldst + nslot * LDS_SLOT;                                                                       \
        __builtin_amdgcn_global_load_lds((const GAS unsigned*)g_, (LAS unsigned*)l_, 16, 0, AUX_SYS);                       \
        __builtin_amdgcn_global_load_lds((const GAS unsigned*)g_, (LAS unsigned*)l_, 16, 1024, AUX_SYS);                    \
        __builtin_amdgcn_global_load_lds((const GAS unsigned*)g_, (LAS unsigned*)l_, 16, 2048, AUX_SYS);                    \
        __builtin_amdgcn_global_load_lds((const GAS unsigned*)g_, (LAS unsigned*)l_, 16, 3072, AUX_SYS);                    \
        ++next; nslot = nslot == NSLOT - 1 ? 0 : nslot + 1; } while (0)
#define POLL_ISSUE() asm volatile("global_load_dwordx2 %0, %1, off sc0 sc1" : "+v"(pvn) : "v"(pollp) : "memory")
#define POLL_TAKE() do { u32x2 f_; asm volatile("s_waitcnt vmcnt(0)\n\tv_mov_b64 %0, %1" : "=v"(f_) : "v"(pvn) : "memory");    \
        known = steps_known(f_[0], f_[1], call); } while (0)
#define AWAIT_NEXT() do { if (next >= known) {                                                                        \
            const u64 t0_ = __builtin_amdgcn_s_memrealtime(); unsigned sp_ = 0;                                       \
            for (;;) { const u64 f_ = ld_sys64(pollp); known = steps_known((unsigned)f_, (unsigned)(f_ >> 32), call); \
                if (next < known) break;                                                                              \
                if (give_up<2>(ws, call, t0_, sp_)) { known = NSTEP; break; } } } } while (0)
#define MFMA_COLS(A_, B_, n0_, n1_) do { __builtin_amdgcn_sched_barrier(0);                                           \
        _Pragma("unroll") for (int n = (n0_); n < (n1_); ++n)                                                         \
            _Pragma("unroll") for (int m = 0; m < 4; ++m)                                                             \
                acc[m][n] = __builtin_amdgcn_mfma_f32_16x16x32_bf16(A_[m], B_[n], acc[m][n], 0, 0, 0);                \
        __builtin_amdgcn_sched_barrier(0); } while (0)
#define RD_A(A_, m_) A_[m_] = *(const LAS bf16x8*)(lb_ + aoff + (m_) * 256)
#define RD_B(B_, n_) B_[n_] = *(const LAS bf16x8*)(lb_ + boff + (n_) * 256)
#define KSTEP(st_, A_, B_, AN_, BN_) do {                                                                             \
        MFMA_COLS(A_, B_, 0, 1);                                                                                      \
        if (polled) POLL_TAKE();                                          \
        const int had_ = next;                                                                                        \
        const int lim_ = min((st_) + NSLOT, known);                                   \
        if (next < lim_) { STAGE_NEXT(); if (next < lim_) STAGE_NEXT(); }                                             \
        const bool full_ = (polled ? had_ + NSLOT - 2 : next) >= (st_) + NSLOT;                                       \
        MFMA_COLS(A_, B_, 1, 4);                                                                                      \
        if (!full_) {                                                              \
            if (next == (st_) + 1 && next < NSTEP) {                                                                  \
                AWAIT_NEXT(); STAGE_NEXT();                                                                           \
                if (next < known) STAGE_NEXT();                                                                       \
            }                                                                                                         \
            const int k_ = next - (st_) - 2;                                                                          \
            if (k_ <= 0) WAIT_VM(0); else if (k_ == 1) WAIT_VM(4); else WAIT_VM(8);                                   \
        }                                                                                                             \
        WAIT_VM(12);                                                                                                  \
        LGKM0_BARRIER();                                                                                              \
        const LAS char* lb_ = lds + rslot * LDS_SLOT;               \
        RD_A(AN_, 0); RD_A(AN_, 1); RD_A(AN_, 2); RD_A(AN_, 3);                                                       \
        polled = known < min(next + 6, NSTEP);                                                                        \
        if (polled) POLL_ISSUE();                                                                                     \
        MFMA_COLS(A_, B_, 4, 5);                                                                                      \
        RD_B(BN_, 0); RD_B(BN_, 1); RD_B(BN_, 2); RD_B(BN_, 3);                                                       \
        MFMA_COLS(A_, B_, 5, 6);                                                                                      \
        RD_B(BN_, 4); RD_B(BN_, 5); RD_B(BN_, 6); RD_B(BN_, 7);                                                       \
        rslot = rslot == NSLOT - 1 ? 0 : rslot + 1;                                                                   \
        MFMA_COLS(A_, B_, 6, 8); } while (0)

    f32x4 acc[4][8];
#pragma unroll
    for (int m = 0; m < 4; ++m)
#pragma unroll
        for (int n = 0; n < 8; ++n) acc[m][n] = (f32x4){0.f, 0.f, 0.f, 0.f};

    AWAIT_NEXT(); STAGE_NEXT();
    WAIT_VM(0);
    LGKM0_BARRIER();
    int rslot = 1;
    bool polled = false;
    bf16x8 a0[4], b0[8], a1[4], b1[8];
    {
        const LAS char* lb_ = lds;
#pragma unroll
        for (int m = 0; m < 4; ++m) RD_A(a0, m);
#pragma unroll
        for (int n = 0; n < 8; ++n) RD_B(b0, n);
    }
    for (int st = 0; st < NSTEP; st += 2) {
        KSTEP(st, a0, b0, a1, b1);
        KSTEP(st + 1, a1, b1, a0, b0);
    }

    int mx = 0;
#pragma unroll
    for (int m = 0; m < 4; ++m)
#pragma unroll
        for (int n = 0; n < 8; ++n)
#pragma unroll
            for (int j = 0; j < 4; ++j) mx = max(mx, __float_as_int(acc[m][n][j]));
    mx = wave_max(mx);
    LAS int* red = (LAS int*)(lds + LDS_MISC);
    if (lane == 0) red[wid] = mx;
    LGKM0_BARRIER();
    if (wid == 0) {
        int t = red[lane & 7];
        ROW_SHR_MAX(t, 1); ROW_SHR_MAX(t, 2); ROW_SHR_MAX(t, 4);
        if (lane == 7) st_sys64(ws + TMAX_OFF + g * 128, (unsigned)t, call);
    }

    const bool remote = owner != rank;
    const __amdgpu_buffer_rsrc_t sls = rsrc_of(ows + OB_OFF + (unsigned)(rank * 128 + tile * 8 + wid) * OB_SLOT, OB_SLOT);
    char* const of = ows + OF_OFF + (rank * 128 + tile * 8 + wid) * 128;
    const int efield = min(268 - (mx >> 23), 253);
    const float sc = __int_as_float(efield << 23);
    const unsigned pos0 = (unsigned)(size_t)(lds + wid * LDS_WAVE);
    const unsigned spill = (unsigned)(size_t)(lds + LDS_MISC + 64);
    unsigned first[16], last[16];
#define VAL(u_, n_) acc[(u_) >> 2][n_][(u_) & 3]
#define PACK1(n_) "v_ashrrev_i32 %[s], 31, %[v" #n_ "]\n\tv_fma_mixlo_f16 %[h], %[v" #n_ "], %[sc], 0\n\t"              \
                  "v_bfi_b32 %[a], %[s], %[sp], %[p]\n\tv_lshl_add_u32 %[p], %[s], 1, %[p]\n\t"                      \
                  "ds_write_b16 %[a], %[h] offset:2*" #n_ "\n\t"
#define PACK8(u_) do { unsigned p_ = pos0 + 2 * first[u_], s_, h_, a_;                                                \
        asm volatile(PACK1(0) PACK1(1) PACK1(2) PACK1(3) PACK1(4) PACK1(5) PACK1(6) PACK1(7)                          \
                     : [p] "+v"(p_), [s] "=&v"(s_), [h] "=&v"(h_), [a] "=&v"(a_)                                      \
                     : [v0] "v"(VAL(u_, 0)), [v1] "v"(VAL(u_, 1)), [v2] "v"(VAL(u_, 2)), [v3] "v"(VAL(u_, 3)),        \
                       [v4] "v"(VAL(u_, 4)), [v5] "v"(VAL(u_, 5)), [v6] "v"(VAL(u_, 6)), [v7] "v"(VAL(u_, 7)),        \
                       [sc] "s"(sc), [sp] "v"(spill) : "memory"); } while (0)
    u32x4 mk = {0u, 0u, 0u, 0u};
    if (remote) {
#pragma unroll
        for (int r = 0; r < 128; ++r) mk[r >> 5] = __builtin_amdgcn_alignbit(mk[r >> 5], __float_as_uint(VAL(r >> 3, r & 7)), 31);
        if (wid != 0) __builtin_amdgcn_raw_buffer_store_b128(mk, sls, lane * 16, 0, AUX_SYS);
        stream_layout(mk, first, last);
        PACK8(0); PACK8(1); PACK8(2); PACK8(3); PACK8(4);
    } else {
#pragma unroll
        for (int m = 0; m < 4; ++m) {
#pragma unroll
            for (int n = 0; n < 8; ++n)
#pragma unroll
                for (int j = 0; j < 4; ++j) acc[m][n][j] = __int_as_float(max(0, __float_as_int(acc[m][n][j])));
            asm volatile("" : "+v"(acc[m][0]), "+v"(acc[m][1]), "+v"(acc[m][2]), "+v"(acc[m][3]),
                              "+v"(acc[m][4]), "+v"(acc[m][5]), "+v"(acc[m][6]), "+v"(acc[m][7]));
        }
    }
    if (wid == 0) {
        const char* ap = lane < NR ? ws + AMAX_OFF + ((g & 7) * NR + lane) * 128 : ws + TMO_OFF - 4;
        const u64 t0 = __builtin_amdgcn_s_memrealtime(); unsigned sp = 0;
        u64 v = 0;
        for (;;) {
            if (lane <= NR) v = ld_sys64(ap);
            if ((__ballot((int)((unsigned)(v >> 32) - call) >= 0) & 0xffu) == 0xffu) break;
            if (give_up(ws, call, t0, sp)) { v = (u64)call << 32; break; }
        }
        int t = lane < NR ? (int)(unsigned)v : ((unsigned)(v >> 32) == call ? NAN_BITS : 0);
        ROW_SHR_MAX(t, 1); ROW_SHR_MAX(t, 2); ROW_SHR_MAX(t, 4); ROW_SHR_MAX(t, 8);
        if (lane == NR) red[8] = t;
    }
    LGKM0_BARRIER();
    const float inv = 448.0f / __int_as_float(red[8]);

    if (remote) {
        const float f = inv * __int_as_float((254 - efield) << 23);
        unsigned sent = 0;
        const unsigned hshift = (lane & 1) * 4, woff = piece_off(lane);
#define SEND(cnt_, upto_) do { __builtin_amdgcn_s_waitcnt(0xC07F); asm volatile("" ::: "memory");                           \
            _Pragma("nounroll") for (; sent < (upto_); sent += 1024) {                                                                    \
                const unsigned o_ = sent + lane * 16;                                                                 \
                const f16x8 h0 = *(const LAS f16x8*)(lds + wid * LDS_WAVE + o_ * 2), h1 = *(const LAS f16x8*)(lds + wid * LDS_WAVE + o_ * 2 + 16); \
                u32x4 q;                                                                                              \
                _Pragma("unroll") for (int d = 0; d < 2; ++d) {                                                       \
                    int w0 = __builtin_amdgcn_cvt_pk_fp8_f32(__builtin_fmaf((float)h0[4 * d], f, 0.f), __builtin_fmaf((float)h0[4 * d + 1], f, 0.f), 0, false); \
                    q[d] = (unsigned)__builtin_amdgcn_cvt_pk_fp8_f32(__builtin_fmaf((float)h0[4 * d + 2], f, 0.f), __builtin_fmaf((float)h0[4 * d + 3], f, 0.f), w0, true); \
                    int w1 = __builtin_amdgcn_cvt_pk_fp8_f32(__builtin_fmaf((float)h1[4 * d], f, 0.f), __builtin_fmaf((float)h1[4 * d + 1], f, 0.f), 0, false); \
                    q[2 + d] = (unsigned)__builtin_amdgcn_cvt_pk_fp8_f32(__builtin_fmaf((float)h1[4 * d + 2], f, 0.f), __builtin_fmaf((float)h1[4 * d + 3], f, 0.f), w1, true); \
                }                                                                                                     \
                  \
                const unsigned h_ = (unsigned)__builtin_amdgcn_update_dpp(0, (int)q[3], 0xf5, 0xf, 0xf, true) >> hshift; \
                _Pragma("unroll") for (int d = 0; d < 4; ++d) q[d] |= (h_ << (7 - d)) & 0x80808080u;                  \
                WAIT_VM(2);                                                                                           \
                if ((o_ & ~31u) < (cnt_)) {                                                                             \
                    const unsigned w_ = 1024 + (sent >> 10) * PIECE_BYTES + woff;                                     \
                    if (lane & 1) __builtin_amdgcn_raw_buffer_store_b96((u32x3){q[0], q[1], q[2]}, sls, w_, 0, AUX_SYS); \
                    else __builtin_amdgcn_raw_buffer_store_b128(q, sls, w_, 0, AUX_SYS);                              \
                }                                                                                                     \
            } } while (0)
        if (wid == 0) __builtin_amdgcn_raw_buffer_store_b128(mk, sls, lane * 16, 0, AUX_SYS);
        SEND(last[4], last[4] & ~1023u);
        PACK8(5); PACK8(6); PACK8(7); PACK8(8);
        if (sent == 1024u) WAIT_VM(2); else WAIT_VM(0);
        if (lane == 0) st_sys(of, call);
        SEND(last[8], last[8] & ~1023u);
        PACK8(9); PACK8(10); PACK8(11); PACK8(12);
        SEND(last[12], last[12] & ~1023u);
        PACK8(13); PACK8(14); PACK8(15);
        SEND(last[15], last[15]);
        WAIT_VM(0);
        if (lane == 0) st_sys64(of + 8, call, red[8] == NAN_BITS ? call : 0u);
    } else {
        const __amdgpu_buffer_rsrc_t ors = rsrc_of(outp, (MM / NR) * NN);
        const unsigned ob = out_base(rank, tile, wid, lane);
#pragma unroll
        for (int m = 0; m < 4; ++m)
#pragma unroll
            for (int j = 0; j < 4; ++j) {
                u32x2 o;
                int lo = __builtin_amdgcn_cvt_pk_fp8_f32(acc[m][0][j] * inv, acc[m][1][j] * inv, 0, false);
                o[0] = (unsigned)__builtin_amdgcn_cvt_pk_fp8_f32(acc[m][2][j] * inv, acc[m][3][j] * inv, lo, true);
                int hi = __builtin_amdgcn_cvt_pk_fp8_f32(acc[m][4][j] * inv, acc[m][5][j] * inv, 0, false);
                o[1] = (unsigned)__builtin_amdgcn_cvt_pk_fp8_f32(acc[m][6][j] * inv, acc[m][7][j] * inv, hi, true);
                __builtin_amdgcn_raw_buffer_store_b64(o, ors, ob + (unsigned)((m * 16 + j) * NN), 0, AUX_SYS);
            }
    }
}

}

extern "C" void kernel_launch(int rank, int nranks, ncclComm_t comm,
                              hipStream_t stream,
                              void* const* d_in, const int* in_sizes, int n_in,
                              void* d_out, int out_size,
                              const int* in_dtypes, int out_dtype,
                              void* const* d_out_all,
                              void* const* d_ws_all, size_t ws_bytes) {
    (void)comm; (void)in_dtypes; (void)out_dtype; (void)d_out_all;
    if (nranks != NR || n_in < 2 || (unsigned)rank >= (unsigned)NR || ws_bytes < WS_NEED ||
        in_sizes[0] != MM * KSH || in_sizes[1] != KSH * NN || out_size != (MM / NR) * NN) {
        fprintf(stderr, "kernel_launch: unexpected problem shape\n");
        return;
    }
    Params p;
    p.x = (const char*)d_in[0];
    p.w = (const char*)d_in[1];
    for (int i = 0; i < NR; ++i) p.ws[i] = (char*)d_ws_all[(rank + i) % NR];
    p.out = (char*)d_out;
    p.rank = rank; p.pad = 0;
    hipLaunchKernelGGL(fused_gemm_relu_q8, dim3(GRID), dim3(BLOCK), 0, stream, p);
    HIP_CHECK(hipGetLastError());
}
```

```cpp
#include <hip/hip_runtime.h>
#include <rccl/rccl.h>
#include <cstdio>
#include <cstdint>

#define HIP_CHECK(expr) do { hipError_t _e = (expr); if (_e != hipSuccess) { \
    fprintf(stderr, "HIP error %s at %s:%d\n", hipGetErrorString(_e), __FILE__, __LINE__); \
    return; } } while (0)

namespace {

typedef short bf16x8 __attribute__((ext_vector_type(8)));
typedef _Float16 f16x8 __attribute__((ext_vector_type(8)));
typedef float f32x4 __attribute__((ext_vector_type(4)));
typedef unsigned u32x4 __attribute__((ext_vector_type(4)));
typedef unsigned u32x3 __attribute__((ext_vector_type(3)));
typedef unsigned u32x2 __attribute__((ext_vector_type(2)));
typedef unsigned long long u64;
#define LAS __attribute__((address_space(3)))
#define GAS __attribute__((address_space(1)))

constexpr int NR = 8;
constexpr int MM = 4096, KSH = 512, NN = 16384, NCOL = NN / NR;
constexpr int NT = KSH / 8;
constexpr int NBLK = 24;
constexpr int PULL_PER_LINK = 18;
constexpr int PACK_DELAY = 100;
constexpr int Q_JOBS = NT * NBLK;
constexpr int Q_TICKETS = Q_JOBS + PULL_PER_LINK;
constexpr int N_COMM = 128;
constexpr int N_PULL = (NR - 1) * PULL_PER_LINK;
constexpr int N_GEMM = 128;
constexpr int GRID = N_COMM + N_GEMM;
constexpr int BLOCK = 512;
constexpr int N_PACKW = N_COMM * 6;
constexpr int NGRP = NT / 8;
static_assert(N_PULL <= 2 * N_COMM, "pull waves are waves 0-1 of the comm workgroups");

__host__ __device__ constexpr int job_block(int k) { return k % 3 == 2 ? 16 + k / 3 : k / 3 * 2 + k % 3; }
__host__ __device__ constexpr unsigned chunk_pos(unsigned src, unsigned blk) { return ((src + blk) & 7u) * 4096u; }

constexpr unsigned XT_T = NR * MM * 16, WT_T = NR * NCOL * 16;
constexpr unsigned XT_OFF = 0;
constexpr unsigned WT_OFF = XT_OFF + NT * XT_T;
constexpr unsigned WS_OFF = WT_OFF + NT * WT_T;
constexpr unsigned OB_SLOT = 1024 + 8192;
constexpr unsigned OB_OFF = WS_OFF + NT * WT_T;
constexpr unsigned CTL_OFF = OB_OFF + NR * 128 * OB_SLOT;
constexpr unsigned PK_OFF = CTL_OFF;
constexpr unsigned FL_OFF = PK_OFF + NR * NGRP * 8192;
constexpr unsigned Q_OFF = FL_OFF + NBLK * NR * NT * 4;
constexpr unsigned TMAX_OFF = Q_OFF + NR * 128;
constexpr unsigned AMAX_OFF = TMAX_OFF + N_GEMM * 128;
constexpr unsigned OF_OFF = AMAX_OFF + 64 * 128;
constexpr unsigned CTR_OFF = OF_OFF + NR * 128 * 128;
constexpr unsigned TMO_OFF = CTR_OFF + GRID * 128 + 4;
constexpr unsigned DBG_OFF = CTR_OFF + GRID * 128 + 128;
constexpr size_t WS_NEED = DBG_OFF + 4096;

constexpr u64 DEADLINE = 50000000ull;
constexpr int AUX_SYS = 17;

constexpr int NSTEP = 2 * NT;
constexpr int NSLOT = 5;
constexpr int LDS_SLOT = 32768;
constexpr int LDS_BYTES = NSLOT * LDS_SLOT;
constexpr int LDS_WAVE = 16384;
constexpr int LDS_MISC = 8 * LDS_WAVE;

struct Params {
    const char* x; const char* w;
    int rank, pad;
    char* ws[NR];
    char* out;
};

__device__ __forceinline__ __amdgpu_buffer_rsrc_t rsrc_of(const void* p, unsigned bytes) {
    return __builtin_amdgcn_make_buffer_rsrc(const_cast<void*>(p), 0, bytes, 0x00020000);
}
__device__ __forceinline__ unsigned ld_sys(const void* p) {
    return __hip_atomic_load((const GAS unsigned*)p, __ATOMIC_RELAXED, __HIP_MEMORY_SCOPE_SYSTEM);
}
__device__ __forceinline__ u64 ld_sys64(const void* p) {
    return __hip_atomic_load((const GAS u64*)p, __ATOMIC_RELAXED, __HIP_MEMORY_SCOPE_SYSTEM);
}
__device__ __forceinline__ void st_sys(void* p, unsigned v) {
    __hip_atomic_store((GAS unsigned*)p, v, __ATOMIC_RELAXED, __HIP_MEMORY_SCOPE_SYSTEM);
}
__device__ __forceinline__ void st_sys64(void* p, unsigned lo, unsigned hi) {
    __hip_atomic_store((GAS u64*)p, ((u64)hi << 32) | lo, __ATOMIC_RELAXED, __HIP_MEMORY_SCOPE_SYSTEM);
}
__device__ __forceinline__ unsigned ld_agent(const void* p) {
    return __hip_atomic_load((const GAS unsigned*)p, __ATOMIC_RELAXED, __HIP_MEMORY_SCOPE_AGENT);
}
__device__ __forceinline__ unsigned add_agent_lane0(void* p, int lane) {
    unsigned v = 0, z = 0;
    asm volatile("" : "+v"(z));
    if (lane == 0) v = __hip_atomic_fetch_add((GAS unsigned*)((char*)p + z), 1u, __ATOMIC_RELAXED, __HIP_MEMORY_SCOPE_AGENT);
    return v;
}
__device__ __forceinline__ u32x4 transposed(const u32x4 (&v)[8], int i) {
    u32x4 o;
#pragma unroll
    for (int d = 0; d < 4; ++d)
        o[d] = __builtin_amdgcn_perm(v[2 * d + 1][i >> 1], v[2 * d][i >> 1], (i & 1) ? 0x07060302u : 0x05040100u);
    return o;
}
constexpr int NAN_BITS = 0x7fc00000;
#define ROW_SHR_MAX(v, n) v = max(v, __builtin_amdgcn_update_dpp(0, v, 0x110 + (n), 0xf, 0xf, true))
__device__ __forceinline__ int wave_max(int v) {
    ROW_SHR_MAX(v, 1); ROW_SHR_MAX(v, 2); ROW_SHR_MAX(v, 4); ROW_SHR_MAX(v, 8);
    return max(max(__builtin_amdgcn_readlane(v, 15), __builtin_amdgcn_readlane(v, 31)),
               max(__builtin_amdgcn_readlane(v, 47), __builtin_amdgcn_readlane(v, 63)));
}
__device__ __forceinline__ unsigned sign_byte(const u32x4& mk, int u) { return (mk[u >> 2] >> (24 - 8 * (u & 3))) & 0xffu; }
__device__ __forceinline__ void stream_layout(const u32x4& mk, unsigned (&first)[16], unsigned (&last)[16]) {
    unsigned base = 0;
#pragma unroll
    for (int u = 0; u < 16; u += 2) {
        const int c = 0x00080008 - (int)(__builtin_popcount(sign_byte(mk, u)) | (__builtin_popcount(sign_byte(mk, u + 1)) << 16));
        int v = c;
        v += __builtin_amdgcn_update_dpp(0, v, 0x111, 0xf, 0xf, true);
        v += __builtin_amdgcn_update_dpp(0, v, 0x112, 0xf, 0xf, true);
        v += __builtin_amdgcn_update_dpp(0, v, 0x114, 0xf, 0xf, true);
        v += __builtin_amdgcn_update_dpp(0, v, 0x118, 0xf, 0xf, true);
        v += __builtin_amdgcn_update_dpp(0, v, 0x142, 0xa, 0xf, false);
        v += __builtin_amdgcn_update_dpp(0, v, 0x143, 0xc, 0xf, false);
        const unsigned tot = (unsigned)__builtin_amdgcn_readlane(v, 63), exc = (unsigned)(v - c);
        first[u] = base + (exc & 0xffffu); base += tot & 0xffffu; last[u] = base;
        first[u + 1] = base + (exc >> 16); base += tot >> 16; last[u + 1] = base;
    }
}
template <int SLEEP = 1>
__device__ __forceinline__ bool give_up(char* ws, unsigned call, u64 t0, unsigned& spins) {
    bool stop = __builtin_amdgcn_s_memrealtime() - t0 > DEADLINE;
    if ((++spins & 63u) == 0u) stop = stop || __builtin_amdgcn_readfirstlane(ld_agent(ws + TMO_OFF)) == call;
    if (stop) {
        __hip_atomic_store((GAS unsigned*)(ws + TMO_OFF), call, __ATOMIC_RELAXED, __HIP_MEMORY_SCOPE_AGENT);
        return true;
    }
    __builtin_amdgcn_s_sleep(SLEEP);
    return false;
}
template <int SLEEP>
__device__ __forceinline__ bool await_word(char* ws, const char* p, unsigned call) {
    const u64 t0 = __builtin_amdgcn_s_memrealtime(); unsigned sp = 0;
    while ((int)(__builtin_amdgcn_readfirstlane(ld_sys(p)) - call) < 0)
        if (give_up<SLEEP>(ws, call, t0, sp)) return false;
    return true;
}
__device__ __forceinline__ unsigned fl_word(int blk, int src, int t) { return FL_OFF + (unsigned)(((blk * 4 + (src & 3)) * NT + t) * 2 + (src >> 2)) * 4u; }
__device__ __forceinline__ int steps_known(unsigned f0, unsigned f1, unsigned call) {
    const u64 lo = __ballot((int)(f0 - call) >= 0), hi = __ballot((int)(f1 - call) >= 0);
    const u64 miss = ~(lo & hi);
    if (miss == 0) return NSTEP;
    const int c = __builtin_ctzll(miss);
    return 2 * c + (int)((lo >> c) & 1);
}
__device__ __forceinline__ unsigned out_base(int src, int tile, int wave, int lane) {
    return (unsigned)(((tile >> 3) * 256 + (wave >> 1) * 64 + (lane >> 4) * 4) * NN + src * NCOL + (tile & 7) * 256 + (wave & 1) * 128 + (lane & 15) * 8);
}
constexpr unsigned PIECE_BYTES = 896;
__device__ __forceinline__ unsigned piece_off(int lane) { return lane & 1 ? 512 + (lane >> 1) * 12 : (lane >> 1) * 16; }
#define WAIT_VM(n) asm volatile("s_waitcnt vmcnt(" #n ")" ::: "memory")
#define LGKM0_BARRIER() do { __builtin_amdgcn_s_waitcnt(0xC07F); asm volatile("" ::: "memory");                       \
        __builtin_amdgcn_s_barrier(); asm volatile("" ::: "memory"); } while (0)

__global__ void __launch_bounds__(BLOCK, 2) fused_gemm_relu_q8(const Params p) {
    __shared__ __attribute__((aligned(1024))) char smem[LDS_BYTES];
    LAS char* const lds = (LAS char*)smem;
    const int hb = blockIdx.x;
    const int lane = threadIdx.x & 63;
    const int wid = __builtin_amdgcn_readfirstlane(threadIdx.x >> 6);
    int rank = p.rank;
    char* ws = p.ws[0];
    const char* px = p.x;
    const char* pw = p.w;
    const u64 t_start = __builtin_amdgcn_s_memrealtime();
    asm volatile("" : "+s"(rank), "+s"(ws), "+s"(px), "+s"(pw));
    const unsigned ticket = add_agent_lane0(ws + CTR_OFF + (size_t)hb * 128 + wid * 4, lane);
    const int b = hb & (N_COMM - 1);

    if (hb < N_COMM) {
        const __amdgpu_buffer_rsrc_t wsr = rsrc_of(ws, CTL_OFF);
        const int cw = wid * N_COMM + b;
        u64 all_t = 0, ev_t = 0;
        if (wid < 2) {
            if (threadIdx.x == 0) *(LAS unsigned*)(lds + LDS_MISC) = 0u;
            LGKM0_BARRIER();
            if (cw == 255) {
                const unsigned call = __builtin_amdgcn_readfirstlane(ticket) + 1u;
                if (lane < NR) st_sys(p.ws[lane & 7] + DBG_OFF + rank * 128, call);
                const u64 t0 = __builtin_amdgcn_s_memrealtime(); unsigned sp = 0;
                for (;;) { const unsigned v = ld_sys(ws + DBG_OFF + (lane & 7) * 128); if (__all((int)(v - call) >= 0)) break; if (give_up(ws, call, t0, sp)) break; }
                all_t = __builtin_amdgcn_s_memrealtime();
                for (;;) { const unsigned v = ld_sys(ws + fl_word((lane >> 3) * 3, lane & 7, 2)); if (__all((int)(v - call) >= 0)) break; if (give_up(ws, call, t0, sp)) break; }
                ev_t = __builtin_amdgcn_s_memrealtime();
            }
            if (cw < N_PULL) {
                const unsigned s = (unsigned)(rank + 1 + cw % (NR - 1)) & 7u;
                const __amdgpu_buffer_rsrc_t srs = rsrc_of(p.ws[1 + cw % (NR - 1)], CTL_OFF);
                char* const queue = ws + Q_OFF + s * 128;
                unsigned tk = add_agent_lane0(queue, lane);
                u32x4 cur[4];
                const unsigned call = __builtin_amdgcn_readfirstlane(ticket) + 1u;
                const unsigned qbase = (call - 1u) * Q_TICKETS;
                const char* const pk = ws + PK_OFF + s * NGRP * 8192 + lane * 128;
                unsigned pv = call - 1u;
                int pgrp = 0;
#define JOB_OF(n_, t_, j_, so_, do_) const int t_ = (int)(n_) / NBLK, j_ = job_block((int)(n_) % NBLK);                           \
                const unsigned do_ = (j_ < 16 ? XT_OFF + t_ * XT_T + j_ * 32768 : WT_OFF + t_ * WT_T + (j_ - 16) * 32768) + chunk_pos(s, j_) + lane * 16; \
                const unsigned so_ = j_ < 16 ? do_ : WS_OFF + t_ * WT_T + rank * 32768 + (j_ - 16) * 4096 + lane * 16
#define WAIT_PACKED(t_, j_) do { const int sl_ = (j_) < 16 ? (j_) : 24 + ((t_) & 7) * 4 + ((j_) >> 1);                 \
                if (((t_) >> 3) != pgrp) { pgrp = (t_) >> 3; pv = call - 1u; }                                         \
                if ((int)(__builtin_amdgcn_readlane(pv, sl_) - call) < 0) {                                            \
                    pv = ld_sys(pk + pgrp * 8192);                 \
                    const u64 t0_ = __builtin_amdgcn_s_memrealtime(); unsigned sp_ = 0;                                \
                    while ((int)(__builtin_amdgcn_readlane(pv, sl_) - call) < 0) {                                     \
                        if (give_up(ws, call, t0_, sp_)) break;                                                        \
                        if (lane == sl_) pv = ld_sys(pk + pgrp * 8192); } } } while (0)
                unsigned n = __builtin_amdgcn_readfirstlane(tk) - qbase;
                if (n < (unsigned)Q_JOBS) {
                    JOB_OF(n, t, j, so, dof);
                    WAIT_PACKED(t, j);
                    tk = add_agent_lane0(queue, lane);
#pragma unroll
                    for (int i = 0; i < 4; ++i) cur[i] = __builtin_amdgcn_raw_buffer_load_b128(srs, so + i * 1024, 0, AUX_SYS);
                    unsigned dcur = dof, fcur = fl_word(j, s, t);
                    for (;;) {
#pragma unroll
                        for (int i = 0; i < 4; ++i) __builtin_amdgcn_raw_buffer_store_b128(cur[i], wsr, dcur + i * 1024, 0, AUX_SYS);
                        asm volatile("" ::: "memory");
                        n = __builtin_amdgcn_readfirstlane(tk) - qbase;
                        char* const flag = ws + fcur;
                        if (n >= (unsigned)Q_JOBS) {
                            WAIT_VM(0);
                            if (lane == 0) st_sys(flag, call);
                            break;
                        }
                        JOB_OF(n, t2, j2, so2, dof2);
                        WAIT_PACKED(t2, j2);
                        tk = add_agent_lane0(queue, lane);
#pragma unroll
                        for (int i = 0; i < 4; ++i) cur[i] = __builtin_amdgcn_raw_buffer_load_b128(srs, so2 + i * 1024, 0, AUX_SYS);
                        dcur = dof2; fcur = fl_word(j2, s, t2);
                        WAIT_VM(5);
                        if (lane == 0) st_sys(flag, call);
                    }
                }
            }
        } else {
            const __amdgpu_buffer_rsrc_t xr = rsrc_of(px, MM * KSH * 2);
            const __amdgpu_buffer_rsrc_t wr_ = rsrc_of(pw, KSH * NN * 2);
            bool met = false;
            bool x_todo = wid >= 4;
            for (int n = (wid - 2) * N_COMM + b; n < NT * 32 || x_todo;) {
                const bool do_x = x_todo && ((b >> 4) <= (n >> 8) || n >= NT * 32);
                const int g = do_x ? b >> 4 : n >> 8;
                const int delay = (do_x ? g * 8 : n >> 5) * PACK_DELAY;
                if (!met && delay > 0) { __builtin_amdgcn_s_barrier(); met = true; }
                while (__builtin_amdgcn_s_memrealtime() - t_start < (u64)delay) __builtin_amdgcn_s_sleep(2);
                u32x4 v[8];
                if (do_x) {
                    const unsigned rb = (wid - 4) * 64 + (lane & 7), piece = lane >> 3;
#pragma unroll
                    for (int i = 0; i < 8; ++i) v[i] = __builtin_amdgcn_raw_buffer_load_b128(xr, ((b & 15) * 256 + rb + i * 8) * (KSH * 2) + g * 128 + piece * 16, 0, 0);
                    if (!met) { __builtin_amdgcn_s_barrier(); met = true; }
                    const unsigned d0 = XT_OFF + (g * 8 + piece) * XT_T + (b & 15) * 32768 + chunk_pos(rank, b & 15) + rb * 16;
#pragma unroll
                    for (int i = 0; i < 8; ++i) __builtin_amdgcn_raw_buffer_store_b128(v[i], wsr, d0 + i * 128, 0, AUX_SYS);
                    char* fl = lane < 8 ? p.ws[lane & 7] + PK_OFF + (rank * NGRP + g) * 8192 + (b & 15) * 128 : ws + fl_word(b & 15, rank, g * 8 + (lane & 7));
                    asm volatile("s_waitcnt vmcnt(0)" : "+v"(fl) :: "memory");
                    unsigned old = 0;
                    if (lane == 0) old = __hip_atomic_fetch_add((LAS unsigned*)(lds + LDS_MISC), 1u, __ATOMIC_RELAXED, __HIP_MEMORY_SCOPE_WORKGROUP);
                    const unsigned call = __builtin_amdgcn_readfirstlane(ticket) + 1u;
                    if (__builtin_amdgcn_readfirstlane(old) == 3u && lane >= 1 && lane < 16) st_sys(fl, call);
                    x_todo = false;
                } else {
                    const int t = n >> 5, cb = n & 31;
                    const unsigned dest = (unsigned)cb >> 2;
#pragma unroll
                    for (int i = 0; i < 8; ++i) v[i] = __builtin_amdgcn_raw_buffer_load_b128(wr_, (unsigned)((t * 8 + i) * (NN * 2) + cb * 1024 + lane * 16), 0, 0);
                    if (!met) { __builtin_amdgcn_s_barrier(); met = true; }
                    const unsigned d0 = (dest == (unsigned)rank ? WT_OFF + t * WT_T + ((cb & 3) * 2 + (lane >> 5)) * 32768 + chunk_pos(rank, (cb & 3) * 2 + (lane >> 5))
                                                                : WS_OFF + t * WT_T + dest * 32768 + ((cb & 3) * 2 + (lane >> 5)) * 4096)
                                      + (((lane >> 4) & 1) * 128 + (lane & 15)) * 16;
#pragma unroll
                    for (int i = 0; i < 8; ++i) __builtin_amdgcn_raw_buffer_store_b128(transposed(v, i), wsr, d0 + i * 256, 0, AUX_SYS);
                    char* const dws = p.ws[(dest - rank) & 7u];
                    WAIT_VM(0);
                    const unsigned call = __builtin_amdgcn_readfirstlane(ticket) + 1u;
                    if (dest == (unsigned)rank) { if (lane < 2) st_sys(ws + fl_word(16 + (cb & 3) * 2 + lane, rank, t), call); }
                    else if (lane == 0) st_sys(dws + PK_OFF + (rank * NGRP + g) * 8192 + (32 + (t & 7) * 4 + (cb & 3)) * 128, call);
                    n += N_PACKW;
                }
            }
        }
        const unsigned call = __builtin_amdgcn_readfirstlane(ticket) + 1u;
        if (wid == 7) {
            if (b == N_COMM - 1) {
                char* amax_out = p.ws[lane & 7] + AMAX_OFF + ((lane >> 3) * NR + rank) * 128;
                asm volatile("" : "+v"(amax_out));
                const char* tm = ws + TMAX_OFF + lane * 256;
                const u64 t0 = __builtin_amdgcn_s_memrealtime(); unsigned sp = 0;
                int mx;
                for (;;) {
                    const u64 v0 = ld_sys64(tm), v1 = ld_sys64(tm + 128);
                    mx = max((int)(unsigned)v0, (int)(unsigned)v1);
                    if (__all((int)((unsigned)(v0 >> 32) - call) >= 0 && (int)((unsigned)(v1 >> 32) - call) >= 0)) break;
                    if (give_up(ws, call, t0, sp)) { mx = NAN_BITS; break; }
                }
                st_sys64(amax_out, (unsigned)wave_max(mx), call);
            }
            return;
        }
        {
            const int src = (rank + 1 + wid) & 7;
            const unsigned slot = OB_OFF + (unsigned)(src * 128 + b) * OB_SLOT;
            const char* const of = ws + OF_OFF + (src * 128 + b) * 128;
            const unsigned lbase = wid * LDS_WAVE;
#pragma unroll
            for (int e = 0; e < 4; ++e) {
                const unsigned idx = lane * 4 + e;
                u32x2 sel = {0u, 0u};
                unsigned rk = 0;
#pragma unroll
                for (int n = 0; n < 8; ++n) {
                    const bool neg = (idx >> (7 - n)) & 1u;
                    sel[n >> 2] |= (neg ? 0x0cu : rk) << (8 * (n & 3));
                    rk += neg ? 0u : 1u;
                }
                *(LAS u32x2*)(lds + lbase + 12288 + idx * 8) = sel;
            }
            char* outp = p.out;
            asm volatile("" : "+s"(outp));
            const unsigned ob = out_base(src, b >> 3, b & 7, lane);
            {
                bool ok = await_word<8>(ws, of, call);
                const u32x4 mk = __builtin_amdgcn_raw_buffer_load_b128(wsr, slot + lane * 16, 0, AUX_SYS);
                unsigned first[16], last[16];
                stream_layout(mk, first, last);
                u32x2 sel[16];
#pragma unroll
                for (int u = 0; u < 16; ++u) {
                    sel[u] = *(const LAS u32x2*)(lds + lbase + 12288 + sign_byte(mk, u) * 8);
                    first[u] += lbase;
                    asm volatile("" : "+v"(first[u]), "+v"(sel[u]));
                }
                const unsigned cnt = last[15];
                const __amdgpu_buffer_rsrc_t ors = rsrc_of(outp, (MM / NR) * NN);
#define TAKE_PIECES(i0_, i1_) do { u32x4 pc[(i1_) - (i0_)];                                                            \
                    _Pragma("unroll") for (int i = (i0_); i < (i1_); ++i)                                              \
                        if (i * 1024u < cnt) pc[i - (i0_)] = __builtin_amdgcn_raw_buffer_load_b128(wsr, slot + 1024 + i * PIECE_BYTES + piece_off(lane), 0, AUX_SYS); \
                    fill = ok && !__any(ld_sys(lane & 1 ? ws + TMO_OFF : of + 12) == call) ? 0u : 0x7f7f7f7fu;          \
                    _Pragma("unroll") for (int i = (i0_); i < (i1_); ++i)                                              \
                        if (i * 1024u < cnt) {                                                                         \
                            u32x4 q = pc[i - (i0_)];                                                                   \
                            unsigned h = ((q[0] & 0x80808080u) >> 7) | ((q[1] & 0x80808080u) >> 6) | ((q[2] & 0x80808080u) >> 5); \
                            const unsigned he = h | ((q[3] & 0x80808080u) >> 4);     \
                            h = (h << 4) | (unsigned)__builtin_amdgcn_update_dpp(0, (int)he, 0xa0, 0xf, 0xf, true);    \
                            q[0] &= 0x7f7f7f7fu; q[1] &= 0x7f7f7f7fu; q[2] &= 0x7f7f7f7fu;                             \
                            q[3] = lane & 1 ? h : q[3] & 0x7f7f7f7fu;                                                  \
                            *(LAS u32x4*)(lds + lbase + i * 1024 + lane * 16) = q;                                     \
                        } } while (0)
                unsigned fill;
                ok = await_word<1>(ws, of + 8, call) && ok;
                TAKE_PIECES(0, 8);
                unsigned d[16][3];
#pragma unroll
                for (int u = 0; u < 16; ++u)
#pragma unroll
                    for (int i = 0; i < 3; ++i) d[u][i] = *(const LAS unsigned*)(lds + (first[u] & ~3u) + i * 4);
                __builtin_amdgcn_sched_barrier(0);
#pragma unroll
                for (int u = 0; u < 16; ++u) {
                    const unsigned lo = __builtin_amdgcn_alignbyte(d[u][1], d[u][0], first[u]), hi = __builtin_amdgcn_alignbyte(d[u][2], d[u][1], first[u]);
                    u32x2 o;
                    o[0] = __builtin_amdgcn_perm(hi, lo, sel[u][0]) | fill;
                    o[1] = __builtin_amdgcn_perm(hi, lo, sel[u][1]) | fill;
                    __builtin_amdgcn_raw_buffer_store_b64(o, ors, ob + (unsigned)(((u >> 2) * 16 + (u & 3)) * NN), 0, AUX_SYS);
                }
            }
            if (cw == 255) { long long c = (long long)(ev_t - all_t) - 200; if (c < 0) c = 0; if (c > 5000) c = 5000;
                while (__builtin_amdgcn_s_memrealtime() - all_t < (u64)(13500 + 10 * c)) __builtin_amdgcn_s_sleep(8); }
        }
        return;
    }

    const int g = b;
    const int pm = (g >> 4) * 2 + (g & 1);
    const int pn = ((g & 7) >> 1) * 2 + ((g >> 3) & 1);
    const int fr = lane & 15, fq = lane >> 4, wr = wid >> 1, wc = wid & 1;
    const int owner = pm >> 1, tile = (pm & 1) * 8 + pn;
    char* ows = p.ws[(owner - rank) & 7];
    char* outp = p.out;
    asm volatile("" : "+s"(ows), "+s"(outp));
    const unsigned call = __builtin_amdgcn_readfirstlane(ticket) + 1u;

    const int op = wid >> 2;
    const char* const pollp = ws + fl_word(op ? 16 + pn : pm, wid & 3, lane);
    const unsigned unit = op ? WT_T : XT_T;
    const char* const gbase = ws + (op ? WT_OFF + pn * 32768 : XT_OFF + pm * 32768) + lane * 16;
    const unsigned slot0 = chunk_pos(wid & 3, op ? pn : pm);
    LAS char* const ldst = lds + wid * 4096;
    const int aoff = fq * 4096 + (wr * 64 + fr) * 16;
    const int boff = 16384 + fq * 4096 + (wc * 128 + fr) * 16;

    int known = 0;
    int next = 0, nslot = 0;
    u32x2 pvn = {0u, 0u};
#define STAGE_NEXT() do { const char* g_ = gbase + ((slot0 ^ ((next & 1) << 14)) + (unsigned)(next >> 1) * unit);              \
        LAS char* l_ = ldst + nslot * LDS_SLOT;                                                                       \
        __builtin_amdgcn_global_load_lds((const GAS unsigned*)g_, (LAS unsigned*)l_, 16, 0, AUX_SYS);                       \
        __builtin_amdgcn_global_load_lds((const GAS unsigned*)g_, (LAS unsigned*)l_, 16, 1024, AUX_SYS);                    \
        __builtin_amdgcn_global_load_lds((const GAS unsigned*)g_, (LAS unsigned*)l_, 16, 2048, AUX_SYS);                    \
        __builtin_amdgcn_global_load_lds((const GAS unsigned*)g_, (LAS unsigned*)l_, 16, 3072, AUX_SYS);                    \
        ++next; nslot = nslot == NSLOT - 1 ? 0 : nslot + 1; } while (0)
#define POLL_ISSUE() asm volatile("global_load_dwordx2 %0, %1, off sc0 sc1" : "+v"(pvn) : "v"(pollp) : "memory")
#define POLL_TAKE() do { u32x2 f_; asm volatile("s_waitcnt vmcnt(0)\n\tv_mov_b64 %0, %1" : "=v"(f_) : "v"(pvn) : "memory");    \
        known = steps_known(f_[0], f_[1], call); } while (0)
#define AWAIT_NEXT() do { if (next >= known) {                                                                        \
            const u64 t0_ = __builtin_amdgcn_s_memrealtime(); unsigned sp_ = 0;                                       \
            for (;;) { const u64 f_ = ld_sys64(pollp); known = steps_known((unsigned)f_, (unsigned)(f_ >> 32), call); \
                if (next < known) break;                                                                              \
                if (give_up<2>(ws, call, t0_, sp_)) { known = NSTEP; break; } } } } while (0)
#define MFMA_COLS(A_, B_, n0_, n1_) do { __builtin_amdgcn_sched_barrier(0);                                           \
        _Pragma("unroll") for (int n = (n0_); n < (n1_); ++n)                                                         \
            _Pragma("unroll") for (int m = 0; m < 4; ++m)                                                             \
                acc[m][n] = __builtin_amdgcn_mfma_f32_16x16x32_bf16(A_[m], B_[n], acc[m][n], 0, 0, 0);                \
        __builtin_amdgcn_sched_barrier(0); } while (0)
#define RD_A(A_, m_) A_[m_] = *(const LAS bf16x8*)(lb_ + aoff + (m_) * 256)
#define RD_B(B_, n_) B_[n_] = *(const LAS bf16x8*)(lb_ + boff + (n_) * 256)
#define KSTEP(st_, A_, B_, AN_, BN_) do {                                                                             \
        MFMA_COLS(A_, B_, 0, 1);                                                                                      \
        if (polled) POLL_TAKE();                                          \
        const int had_ = next;                                                                                        \
        const int lim_ = min((st_) + NSLOT, known);                                   \
        if (next < lim_) { STAGE_NEXT(); if (next < lim_) STAGE_NEXT(); }                                             \
        const bool full_ = (polled ? had_ + NSLOT - 2 : next) >= (st_) + NSLOT;                                       \
        MFMA_COLS(A_, B_, 1, 4);                                                                                      \
        if (!full_) {                                                              \
            if (next == (st_) + 1 && next < NSTEP) {                                                                  \
                AWAIT_NEXT(); STAGE_NEXT();                                                                           \
                if (next < known) STAGE_NEXT();                                                                       \
            }                                                                                                         \
            const int k_ = next - (st_) - 2;                                                                          \
            if (k_ <= 0) WAIT_VM(0); else if (k_ == 1) WAIT_VM(4); else WAIT_VM(8);                                   \
        }                                                                                                             \
        WAIT_VM(12);                                                                                                  \
        LGKM0_BARRIER();                                                                                              \
        const LAS char* lb_ = lds + rslot * LDS_SLOT;               \
        RD_A(AN_, 0); RD_A(AN_, 1); RD_A(AN_, 2); RD_A(AN_, 3);                                                       \
        polled = known < min(next + 6, NSTEP);                                                                        \
        if (polled) POLL_ISSUE();                                                                                     \
        MFMA_COLS(A_, B_, 4, 5);                                                                                      \
        RD_B(BN_, 0); RD_B(BN_, 1); RD_B(BN_, 2); RD_B(BN_, 3);                                                       \
        MFMA_COLS(A_, B_, 5, 6);                                                                                      \
        RD_B(BN_, 4); RD_B(BN_, 5); RD_B(BN_, 6); RD_B(BN_, 7);                                                       \
        rslot = rslot == NSLOT - 1 ? 0 : rslot + 1;                                                                   \
        MFMA_COLS(A_, B_, 6, 8); } while (0)

    f32x4 acc[4][8];
#pragma unroll
    for (int m = 0; m < 4; ++m)
#pragma unroll
        for (int n = 0; n < 8; ++n) acc[m][n] = (f32x4){0.f, 0.f, 0.f, 0.f};

    AWAIT_NEXT(); STAGE_NEXT();
    WAIT_VM(0);
    LGKM0_BARRIER();
    int rslot = 1;
    bool polled = false;
    bf16x8 a0[4], b0[8], a1[4], b1[8];
    {
        const LAS char* lb_ = lds;
#pragma unroll
        for (int m = 0; m < 4; ++m) RD_A(a0, m);
#pragma unroll
        for (int n = 0; n < 8; ++n) RD_B(b0, n);
    }
    for (int st = 0; st < NSTEP; st += 2) {
        KSTEP(st, a0, b0, a1, b1);
        KSTEP(st + 1, a1, b1, a0, b0);
    }

    int mx = 0;
#pragma unroll
    for (int m = 0; m < 4; ++m)
#pragma unroll
        for (int n = 0; n < 8; ++n)
#pragma unroll
            for (int j = 0; j < 4; ++j) mx = max(mx, __float_as_int(acc[m][n][j]));
    mx = wave_max(mx);
    LAS int* red = (LAS int*)(lds + LDS_MISC);
    if (lane == 0) red[wid] = mx;
    LGKM0_BARRIER();
    if (wid == 0) {
        int t = red[lane & 7];
        ROW_SHR_MAX(t, 1); ROW_SHR_MAX(t, 2); ROW_SHR_MAX(t, 4);
        if (lane == 7) st_sys64(ws + TMAX_OFF + g * 128, (unsigned)t, call);
    }

    const bool remote = owner != rank;
    const __amdgpu_buffer_rsrc_t sls = rsrc_of(ows + OB_OFF + (unsigned)(rank * 128 + tile * 8 + wid) * OB_SLOT, OB_SLOT);
    char* const of = ows + OF_OFF + (rank * 128 + tile * 8 + wid) * 128;
    const int efield = min(268 - (mx >> 23), 253);
    const float sc = __int_as_float(efield << 23);
    const unsigned pos0 = (unsigned)(size_t)(lds + wid * LDS_WAVE);
    const unsigned spill = (unsigned)(size_t)(lds + LDS_MISC + 64);
    unsigned first[16], last[16];
#define VAL(u_, n_) acc[(u_) >> 2][n_][(u_) & 3]
#define PACK1(n_) "v_ashrrev_i32 %[s], 31, %[v" #n_ "]\n\tv_fma_mixlo_f16 %[h], %[v" #n_ "], %[sc], 0\n\t"              \
                  "v_bfi_b32 %[a], %[s], %[sp], %[p]\n\tv_lshl_add_u32 %[p], %[s], 1, %[p]\n\t"                      \
                  "ds_write_b16 %[a], %[h] offset:2*" #n_ "\n\t"
#define PACK8(u_) do { unsigned p_ = pos0 + 2 * first[u_], s_, h_, a_;                                                \
        asm volatile(PACK1(0) PACK1(1) PACK1(2) PACK1(3) PACK1(4) PACK1(5) PACK1(6) PACK1(7)                          \
                     : [p] "+v"(p_), [s] "=&v"(s_), [h] "=&v"(h_), [a] "=&v"(a_)                                      \
                     : [v0] "v"(VAL(u_, 0)), [v1] "v"(VAL(u_, 1)), [v2] "v"(VAL(u_, 2)), [v3] "v"(VAL(u_, 3)),        \
                       [v4] "v"(VAL(u_, 4)), [v5] "v"(VAL(u_, 5)), [v6] "v"(VAL(u_, 6)), [v7] "v"(VAL(u_, 7)),        \
                       [sc] "s"(sc), [sp] "v"(spill) : "memory"); } while (0)
    u32x4 mk = {0u, 0u, 0u, 0u};
    if (remote) {
#pragma unroll
        for (int r = 0; r < 128; ++r) mk[r >> 5] = __builtin_amdgcn_alignbit(mk[r >> 5], __float_as_uint(VAL(r >> 3, r & 7)), 31);
        if (wid != 0) __builtin_amdgcn_raw_buffer_store_b128(mk, sls, lane * 16, 0, AUX_SYS);
        stream_layout(mk, first, last);
        PACK8(0); PACK8(1); PACK8(2); PACK8(3); PACK8(4);
    } else {
#pragma unroll
        for (int m = 0; m < 4; ++m) {
#pragma unroll
            for (int n = 0; n < 8; ++n)
#pragma unroll
                for (int j = 0; j < 4; ++j) acc[m][n][j] = __int_as_float(max(0, __float_as_int(acc[m][n][j])));
            asm volatile("" : "+v"(acc[m][0]), "+v"(acc[m][1]), "+v"(acc[m][2]), "+v"(acc[m][3]),
                              "+v"(acc[m][4]), "+v"(acc[m][5]), "+v"(acc[m][6]), "+v"(acc[m][7]));
        }
    }
    if (wid == 0) {
        const char* ap = lane < NR ? ws + AMAX_OFF + ((g & 7) * NR + lane) * 128 : ws + TMO_OFF - 4;
        const u64 t0 = __builtin_amdgcn_s_memrealtime(); unsigned sp = 0;
        u64 v = 0;
        for (;;) {
            if (lane <= NR) v = ld_sys64(ap);
            if ((__ballot((int)((unsigned)(v >> 32) - call) >= 0) & 0xffu) == 0xffu) break;
            if (give_up(ws, call, t0, sp)) { v = (u64)call << 32; break; }
        }
        int t = lane < NR ? (int)(unsigned)v : ((unsigned)(v >> 32) == call ? NAN_BITS : 0);
        ROW_SHR_MAX(t, 1); ROW_SHR_MAX(t, 2); ROW_SHR_MAX(t, 4); ROW_SHR_MAX(t, 8);
        if (lane == NR) red[8] = t;
    }
    LGKM0_BARRIER();
    const float inv = 448.0f / __int_as_float(red[8]);

    if (remote) {
        const float f = inv * __int_as_float((254 - efield) << 23);
        unsigned sent = 0;
        const unsigned hshift = (lane & 1) * 4, woff = piece_off(lane);
#define SEND(cnt_, upto_) do { __builtin_amdgcn_s_waitcnt(0xC07F); asm volatile("" ::: "memory");                           \
            _Pragma("nounroll") for (; sent < (upto_); sent += 1024) {                                                                    \
                const unsigned o_ = sent + lane * 16;                                                                 \
                const f16x8 h0 = *(const LAS f16x8*)(lds + wid * LDS_WAVE + o_ * 2), h1 = *(const LAS f16x8*)(lds + wid * LDS_WAVE + o_ * 2 + 16); \
                u32x4 q;                                                                                              \
                _Pragma("unroll") for (int d = 0; d < 2; ++d) {                                                       \
                    int w0 = __builtin_amdgcn_cvt_pk_fp8_f32(__builtin_fmaf((float)h0[4 * d], f, 0.f), __builtin_fmaf((float)h0[4 * d + 1], f, 0.f), 0, false); \
                    q[d] = (unsigned)__builtin_amdgcn_cvt_pk_fp8_f32(__builtin_fmaf((float)h0[4 * d + 2], f, 0.f), __builtin_fmaf((float)h0[4 * d + 3], f, 0.f), w0, true); \
                    int w1 = __builtin_amdgcn_cvt_pk_fp8_f32(__builtin_fmaf((float)h1[4 * d], f, 0.f), __builtin_fmaf((float)h1[4 * d + 1], f, 0.f), 0, false); \
                    q[2 + d] = (unsigned)__builtin_amdgcn_cvt_pk_fp8_f32(__builtin_fmaf((float)h1[4 * d + 2], f, 0.f), __builtin_fmaf((float)h1[4 * d + 3], f, 0.f), w1, true); \
                }                                                                                                     \
                  \
                const unsigned h_ = (unsigned)__builtin_amdgcn_update_dpp(0, (int)q[3], 0xf5, 0xf, 0xf, true) >> hshift; \
                _Pragma("unroll") for (int d = 0; d < 4; ++d) q[d] |= (h_ << (7 - d)) & 0x80808080u;                  \
                WAIT_VM(2);                                                                                           \
                if ((o_ & ~31u) < (cnt_)) {                                                                             \
                    const unsigned w_ = 1024 + (sent >> 10) * PIECE_BYTES + woff;                                     \
                    if (lane & 1) __builtin_amdgcn_raw_buffer_store_b96((u32x3){q[0], q[1], q[2]}, sls, w_, 0, AUX_SYS); \
                    else __builtin_amdgcn_raw_buffer_store_b128(q, sls, w_, 0, AUX_SYS);                              \
                }                                                                                                     \
            } } while (0)
        if (wid == 0) __builtin_amdgcn_raw_buffer_store_b128(mk, sls, lane * 16, 0, AUX_SYS);
        SEND(last[4], last[4] & ~1023u);
        PACK8(5); PACK8(6); PACK8(7); PACK8(8);
        if (sent == 1024u) WAIT_VM(2); else WAIT_VM(0);
        if (lane == 0) st_sys(of, call);
        SEND(last[8], last[8] & ~1023u);
        PACK8(9); PACK8(10); PACK8(11); PACK8(12);
        SEND(last[12], last[12] & ~1023u);
        PACK8(13); PACK8(14); PACK8(15);
        SEND(last[15], last[15]);
        WAIT_VM(0);
        if (lane == 0) st_sys64(of + 8, call, red[8] == NAN_BITS ? call : 0u);
    } else {
        const __amdgpu_buffer_rsrc_t ors = rsrc_of(outp, (MM / NR) * NN);
        const unsigned ob = out_base(rank, tile, wid, lane);
#pragma unroll
        for (int m = 0; m < 4; ++m)
#pragma unroll
            for (int j = 0; j < 4; ++j) {
                u32x2 o;
                int lo = __builtin_amdgcn_cvt_pk_fp8_f32(acc[m][0][j] * inv, acc[m][1][j] * inv, 0, false);
                o[0] = (unsigned)__builtin_amdgcn_cvt_pk_fp8_f32(acc[m][2][j] * inv, acc[m][3][j] * inv, lo, true);
                int hi = __builtin_amdgcn_cvt_pk_fp8_f32(acc[m][4][j] * inv, acc[m][5][j] * inv, 0, false);
                o[1] = (unsigned)__builtin_amdgcn_cvt_pk_fp8_f32(acc[m][6][j] * inv, acc[m][7][j] * inv, hi, true);
                __builtin_amdgcn_raw_buffer_store_b64(o, ors, ob + (unsigned)((m * 16 + j) * NN), 0, AUX_SYS);
            }
    }
}

}

extern "C" void kernel_launch(int rank, int nranks, ncclComm_t comm,
                              hipStream_t stream,
                              void* const* d_in, const int* in_sizes, int n_in,
                              void* d_out, int out_size,
                              const int* in_dtypes, int out_dtype,
                              void* const* d_out_all,
                              void* const* d_ws_all, size_t ws_bytes) {
    (void)comm; (void)in_dtypes; (void)out_dtype; (void)d_out_all;
    if (nranks != NR || n_in < 2 || (unsigned)rank >= (unsigned)NR || ws_bytes < WS_NEED ||
        in_sizes[0] != MM * KSH || in_sizes[1] != KSH * NN || out_size != (MM / NR) * NN) {
        fprintf(stderr, "kernel_launch: unexpected problem shape\n");
        return;
    }
    Params p;
    p.x = (const char*)d_in[0];
    p.w = (const char*)d_in[1];
    for (int i = 0; i < NR; ++i) p.ws[i] = (char*)d_ws_all[(rank + i) % NR];
    p.out = (char*)d_out;
    p.rank = rank; p.pad = 0;
    hipLaunchKernelGGL(fused_gemm_relu_q8, dim3(GRID), dim3(BLOCK), 0, stream, p);
    HIP_CHECK(hipGetLastError());
}
```

```cpp
#include <hip/hip_runtime.h>
#include <rccl/rccl.h>
#include <cstdio>
#include <cstdint>

#define HIP_CHECK(expr) do { hipError_t _e = (expr); if (_e != hipSuccess) { \
    fprintf(stderr, "HIP error %s at %s:%d\n", hipGetErrorString(_e), __FILE__, __LINE__); \
    return; } } while (0)

namespace {

typedef short bf16x8 __attribute__((ext_vector_type(8)));
typedef _Float16 f16x8 __attribute__((ext_vector_type(8)));
typedef float f32x4 __attribute__((ext_vector_type(4)));
typedef unsigned u32x4 __attribute__((ext_vector_type(4)));
typedef unsigned u32x3 __attribute__((ext_vector_type(3)));
typedef unsigned u32x2 __attribute__((ext_vector_type(2)));
typedef unsigned long long u64;
#define LAS __attribute__((address_space(3)))
#define GAS __attribute__((address_space(1)))

constexpr int NR = 8;
constexpr int MM = 4096, KSH = 512, NN = 16384, NCOL = NN / NR;
constexpr int NT = KSH / 8;
constexpr int NBLK = 24;
#define PULL_PER_LINK 18
#define AMAX_REP 8
#define AUX_OUT 17
#define SEND_DEPTH 2
#define JMODE 2
#define JORDER(k) ((k) % 3 == 2 ? 16 + (k) / 3 : (k) / 3 * 2 + (k) % 3)
#define SLEEP_K 2
#define ROT 1
#define SLOT(src, blk) ((((src) + ROT * (blk)) & 7) * 4096)
#define LATE_FLAG 1
#define STAGE_AUX 17
#define POLL_POS 0
#define TAKE_COL 1
#define POLL_AHEAD 6
#define PACK_DELAY 60
constexpr int Q_JOBS = NT * NBLK;
constexpr int Q_TICKETS = Q_JOBS + PULL_PER_LINK;
constexpr int N_COMM = 128;
constexpr int N_PULL = (NR - 1) * PULL_PER_LINK;
constexpr int N_GEMM = 128;
constexpr int GRID = N_COMM + N_GEMM;
constexpr int BLOCK = 512;
constexpr int N_PACKW = N_COMM * 6;
constexpr int NGRP = NT / 8;
static_assert(N_PULL <= 2 * N_COMM, "pull waves are waves 0-1 of the comm workgroups");

constexpr unsigned XT_T = NR * MM * 16, WT_T = NR * NCOL * 16;
constexpr unsigned XT_OFF = 0;
constexpr unsigned WT_OFF = XT_OFF + NT * XT_T;
constexpr unsigned WS_OFF = WT_OFF + NT * WT_T;
constexpr unsigned OB_SLOT = 1024 + 8192;
constexpr unsigned OB_OFF = WS_OFF + NT * WT_T;
constexpr unsigned CTL_OFF = OB_OFF + NR * 128 * OB_SLOT;
constexpr unsigned PK_OFF = CTL_OFF;
constexpr unsigned CNT_OFF = PK_OFF + NR * 8192;
constexpr unsigned FL_OFF = CNT_OFF + NGRP * 9 * 128;
constexpr unsigned Q_OFF = FL_OFF + NBLK * NR * NT * 4;
constexpr unsigned TMAX_OFF = Q_OFF + NR * 128;
constexpr unsigned AMAX_OFF = TMAX_OFF + N_GEMM * 128;
constexpr unsigned OF_OFF = AMAX_OFF + 64 * 128;
constexpr unsigned CTR_OFF = OF_OFF + NR * 128 * 128;
constexpr unsigned TMO_OFF = CTR_OFF + GRID * 128 + 4;
constexpr unsigned DBG_OFF = CTR_OFF + GRID * 128 + 128;
constexpr size_t WS_NEED = DBG_OFF + 524288;

constexpr u64 DEADLINE = 50000000ull;
constexpr int AUX_SYS = 17;

constexpr int NSTEP = 2 * NT;
constexpr int NSLOT = 5;
constexpr int LDS_SLOT = 32768;
constexpr int LDS_BYTES = NSLOT * LDS_SLOT;
constexpr int LDS_WAVE = 16384;
constexpr int LDS_MISC = 8 * LDS_WAVE;

struct Params {
    const char* x; const char* w;
    int rank, pad;
    char* ws[NR];
    char* out;
};

__device__ __forceinline__ __amdgpu_buffer_rsrc_t rsrc_of(const void* p, unsigned bytes) {
    return __builtin_amdgcn_make_buffer_rsrc(const_cast<void*>(p), 0, bytes, 0x00020000);
}
__device__ __forceinline__ unsigned ld_sys(const void* p) {
    return __hip_atomic_load((const GAS unsigned*)p, __ATOMIC_RELAXED, __HIP_MEMORY_SCOPE_SYSTEM);
}
__device__ __forceinline__ u64 ld_sys64(const void* p) {
    return __hip_atomic_load((const GAS u64*)p, __ATOMIC_RELAXED, __HIP_MEMORY_SCOPE_SYSTEM);
}
__device__ __forceinline__ void st_sys(void* p, unsigned v) {
    __hip_atomic_store((GAS unsigned*)p, v, __ATOMIC_RELAXED, __HIP_MEMORY_SCOPE_SYSTEM);
}
__device__ __forceinline__ void st_sys64(void* p, unsigned lo, unsigned hi) {
    __hip_atomic_store((GAS u64*)p, ((u64)hi << 32) | lo, __ATOMIC_RELAXED, __HIP_MEMORY_SCOPE_SYSTEM);
}
__device__ __forceinline__ unsigned ld_agent(const void* p) {
    return __hip_atomic_load((const GAS unsigned*)p, __ATOMIC_RELAXED, __HIP_MEMORY_SCOPE_AGENT);
}
__device__ __forceinline__ unsigned add_agent_lane0(void* p, int lane) {
    unsigned v = 0, z = 0;
    asm volatile("" : "+v"(z));
    if (lane == 0) v = __hip_atomic_fetch_add((GAS unsigned*)((char*)p + z), 1u, __ATOMIC_RELAXED, __HIP_MEMORY_SCOPE_AGENT);
    return v;
}
__device__ __forceinline__ u32x4 transposed(const u32x4 (&v)[8], int i) {
    u32x4 o;
#pragma unroll
    for (int d = 0; d < 4; ++d)
        o[d] = __builtin_amdgcn_perm(v[2 * d + 1][i >> 1], v[2 * d][i >> 1], (i & 1) ? 0x07060302u : 0x05040100u);
    return o;
}
constexpr int NAN_BITS = 0x7fc00000;
#define ROW_SHR_MAX(v, n) v = max(v, __builtin_amdgcn_update_dpp(0, v, 0x110 + (n), 0xf, 0xf, true))
__device__ __forceinline__ int wave_max(int v) {
    ROW_SHR_MAX(v, 1); ROW_SHR_MAX(v, 2); ROW_SHR_MAX(v, 4); ROW_SHR_MAX(v, 8);
    return max(max(__builtin_amdgcn_readlane(v, 15), __builtin_amdgcn_readlane(v, 31)),
               max(__builtin_amdgcn_readlane(v, 47), __builtin_amdgcn_readlane(v, 63)));
}
__device__ __forceinline__ unsigned sign_byte(const u32x4& mk, int u) { return (mk[u >> 2] >> (24 - 8 * (u & 3))) & 0xffu; }
__device__ __forceinline__ void stream_layout(const u32x4& mk, unsigned (&first)[16], unsigned (&last)[16]) {
    unsigned base = 0;
#pragma unroll
    for (int u = 0; u < 16; u += 2) {
        const int c = 0x00080008 - (int)(__builtin_popcount(sign_byte(mk, u)) | (__builtin_popcount(sign_byte(mk, u + 1)) << 16));
        int v = c;
        v += __builtin_amdgcn_update_dpp(0, v, 0x111, 0xf, 0xf, true);
        v += __builtin_amdgcn_update_dpp(0, v, 0x112, 0xf, 0xf, true);
        v += __builtin_amdgcn_update_dpp(0, v, 0x114, 0xf, 0xf, true);
        v += __builtin_amdgcn_update_dpp(0, v, 0x118, 0xf, 0xf, true);
        v += __builtin_amdgcn_update_dpp(0, v, 0x142, 0xa, 0xf, false);
        v += __builtin_amdgcn_update_dpp(0, v, 0x143, 0xc, 0xf, false);
        const unsigned tot = (unsigned)__builtin_amdgcn_readlane(v, 63), exc = (unsigned)(v - c);
        first[u] = base + (exc & 0xffffu); base += tot & 0xffffu; last[u] = base;
        first[u + 1] = base + (exc >> 16); base += tot >> 16; last[u + 1] = base;
    }
}
template <int SLEEP = 1>
__device__ __forceinline__ bool give_up(char* ws, unsigned call, u64 t0, unsigned& spins) {
    bool stop = __builtin_amdgcn_s_memrealtime() - t0 > DEADLINE;
    if ((++spins & 63u) == 0u) stop = stop || __builtin_amdgcn_readfirstlane(ld_agent(ws + TMO_OFF)) == call;
    if (stop) {
        __hip_atomic_store((GAS unsigned*)(ws + TMO_OFF), call, __ATOMIC_RELAXED, __HIP_MEMORY_SCOPE_AGENT);
        return true;
    }
    __builtin_amdgcn_s_sleep(SLEEP);
    return false;
}
template <int SLEEP>
__device__ __forceinline__ bool await_word(char* ws, const char* p, unsigned call) {
    const u64 t0 = __builtin_amdgcn_s_memrealtime(); unsigned sp = 0;
    while ((int)(__builtin_amdgcn_readfirstlane(ld_sys(p)) - call) < 0)
        if (give_up<SLEEP>(ws, call, t0, sp)) return false;
    return true;
}
__device__ __forceinline__ unsigned fl_word(int blk, int src, int t) { return FL_OFF + (unsigned)(((blk * 4 + (src & 3)) * NT + t) * 2 + (src >> 2)) * 4u; }
__device__ __forceinline__ int steps_known(unsigned f0, unsigned f1, unsigned call) {
    const u64 lo = __ballot((int)(f0 - call) >= 0), hi = __ballot((int)(f1 - call) >= 0);
    const u64 miss = ~(lo & hi);
    if (miss == 0) return NSTEP;
    const int c = __builtin_ctzll(miss);
    return 2 * c + (int)((lo >> c) & 1);
}
__device__ __forceinline__ unsigned out_base(int src, int tile, int wave, int lane) {
    return (unsigned)(((tile >> 3) * 256 + (wave >> 1) * 64 + (lane >> 4) * 4) * NN + src * NCOL + (tile & 7) * 256 + (wave & 1) * 128 + (lane & 15) * 8);
}
constexpr unsigned PIECE_BYTES = 896;
__device__ __forceinline__ unsigned piece_off(int lane) { return lane & 1 ? 512 + (lane >> 1) * 12 : (lane >> 1) * 16; }
#define WAIT_VM(n) asm volatile("s_waitcnt vmcnt(" #n ")" ::: "memory")
#define LGKM0_BARRIER() do { __builtin_amdgcn_s_waitcnt(0xC07F); asm volatile("" ::: "memory");                       \
        __builtin_amdgcn_s_barrier(); asm volatile("" ::: "memory"); } while (0)
#define STAMP(i) do {} while (0)

__global__ void __launch_bounds__(BLOCK, 2) fused_gemm_relu_q8(const Params p) {
    __shared__ __attribute__((aligned(1024))) char smem[LDS_BYTES];
    LAS char* const lds = (LAS char*)smem;
    const int hb = blockIdx.x;
    const int lane = threadIdx.x & 63;
    const int wid = __builtin_amdgcn_readfirstlane(threadIdx.x >> 6);
    int rank = p.rank;
    char* ws = p.ws[0];
    const char* px = p.x;
    const char* pw = p.w;
    const u64 t_start = __builtin_amdgcn_s_memrealtime();
    asm volatile("" : "+s"(rank), "+s"(ws), "+s"(px), "+s"(pw));
    const unsigned ticket = add_agent_lane0(ws + CTR_OFF + (size_t)hb * 128 + wid * 4, lane);
    const int b = hb & (N_COMM - 1);

    if (hb < N_COMM) {
        const __amdgpu_buffer_rsrc_t wsr = rsrc_of(ws, CTL_OFF);
        const int cw = wid * N_COMM + b;
        const int dbg = 128 + cw;
        (void)dbg;
        STAMP(0);
        if (wid < 2) {
            if (threadIdx.x == 0) *(LAS unsigned*)(lds + LDS_MISC) = 0u;
            LGKM0_BARRIER();
            if (cw < N_PULL) {
                const unsigned s = (unsigned)(rank + 1 + cw % (NR - 1)) & 7u;
                const __amdgpu_buffer_rsrc_t srs = rsrc_of(p.ws[1 + cw % (NR - 1)], CTL_OFF);
                char* const queue = ws + Q_OFF + s * 128;
                unsigned tk = add_agent_lane0(queue, lane);
                u32x4 cur[4];
                const unsigned call = __builtin_amdgcn_readfirstlane(ticket) + 1u;
                const unsigned qbase = (call - 1u) * Q_TICKETS;
                const char* const pk = ws + PK_OFF + s * 8192;
                unsigned pv = call - 1u, pg = call - 1u;
#define JOB_OF(n_, t_, j_, so_, do_) const int t_ = (int)(n_) / NBLK, j_ = JORDER((int)(n_) % NBLK);                           \
                const unsigned do_ = (j_ < 16 ? XT_OFF + t_ * XT_T + j_ * 32768 : WT_OFF + t_ * WT_T + (j_ - 16) * 32768) + SLOT(s, j_) + lane * 16; \
                const unsigned so_ = j_ < 16 ? do_ : WS_OFF + t_ * WT_T + rank * 32768 + (j_ - 16) * 4096 + lane * 16
#define SEEN(v_, l_) ((int)(__builtin_amdgcn_readlane(v_, l_) - call) >= 0)
#define WAIT_PACKED(t_, j_) do { const int sg_ = ((t_) >> 3) + ((j_) < 16 ? 0 : 8);                                    \
                if (!SEEN(pg, sg_)) {                                                                                  \
                    const u64 t0_ = __builtin_amdgcn_s_memrealtime(); unsigned sp_ = 0;                                \
                    if ((t_) < 8) { const int sl_ = (j_) < 16 ? (j_) : 24 + (t_) * 4 + ((j_) >> 1);                    \
                        if (!SEEN(pv, sl_)) {                            \
                            pv = ld_sys(pk + lane * 128); pg = ld_sys(pk + 16 * 128 + (lane & 15) * 4);                \
                            while (!SEEN(pv, sl_)) {                                                                   \
                                if (give_up(ws, call, t0_, sp_)) break;                                                \
                                if (lane == sl_) pv = ld_sys(pk + lane * 128); } }                                     \
                    } else {                                                                                           \
                        for (;;) { pg = ld_sys(pk + 16 * 128 + (lane & 15) * 4);                                       \
                            if (SEEN(pg, sg_) || give_up(ws, call, t0_, sp_)) break; } }                               \
                } } while (0)
#define PTAIL() do {} while (0)
                unsigned n = __builtin_amdgcn_readfirstlane(tk) - qbase;
                if (n < (unsigned)Q_JOBS) {
                    JOB_OF(n, t, j, so, dof);
                    WAIT_PACKED(t, j);
                    STAMP(2);
                    tk = add_agent_lane0(queue, lane);
#pragma unroll
                    for (int i = 0; i < 4; ++i) cur[i] = __builtin_amdgcn_raw_buffer_load_b128(srs, so + i * 1024, 0, AUX_SYS);
                    unsigned dcur = dof, fcur = fl_word(j, s, t);
                    for (;;) {
#pragma unroll
                        for (int i = 0; i < 4; ++i) __builtin_amdgcn_raw_buffer_store_b128(cur[i], wsr, dcur + i * 1024, 0, AUX_SYS);
                        asm volatile("" ::: "memory");
                        n = __builtin_amdgcn_readfirstlane(tk) - qbase;
                        char* const flag = ws + fcur;
                        if (n >= (unsigned)Q_JOBS) {
                            WAIT_VM(0);
                            if (lane == 0) st_sys(flag, call);
                            PTAIL();
                            break;
                        }
                        JOB_OF(n, t2, j2, so2, dof2);
                        WAIT_PACKED(t2, j2);
                        tk = add_agent_lane0(queue, lane);
#pragma unroll
                        for (int i = 0; i < 4; ++i) cur[i] = __builtin_amdgcn_raw_buffer_load_b128(srs, so2 + i * 1024, 0, AUX_SYS);
                        dcur = dof2; fcur = fl_word(j2, s, t2);
                        WAIT_VM(5);
                        if (lane == 0) st_sys(flag, call);
                        PTAIL();
                    }
                }
                STAMP(1);
            }
        } else {
            const __amdgpu_buffer_rsrc_t xr = rsrc_of(px, MM * KSH * 2);
            const __amdgpu_buffer_rsrc_t wr_ = rsrc_of(pw, KSH * NN * 2);
            bool met = false;
            bool x_todo = wid >= 4;
            for (int n = (wid - 2) * N_COMM + b; n < NT * 32 || x_todo;) {
                const bool do_x = x_todo && ((b >> 4) <= (n >> 8) || n >= NT * 32);
                const int g = do_x ? b >> 4 : n >> 8;
                const int delay = (do_x ? g * 8 : n >> 5) * PACK_DELAY;
                if (!met && delay > 0) { __builtin_amdgcn_s_barrier(); met = true; }
                while (__builtin_amdgcn_s_memrealtime() - t_start < (u64)delay) __builtin_amdgcn_s_sleep(2);
                u32x4 v[8];
                if (do_x) {
                    const unsigned rb = (wid - 4) * 64 + (lane & 7), piece = lane >> 3;
#pragma unroll
                    for (int i = 0; i < 8; ++i) v[i] = __builtin_amdgcn_raw_buffer_load_b128(xr, ((b & 15) * 256 + rb + i * 8) * (KSH * 2) + g * 128 + piece * 16, 0, 0);
                    if (!met) { __builtin_amdgcn_s_barrier(); met = true; }
                    const unsigned d0 = XT_OFF + (g * 8 + piece) * XT_T + (b & 15) * 32768 + SLOT(rank, b & 15) + rb * 16;
#pragma unroll
                    for (int i = 0; i < 8; ++i) __builtin_amdgcn_raw_buffer_store_b128(v[i], wsr, d0 + i * 128, 0, AUX_SYS);
                    char* fl = lane < 8 ? p.ws[lane & 7] + PK_OFF + rank * 8192 + (b & 15) * 128
                                        : ws + fl_word(b & 15, rank, g * 8 + (lane & 7));
                    asm volatile("s_waitcnt vmcnt(0)" : "+v"(fl) :: "memory");
                    unsigned old = 0;
                    if (lane == 0) old = __hip_atomic_fetch_add((LAS unsigned*)(lds + LDS_MISC), 1u, __ATOMIC_RELAXED, __HIP_MEMORY_SCOPE_WORKGROUP);
                    const unsigned call = __builtin_amdgcn_readfirstlane(ticket) + 1u;
                    if (__builtin_amdgcn_readfirstlane(old) == 3u) {
                        if (lane >= (g == 0 ? 1 : 8) && lane < 16) st_sys(fl, call);
                        if (__builtin_amdgcn_readfirstlane(add_agent_lane0(ws + CNT_OFF + g * 9 * 128, lane)) + 1u == call * 16u && lane >= 1 && lane < 8)
                            st_sys(fl + (16 - (b & 15)) * 128 + g * 4, call);
                    }
                    x_todo = false;
                } else {
                    const int t = n >> 5, cb = n & 31;
                    const unsigned dest = (unsigned)cb >> 2;
#pragma unroll
                    for (int i = 0; i < 8; ++i) v[i] = __builtin_amdgcn_raw_buffer_load_b128(wr_, (unsigned)((t * 8 + i) * (NN * 2) + cb * 1024 + lane * 16), 0, 0);
                    if (!met) { __builtin_amdgcn_s_barrier(); met = true; }
                    const unsigned d0 = (dest == (unsigned)rank ? WT_OFF + t * WT_T + ((cb & 3) * 2 + (lane >> 5)) * 32768 + SLOT(rank, (cb & 3) * 2 + (lane >> 5))
                                                                : WS_OFF + t * WT_T + dest * 32768 + ((cb & 3) * 2 + (lane >> 5)) * 4096)
                                      + (((lane >> 4) & 1) * 128 + (lane & 15)) * 16;
#pragma unroll
                    for (int i = 0; i < 8; ++i) __builtin_amdgcn_raw_buffer_store_b128(transposed(v, i), wsr, d0 + i * 256, 0, AUX_SYS);
                    char* const dws = p.ws[(dest - rank) & 7u];
                    WAIT_VM(0);
                    const unsigned call = __builtin_amdgcn_readfirstlane(ticket) + 1u;
                    if (dest == (unsigned)rank) { if (lane < 2) st_sys(ws + fl_word(16 + (cb & 3) * 2 + lane, rank, t), call); }
                    else {
                        if (g == 0 && lane == 0) st_sys(dws + PK_OFF + rank * 8192 + (32 + t * 4 + (cb & 3)) * 128, call);
                        if (__builtin_amdgcn_readfirstlane(add_agent_lane0(ws + CNT_OFF + (g * 9 + 1 + dest) * 128, lane)) + 1u == call * 32u && lane == 0)
                            st_sys(dws + PK_OFF + rank * 8192 + 16 * 128 + (8 + g) * 4, call);
                    }
                    n += N_PACKW;
                }
            }
        }
        const unsigned call = __builtin_amdgcn_readfirstlane(ticket) + 1u;
        if (wid == 7) {
            if (b == N_COMM - 1) {
                char* amax_out = p.ws[lane & 7] + AMAX_OFF + ((lane >> 3) * NR + rank) * 128;
                asm volatile("" : "+v"(amax_out));
                const char* tm = ws + TMAX_OFF + lane * 256;
                const u64 t0 = __builtin_amdgcn_s_memrealtime(); unsigned sp = 0;
                int mx;
                for (;;) {
                    const u64 v0 = ld_sys64(tm), v1 = ld_sys64(tm + 128);
                    mx = max((int)(unsigned)v0, (int)(unsigned)v1);
                    if (__all((int)((unsigned)(v0 >> 32) - call) >= 0 && (int)((unsigned)(v1 >> 32) - call) >= 0)) break;
                    if (give_up(ws, call, t0, sp)) { mx = NAN_BITS; break; }
                }
                if (lane < 8 * AMAX_REP) st_sys64(amax_out, (unsigned)wave_max(mx), call);
                STAMP(6);
            }
            return;
        }
        {
            const int src = (rank + 1 + wid) & 7;
            const unsigned slot = OB_OFF + (unsigned)(src * 128 + b) * OB_SLOT;
            const char* const of = ws + OF_OFF + (src * 128 + b) * 128;
            const unsigned lbase = wid * LDS_WAVE;
#pragma unroll
            for (int e = 0; e < 4; ++e) {
                const unsigned idx = lane * 4 + e;
                u32x2 sel = {0u, 0u};
                unsigned rk = 0;
#pragma unroll
                for (int n = 0; n < 8; ++n) {
                    const bool neg = (idx >> (7 - n)) & 1u;
                    sel[n >> 2] |= (neg ? 0x0cu : rk) << (8 * (n & 3));
                    rk += neg ? 0u : 1u;
                }
                *(LAS u32x2*)(lds + lbase + 12288 + idx * 8) = sel;
            }
            char* outp = p.out;
            asm volatile("" : "+s"(outp));
            const unsigned ob = out_base(src, b >> 3, b & 7, lane);
            {
                bool ok = await_word<8>(ws, of, call);
                const u32x4 mk = __builtin_amdgcn_raw_buffer_load_b128(wsr, slot + lane * 16, 0, AUX_SYS);
                unsigned first[16], last[16];
                stream_layout(mk, first, last);
                u32x2 sel[16];
#pragma unroll
                for (int u = 0; u < 16; ++u) {
                    sel[u] = *(const LAS u32x2*)(lds + lbase + 12288 + sign_byte(mk, u) * 8);
                    first[u] += lbase;
                    asm volatile("" : "+v"(first[u]), "+v"(sel[u]));
                }
                STAMP(3);
                const unsigned cnt = last[15];
                const __amdgpu_buffer_rsrc_t ors = rsrc_of(outp, (MM / NR) * NN);
#define TAKE_PIECES(i0_, i1_) do { u32x4 pc[(i1_) - (i0_)];                                                            \
                    _Pragma("unroll") for (int i = (i0_); i < (i1_); ++i)                                              \
                        if (i * 1024u < cnt) pc[i - (i0_)] = __builtin_amdgcn_raw_buffer_load_b128(wsr, slot + 1024 + i * PIECE_BYTES + piece_off(lane), 0, AUX_SYS); \
                    fill = ok && !__any(ld_sys(lane & 1 ? ws + TMO_OFF : of + 12) == call) ? 0u : 0x7f7f7f7fu;          \
                    _Pragma("unroll") for (int i = (i0_); i < (i1_); ++i)                                              \
                        if (i * 1024u < cnt) {                                                                         \
                            u32x4 q = pc[i - (i0_)];                                                                   \
                            unsigned h = ((q[0] & 0x80808080u) >> 7) | ((q[1] & 0x80808080u) >> 6) | ((q[2] & 0x80808080u) >> 5); \
                            const unsigned he = h | ((q[3] & 0x80808080u) >> 4);     \
                            h = (h << 4) | (unsigned)__builtin_amdgcn_update_dpp(0, (int)he, 0xa0, 0xf, 0xf, true);    \
                            q[0] &= 0x7f7f7f7fu; q[1] &= 0x7f7f7f7fu; q[2] &= 0x7f7f7f7fu;                             \
                            q[3] = lane & 1 ? h : q[3] & 0x7f7f7f7fu;                                                  \
                            *(LAS u32x4*)(lds + lbase + i * 1024 + lane * 16) = q;                                     \
                        } } while (0)
                unsigned fill;
                ok = await_word<1>(ws, of + 8, call) && ok;
                STAMP(4);
                TAKE_PIECES(0, 8);
                unsigned d[16][3];
#pragma unroll
                for (int u = 0; u < 16; ++u)
#pragma unroll
                    for (int i = 0; i < 3; ++i) d[u][i] = *(const LAS unsigned*)(lds + (first[u] & ~3u) + i * 4);
                __builtin_amdgcn_sched_barrier(0);
#pragma unroll
                for (int u = 0; u < 16; ++u) {
                    const unsigned lo = __builtin_amdgcn_alignbyte(d[u][1], d[u][0], first[u]), hi = __builtin_amdgcn_alignbyte(d[u][2], d[u][1], first[u]);
                    u32x2 o;
                    o[0] = __builtin_amdgcn_perm(hi, lo, sel[u][0]) | fill;
                    o[1] = __builtin_amdgcn_perm(hi, lo, sel[u][1]) | fill;
                    __builtin_amdgcn_raw_buffer_store_b64(o, ors, ob + (unsigned)(((u >> 2) * 16 + (u & 3)) * NN), 0, AUX_OUT);
                }
            }
        }
        return;
    }

    const int g = b;
    const int dbg = g;
    (void)dbg;
    const int pm = (g >> 4) * 2 + (g & 1);
    const int pn = ((g & 7) >> 1) * 2 + ((g >> 3) & 1);
    const int fr = lane & 15, fq = lane >> 4, wr = wid >> 1, wc = wid & 1;
    if (wid == 0) STAMP(0);
    const int owner = pm >> 1, tile = (pm & 1) * 8 + pn;
    char* ows = p.ws[(owner - rank) & 7];
    char* outp = p.out;
    asm volatile("" : "+s"(ows), "+s"(outp));
    const unsigned call = __builtin_amdgcn_readfirstlane(ticket) + 1u;

    const int op = wid >> 2;
    const char* const pollp = ws + fl_word(op ? 16 + pn : pm, wid & 3, lane);
    const unsigned unit = op ? WT_T : XT_T;
    const char* const gbase = ws + (op ? WT_OFF + pn * 32768 : XT_OFF + pm * 32768) + lane * 16;
    const unsigned slot0 = SLOT(wid & 3, op ? pn : pm);
    LAS char* const ldst = lds + wid * 4096;
    const int aoff = fq * 4096 + (wr * 64 + fr) * 16;
    const int boff = 16384 + fq * 4096 + (wc * 128 + fr) * 16;

    int known = 0;
    int next = 0, nslot = 0;
    u32x2 pvn = {0u, 0u};
#define STAGE_NEXT() do { const char* g_ = gbase + ((slot0 ^ ((next & 1) << 14)) + (unsigned)(next >> 1) * unit);              \
        LAS char* l_ = ldst + nslot * LDS_SLOT;                                                                       \
        __builtin_amdgcn_global_load_lds((const GAS unsigned*)g_, (LAS unsigned*)l_, 16, 0, STAGE_AUX);                       \
        __builtin_amdgcn_global_load_lds((const GAS unsigned*)g_, (LAS unsigned*)l_, 16, 1024, STAGE_AUX);                    \
        __builtin_amdgcn_global_load_lds((const GAS unsigned*)g_, (LAS unsigned*)l_, 16, 2048, STAGE_AUX);                    \
        __builtin_amdgcn_global_load_lds((const GAS unsigned*)g_, (LAS unsigned*)l_, 16, 3072, STAGE_AUX);                    \
        ++next; nslot = nslot == NSLOT - 1 ? 0 : nslot + 1; } while (0)
#define POLL_ISSUE() asm volatile("global_load_dwordx2 %0, %1, off sc0 sc1" : "+v"(pvn) : "v"(pollp) : "memory")
#define POLL_TAKE() do { u32x2 f_; asm volatile("s_waitcnt vmcnt(0)\n\tv_mov_b64 %0, %1" : "=v"(f_) : "v"(pvn) : "memory");    \
        known = steps_known(f_[0], f_[1], call); } while (0)
#define AWAIT_NEXT() do { if (next >= known) {                                                                        \
            const u64 t0_ = __builtin_amdgcn_s_memrealtime(); unsigned sp_ = 0;                                       \
            for (;;) { const u64 f_ = ld_sys64(pollp); known = steps_known((unsigned)f_, (unsigned)(f_ >> 32), call); \
                if (next < known) break;                                                                              \
                if (give_up<SLEEP_K>(ws, call, t0_, sp_)) { known = NSTEP; break; } } } } while (0)
#define MFMA_COLS(A_, B_, n0_, n1_) do { __builtin_amdgcn_sched_barrier(0);                                           \
        _Pragma("unroll") for (int n = (n0_); n < (n1_); ++n)                                                         \
            _Pragma("unroll") for (int m = 0; m < 4; ++m)                                                             \
                acc[m][n] = __builtin_amdgcn_mfma_f32_16x16x32_bf16(A_[m], B_[n], acc[m][n], 0, 0, 0);                \
        __builtin_amdgcn_sched_barrier(0); } while (0)
#define RD_A(A_, m_) A_[m_] = *(const LAS bf16x8*)(lb_ + aoff + (m_) * 256)
#define RD_B(B_, n_) B_[n_] = *(const LAS bf16x8*)(lb_ + boff + (n_) * 256)
#define TAIL_REC(st_) do {} while (0)
#define KSTEP(st_, A_, B_, AN_, BN_) do {                                                                             \
        MFMA_COLS(A_, B_, 0, TAKE_COL);                                                                               \
        if (polled) POLL_TAKE();                                          \
        const int had_ = next;                                                                                        \
        const int lim_ = min((st_) + NSLOT, known);                                   \
        if (next < lim_) { STAGE_NEXT(); if (next < lim_) STAGE_NEXT(); }                                             \
        const bool full_ = (polled ? had_ + NSLOT - 2 : next) >= (st_) + NSLOT;                                       \
        MFMA_COLS(A_, B_, TAKE_COL, 4);                                                                                      \
        if (!full_) {                                                              \
            if (next == (st_) + 1 && next < NSTEP) {                                                                  \
                AWAIT_NEXT(); STAGE_NEXT();                                                                           \
                if (next < known) STAGE_NEXT();                                                                       \
            }                                                                                                         \
            const int k_ = next - (st_) - 2;                                                                          \
            if (k_ <= 0) WAIT_VM(0); else if (k_ == 1) WAIT_VM(4); else WAIT_VM(8);                                   \
        }                                                                                                             \
        WAIT_VM(12);                                                                                                  \
        LGKM0_BARRIER();                                                                                              \
        TAIL_REC(st_);                                                                                                \
        const LAS char* lb_ = lds + rslot * LDS_SLOT;               \
        RD_A(AN_, 0); RD_A(AN_, 1); RD_A(AN_, 2); RD_A(AN_, 3);                                                       \
        if (POLL_POS == 0) { polled = known < min(next + POLL_AHEAD, NSTEP); if (polled) POLL_ISSUE(); }              \
        MFMA_COLS(A_, B_, 4, 5);                                                                                      \
        RD_B(BN_, 0); RD_B(BN_, 1); RD_B(BN_, 2); RD_B(BN_, 3);                                                       \
        MFMA_COLS(A_, B_, 5, 6);                                                                                      \
        RD_B(BN_, 4); RD_B(BN_, 5); RD_B(BN_, 6); RD_B(BN_, 7);                                                       \
        rslot = rslot == NSLOT - 1 ? 0 : rslot + 1;                                                                   \
        if (POLL_POS == 2) { polled = known < min(next + POLL_AHEAD, NSTEP); if (polled) POLL_ISSUE(); }              \
        MFMA_COLS(A_, B_, 6, 8); } while (0)

    f32x4 acc[4][8];
#pragma unroll
    for (int m = 0; m < 4; ++m)
#pragma unroll
        for (int n = 0; n < 8; ++n) acc[m][n] = (f32x4){0.f, 0.f, 0.f, 0.f};

    AWAIT_NEXT(); STAGE_NEXT();
    WAIT_VM(0);
    LGKM0_BARRIER();
    int rslot = 1;
    bool polled = false;
    bf16x8 a0[4], b0[8], a1[4], b1[8];
    {
        const LAS char* lb_ = lds;
#pragma unroll
        for (int m = 0; m < 4; ++m) RD_A(a0, m);
#pragma unroll
        for (int n = 0; n < 8; ++n) RD_B(b0, n);
    }
    if (wid == 0) STAMP(1);
    for (int st = 0; st < NSTEP; st += 2) {
        KSTEP(st, a0, b0, a1, b1);
        KSTEP(st + 1, a1, b1, a0, b0);
    }
    if (wid == 0) STAMP(2);

    int mx = 0;
#pragma unroll
    for (int m = 0; m < 4; ++m)
#pragma unroll
        for (int n = 0; n < 8; ++n)
#pragma unroll
            for (int j = 0; j < 4; ++j) mx = max(mx, __float_as_int(acc[m][n][j]));
    mx = wave_max(mx);
    LAS int* red = (LAS int*)(lds + LDS_MISC);
    if (lane == 0) red[wid] = mx;
    LGKM0_BARRIER();
    if (wid == 0) {
        int t = red[lane & 7];
        ROW_SHR_MAX(t, 1); ROW_SHR_MAX(t, 2); ROW_SHR_MAX(t, 4);
        if (lane == 7) st_sys64(ws + TMAX_OFF + g * 128, (unsigned)t, call);
        STAMP(7);
    }

    const bool remote = owner != rank;
    const __amdgpu_buffer_rsrc_t sls = rsrc_of(ows + OB_OFF + (unsigned)(rank * 128 + tile * 8 + wid) * OB_SLOT, OB_SLOT);
    char* const of = ows + OF_OFF + (rank * 128 + tile * 8 + wid) * 128;
    const int efield = min(268 - (mx >> 23), 253);
    const float sc = __int_as_float(efield << 23);
    const unsigned pos0 = (unsigned)(size_t)(lds + wid * LDS_WAVE);
    const unsigned spill = (unsigned)(size_t)(lds + LDS_MISC + 64);
    unsigned first[16], last[16];
#define VAL(u_, n_) acc[(u_) >> 2][n_][(u_) & 3]
#define PACK1(n_) "v_ashrrev_i32 %[s], 31, %[v" #n_ "]\n\tv_fma_mixlo_f16 %[h], %[v" #n_ "], %[sc], 0\n\t"              \
                  "v_bfi_b32 %[a], %[s], %[sp], %[p]\n\tv_lshl_add_u32 %[p], %[s], 1, %[p]\n\t"                      \
                  "ds_write_b16 %[a], %[h] offset:2*" #n_ "\n\t"
#define PACK8(u_) do { unsigned p_ = pos0 + 2 * first[u_], s_, h_, a_;                                                \
        asm volatile(PACK1(0) PACK1(1) PACK1(2) PACK1(3) PACK1(4) PACK1(5) PACK1(6) PACK1(7)                          \
                     : [p] "+v"(p_), [s] "=&v"(s_), [h] "=&v"(h_), [a] "=&v"(a_)                                      \
                     : [v0] "v"(VAL(u_, 0)), [v1] "v"(VAL(u_, 1)), [v2] "v"(VAL(u_, 2)), [v3] "v"(VAL(u_, 3)),        \
                       [v4] "v"(VAL(u_, 4)), [v5] "v"(VAL(u_, 5)), [v6] "v"(VAL(u_, 6)), [v7] "v"(VAL(u_, 7)),        \
                       [sc] "s"(sc), [sp] "v"(spill) : "memory"); } while (0)
    u32x4 mk = {0u, 0u, 0u, 0u};
    if (remote) {
#pragma unroll
        for (int r = 0; r < 128; ++r) mk[r >> 5] = __builtin_amdgcn_alignbit(mk[r >> 5], __float_as_uint(VAL(r >> 3, r & 7)), 31);
        if (!LATE_FLAG || wid != 0) __builtin_amdgcn_raw_buffer_store_b128(mk, sls, lane * 16, 0, AUX_SYS);
        stream_layout(mk, first, last);
        PACK8(0); PACK8(1); PACK8(2); PACK8(3); PACK8(4);
        if (!LATE_FLAG) { WAIT_VM(0); if (lane == 0) st_sys(of, call); }
    } else {
#pragma unroll
        for (int m = 0; m < 4; ++m) {
#pragma unroll
            for (int n = 0; n < 8; ++n)
#pragma unroll
                for (int j = 0; j < 4; ++j) acc[m][n][j] = __int_as_float(max(0, __float_as_int(acc[m][n][j])));
            asm volatile("" : "+v"(acc[m][0]), "+v"(acc[m][1]), "+v"(acc[m][2]), "+v"(acc[m][3]),
                              "+v"(acc[m][4]), "+v"(acc[m][5]), "+v"(acc[m][6]), "+v"(acc[m][7]));
        }
    }
    if (wid == 1) STAMP(6);
    if (wid == 0) {
        const char* ap = lane < NR ? ws + AMAX_OFF + ((g & (AMAX_REP - 1)) * NR + lane) * 128 : ws + TMO_OFF - 4;
        const u64 t0 = __builtin_amdgcn_s_memrealtime(); unsigned sp = 0;
        u64 v = 0;
        for (;;) {
            if (lane <= NR) v = ld_sys64(ap);
            if ((__ballot((int)((unsigned)(v >> 32) - call) >= 0) & 0xffu) == 0xffu) break;
            if (give_up(ws, call, t0, sp)) { v = (u64)call << 32; break; }
        }
        STAMP(8);
        int t = lane < NR ? (int)(unsigned)v : ((unsigned)(v >> 32) == call ? NAN_BITS : 0);
        ROW_SHR_MAX(t, 1); ROW_SHR_MAX(t, 2); ROW_SHR_MAX(t, 4); ROW_SHR_MAX(t, 8);
        if (lane == NR) red[8] = t;
    }
    LGKM0_BARRIER();
    const float inv = 448.0f / __int_as_float(red[8]);
    if (wid == 0) STAMP(3);

    if (remote) {
        const float f = inv * __int_as_float((254 - efield) << 23);
        unsigned sent = 0;
        const unsigned hshift = (lane & 1) * 4, woff = piece_off(lane);
#define SEND_GATE() WAIT_VM(2)
#define SEND(cnt_, upto_) do { __builtin_amdgcn_s_waitcnt(0xC07F); asm volatile("" ::: "memory");                           \
            _Pragma("nounroll") for (; sent < (upto_); sent += 1024) {                                                                    \
                const unsigned o_ = sent + lane * 16;                                                                 \
                const f16x8 h0 = *(const LAS f16x8*)(lds + wid * LDS_WAVE + o_ * 2), h1 = *(const LAS f16x8*)(lds + wid * LDS_WAVE + o_ * 2 + 16); \
                u32x4 q;                                                                                              \
                _Pragma("unroll") for (int d = 0; d < 2; ++d) {                                                       \
                    int w0 = __builtin_amdgcn_cvt_pk_fp8_f32(__builtin_fmaf((float)h0[4 * d], f, 0.f), __builtin_fmaf((float)h0[4 * d + 1], f, 0.f), 0, false); \
                    q[d] = (unsigned)__builtin_amdgcn_cvt_pk_fp8_f32(__builtin_fmaf((float)h0[4 * d + 2], f, 0.f), __builtin_fmaf((float)h0[4 * d + 3], f, 0.f), w0, true); \
                    int w1 = __builtin_amdgcn_cvt_pk_fp8_f32(__builtin_fmaf((float)h1[4 * d], f, 0.f), __builtin_fmaf((float)h1[4 * d + 1], f, 0.f), 0, false); \
                    q[2 + d] = (unsigned)__builtin_amdgcn_cvt_pk_fp8_f32(__builtin_fmaf((float)h1[4 * d + 2], f, 0.f), __builtin_fmaf((float)h1[4 * d + 3], f, 0.f), w1, true); \
                }                                                                                                     \
                  \
                const unsigned h_ = (unsigned)__builtin_amdgcn_update_dpp(0, (int)q[3], 0xf5, 0xf, 0xf, true) >> hshift; \
                _Pragma("unroll") for (int d = 0; d < 4; ++d) q[d] |= (h_ << (7 - d)) & 0x80808080u;                  \
                SEND_GATE();                                                                                          \
                if ((o_ & ~31u) < (cnt_)) {                                                                             \
                    const unsigned w_ = 1024 + (sent >> 10) * PIECE_BYTES + woff;                                     \
                    if (lane & 1) __builtin_amdgcn_raw_buffer_store_b96((u32x3){q[0], q[1], q[2]}, sls, w_, 0, AUX_SYS); \
                    else __builtin_amdgcn_raw_buffer_store_b128(q, sls, w_, 0, AUX_SYS);                              \
                }                                                                                                     \
            } } while (0)
        if (LATE_FLAG && wid == 0) __builtin_amdgcn_raw_buffer_store_b128(mk, sls, lane * 16, 0, AUX_SYS);
        SEND(last[4], last[4] & ~1023u);
        PACK8(5); PACK8(6); PACK8(7); PACK8(8);
        if (LATE_FLAG) { if (sent == 1024u) WAIT_VM(2); else WAIT_VM(0); if (lane == 0) st_sys(of, call); }
        SEND(last[8], last[8] & ~1023u);
        PACK8(9); PACK8(10); PACK8(11); PACK8(12);
        SEND(last[12], last[12] & ~1023u);
        PACK8(13); PACK8(14); PACK8(15);
        SEND(last[15], last[15]);
        WAIT_VM(0);
        if (lane == 0) st_sys64(of + 8, call, red[8] == NAN_BITS ? call : 0u);
    } else {
        const __amdgpu_buffer_rsrc_t ors = rsrc_of(outp, (MM / NR) * NN);
        const unsigned ob = out_base(rank, tile, wid, lane);
#pragma unroll
        for (int m = 0; m < 4; ++m)
#pragma unroll
            for (int j = 0; j < 4; ++j) {
                u32x2 o;
                int lo = __builtin_amdgcn_cvt_pk_fp8_f32(acc[m][0][j] * inv, acc[m][1][j] * inv, 0, false);
                o[0] = (unsigned)__builtin_amdgcn_cvt_pk_fp8_f32(acc[m][2][j] * inv, acc[m][3][j] * inv, lo, true);
                int hi = __builtin_amdgcn_cvt_pk_fp8_f32(acc[m][4][j] * inv, acc[m][5][j] * inv, 0, false);
                o[1] = (unsigned)__builtin_amdgcn_cvt_pk_fp8_f32(acc[m][6][j] * inv, acc[m][7][j] * inv, hi, true);
                __builtin_amdgcn_raw_buffer_store_b64(o, ors, ob + (unsigned)((m * 16 + j) * NN), 0, AUX_OUT);
            }
    }
    if (wid == 0) STAMP(4);
    if (wid == 7) STAMP(5);
}

}

extern "C" void kernel_launch(int rank, int nranks, ncclComm_t comm,
                              hipStream_t stream,
                              void* const* d_in, const int* in_sizes, int n_in,
                              void* d_out, int out_size,
                              const int* in_dtypes, int out_dtype,
                              void* const* d_out_all,
                              void* const* d_ws_all, size_t ws_bytes) {
    (void)comm; (void)in_dtypes; (void)out_dtype; (void)d_out_all;
    if (nranks != NR || n_in < 2 || (unsigned)rank >= (unsigned)NR || ws_bytes < WS_NEED ||
        in_sizes[0] != MM * KSH || in_sizes[1] != KSH * NN || out_size != (MM / NR) * NN) {
        fprintf(stderr, "kernel_launch: unexpected problem shape\n");
        return;
    }
    Params p;
    p.x = (const char*)d_in[0];
    p.w = (const char*)d_in[1];
    for (int i = 0; i < NR; ++i) p.ws[i] = (char*)d_ws_all[(rank + i) % NR];
    p.out = (char*)d_out;
    p.rank = rank; p.pad = 0;
    hipLaunchKernelGGL(fused_gemm_relu_q8, dim3(GRID), dim3(BLOCK), 0, stream, p);
    HIP_CHECK(hipGetLastError());
}
```

```cpp
#include <hip/hip_runtime.h>
#include <rccl/rccl.h>
#include <cstdio>
#include <cstdint>

#define HIP_CHECK(expr) do { hipError_t _e = (expr); if (_e != hipSuccess) { \
    fprintf(stderr, "HIP error %s at %s:%d\n", hipGetErrorString(_e), __FILE__, __LINE__); \
    return; } } while (0)

namespace {

typedef short bf16x8 __attribute__((ext_vector_type(8)));
typedef _Float16 f16x8 __attribute__((ext_vector_type(8)));
typedef float f32x4 __attribute__((ext_vector_type(4)));
typedef unsigned u32x4 __attribute__((ext_vector_type(4)));
typedef unsigned u32x3 __attribute__((ext_vector_type(3)));
typedef unsigned u32x2 __attribute__((ext_vector_type(2)));
typedef unsigned long long u64;
#define LAS __attribute__((address_space(3)))
#define GAS __attribute__((address_space(1)))

constexpr int NR = 8;
constexpr int MM = 4096, KSH = 512, NN = 16384, NCOL = NN / NR;
constexpr int NT = KSH / 8;
constexpr int NBLK = 24;
#define PULL_PER_LINK 18
#define AMAX_REP 8
#define AUX_OUT 17
#define SEND_DEPTH 2
#define JMODE 2
#define JORDER(k) ((k) % 3 == 2 ? 16 + (k) / 3 : (k) / 3 * 2 + (k) % 3)
#define SLEEP_K 2
#define ROT 1
#define SLOT(src, blk) ((((src) + ROT * (blk)) & 7) * 4096)
#define LATE_FLAG 1
#define STAGE_AUX 17
#define POLL_POS 0
#define TAKE_COL 1
#define POLL_AHEAD 6
#define PACK_DELAY 60
constexpr int Q_JOBS = NT * NBLK;
constexpr int Q_TICKETS = Q_JOBS + PULL_PER_LINK;
constexpr int N_COMM = 128;
constexpr int N_PULL = (NR - 1) * PULL_PER_LINK;
constexpr int N_GEMM = 128;
constexpr int GRID = N_COMM + N_GEMM;
constexpr int BLOCK = 512;
constexpr int N_PACKW = N_COMM * 6;
constexpr int NGRP = NT / 8;
static_assert(N_PULL <= 2 * N_COMM, "pull waves are waves 0-1 of the comm workgroups");

constexpr unsigned XT_T = NR * MM * 16, WT_T = NR * NCOL * 16;
constexpr unsigned XT_OFF = 0;
constexpr unsigned WT_OFF = XT_OFF + NT * XT_T;
constexpr unsigned WS_OFF = WT_OFF + NT * WT_T;
constexpr unsigned OB_SLOT = 1024 + 8192;
constexpr unsigned OB_OFF = WS_OFF + NT * WT_T;
constexpr unsigned CTL_OFF = OB_OFF + NR * 128 * OB_SLOT;
constexpr unsigned PK_OFF = CTL_OFF;
constexpr unsigned CNT_OFF = PK_OFF + NR * 8192;
constexpr unsigned FL_OFF = CNT_OFF + NGRP * 9 * 128;
constexpr unsigned Q_OFF = FL_OFF + NBLK * NR * NT * 4;
constexpr unsigned TMAX_OFF = Q_OFF + NR * 128;
constexpr unsigned AMAX_OFF = TMAX_OFF + N_GEMM * 128;
constexpr unsigned OF_OFF = AMAX_OFF + 64 * 128;
constexpr unsigned CTR_OFF = OF_OFF + NR * 128 * 128;
constexpr unsigned TMO_OFF = CTR_OFF + GRID * 128 + 4;
constexpr unsigned DBG_OFF = CTR_OFF + GRID * 128 + 128;
constexpr size_t WS_NEED = DBG_OFF + 524288;

constexpr u64 DEADLINE = 50000000ull;
constexpr int AUX_SYS = 17;

constexpr int NSTEP = 2 * NT;
constexpr int NSLOT = 5;
constexpr int LDS_SLOT = 32768;
constexpr int LDS_BYTES = NSLOT * LDS_SLOT;
constexpr int LDS_WAVE = 16384;
constexpr int LDS_MISC = 8 * LDS_WAVE;

struct Params {
    const char* x; const char* w;
    int rank, pad;
    char* ws[NR];
    char* out;
};

__device__ __forceinline__ __amdgpu_buffer_rsrc_t rsrc_of(const void* p, unsigned bytes) {
    return __builtin_amdgcn_make_buffer_rsrc(const_cast<void*>(p), 0, bytes, 0x00020000);
}
__device__ __forceinline__ unsigned ld_sys(const void* p) {
    return __hip_atomic_load((const GAS unsigned*)p, __ATOMIC_RELAXED, __HIP_MEMORY_SCOPE_SYSTEM);
}
__device__ __forceinline__ u64 ld_sys64(const void* p) {
    return __hip_atomic_load((const GAS u64*)p, __ATOMIC_RELAXED, __HIP_MEMORY_SCOPE_SYSTEM);
}
__device__ __forceinline__ void st_sys(void* p, unsigned v) {
    __hip_atomic_store((GAS unsigned*)p, v, __ATOMIC_RELAXED, __HIP_MEMORY_SCOPE_SYSTEM);
}
__device__ __forceinline__ void st_sys64(void* p, unsigned lo, unsigned hi) {
    __hip_atomic_store((GAS u64*)p, ((u64)hi << 32) | lo, __ATOMIC_RELAXED, __HIP_MEMORY_SCOPE_SYSTEM);
}
__device__ __forceinline__ unsigned ld_agent(const void* p) {
    return __hip_atomic_load((const GAS unsigned*)p, __ATOMIC_RELAXED, __HIP_MEMORY_SCOPE_AGENT);
}
__device__ __forceinline__ unsigned add_agent_lane0(void* p, int lane) {
    unsigned v = 0, z = 0;
    asm volatile("" : "+v"(z));
    if (lane == 0) v = __hip_atomic_fetch_add((GAS unsigned*)((char*)p + z), 1u, __ATOMIC_RELAXED, __HIP_MEMORY_SCOPE_AGENT);
    return v;
}
__device__ __forceinline__ u32x4 transposed(const u32x4 (&v)[8], int i) {
    u32x4 o;
#pragma unroll
    for (int d = 0; d < 4; ++d)
        o[d] = __builtin_amdgcn_perm(v[2 * d + 1][i >> 1], v[2 * d][i >> 1], (i & 1) ? 0x07060302u : 0x05040100u);
    return o;
}
constexpr int NAN_BITS = 0x7fc00000;
#define ROW_SHR_MAX(v, n) v = max(v, __builtin_amdgcn_update_dpp(0, v, 0x110 + (n), 0xf, 0xf, true))
__device__ __forceinline__ int wave_max(int v) {
    ROW_SHR_MAX(v, 1); ROW_SHR_MAX(v, 2); ROW_SHR_MAX(v, 4); ROW_SHR_MAX(v, 8);
    return max(max(__builtin_amdgcn_readlane(v, 15), __builtin_amdgcn_readlane(v, 31)),
               max(__builtin_amdgcn_readlane(v, 47), __builtin_amdgcn_readlane(v, 63)));
}
__device__ __forceinline__ unsigned sign_byte(const u32x4& mk, int u) { return (mk[u >> 2] >> (24 - 8 * (u & 3))) & 0xffu; }
__device__ __forceinline__ void stream_layout(const u32x4& mk, unsigned (&first)[16], unsigned (&last)[16]) {
    unsigned base = 0;
#pragma unroll
    for (int u = 0; u < 16; u += 2) {
        const int c = 0x00080008 - (int)(__builtin_popcount(sign_byte(mk, u)) | (__builtin_popcount(sign_byte(mk, u + 1)) << 16));
        int v = c;
        v += __builtin_amdgcn_update_dpp(0, v, 0x111, 0xf, 0xf, true);
        v += __builtin_amdgcn_update_dpp(0, v, 0x112, 0xf, 0xf, true);
        v += __builtin_amdgcn_update_dpp(0, v, 0x114, 0xf, 0xf, true);
        v += __builtin_amdgcn_update_dpp(0, v, 0x118, 0xf, 0xf, true);
        v += __builtin_amdgcn_update_dpp(0, v, 0x142, 0xa, 0xf, false);
        v += __builtin_amdgcn_update_dpp(0, v, 0x143, 0xc, 0xf, false);
        const unsigned tot = (unsigned)__builtin_amdgcn_readlane(v, 63), exc = (unsigned)(v - c);
        first[u] = base + (exc & 0xffffu); base += tot & 0xffffu; last[u] = base;
        first[u + 1] = base + (exc >> 16); base += tot >> 16; last[u + 1] = base;
    }
}
template <int SLEEP = 1>
__device__ __forceinline__ bool give_up(char* ws, unsigned call, u64 t0, unsigned& spins) {
    bool stop = __builtin_amdgcn_s_memrealtime() - t0 > DEADLINE;
    if ((++spins & 63u) == 0u) stop = stop || __builtin_amdgcn_readfirstlane(ld_agent(ws + TMO_OFF)) == call;
    if (stop) {
        __hip_atomic_store((GAS unsigned*)(ws + TMO_OFF), call, __ATOMIC_RELAXED, __HIP_MEMORY_SCOPE_AGENT);
        return true;
    }
    __builtin_amdgcn_s_sleep(SLEEP);
    return false;
}
template <int SLEEP>
__device__ __forceinline__ bool await_word(char* ws, const char* p, unsigned call) {
    const u64 t0 = __builtin_amdgcn_s_memrealtime(); unsigned sp = 0;
    while ((int)(__builtin_amdgcn_readfirstlane(ld_sys(p)) - call) < 0)
        if (give_up<SLEEP>(ws, call, t0, sp)) return false;
    return true;
}
__device__ __forceinline__ unsigned fl_word(int blk, int src, int t) { return FL_OFF + (unsigned)(((blk * 4 + (src & 3)) * NT + t) * 2 + (src >> 2)) * 4u; }
__device__ __forceinline__ int steps_known(unsigned f0, unsigned f1, unsigned call) {
    const u64 lo = __ballot((int)(f0 - call) >= 0), hi = __ballot((int)(f1 - call) >= 0);
    const u64 miss = ~(lo & hi);
    if (miss == 0) return NSTEP;
    const int c = __builtin_ctzll(miss);
    return 2 * c + (int)((lo >> c) & 1);
}
__device__ __forceinline__ unsigned out_base(int src, int tile, int wave, int lane) {
    return (unsigned)(((tile >> 3) * 256 + (wave >> 1) * 64 + (lane >> 4) * 4) * NN + src * NCOL + (tile & 7) * 256 + (wave & 1) * 128 + (lane & 15) * 8);
}
constexpr unsigned PIECE_BYTES = 896;
__device__ __forceinline__ unsigned piece_off(int lane) { return lane & 1 ? 512 + (lane >> 1) * 12 : (lane >> 1) * 16; }
#define WAIT_VM(n) asm volatile("s_waitcnt vmcnt(" #n ")" ::: "memory")
#define LGKM0_BARRIER() do { __builtin_amdgcn_s_waitcnt(0xC07F); asm volatile("" ::: "memory");                       \
        __builtin_amdgcn_s_barrier(); asm volatile("" ::: "memory"); } while (0)
#define STAMP(i) do {} while (0)

__global__ void __launch_bounds__(BLOCK, 2) fused_gemm_relu_q8(const Params p) {
    __shared__ __attribute__((aligned(1024))) char smem[LDS_BYTES];
    LAS char* const lds = (LAS char*)smem;
    const int hb = blockIdx.x;
    const int lane = threadIdx.x & 63;
    const int wid = __builtin_amdgcn_readfirstlane(threadIdx.x >> 6);
    int rank = p.rank;
    char* ws = p.ws[0];
    const char* px = p.x;
    const char* pw = p.w;
    const u64 t_start = __builtin_amdgcn_s_memrealtime();
    asm volatile("" : "+s"(rank), "+s"(ws), "+s"(px), "+s"(pw));
    const unsigned ticket = add_agent_lane0(ws + CTR_OFF + (size_t)hb * 128 + wid * 4, lane);
    const int b = hb & (N_COMM - 1);

    if (hb < N_COMM) {
        const __amdgpu_buffer_rsrc_t wsr = rsrc_of(ws, CTL_OFF);
        const int cw = wid * N_COMM + b;
        const int dbg = 128 + cw;
        (void)dbg;
        u64 all_t = 0, ev_t = 0;
        if (wid < 2) {
            if (threadIdx.x == 0) *(LAS unsigned*)(lds + LDS_MISC) = 0u;
            LGKM0_BARRIER();
            if (cw == 255) {
                const unsigned call = __builtin_amdgcn_readfirstlane(ticket) + 1u;
                if (lane < NR) st_sys(p.ws[lane & 7] + DBG_OFF + rank * 128, call);
                const u64 t0 = __builtin_amdgcn_s_memrealtime(); unsigned sp = 0;
                for (;;) { const unsigned v = ld_sys(ws + DBG_OFF + (lane & 7) * 128); if (__all((int)(v - call) >= 0)) break; if (give_up(ws, call, t0, sp)) break; }
                all_t = __builtin_amdgcn_s_memrealtime();
                for (;;) { const unsigned v = ld_sys(ws + fl_word((lane >> 3) * 3, lane & 7, 63)); if (__all((int)(v - call) >= 0)) break; if (give_up(ws, call, t0, sp)) break; }
                ev_t = __builtin_amdgcn_s_memrealtime();
            }
            if (cw < N_PULL) {
                const unsigned s = (unsigned)(rank + 1 + cw % (NR - 1)) & 7u;
                const __amdgpu_buffer_rsrc_t srs = rsrc_of(p.ws[1 + cw % (NR - 1)], CTL_OFF);
                char* const queue = ws + Q_OFF + s * 128;
                unsigned tk = add_agent_lane0(queue, lane);
                u32x4 cur[4];
                const unsigned call = __builtin_amdgcn_readfirstlane(ticket) + 1u;
                const unsigned qbase = (call - 1u) * Q_TICKETS;
                const char* const pk = ws + PK_OFF + s * 8192;
                unsigned pv = call - 1u, pg = call - 1u;
#define JOB_OF(n_, t_, j_, so_, do_) const int t_ = (int)(n_) / NBLK, j_ = JORDER((int)(n_) % NBLK);                           \
                const unsigned do_ = (j_ < 16 ? XT_OFF + t_ * XT_T + j_ * 32768 : WT_OFF + t_ * WT_T + (j_ - 16) * 32768) + SLOT(s, j_) + lane * 16; \
                const unsigned so_ = j_ < 16 ? do_ : WS_OFF + t_ * WT_T + rank * 32768 + (j_ - 16) * 4096 + lane * 16
#define SEEN(v_, l_) ((int)(__builtin_amdgcn_readlane(v_, l_) - call) >= 0)
#define WAIT_PACKED(t_, j_) do { const int sg_ = ((t_) >> 3) + ((j_) < 16 ? 0 : 8);                                    \
                if (!SEEN(pg, sg_)) {                                                                                  \
                    const u64 t0_ = __builtin_amdgcn_s_memrealtime(); unsigned sp_ = 0;                                \
                    if ((t_) < 8) { const int sl_ = (j_) < 16 ? (j_) : 24 + (t_) * 4 + ((j_) >> 1);                    \
                        if (!SEEN(pv, sl_)) {                            \
                            pv = ld_sys(pk + lane * 128); pg = ld_sys(pk + 16 * 128 + (lane & 15) * 4);                \
                            while (!SEEN(pv, sl_)) {                                                                   \
                                if (give_up(ws, call, t0_, sp_)) break;                                                \
                                if (lane == sl_) pv = ld_sys(pk + lane * 128); } }                                     \
                    } else {                                                                                           \
                        for (;;) { pg = ld_sys(pk + 16 * 128 + (lane & 15) * 4);                                       \
                            if (SEEN(pg, sg_) || give_up(ws, call, t0_, sp_)) break; } }                               \
                } } while (0)
#define PTAIL() do {} while (0)
                unsigned n = __builtin_amdgcn_readfirstlane(tk) - qbase;
                if (n < (unsigned)Q_JOBS) {
                    JOB_OF(n, t, j, so, dof);
                    WAIT_PACKED(t, j);
                    STAMP(2);
                    tk = add_agent_lane0(queue, lane);
#pragma unroll
                    for (int i = 0; i < 4; ++i) cur[i] = __builtin_amdgcn_raw_buffer_load_b128(srs, so + i * 1024, 0, AUX_SYS);
                    unsigned dcur = dof, fcur = fl_word(j, s, t);
                    for (;;) {
#pragma unroll
                        for (int i = 0; i < 4; ++i) __builtin_amdgcn_raw_buffer_store_b128(cur[i], wsr, dcur + i * 1024, 0, AUX_SYS);
                        asm volatile("" ::: "memory");
                        n = __builtin_amdgcn_readfirstlane(tk) - qbase;
                        char* const flag = ws + fcur;
                        if (n >= (unsigned)Q_JOBS) {
                            WAIT_VM(0);
                            if (lane == 0) st_sys(flag, call);
                            PTAIL();
                            break;
                        }
                        JOB_OF(n, t2, j2, so2, dof2);
                        WAIT_PACKED(t2, j2);
                        tk = add_agent_lane0(queue, lane);
#pragma unroll
                        for (int i = 0; i < 4; ++i) cur[i] = __builtin_amdgcn_raw_buffer_load_b128(srs, so2 + i * 1024, 0, AUX_SYS);
                        dcur = dof2; fcur = fl_word(j2, s, t2);
                        WAIT_VM(5);
                        if (lane == 0) st_sys(flag, call);
                        PTAIL();
                    }
                }
                STAMP(1);
            }
        } else {
            const __amdgpu_buffer_rsrc_t xr = rsrc_of(px, MM * KSH * 2);
            const __amdgpu_buffer_rsrc_t wr_ = rsrc_of(pw, KSH * NN * 2);
            bool met = false;
            bool x_todo = wid >= 4;
            for (int n = (wid - 2) * N_COMM + b; n < NT * 32 || x_todo;) {
                const bool do_x = x_todo && ((b >> 4) <= (n >> 8) || n >= NT * 32);
                const int g = do_x ? b >> 4 : n >> 8;
                const int delay = (do_x ? g * 8 : n >> 5) * PACK_DELAY;
                if (!met && delay > 0) { __builtin_amdgcn_s_barrier(); met = true; }
                while (__builtin_amdgcn_s_memrealtime() - t_start < (u64)delay) __builtin_amdgcn_s_sleep(2);
                u32x4 v[8];
                if (do_x) {
                    const unsigned rb = (wid - 4) * 64 + (lane & 7), piece = lane >> 3;
#pragma unroll
                    for (int i = 0; i < 8; ++i) v[i] = __builtin_amdgcn_raw_buffer_load_b128(xr, ((b & 15) * 256 + rb + i * 8) * (KSH * 2) + g * 128 + piece * 16, 0, 0);
                    if (!met) { __builtin_amdgcn_s_barrier(); met = true; }
                    const unsigned d0 = XT_OFF + (g * 8 + piece) * XT_T + (b & 15) * 32768 + SLOT(rank, b & 15) + rb * 16;
#pragma unroll
                    for (int i = 0; i < 8; ++i) __builtin_amdgcn_raw_buffer_store_b128(v[i], wsr, d0 + i * 128, 0, AUX_SYS);
                    char* fl = lane < 8 ? p.ws[lane & 7] + PK_OFF + rank * 8192 + (b & 15) * 128
                                        : ws + fl_word(b & 15, rank, g * 8 + (lane & 7));
                    asm volatile("s_waitcnt vmcnt(0)" : "+v"(fl) :: "memory");
                    unsigned old = 0;
                    if (lane == 0) old = __hip_atomic_fetch_add((LAS unsigned*)(lds + LDS_MISC), 1u, __ATOMIC_RELAXED, __HIP_MEMORY_SCOPE_WORKGROUP);
                    const unsigned call = __builtin_amdgcn_readfirstlane(ticket) + 1u;
                    if (__builtin_amdgcn_readfirstlane(old) == 3u) {
                        if (lane >= (g == 0 ? 1 : 8) && lane < 16) st_sys(fl, call);
                        if (__builtin_amdgcn_readfirstlane(add_agent_lane0(ws + CNT_OFF + g * 9 * 128, lane)) + 1u == call * 16u && lane >= 1 && lane < 8)
                            st_sys(fl + (16 - (b & 15)) * 128 + g * 4, call);
                    }
                    x_todo = false;
                } else {
                    const int t = n >> 5, cb = n & 31;
                    const unsigned dest = (unsigned)cb >> 2;
#pragma unroll
                    for (int i = 0; i < 8; ++i) v[i] = __builtin_amdgcn_raw_buffer_load_b128(wr_, (unsigned)((t * 8 + i) * (NN * 2) + cb * 1024 + lane * 16), 0, 0);
                    if (!met) { __builtin_amdgcn_s_barrier(); met = true; }
                    const unsigned d0 = (dest == (unsigned)rank ? WT_OFF + t * WT_T + ((cb & 3) * 2 + (lane >> 5)) * 32768 + SLOT(rank, (cb & 3) * 2 + (lane >> 5))
                                                                : WS_OFF + t * WT_T + dest * 32768 + ((cb & 3) * 2 + (lane >> 5)) * 4096)
                                      + (((lane >> 4) & 1) * 128 + (lane & 15)) * 16;
#pragma unroll
                    for (int i = 0; i < 8; ++i) __builtin_amdgcn_raw_buffer_store_b128(transposed(v, i), wsr, d0 + i * 256, 0, AUX_SYS);
                    char* const dws = p.ws[(dest - rank) & 7u];
                    WAIT_VM(0);
                    const unsigned call = __builtin_amdgcn_readfirstlane(ticket) + 1u;
                    if (dest == (unsigned)rank) { if (lane < 2) st_sys(ws + fl_word(16 + (cb & 3) * 2 + lane, rank, t), call); }
                    else {
                        if (g == 0 && lane == 0) st_sys(dws + PK_OFF + rank * 8192 + (32 + t * 4 + (cb & 3)) * 128, call);
                        if (__builtin_amdgcn_readfirstlane(add_agent_lane0(ws + CNT_OFF + (g * 9 + 1 + dest) * 128, lane)) + 1u == call * 32u && lane == 0)
                            st_sys(dws + PK_OFF + rank * 8192 + 16 * 128 + (8 + g) * 4, call);
                    }
                    n += N_PACKW;
                }
            }
        }
        const unsigned call = __builtin_amdgcn_readfirstlane(ticket) + 1u;
        if (wid == 7) {
            if (b == N_COMM - 1) {
                char* amax_out = p.ws[lane & 7] + AMAX_OFF + ((lane >> 3) * NR + rank) * 128;
                asm volatile("" : "+v"(amax_out));
                const char* tm = ws + TMAX_OFF + lane * 256;
                const u64 t0 = __builtin_amdgcn_s_memrealtime(); unsigned sp = 0;
                int mx;
                for (;;) {
                    const u64 v0 = ld_sys64(tm), v1 = ld_sys64(tm + 128);
                    mx = max((int)(unsigned)v0, (int)(unsigned)v1);
                    if (__all((int)((unsigned)(v0 >> 32) - call) >= 0 && (int)((unsigned)(v1 >> 32) - call) >= 0)) break;
                    if (give_up(ws, call, t0, sp)) { mx = NAN_BITS; break; }
                }
                if (lane < 8 * AMAX_REP) st_sys64(amax_out, (unsigned)wave_max(mx), call);
                STAMP(6);
            }
            return;
        }
        {
            const int src = (rank + 1 + wid) & 7;
            const unsigned slot = OB_OFF + (unsigned)(src * 128 + b) * OB_SLOT;
            const char* const of = ws + OF_OFF + (src * 128 + b) * 128;
            const unsigned lbase = wid * LDS_WAVE;
#pragma unroll
            for (int e = 0; e < 4; ++e) {
                const unsigned idx = lane * 4 + e;
                u32x2 sel = {0u, 0u};
                unsigned rk = 0;
#pragma unroll
                for (int n = 0; n < 8; ++n) {
                    const bool neg = (idx >> (7 - n)) & 1u;
                    sel[n >> 2] |= (neg ? 0x0cu : rk) << (8 * (n & 3));
                    rk += neg ? 0u : 1u;
                }
                *(LAS u32x2*)(lds + lbase + 12288 + idx * 8) = sel;
            }
            char* outp = p.out;
            asm volatile("" : "+s"(outp));
            const unsigned ob = out_base(src, b >> 3, b & 7, lane);
            {
                bool ok = await_word<8>(ws, of, call);
                const u32x4 mk = __builtin_amdgcn_raw_buffer_load_b128(wsr, slot + lane * 16, 0, AUX_SYS);
                unsigned first[16], last[16];
                stream_layout(mk, first, last);
                u32x2 sel[16];
#pragma unroll
                for (int u = 0; u < 16; ++u) {
                    sel[u] = *(const LAS u32x2*)(lds + lbase + 12288 + sign_byte(mk, u) * 8);
                    first[u] += lbase;
                    asm volatile("" : "+v"(first[u]), "+v"(sel[u]));
                }
                STAMP(3);
                const unsigned cnt = last[15];
                const __amdgpu_buffer_rsrc_t ors = rsrc_of(outp, (MM / NR) * NN);
#define TAKE_PIECES(i0_, i1_) do { u32x4 pc[(i1_) - (i0_)];                                                            \
                    _Pragma("unroll") for (int i = (i0_); i < (i1_); ++i)                                              \
                        if (i * 1024u < cnt) pc[i - (i0_)] = __builtin_amdgcn_raw_buffer_load_b128(wsr, slot + 1024 + i * PIECE_BYTES + piece_off(lane), 0, AUX_SYS); \
                    fill = ok && !__any(ld_sys(lane & 1 ? ws + TMO_OFF : of + 12) == call) ? 0u : 0x7f7f7f7fu;          \
                    _Pragma("unroll") for (int i = (i0_); i < (i1_); ++i)                                              \
                        if (i * 1024u < cnt) {                                                                         \
                            u32x4 q = pc[i - (i0_)];                                                                   \
                            unsigned h = ((q[0] & 0x80808080u) >> 7) | ((q[1] & 0x80808080u) >> 6) | ((q[2] & 0x80808080u) >> 5); \
                            const unsigned he = h | ((q[3] & 0x80808080u) >> 4);     \
                            h = (h << 4) | (unsigned)__builtin_amdgcn_update_dpp(0, (int)he, 0xa0, 0xf, 0xf, true);    \
                            q[0] &= 0x7f7f7f7fu; q[1] &= 0x7f7f7f7fu; q[2] &= 0x7f7f7f7fu;                             \
                            q[3] = lane & 1 ? h : q[3] & 0x7f7f7f7fu;                                                  \
                            *(LAS u32x4*)(lds + lbase + i * 1024 + lane * 16) = q;                                     \
                        } } while (0)
                unsigned fill;
                ok = await_word<1>(ws, of + 8, call) && ok;
                STAMP(4);
                TAKE_PIECES(0, 8);
                unsigned d[16][3];
#pragma unroll
                for (int u = 0; u < 16; ++u)
#pragma unroll
                    for (int i = 0; i < 3; ++i) d[u][i] = *(const LAS unsigned*)(lds + (first[u] & ~3u) + i * 4);
                __builtin_amdgcn_sched_barrier(0);
#pragma unroll
                for (int u = 0; u < 16; ++u) {
                    const unsigned lo = __builtin_amdgcn_alignbyte(d[u][1], d[u][0], first[u]), hi = __builtin_amdgcn_alignbyte(d[u][2], d[u][1], first[u]);
                    u32x2 o;
                    o[0] = __builtin_amdgcn_perm(hi, lo, sel[u][0]) | fill;
                    o[1] = __builtin_amdgcn_perm(hi, lo, sel[u][1]) | fill;
                    __builtin_amdgcn_raw_buffer_store_b64(o, ors, ob + (unsigned)(((u >> 2) * 16 + (u & 3)) * NN), 0, AUX_OUT);
                }
            }
            if (cw == 255) { long long c = (long long)(ev_t - all_t) - 10200; if (c < 0) c = 0; if (c > 5000) c = 5000;
                while (__builtin_amdgcn_s_memrealtime() - all_t < (u64)(13500 + 10 * c)) __builtin_amdgcn_s_sleep(8); }
        }
        return;
    }

    const int g = b;
    const int dbg = g;
    (void)dbg;
    const int pm = (g >> 4) * 2 + (g & 1);
    const int pn = ((g & 7) >> 1) * 2 + ((g >> 3) & 1);
    const int fr = lane & 15, fq = lane >> 4, wr = wid >> 1, wc = wid & 1;
    if (wid == 0) STAMP(0);
    const int owner = pm >> 1, tile = (pm & 1) * 8 + pn;
    char* ows = p.ws[(owner - rank) & 7];
    char* outp = p.out;
    asm volatile("" : "+s"(ows), "+s"(outp));
    const unsigned call = __builtin_amdgcn_readfirstlane(ticket) + 1u;

    const int op = wid >> 2;
    const char* const pollp = ws + fl_word(op ? 16 + pn : pm, wid & 3, lane);
    const unsigned unit = op ? WT_T : XT_T;
    const char* const gbase = ws + (op ? WT_OFF + pn * 32768 : XT_OFF + pm * 32768) + lane * 16;
    const unsigned slot0 = SLOT(wid & 3, op ? pn : pm);
    LAS char* const ldst = lds + wid * 4096;
    const int aoff = fq * 4096 + (wr * 64 + fr) * 16;
    const int boff = 16384 + fq * 4096 + (wc * 128 + fr) * 16;

    int known = 0;
    int next = 0, nslot = 0;
    u32x2 pvn = {0u, 0u};
#define STAGE_NEXT() do { const char* g_ = gbase + ((slot0 ^ ((next & 1) << 14)) + (unsigned)(next >> 1) * unit);              \
        LAS char* l_ = ldst + nslot * LDS_SLOT;                                                                       \
        __builtin_amdgcn_global_load_lds((const GAS unsigned*)g_, (LAS unsigned*)l_, 16, 0, STAGE_AUX);                       \
        __builtin_amdgcn_global_load_lds((const GAS unsigned*)g_, (LAS unsigned*)l_, 16, 1024, STAGE_AUX);                    \
        __builtin_amdgcn_global_load_lds((const GAS unsigned*)g_, (LAS unsigned*)l_, 16, 2048, STAGE_AUX);                    \
        __builtin_amdgcn_global_load_lds((const GAS unsigned*)g_, (LAS unsigned*)l_, 16, 3072, STAGE_AUX);                    \
        ++next; nslot = nslot == NSLOT - 1 ? 0 : nslot + 1; } while (0)
#define POLL_ISSUE() asm volatile("global_load_dwordx2 %0, %1, off sc0 sc1" : "+v"(pvn) : "v"(pollp) : "memory")
#define POLL_TAKE() do { u32x2 f_; asm volatile("s_waitcnt vmcnt(0)\n\tv_mov_b64 %0, %1" : "=v"(f_) : "v"(pvn) : "memory");    \
        known = steps_known(f_[0], f_[1], call); } while (0)
#define AWAIT_NEXT() do { if (next >= known) {                                                                        \
            const u64 t0_ = __builtin_amdgcn_s_memrealtime(); unsigned sp_ = 0;                                       \
            for (;;) { const u64 f_ = ld_sys64(pollp); known = steps_known((unsigned)f_, (unsigned)(f_ >> 32), call); \
                if (next < known) break;                                                                              \
                if (give_up<SLEEP_K>(ws, call, t0_, sp_)) { known = NSTEP; break; } } } } while (0)
#define MFMA_COLS(A_, B_, n0_, n1_) do { __builtin_amdgcn_sched_barrier(0);                                           \
        _Pragma("unroll") for (int n = (n0_); n < (n1_); ++n)                                                         \
            _Pragma("unroll") for (int m = 0; m < 4; ++m)                                                             \
                acc[m][n] = __builtin_amdgcn_mfma_f32_16x16x32_bf16(A_[m], B_[n], acc[m][n], 0, 0, 0);                \
        __builtin_amdgcn_sched_barrier(0); } while (0)
#define RD_A(A_, m_) A_[m_] = *(const LAS bf16x8*)(lb_ + aoff + (m_) * 256)
#define RD_B(B_, n_) B_[n_] = *(const LAS bf16x8*)(lb_ + boff + (n_) * 256)
#define TAIL_REC(st_) do {} while (0)
#define KSTEP(st_, A_, B_, AN_, BN_) do {                                                                             \
        MFMA_COLS(A_, B_, 0, TAKE_COL);                                                                               \
        if (polled) POLL_TAKE();                                          \
        const int had_ = next;                                                                                        \
        const int lim_ = min((st_) + NSLOT, known);                                   \
        if (next < lim_) { STAGE_NEXT(); if (next < lim_) STAGE_NEXT(); }                                             \
        const bool full_ = (polled ? had_ + NSLOT - 2 : next) >= (st_) + NSLOT;                                       \
        MFMA_COLS(A_, B_, TAKE_COL, 4);                                                                                      \
        if (!full_) {                                                              \
            if (next == (st_) + 1 && next < NSTEP) {                                                                  \
                AWAIT_NEXT(); STAGE_NEXT();                                                                           \
                if (next < known) STAGE_NEXT();                                                                       \
            }                                                                                                         \
            const int k_ = next - (st_) - 2;                                                                          \
            if (k_ <= 0) WAIT_VM(0); else if (k_ == 1) WAIT_VM(4); else WAIT_VM(8);                                   \
        }                                                                                                             \
        WAIT_VM(12);                                                                                                  \
        LGKM0_BARRIER();                                                                                              \
        TAIL_REC(st_);                                                                                                \
        const LAS char* lb_ = lds + rslot * LDS_SLOT;               \
        RD_A(AN_, 0); RD_A(AN_, 1); RD_A(AN_, 2); RD_A(AN_, 3);                                                       \
        if (POLL_POS == 0) { polled = known < min(next + POLL_AHEAD, NSTEP); if (polled) POLL_ISSUE(); }              \
        MFMA_COLS(A_, B_, 4, 5);                                                                                      \
        RD_B(BN_, 0); RD_B(BN_, 1); RD_B(BN_, 2); RD_B(BN_, 3);                                                       \
        MFMA_COLS(A_, B_, 5, 6);                                                                                      \
        RD_B(BN_, 4); RD_B(BN_, 5); RD_B(BN_, 6); RD_B(BN_, 7);                                                       \
        rslot = rslot == NSLOT - 1 ? 0 : rslot + 1;                                                                   \
        if (POLL_POS == 2) { polled = known < min(next + POLL_AHEAD, NSTEP); if (polled) POLL_ISSUE(); }              \
        MFMA_COLS(A_, B_, 6, 8); } while (0)

    f32x4 acc[4][8];
#pragma unroll
    for (int m = 0; m < 4; ++m)
#pragma unroll
        for (int n = 0; n < 8; ++n) acc[m][n] = (f32x4){0.f, 0.f, 0.f, 0.f};

    AWAIT_NEXT(); STAGE_NEXT();
    WAIT_VM(0);
    LGKM0_BARRIER();
    int rslot = 1;
    bool polled = false;
    bf16x8 a0[4], b0[8], a1[4], b1[8];
    {
        const LAS char* lb_ = lds;
#pragma unroll
        for (int m = 0; m < 4; ++m) RD_A(a0, m);
#pragma unroll
        for (int n = 0; n < 8; ++n) RD_B(b0, n);
    }
    if (wid == 0) STAMP(1);
    for (int st = 0; st < NSTEP; st += 2) {
        KSTEP(st, a0, b0, a1, b1);
        KSTEP(st + 1, a1, b1, a0, b0);
    }
    if (wid == 0) STAMP(2);

    int mx = 0;
#pragma unroll
    for (int m = 0; m < 4; ++m)
#pragma unroll
        for (int n = 0; n < 8; ++n)
#pragma unroll
            for (int j = 0; j < 4; ++j) mx = max(mx, __float_as_int(acc[m][n][j]));
    mx = wave_max(mx);
    LAS int* red = (LAS int*)(lds + LDS_MISC);
    if (lane == 0) red[wid] = mx;
    LGKM0_BARRIER();
    if (wid == 0) {
        int t = red[lane & 7];
        ROW_SHR_MAX(t, 1); ROW_SHR_MAX(t, 2); ROW_SHR_MAX(t, 4);
        if (lane == 7) st_sys64(ws + TMAX_OFF + g * 128, (unsigned)t, call);
        STAMP(7);
    }

    const bool remote = owner != rank;
    const __amdgpu_buffer_rsrc_t sls = rsrc_of(ows + OB_OFF + (unsigned)(rank * 128 + tile * 8 + wid) * OB_SLOT, OB_SLOT);
    char* const of = ows + OF_OFF + (rank * 128 + tile * 8 + wid) * 128;
    const int efield = min(268 - (mx >> 23), 253);
    const float sc = __int_as_float(efield << 23);
    const unsigned pos0 = (unsigned)(size_t)(lds + wid * LDS_WAVE);
    const unsigned spill = (unsigned)(size_t)(lds + LDS_MISC + 64);
    unsigned first[16], last[16];
#define VAL(u_, n_) acc[(u_) >> 2][n_][(u_) & 3]
#define PACK1(n_) "v_ashrrev_i32 %[s], 31, %[v" #n_ "]\n\tv_fma_mixlo_f16 %[h], %[v" #n_ "], %[sc], 0\n\t"              \
                  "v_bfi_b32 %[a], %[s], %[sp], %[p]\n\tv_lshl_add_u32 %[p], %[s], 1, %[p]\n\t"                      \
                  "ds_write_b16 %[a], %[h] offset:2*" #n_ "\n\t"
#define PACK8(u_) do { unsigned p_ = pos0 + 2 * first[u_], s_, h_, a_;                                                \
        asm volatile(PACK1(0) PACK1(1) PACK1(2) PACK1(3) PACK1(4) PACK1(5) PACK1(6) PACK1(7)                          \
                     : [p] "+v"(p_), [s] "=&v"(s_), [h] "=&v"(h_), [a] "=&v"(a_)                                      \
                     : [v0] "v"(VAL(u_, 0)), [v1] "v"(VAL(u_, 1)), [v2] "v"(VAL(u_, 2)), [v3] "v"(VAL(u_, 3)),        \
                       [v4] "v"(VAL(u_, 4)), [v5] "v"(VAL(u_, 5)), [v6] "v"(VAL(u_, 6)), [v7] "v"(VAL(u_, 7)),        \
                       [sc] "s"(sc), [sp] "v"(spill) : "memory"); } while (0)
    u32x4 mk = {0u, 0u, 0u, 0u};
    if (remote) {
#pragma unroll
        for (int r = 0; r < 128; ++r) mk[r >> 5] = __builtin_amdgcn_alignbit(mk[r >> 5], __float_as_uint(VAL(r >> 3, r & 7)), 31);
        if (!LATE_FLAG || wid != 0) __builtin_amdgcn_raw_buffer_store_b128(mk, sls, lane * 16, 0, AUX_SYS);
        stream_layout(mk, first, last);
        PACK8(0); PACK8(1); PACK8(2); PACK8(3); PACK8(4);
        if (!LATE_FLAG) { WAIT_VM(0); if (lane == 0) st_sys(of, call); }
    } else {
#pragma unroll
        for (int m = 0; m < 4; ++m) {
#pragma unroll
            for (int n = 0; n < 8; ++n)
#pragma unroll
                for (int j = 0; j < 4; ++j) acc[m][n][j] = __int_as_float(max(0, __float_as_int(acc[m][n][j])));
            asm volatile("" : "+v"(acc[m][0]), "+v"(acc[m][1]), "+v"(acc[m][2]), "+v"(acc[m][3]),
                              "+v"(acc[m][4]), "+v"(acc[m][5]), "+v"(acc[m][6]), "+v"(acc[m][7]));
        }
    }
    if (wid == 1) STAMP(6);
    if (wid == 0) {
        const char* ap = lane < NR ? ws + AMAX_OFF + ((g & (AMAX_REP - 1)) * NR + lane) * 128 : ws + TMO_OFF - 4;
        const u64 t0 = __builtin_amdgcn_s_memrealtime(); unsigned sp = 0;
        u64 v = 0;
        for (;;) {
            if (lane <= NR) v = ld_sys64(ap);
            if ((__ballot((int)((unsigned)(v >> 32) - call) >= 0) & 0xffu) == 0xffu) break;
            if (give_up(ws, call, t0, sp)) { v = (u64)call << 32; break; }
        }
        STAMP(8);
        int t = lane < NR ? (int)(unsigned)v : ((unsigned)(v >> 32) == call ? NAN_BITS : 0);
        ROW_SHR_MAX(t, 1); ROW_SHR_MAX(t, 2); ROW_SHR_MAX(t, 4); ROW_SHR_MAX(t, 8);
        if (lane == NR) red[8] = t;
    }
    LGKM0_BARRIER();
    const float inv = 448.0f / __int_as_float(red[8]);
    if (wid == 0) STAMP(3);

    if (remote) {
        const float f = inv * __int_as_float((254 - efield) << 23);
        unsigned sent = 0;
        const unsigned hshift = (lane & 1) * 4, woff = piece_off(lane);
#define SEND_GATE() WAIT_VM(2)
#define SEND(cnt_, upto_) do { __builtin_amdgcn_s_waitcnt(0xC07F); asm volatile("" ::: "memory");                           \
            _Pragma("nounroll") for (; sent < (upto_); sent += 1024) {                                                                    \
                const unsigned o_ = sent + lane * 16;                                                                 \
                const f16x8 h0 = *(const LAS f16x8*)(lds + wid * LDS_WAVE + o_ * 2), h1 = *(const LAS f16x8*)(lds + wid * LDS_WAVE + o_ * 2 + 16); \
                u32x4 q;                                                                                              \
                _Pragma("unroll") for (int d = 0; d < 2; ++d) {                                                       \
                    int w0 = __builtin_amdgcn_cvt_pk_fp8_f32(__builtin_fmaf((float)h0[4 * d], f, 0.f), __builtin_fmaf((float)h0[4 * d + 1], f, 0.f), 0, false); \
                    q[d] = (unsigned)__builtin_amdgcn_cvt_pk_fp8_f32(__builtin_fmaf((float)h0[4 * d + 2], f, 0.f), __builtin_fmaf((float)h0[4 * d + 3], f, 0.f), w0, true); \
                    int w1 = __builtin_amdgcn_cvt_pk_fp8_f32(__builtin_fmaf((float)h1[4 * d], f, 0.f), __builtin_fmaf((float)h1[4 * d + 1], f, 0.f), 0, false); \
                    q[2 + d] = (unsigned)__builtin_amdgcn_cvt_pk_fp8_f32(__builtin_fmaf((float)h1[4 * d + 2], f, 0.f), __builtin_fmaf((float)h1[4 * d + 3], f, 0.f), w1, true); \
                }                                                                                                     \
                  \
                const unsigned h_ = (unsigned)__builtin_amdgcn_update_dpp(0, (int)q[3], 0xf5, 0xf, 0xf, true) >> hshift; \
                _Pragma("unroll") for (int d = 0; d < 4; ++d) q[d] |= (h_ << (7 - d)) & 0x80808080u;                  \
                SEND_GATE();                                                                                          \
                if ((o_ & ~31u) < (cnt_)) {                                                                             \
                    const unsigned w_ = 1024 + (sent >> 10) * PIECE_BYTES + woff;                                     \
                    if (lane & 1) __builtin_amdgcn_raw_buffer_store_b96((u32x3){q[0], q[1], q[2]}, sls, w_, 0, AUX_SYS); \
                    else __builtin_amdgcn_raw_buffer_store_b128(q, sls, w_, 0, AUX_SYS);                              \
                }                                                                                                     \
            } } while (0)
        if (LATE_FLAG && wid == 0) __builtin_amdgcn_raw_buffer_store_b128(mk, sls, lane * 16, 0, AUX_SYS);
        SEND(last[4], last[4] & ~1023u);
        PACK8(5); PACK8(6); PACK8(7); PACK8(8);
        if (LATE_FLAG) { if (sent == 1024u) WAIT_VM(2); else WAIT_VM(0); if (lane == 0) st_sys(of, call); }
        SEND(last[8], last[8] & ~1023u);
        PACK8(9); PACK8(10); PACK8(11); PACK8(12);
        SEND(last[12], last[12] & ~1023u);
        PACK8(13); PACK8(14); PACK8(15);
        SEND(last[15], last[15]);
        WAIT_VM(0);
        if (lane == 0) st_sys64(of + 8, call, red[8] == NAN_BITS ? call : 0u);
    } else {
        const __amdgpu_buffer_rsrc_t ors = rsrc_of(outp, (MM / NR) * NN);
        const unsigned ob = out_base(rank, tile, wid, lane);
#pragma unroll
        for (int m = 0; m < 4; ++m)
#pragma unroll
            for (int j = 0; j < 4; ++j) {
                u32x2 o;
                int lo = __builtin_amdgcn_cvt_pk_fp8_f32(acc[m][0][j] * inv, acc[m][1][j] * inv, 0, false);
                o[0] = (unsigned)__builtin_amdgcn_cvt_pk_fp8_f32(acc[m][2][j] * inv, acc[m][3][j] * inv, lo, true);
                int hi = __builtin_amdgcn_cvt_pk_fp8_f32(acc[m][4][j] * inv, acc[m][5][j] * inv, 0, false);
                o[1] = (unsigned)__builtin_amdgcn_cvt_pk_fp8_f32(acc[m][6][j] * inv, acc[m][7][j] * inv, hi, true);
                __builtin_amdgcn_raw_buffer_store_b64(o, ors, ob + (unsigned)((m * 16 + j) * NN), 0, AUX_OUT);
            }
    }
    if (wid == 0) STAMP(4);
    if (wid == 7) STAMP(5);
}

}

extern "C" void kernel_launch(int rank, int nranks, ncclComm_t comm,
                              hipStream_t stream,
                              void* const* d_in, const int* in_sizes, int n_in,
                              void* d_out, int out_size,
                              const int* in_dtypes, int out_dtype,
                              void* const* d_out_all,
                              void* const* d_ws_all, size_t ws_bytes) {
    (void)comm; (void)in_dtypes; (void)out_dtype; (void)d_out_all;
    if (nranks != NR || n_in < 2 || (unsigned)rank >= (unsigned)NR || ws_bytes < WS_NEED ||
        in_sizes[0] != MM * KSH || in_sizes[1] != KSH * NN || out_size != (MM / NR) * NN) {
        fprintf(stderr, "kernel_launch: unexpected problem shape\n");
        return;
    }
    Params p;
    p.x = (const char*)d_in[0];
    p.w = (const char*)d_in[1];
    for (int i = 0; i < NR; ++i) p.ws[i] = (char*)d_ws_all[(rank + i) % NR];
    p.out = (char*)d_out;
    p.rank = rank; p.pad = 0;
    hipLaunchKernelGGL(fused_gemm_relu_q8, dim3(GRID), dim3(BLOCK), 0, stream, p);
    HIP_CHECK(hipGetLastError());
}
```

```cpp
#include <hip/hip_runtime.h>
#include <rccl/rccl.h>
#include <cstdio>
#include <cstdint>

#define HIP_CHECK(expr) do { hipError_t _e = (expr); if (_e != hipSuccess) { \
    fprintf(stderr, "HIP error %s at %s:%d\n", hipGetErrorString(_e), __FILE__, __LINE__); \
    return; } } while (0)

namespace {

typedef short bf16x8 __attribute__((ext_vector_type(8)));
typedef _Float16 f16x8 __attribute__((ext_vector_type(8)));
typedef float f32x4 __attribute__((ext_vector_type(4)));
typedef unsigned u32x4 __attribute__((ext_vector_type(4)));
typedef unsigned u32x3 __attribute__((ext_vector_type(3)));
typedef unsigned u32x2 __attribute__((ext_vector_type(2)));
typedef unsigned long long u64;
#define LAS __attribute__((address_space(3)))
#define GAS __attribute__((address_space(1)))

constexpr int NR = 8;
constexpr int MM = 4096, KSH = 512, NN = 16384, NCOL = NN / NR;
constexpr int NT = KSH / 8;
constexpr int NBLK = 24;
#define PULL_PER_LINK 18
#define AMAX_REP 8
#define AUX_OUT 17
#define SEND_DEPTH 2
#define JMODE 2
#define JORDER(k) ((k) % 3 == 2 ? 16 + (k) / 3 : (k) / 3 * 2 + (k) % 3)
#define SLEEP_K 2
#define ROT 1
#define SLOT(src, blk) ((((src) + ROT * (blk)) & 7) * 4096)
#define LATE_FLAG 1
#define STAGE_AUX 17
#define POLL_POS 0
#define TAKE_COL 1
#define POLL_AHEAD 6
#define PRETOUCH 6000
#define PRE_OUT 0
#define PACK_DELAY 60
constexpr int Q_JOBS = NT * NBLK;
constexpr int Q_TICKETS = Q_JOBS + PULL_PER_LINK;
constexpr int N_COMM = 128;
constexpr int N_PULL = (NR - 1) * PULL_PER_LINK;
constexpr int N_GEMM = 128;
constexpr int GRID = N_COMM + N_GEMM;
constexpr int BLOCK = 512;
constexpr int N_PACKW = N_COMM * 6;
constexpr int NGRP = NT / 8;
static_assert(N_PULL <= 2 * N_COMM, "pull waves are waves 0-1 of the comm workgroups");

constexpr unsigned XT_T = NR * MM * 16, WT_T = NR * NCOL * 16;
constexpr unsigned XT_OFF = 0;
constexpr unsigned WT_OFF = XT_OFF + NT * XT_T;
constexpr unsigned WS_OFF = WT_OFF + NT * WT_T;
constexpr unsigned OB_SLOT = 1024 + 8192;
constexpr unsigned OB_OFF = WS_OFF + NT * WT_T;
constexpr unsigned CTL_OFF = OB_OFF + NR * 128 * OB_SLOT;
constexpr unsigned PK_OFF = CTL_OFF;
constexpr unsigned CNT_OFF = PK_OFF + NR * 8192;
constexpr unsigned FL_OFF = CNT_OFF + NGRP * 9 * 128;
constexpr unsigned Q_OFF = FL_OFF + NBLK * NR * NT * 4;
constexpr unsigned TMAX_OFF = Q_OFF + NR * 128;
constexpr unsigned AMAX_OFF = TMAX_OFF + N_GEMM * 128;
constexpr unsigned OF_OFF = AMAX_OFF + 64 * 128;
constexpr unsigned CTR_OFF = OF_OFF + NR * 128 * 128;
constexpr unsigned TMO_OFF = CTR_OFF + GRID * 128 + 4;
constexpr unsigned DBG_OFF = CTR_OFF + GRID * 128 + 128;
constexpr size_t WS_NEED = DBG_OFF + 524288;

constexpr u64 DEADLINE = 50000000ull;
constexpr int AUX_SYS = 17;

constexpr int NSTEP = 2 * NT;
constexpr int NSLOT = 5;
constexpr int LDS_SLOT = 32768;
constexpr int LDS_BYTES = NSLOT * LDS_SLOT;
constexpr int LDS_WAVE = 16384;
constexpr int LDS_MISC = 8 * LDS_WAVE;

struct Params {
    const char* x; const char* w;
    int rank, pad;
    char* ws[NR];
    char* out;
};

__device__ __forceinline__ __amdgpu_buffer_rsrc_t rsrc_of(const void* p, unsigned bytes) {
    return __builtin_amdgcn_make_buffer_rsrc(const_cast<void*>(p), 0, bytes, 0x00020000);
}
__device__ __forceinline__ unsigned ld_sys(const void* p) {
    return __hip_atomic_load((const GAS unsigned*)p, __ATOMIC_RELAXED, __HIP_MEMORY_SCOPE_SYSTEM);
}
__device__ __forceinline__ u64 ld_sys64(const void* p) {
    return __hip_atomic_load((const GAS u64*)p, __ATOMIC_RELAXED, __HIP_MEMORY_SCOPE_SYSTEM);
}
__device__ __forceinline__ void st_sys(void* p, unsigned v) {
    __hip_atomic_store((GAS unsigned*)p, v, __ATOMIC_RELAXED, __HIP_MEMORY_SCOPE_SYSTEM);
}
__device__ __forceinline__ void st_sys64(void* p, unsigned lo, unsigned hi) {
    __hip_atomic_store((GAS u64*)p, ((u64)hi << 32) | lo, __ATOMIC_RELAXED, __HIP_MEMORY_SCOPE_SYSTEM);
}
__device__ __forceinline__ unsigned ld_agent(const void* p) {
    return __hip_atomic_load((const GAS unsigned*)p, __ATOMIC_RELAXED, __HIP_MEMORY_SCOPE_AGENT);
}
__device__ __forceinline__ unsigned add_agent_lane0(void* p, int lane) {
    unsigned v = 0, z = 0;
    asm volatile("" : "+v"(z));
    if (lane == 0) v = __hip_atomic_fetch_add((GAS unsigned*)((char*)p + z), 1u, __ATOMIC_RELAXED, __HIP_MEMORY_SCOPE_AGENT);
    return v;
}
__device__ __forceinline__ u32x4 transposed(const u32x4 (&v)[8], int i) {
    u32x4 o;
#pragma unroll
    for (int d = 0; d < 4; ++d)
        o[d] = __builtin_amdgcn_perm(v[2 * d + 1][i >> 1], v[2 * d][i >> 1], (i & 1) ? 0x07060302u : 0x05040100u);
    return o;
}
constexpr int NAN_BITS = 0x7fc00000;
#define ROW_SHR_MAX(v, n) v = max(v, __builtin_amdgcn_update_dpp(0, v, 0x110 + (n), 0xf, 0xf, true))
__device__ __forceinline__ int wave_max(int v) {
    ROW_SHR_MAX(v, 1); ROW_SHR_MAX(v, 2); ROW_SHR_MAX(v, 4); ROW_SHR_MAX(v, 8);
    return max(max(__builtin_amdgcn_readlane(v, 15), __builtin_amdgcn_readlane(v, 31)),
               max(__builtin_amdgcn_readlane(v, 47), __builtin_amdgcn_readlane(v, 63)));
}
__device__ __forceinline__ unsigned sign_byte(const u32x4& mk, int u) { return (mk[u >> 2] >> (24 - 8 * (u & 3))) & 0xffu; }
__device__ __forceinline__ void stream_layout(const u32x4& mk, unsigned (&first)[16], unsigned (&last)[16]) {
    unsigned base = 0;
#pragma unroll
    for (int u = 0; u < 16; u += 2) {
        const int c = 0x00080008 - (int)(__builtin_popcount(sign_byte(mk, u)) | (__builtin_popcount(sign_byte(mk, u + 1)) << 16));
        int v = c;
        v += __builtin_amdgcn_update_dpp(0, v, 0x111, 0xf, 0xf, true);
        v += __builtin_amdgcn_update_dpp(0, v, 0x112, 0xf, 0xf, true);
        v += __builtin_amdgcn_update_dpp(0, v, 0x114, 0xf, 0xf, true);
        v += __builtin_amdgcn_update_dpp(0, v, 0x118, 0xf, 0xf, true);
        v += __builtin_amdgcn_update_dpp(0, v, 0x142, 0xa, 0xf, false);
        v += __builtin_amdgcn_update_dpp(0, v, 0x143, 0xc, 0xf, false);
        const unsigned tot = (unsigned)__builtin_amdgcn_readlane(v, 63), exc = (unsigned)(v - c);
        first[u] = base + (exc & 0xffffu); base += tot & 0xffffu; last[u] = base;
        first[u + 1] = base + (exc >> 16); base += tot >> 16; last[u + 1] = base;
    }
}
template <int SLEEP = 1>
__device__ __forceinline__ bool give_up(char* ws, unsigned call, u64 t0, unsigned& spins) {
    bool stop = __builtin_amdgcn_s_memrealtime() - t0 > DEADLINE;
    if ((++spins & 63u) == 0u) stop = stop || __builtin_amdgcn_readfirstlane(ld_agent(ws + TMO_OFF)) == call;
    if (stop) {
        __hip_atomic_store((GAS unsigned*)(ws + TMO_OFF), call, __ATOMIC_RELAXED, __HIP_MEMORY_SCOPE_AGENT);
        return true;
    }
    __builtin_amdgcn_s_sleep(SLEEP);
    return false;
}
template <int SLEEP>
__device__ __forceinline__ bool await_word(char* ws, const char* p, unsigned call) {
    const u64 t0 = __builtin_amdgcn_s_memrealtime(); unsigned sp = 0;
    while ((int)(__builtin_amdgcn_readfirstlane(ld_sys(p)) - call) < 0)
        if (give_up<SLEEP>(ws, call, t0, sp)) return false;
    return true;
}
__device__ __forceinline__ unsigned fl_word(int blk, int src, int t) { return FL_OFF + (unsigned)(((blk * 4 + (src & 3)) * NT + t) * 2 + (src >> 2)) * 4u; }
__device__ __forceinline__ int steps_known(unsigned f0, unsigned f1, unsigned call) {
    const u64 lo = __ballot((int)(f0 - call) >= 0), hi = __ballot((int)(f1 - call) >= 0);
    const u64 miss = ~(lo & hi);
    if (miss == 0) return NSTEP;
    const int c = __builtin_ctzll(miss);
    return 2 * c + (int)((lo >> c) & 1);
}
__device__ __forceinline__ unsigned out_base(int src, int tile, int wave, int lane) {
    return (unsigned)(((tile >> 3) * 256 + (wave >> 1) * 64 + (lane >> 4) * 4) * NN + src * NCOL + (tile & 7) * 256 + (wave & 1) * 128 + (lane & 15) * 8);
}
constexpr unsigned PIECE_BYTES = 896;
__device__ __forceinline__ unsigned piece_off(int lane) { return lane & 1 ? 512 + (lane >> 1) * 12 : (lane >> 1) * 16; }
#define WAIT_VM(n) asm volatile("s_waitcnt vmcnt(" #n ")" ::: "memory")
#define LGKM0_BARRIER() do { __builtin_amdgcn_s_waitcnt(0xC07F); asm volatile("" ::: "memory");                       \
        __builtin_amdgcn_s_barrier(); asm volatile("" ::: "memory"); } while (0)
#define STAMP(i) do {} while (0)

__global__ void __launch_bounds__(BLOCK, 2) fused_gemm_relu_q8(const Params p) {
    __shared__ __attribute__((aligned(1024))) char smem[LDS_BYTES];
    LAS char* const lds = (LAS char*)smem;
    const int hb = blockIdx.x;
    const int lane = threadIdx.x & 63;
    const int wid = __builtin_amdgcn_readfirstlane(threadIdx.x >> 6);
    int rank = p.rank;
    char* ws = p.ws[0];
    const char* px = p.x;
    const char* pw = p.w;
    const u64 t_start = __builtin_amdgcn_s_memrealtime();
    asm volatile("" : "+s"(rank), "+s"(ws), "+s"(px), "+s"(pw));
    const unsigned ticket = add_agent_lane0(ws + CTR_OFF + (size_t)hb * 128 + wid * 4, lane);
    const int b = hb & (N_COMM - 1);

    if (hb < N_COMM) {
        const __amdgpu_buffer_rsrc_t wsr = rsrc_of(ws, CTL_OFF);
        const int cw = wid * N_COMM + b;
        const int dbg = 128 + cw;
        (void)dbg;
        STAMP(0);
        if (wid < 2) {
            if (threadIdx.x == 0) *(LAS unsigned*)(lds + LDS_MISC) = 0u;
            LGKM0_BARRIER();
            if (cw < N_PULL) {
                const unsigned s = (unsigned)(rank + 1 + cw % (NR - 1)) & 7u;
                const __amdgpu_buffer_rsrc_t srs = rsrc_of(p.ws[1 + cw % (NR - 1)], CTL_OFF);
                char* const queue = ws + Q_OFF + s * 128;
                unsigned tk = add_agent_lane0(queue, lane);
                u32x4 cur[4];
                const unsigned call = __builtin_amdgcn_readfirstlane(ticket) + 1u;
                const unsigned qbase = (call - 1u) * Q_TICKETS;
                const char* const pk = ws + PK_OFF + s * 8192;
                unsigned pv = call - 1u, pg = call - 1u;
#define JOB_OF(n_, t_, j_, so_, do_) const int t_ = (int)(n_) / NBLK, j_ = JORDER((int)(n_) % NBLK);                           \
                const unsigned do_ = (j_ < 16 ? XT_OFF + t_ * XT_T + j_ * 32768 : WT_OFF + t_ * WT_T + (j_ - 16) * 32768) + SLOT(s, j_) + lane * 16; \
                const unsigned so_ = j_ < 16 ? do_ : WS_OFF + t_ * WT_T + rank * 32768 + (j_ - 16) * 4096 + lane * 16
#define SEEN(v_, l_) ((int)(__builtin_amdgcn_readlane(v_, l_) - call) >= 0)
#define WAIT_PACKED(t_, j_) do { const int sg_ = ((t_) >> 3) + ((j_) < 16 ? 0 : 8);                                    \
                if (!SEEN(pg, sg_)) {                                                                                  \
                    const u64 t0_ = __builtin_amdgcn_s_memrealtime(); unsigned sp_ = 0;                                \
                    if ((t_) < 8) { const int sl_ = (j_) < 16 ? (j_) : 24 + (t_) * 4 + ((j_) >> 1);                    \
                        if (!SEEN(pv, sl_)) {                            \
                            pv = ld_sys(pk + lane * 128); pg = ld_sys(pk + 16 * 128 + (lane & 15) * 4);                \
                            while (!SEEN(pv, sl_)) {                                                                   \
                                if (give_up(ws, call, t0_, sp_)) break;                                                \
                                if (lane == sl_) pv = ld_sys(pk + lane * 128); } }                                     \
                    } else {                                                                                           \
                        for (;;) { pg = ld_sys(pk + 16 * 128 + (lane & 15) * 4);                                       \
                            if (SEEN(pg, sg_) || give_up(ws, call, t0_, sp_)) break; } }                               \
                } } while (0)
#define PTAIL() do {} while (0)
                unsigned n = __builtin_amdgcn_readfirstlane(tk) - qbase;
                if (n < (unsigned)Q_JOBS) {
                    JOB_OF(n, t, j, so, dof);
                    WAIT_PACKED(t, j);
                    STAMP(2);
                    tk = add_agent_lane0(queue, lane);
#pragma unroll
                    for (int i = 0; i < 4; ++i) cur[i] = __builtin_amdgcn_raw_buffer_load_b128(srs, so + i * 1024, 0, AUX_SYS);
                    unsigned dcur = dof, fcur = fl_word(j, s, t);
                    for (;;) {
#pragma unroll
                        for (int i = 0; i < 4; ++i) __builtin_amdgcn_raw_buffer_store_b128(cur[i], wsr, dcur + i * 1024, 0, AUX_SYS);
                        asm volatile("" ::: "memory");
                        n = __builtin_amdgcn_readfirstlane(tk) - qbase;
                        char* const flag = ws + fcur;
                        if (n >= (unsigned)Q_JOBS) {
                            WAIT_VM(0);
                            if (lane == 0) st_sys(flag, call);
                            PTAIL();
                            break;
                        }
                        JOB_OF(n, t2, j2, so2, dof2);
                        WAIT_PACKED(t2, j2);
                        tk = add_agent_lane0(queue, lane);
#pragma unroll
                        for (int i = 0; i < 4; ++i) cur[i] = __builtin_amdgcn_raw_buffer_load_b128(srs, so2 + i * 1024, 0, AUX_SYS);
                        dcur = dof2; fcur = fl_word(j2, s, t2);
                        WAIT_VM(5);
                        if (lane == 0) st_sys(flag, call);
                        PTAIL();
                    }
                }
                STAMP(1);
            }
        } else {
            const __amdgpu_buffer_rsrc_t xr = rsrc_of(px, MM * KSH * 2);
            const __amdgpu_buffer_rsrc_t wr_ = rsrc_of(pw, KSH * NN * 2);
            bool met = false;
            bool x_todo = wid >= 4;
            for (int n = (wid - 2) * N_COMM + b; n < NT * 32 || x_todo;) {
                const bool do_x = x_todo && ((b >> 4) <= (n >> 8) || n >= NT * 32);
                const int g = do_x ? b >> 4 : n >> 8;
                const int delay = (do_x ? g * 8 : n >> 5) * PACK_DELAY;
                if (!met && delay > 0) { __builtin_amdgcn_s_barrier(); met = true; }
                while (__builtin_amdgcn_s_memrealtime() - t_start < (u64)delay) __builtin_amdgcn_s_sleep(2);
                u32x4 v[8];
                if (do_x) {
                    const unsigned rb = (wid - 4) * 64 + (lane & 7), piece = lane >> 3;
#pragma unroll
                    for (int i = 0; i < 8; ++i) v[i] = __builtin_amdgcn_raw_buffer_load_b128(xr, ((b & 15) * 256 + rb + i * 8) * (KSH * 2) + g * 128 + piece * 16, 0, 0);
                    if (!met) { __builtin_amdgcn_s_barrier(); met = true; }
                    const unsigned d0 = XT_OFF + (g * 8 + piece) * XT_T + (b & 15) * 32768 + SLOT(rank, b & 15) + rb * 16;
#pragma unroll
                    for (int i = 0; i < 8; ++i) __builtin_amdgcn_raw_buffer_store_b128(v[i], wsr, d0 + i * 128, 0, AUX_SYS);
                    char* fl = lane < 8 ? p.ws[lane & 7] + PK_OFF + rank * 8192 + (b & 15) * 128
                                        : ws + fl_word(b & 15, rank, g * 8 + (lane & 7));
                    asm volatile("s_waitcnt vmcnt(0)" : "+v"(fl) :: "memory");
                    unsigned old = 0;
                    if (lane == 0) old = __hip_atomic_fetch_add((LAS unsigned*)(lds + LDS_MISC), 1u, __ATOMIC_RELAXED, __HIP_MEMORY_SCOPE_WORKGROUP);
                    const unsigned call = __builtin_amdgcn_readfirstlane(ticket) + 1u;
                    if (__builtin_amdgcn_readfirstlane(old) == 3u) {
                        if (lane >= (g == 0 ? 1 : 8) && lane < 16) st_sys(fl, call);
                        if (__builtin_amdgcn_readfirstlane(add_agent_lane0(ws + CNT_OFF + g * 9 * 128, lane)) + 1u == call * 16u && lane >= 1 && lane < 8)
                            st_sys(fl + (16 - (b & 15)) * 128 + g * 4, call);
                    }
                    x_todo = false;
                } else {
                    const int t = n >> 5, cb = n & 31;
                    const unsigned dest = (unsigned)cb >> 2;
#pragma unroll
                    for (int i = 0; i < 8; ++i) v[i] = __builtin_amdgcn_raw_buffer_load_b128(wr_, (unsigned)((t * 8 + i) * (NN * 2) + cb * 1024 + lane * 16), 0, 0);
                    if (!met) { __builtin_amdgcn_s_barrier(); met = true; }
                    const unsigned d0 = (dest == (unsigned)rank ? WT_OFF + t * WT_T + ((cb & 3) * 2 + (lane >> 5)) * 32768 + SLOT(rank, (cb & 3) * 2 + (lane >> 5))
                                                                : WS_OFF + t * WT_T + dest * 32768 + ((cb & 3) * 2 + (lane >> 5)) * 4096)
                                      + (((lane >> 4) & 1) * 128 + (lane & 15)) * 16;
#pragma unroll
                    for (int i = 0; i < 8; ++i) __builtin_amdgcn_raw_buffer_store_b128(transposed(v, i), wsr, d0 + i * 256, 0, AUX_SYS);
                    char* const dws = p.ws[(dest - rank) & 7u];
                    WAIT_VM(0);
                    const unsigned call = __builtin_amdgcn_readfirstlane(ticket) + 1u;
                    if (dest == (unsigned)rank) { if (lane < 2) st_sys(ws + fl_word(16 + (cb & 3) * 2 + lane, rank, t), call); }
                    else {
                        if (g == 0 && lane == 0) st_sys(dws + PK_OFF + rank * 8192 + (32 + t * 4 + (cb & 3)) * 128, call);
                        if (__builtin_amdgcn_readfirstlane(add_agent_lane0(ws + CNT_OFF + (g * 9 + 1 + dest) * 128, lane)) + 1u == call * 32u && lane == 0)
                            st_sys(dws + PK_OFF + rank * 8192 + 16 * 128 + (8 + g) * 4, call);
                    }
                    n += N_PACKW;
                }
            }
        }
        const unsigned call = __builtin_amdgcn_readfirstlane(ticket) + 1u;
        if (wid == 7) {
            if (PRETOUCH) {
                while (__builtin_amdgcn_s_memrealtime() - t_start < (u64)(PRETOUCH + b * 16)) __builtin_amdgcn_s_sleep(8);
                unsigned tv = 0;
#pragma unroll
                for (int i = 1; i < NR; ++i) tv += ld_sys(ws + OB_OFF + (unsigned)(((rank + i) & 7) * 128 + b) * OB_SLOT + lane * 128);
                if (PRE_OUT) {
#pragma unroll
                    for (int i = 0; i < NR; ++i) tv += ld_sys(p.out + out_base(i, b >> 3, b & 7, 0) + (size_t)lane * NN);
                }
                asm volatile("" :: "v"(tv));
            }
            if (b == N_COMM - 1) {
                char* amax_out = p.ws[lane & 7] + AMAX_OFF + ((lane >> 3) * NR + rank) * 128;
                asm volatile("" : "+v"(amax_out));
                const char* tm = ws + TMAX_OFF + lane * 256;
                const u64 t0 = __builtin_amdgcn_s_memrealtime(); unsigned sp = 0;
                int mx;
                for (;;) {
                    const u64 v0 = ld_sys64(tm), v1 = ld_sys64(tm + 128);
                    mx = max((int)(unsigned)v0, (int)(unsigned)v1);
                    if (__all((int)((unsigned)(v0 >> 32) - call) >= 0 && (int)((unsigned)(v1 >> 32) - call) >= 0)) break;
                    if (give_up(ws, call, t0, sp)) { mx = NAN_BITS; break; }
                }
                if (lane < 8 * AMAX_REP) st_sys64(amax_out, (unsigned)wave_max(mx), call);
                STAMP(6);
            }
            return;
        }
        {
            const int src = (rank + 1 + wid) & 7;
            const unsigned slot = OB_OFF + (unsigned)(src * 128 + b) * OB_SLOT;
            const char* const of = ws + OF_OFF + (src * 128 + b) * 128;
            const unsigned lbase = wid * LDS_WAVE;
#pragma unroll
            for (int e = 0; e < 4; ++e) {
                const unsigned idx = lane * 4 + e;
                u32x2 sel = {0u, 0u};
                unsigned rk = 0;
#pragma unroll
                for (int n = 0; n < 8; ++n) {
                    const bool neg = (idx >> (7 - n)) & 1u;
                    sel[n >> 2] |= (neg ? 0x0cu : rk) << (8 * (n & 3));
                    rk += neg ? 0u : 1u;
                }
                *(LAS u32x2*)(lds + lbase + 12288 + idx * 8) = sel;
            }
            char* outp = p.out;
            asm volatile("" : "+s"(outp));
            const unsigned ob = out_base(src, b >> 3, b & 7, lane);
            {
                bool ok = await_word<8>(ws, of, call);
                const u32x4 mk = __builtin_amdgcn_raw_buffer_load_b128(wsr, slot + lane * 16, 0, AUX_SYS);
                unsigned first[16], last[16];
                stream_layout(mk, first, last);
                u32x2 sel[16];
#pragma unroll
                for (int u = 0; u < 16; ++u) {
                    sel[u] = *(const LAS u32x2*)(lds + lbase + 12288 + sign_byte(mk, u) * 8);
                    first[u] += lbase;
                    asm volatile("" : "+v"(first[u]), "+v"(sel[u]));
                }
                STAMP(3);
                const unsigned cnt = last[15];
                const __amdgpu_buffer_rsrc_t ors = rsrc_of(outp, (MM / NR) * NN);
#define TAKE_PIECES(i0_, i1_) do { u32x4 pc[(i1_) - (i0_)];                                                            \
                    _Pragma("unroll") for (int i = (i0_); i < (i1_); ++i)                                              \
                        if (i * 1024u < cnt) pc[i - (i0_)] = __builtin_amdgcn_raw_buffer_load_b128(wsr, slot + 1024 + i * PIECE_BYTES + piece_off(lane), 0, AUX_SYS); \
                    fill = ok && !__any(ld_sys(lane & 1 ? ws + TMO_OFF : of + 12) == call) ? 0u : 0x7f7f7f7fu;          \
                    _Pragma("unroll") for (int i = (i0_); i < (i1_); ++i)                                              \
                        if (i * 1024u < cnt) {                                                                         \
                            u32x4 q = pc[i - (i0_)];                                                                   \
                            unsigned h = ((q[0] & 0x80808080u) >> 7) | ((q[1] & 0x80808080u) >> 6) | ((q[2] & 0x80808080u) >> 5); \
                            const unsigned he = h | ((q[3] & 0x80808080u) >> 4);     \
                            h = (h << 4) | (unsigned)__builtin_amdgcn_update_dpp(0, (int)he, 0xa0, 0xf, 0xf, true);    \
                            q[0] &= 0x7f7f7f7fu; q[1] &= 0x7f7f7f7fu; q[2] &= 0x7f7f7f7fu;                             \
                            q[3] = lane & 1 ? h : q[3] & 0x7f7f7f7fu;                                                  \
                            *(LAS u32x4*)(lds + lbase + i * 1024 + lane * 16) = q;                                     \
                        } } while (0)
                unsigned fill;
                ok = await_word<1>(ws, of + 8, call) && ok;
                STAMP(4);
                TAKE_PIECES(0, 8);
                unsigned d[16][3];
#pragma unroll
                for (int u = 0; u < 16; ++u)
#pragma unroll
                    for (int i = 0; i < 3; ++i) d[u][i] = *(const LAS unsigned*)(lds + (first[u] & ~3u) + i * 4);
                __builtin_amdgcn_sched_barrier(0);
#pragma unroll
                for (int u = 0; u < 16; ++u) {
                    const unsigned lo = __builtin_amdgcn_alignbyte(d[u][1], d[u][0], first[u]), hi = __builtin_amdgcn_alignbyte(d[u][2], d[u][1], first[u]);
                    u32x2 o;
                    o[0] = __builtin_amdgcn_perm(hi, lo, sel[u][0]) | fill;
                    o[1] = __builtin_amdgcn_perm(hi, lo, sel[u][1]) | fill;
                    __builtin_amdgcn_raw_buffer_store_b64(o, ors, ob + (unsigned)(((u >> 2) * 16 + (u & 3)) * NN), 0, AUX_OUT);
                }
            }
        }
        return;
    }

    const int g = b;
    const int dbg = g;
    (void)dbg;
    const int pm = (g >> 4) * 2 + (g & 1);
    const int pn = ((g & 7) >> 1) * 2 + ((g >> 3) & 1);
    const int fr = lane & 15, fq = lane >> 4, wr = wid >> 1, wc = wid & 1;
    if (wid == 0) STAMP(0);
    const int owner = pm >> 1, tile = (pm & 1) * 8 + pn;
    char* ows = p.ws[(owner - rank) & 7];
    char* outp = p.out;
    asm volatile("" : "+s"(ows), "+s"(outp));
    const unsigned call = __builtin_amdgcn_readfirstlane(ticket) + 1u;

    const int op = wid >> 2;
    const char* const pollp = ws + fl_word(op ? 16 + pn : pm, wid & 3, lane);
    const unsigned unit = op ? WT_T : XT_T;
    const char* const gbase = ws + (op ? WT_OFF + pn * 32768 : XT_OFF + pm * 32768) + lane * 16;
    const unsigned slot0 = SLOT(wid & 3, op ? pn : pm);
    LAS char* const ldst = lds + wid * 4096;
    const int aoff = fq * 4096 + (wr * 64 + fr) * 16;
    const int boff = 16384 + fq * 4096 + (wc * 128 + fr) * 16;

    int known = 0;
    int next = 0, nslot = 0;
    u32x2 pvn = {0u, 0u};
#define STAGE_NEXT() do { const char* g_ = gbase + ((slot0 ^ ((next & 1) << 14)) + (unsigned)(next >> 1) * unit);              \
        LAS char* l_ = ldst + nslot * LDS_SLOT;                                                                       \
        __builtin_amdgcn_global_load_lds((const GAS unsigned*)g_, (LAS unsigned*)l_, 16, 0, STAGE_AUX);                       \
        __builtin_amdgcn_global_load_lds((const GAS unsigned*)g_, (LAS unsigned*)l_, 16, 1024, STAGE_AUX);                    \
        __builtin_amdgcn_global_load_lds((const GAS unsigned*)g_, (LAS unsigned*)l_, 16, 2048, STAGE_AUX);                    \
        __builtin_amdgcn_global_load_lds((const GAS unsigned*)g_, (LAS unsigned*)l_, 16, 3072, STAGE_AUX);                    \
        ++next; nslot = nslot == NSLOT - 1 ? 0 : nslot + 1; } while (0)
#define POLL_ISSUE() asm volatile("global_load_dwordx2 %0, %1, off sc0 sc1" : "+v"(pvn) : "v"(pollp) : "memory")
#define POLL_TAKE() do { u32x2 f_; asm volatile("s_waitcnt vmcnt(0)\n\tv_mov_b64 %0, %1" : "=v"(f_) : "v"(pvn) : "memory");    \
        known = steps_known(f_[0], f_[1], call); } while (0)
#define AWAIT_NEXT() do { if (next >= known) {                                                                        \
            const u64 t0_ = __builtin_amdgcn_s_memrealtime(); unsigned sp_ = 0;                                       \
            for (;;) { const u64 f_ = ld_sys64(pollp); known = steps_known((unsigned)f_, (unsigned)(f_ >> 32), call); \
                if (next < known) break;                                                                              \
                if (give_up<SLEEP_K>(ws, call, t0_, sp_)) { known = NSTEP; break; } } } } while (0)
#define MFMA_COLS(A_, B_, n0_, n1_) do { __builtin_amdgcn_sched_barrier(0);                                           \
        _Pragma("unroll") for (int n = (n0_); n < (n1_); ++n)                                                         \
            _Pragma("unroll") for (int m = 0; m < 4; ++m)                                                             \
                acc[m][n] = __builtin_amdgcn_mfma_f32_16x16x32_bf16(A_[m], B_[n], acc[m][n], 0, 0, 0);                \
        __builtin_amdgcn_sched_barrier(0); } while (0)
#define RD_A(A_, m_) A_[m_] = *(const LAS bf16x8*)(lb_ + aoff + (m_) * 256)
#define RD_B(B_, n_) B_[n_] = *(const LAS bf16x8*)(lb_ + boff + (n_) * 256)
#define TAIL_REC(st_) do {} while (0)
#define KSTEP(st_, A_, B_, AN_, BN_) do {                                                                             \
        MFMA_COLS(A_, B_, 0, TAKE_COL);                                                                               \
        if (polled) POLL_TAKE();                                          \
        const int had_ = next;                                                                                        \
        const int lim_ = min((st_) + NSLOT, known);                                   \
        if (next < lim_) { STAGE_NEXT(); if (next < lim_) STAGE_NEXT(); }                                             \
        const bool full_ = (polled ? had_ + NSLOT - 2 : next) >= (st_) + NSLOT;                                       \
        MFMA_COLS(A_, B_, TAKE_COL, 4);                                                                                      \
        if (!full_) {                                                              \
            if (next == (st_) + 1 && next < NSTEP) {                                                                  \
                AWAIT_NEXT(); STAGE_NEXT();                                                                           \
                if (next < known) STAGE_NEXT();                                                                       \
            }                                                                                                         \
            const int k_ = next - (st_) - 2;                                                                          \
            if (k_ <= 0) WAIT_VM(0); else if (k_ == 1) WAIT_VM(4); else WAIT_VM(8);                                   \
        }                                                                                                             \
        WAIT_VM(12);                                                                                                  \
        LGKM0_BARRIER();                                                                                              \
        TAIL_REC(st_);                                                                                                \
        const LAS char* lb_ = lds + rslot * LDS_SLOT;               \
        RD_A(AN_, 0); RD_A(AN_, 1); RD_A(AN_, 2); RD_A(AN_, 3);                                                       \
        if (POLL_POS == 0) { polled = known < min(next + POLL_AHEAD, NSTEP); if (polled) POLL_ISSUE(); }              \
        MFMA_COLS(A_, B_, 4, 5);                                                                                      \
        RD_B(BN_, 0); RD_B(BN_, 1); RD_B(BN_, 2); RD_B(BN_, 3);                                                       \
        MFMA_COLS(A_, B_, 5, 6);                                                                                      \
        RD_B(BN_, 4); RD_B(BN_, 5); RD_B(BN_, 6); RD_B(BN_, 7);                                                       \
        rslot = rslot == NSLOT - 1 ? 0 : rslot + 1;                                                                   \
        if (POLL_POS == 2) { polled = known < min(next + POLL_AHEAD, NSTEP); if (polled) POLL_ISSUE(); }              \
        MFMA_COLS(A_, B_, 6, 8); } while (0)

    f32x4 acc[4][8];
#pragma unroll
    for (int m = 0; m < 4; ++m)
#pragma unroll
        for (int n = 0; n < 8; ++n) acc[m][n] = (f32x4){0.f, 0.f, 0.f, 0.f};

    AWAIT_NEXT(); STAGE_NEXT();
    WAIT_VM(0);
    LGKM0_BARRIER();
    int rslot = 1;
    bool polled = false;
    bf16x8 a0[4], b0[8], a1[4], b1[8];
    {
        const LAS char* lb_ = lds;
#pragma unroll
        for (int m = 0; m < 4; ++m) RD_A(a0, m);
#pragma unroll
        for (int n = 0; n < 8; ++n) RD_B(b0, n);
    }
    if (wid == 0) STAMP(1);
    for (int st = 0; st < NSTEP; st += 2) {
        KSTEP(st, a0, b0, a1, b1);
        KSTEP(st + 1, a1, b1, a0, b0);
    }
    if (wid == 0) STAMP(2);

    int mx = 0;
#pragma unroll
    for (int m = 0; m < 4; ++m)
#pragma unroll
        for (int n = 0; n < 8; ++n)
#pragma unroll
            for (int j = 0; j < 4; ++j) mx = max(mx, __float_as_int(acc[m][n][j]));
    mx = wave_max(mx);
    LAS int* red = (LAS int*)(lds + LDS_MISC);
    if (lane == 0) red[wid] = mx;
    LGKM0_BARRIER();
    if (wid == 0) {
        int t = red[lane & 7];
        ROW_SHR_MAX(t, 1); ROW_SHR_MAX(t, 2); ROW_SHR_MAX(t, 4);
        if (lane == 7) st_sys64(ws + TMAX_OFF + g * 128, (unsigned)t, call);
        STAMP(7);
    }

    const bool remote = owner != rank;
    const __amdgpu_buffer_rsrc_t sls = rsrc_of(ows + OB_OFF + (unsigned)(rank * 128 + tile * 8 + wid) * OB_SLOT, OB_SLOT);
    char* const of = ows + OF_OFF + (rank * 128 + tile * 8 + wid) * 128;
    const int efield = min(268 - (mx >> 23), 253);
    const float sc = __int_as_float(efield << 23);
    const unsigned pos0 = (unsigned)(size_t)(lds + wid * LDS_WAVE);
    const unsigned spill = (unsigned)(size_t)(lds + LDS_MISC + 64);
    unsigned first[16], last[16];
#define VAL(u_, n_) acc[(u_) >> 2][n_][(u_) & 3]
#define PACK1(n_) "v_ashrrev_i32 %[s], 31, %[v" #n_ "]\n\tv_fma_mixlo_f16 %[h], %[v" #n_ "], %[sc], 0\n\t"              \
                  "v_bfi_b32 %[a], %[s], %[sp], %[p]\n\tv_lshl_add_u32 %[p], %[s], 1, %[p]\n\t"                      \
                  "ds_write_b16 %[a], %[h] offset:2*" #n_ "\n\t"
#define PACK8(u_) do { unsigned p_ = pos0 + 2 * first[u_], s_, h_, a_;                                                \
        asm volatile(PACK1(0) PACK1(1) PACK1(2) PACK1(3) PACK1(4) PACK1(5) PACK1(6) PACK1(7)                          \
                     : [p] "+v"(p_), [s] "=&v"(s_), [h] "=&v"(h_), [a] "=&v"(a_)                                      \
                     : [v0] "v"(VAL(u_, 0)), [v1] "v"(VAL(u_, 1)), [v2] "v"(VAL(u_, 2)), [v3] "v"(VAL(u_, 3)),        \
                       [v4] "v"(VAL(u_, 4)), [v5] "v"(VAL(u_, 5)), [v6] "v"(VAL(u_, 6)), [v7] "v"(VAL(u_, 7)),        \
                       [sc] "s"(sc), [sp] "v"(spill) : "memory"); } while (0)
    u32x4 mk = {0u, 0u, 0u, 0u};
    if (remote) {
#pragma unroll
        for (int r = 0; r < 128; ++r) mk[r >> 5] = __builtin_amdgcn_alignbit(mk[r >> 5], __float_as_uint(VAL(r >> 3, r & 7)), 31);
        if (!LATE_FLAG || wid != 0) __builtin_amdgcn_raw_buffer_store_b128(mk, sls, lane * 16, 0, AUX_SYS);
        stream_layout(mk, first, last);
        PACK8(0); PACK8(1); PACK8(2); PACK8(3); PACK8(4);
        if (!LATE_FLAG) { WAIT_VM(0); if (lane == 0) st_sys(of, call); }
    } else {
#pragma unroll
        for (int m = 0; m < 4; ++m) {
#pragma unroll
            for (int n = 0; n < 8; ++n)
#pragma unroll
                for (int j = 0; j < 4; ++j) acc[m][n][j] = __int_as_float(max(0, __float_as_int(acc[m][n][j])));
            asm volatile("" : "+v"(acc[m][0]), "+v"(acc[m][1]), "+v"(acc[m][2]), "+v"(acc[m][3]),
                              "+v"(acc[m][4]), "+v"(acc[m][5]), "+v"(acc[m][6]), "+v"(acc[m][7]));
        }
    }
    if (wid == 1) STAMP(6);
    if (wid == 0) {
        const char* ap = lane < NR ? ws + AMAX_OFF + ((g & (AMAX_REP - 1)) * NR + lane) * 128 : ws + TMO_OFF - 4;
        const u64 t0 = __builtin_amdgcn_s_memrealtime(); unsigned sp = 0;
        u64 v = 0;
        for (;;) {
            if (lane <= NR) v = ld_sys64(ap);
            if ((__ballot((int)((unsigned)(v >> 32) - call) >= 0) & 0xffu) == 0xffu) break;
            if (give_up(ws, call, t0, sp)) { v = (u64)call << 32; break; }
        }
        STAMP(8);
        int t = lane < NR ? (int)(unsigned)v : ((unsigned)(v >> 32) == call ? NAN_BITS : 0);
        ROW_SHR_MAX(t, 1); ROW_SHR_MAX(t, 2); ROW_SHR_MAX(t, 4); ROW_SHR_MAX(t, 8);
        if (lane == NR) red[8] = t;
    }
    LGKM0_BARRIER();
    const float inv = 448.0f / __int_as_float(red[8]);
    if (wid == 0) STAMP(3);

    if (remote) {
        const float f = inv * __int_as_float((254 - efield) << 23);
        unsigned sent = 0;
        const unsigned hshift = (lane & 1) * 4, woff = piece_off(lane);
#define SEND_GATE() WAIT_VM(2)
#define SEND(cnt_, upto_) do { __builtin_amdgcn_s_waitcnt(0xC07F); asm volatile("" ::: "memory");                           \
            _Pragma("nounroll") for (; sent < (upto_); sent += 1024) {                                                                    \
                const unsigned o_ = sent + lane * 16;                                                                 \
                const f16x8 h0 = *(const LAS f16x8*)(lds + wid * LDS_WAVE + o_ * 2), h1 = *(const LAS f16x8*)(lds + wid * LDS_WAVE + o_ * 2 + 16); \
                u32x4 q;                                                                                              \
                _Pragma("unroll") for (int d = 0; d < 2; ++d) {                                                       \
                    int w0 = __builtin_amdgcn_cvt_pk_fp8_f32(__builtin_fmaf((float)h0[4 * d], f, 0.f), __builtin_fmaf((float)h0[4 * d + 1], f, 0.f), 0, false); \
                    q[d] = (unsigned)__builtin_amdgcn_cvt_pk_fp8_f32(__builtin_fmaf((float)h0[4 * d + 2], f, 0.f), __builtin_fmaf((float)h0[4 * d + 3], f, 0.f), w0, true); \
                    int w1 = __builtin_amdgcn_cvt_pk_fp8_f32(__builtin_fmaf((float)h1[4 * d], f, 0.f), __builtin_fmaf((float)h1[4 * d + 1], f, 0.f), 0, false); \
                    q[2 + d] = (unsigned)__builtin_amdgcn_cvt_pk_fp8_f32(__builtin_fmaf((float)h1[4 * d + 2], f, 0.f), __builtin_fmaf((float)h1[4 * d + 3], f, 0.f), w1, true); \
                }                                                                                                     \
                  \
                const unsigned h_ = (unsigned)__builtin_amdgcn_update_dpp(0, (int)q[3], 0xf5, 0xf, 0xf, true) >> hshift; \
                _Pragma("unroll") for (int d = 0; d < 4; ++d) q[d] |= (h_ << (7 - d)) & 0x80808080u;                  \
                SEND_GATE();                                                                                          \
                if ((o_ & ~31u) < (cnt_)) {                                                                             \
                    const unsigned w_ = 1024 + (sent >> 10) * PIECE_BYTES + woff;                                     \
                    if (lane & 1) __builtin_amdgcn_raw_buffer_store_b96((u32x3){q[0], q[1], q[2]}, sls, w_, 0, AUX_SYS); \
                    else __builtin_amdgcn_raw_buffer_store_b128(q, sls, w_, 0, AUX_SYS);                              \
                }                                                                                                     \
            } } while (0)
        if (LATE_FLAG && wid == 0) __builtin_amdgcn_raw_buffer_store_b128(mk, sls, lane * 16, 0, AUX_SYS);
        SEND(last[4], last[4] & ~1023u);
        PACK8(5); PACK8(6); PACK8(7); PACK8(8);
        if (LATE_FLAG) { if (sent == 1024u) WAIT_VM(2); else WAIT_VM(0); if (lane == 0) st_sys(of, call); }
        SEND(last[8], last[8] & ~1023u);
        PACK8(9); PACK8(10); PACK8(11); PACK8(12);
        SEND(last[12], last[12] & ~1023u);
        PACK8(13); PACK8(14); PACK8(15);
        SEND(last[15], last[15]);
        WAIT_VM(0);
        if (lane == 0) st_sys64(of + 8, call, red[8] == NAN_BITS ? call : 0u);
    } else {
        const __amdgpu_buffer_rsrc_t ors = rsrc_of(outp, (MM / NR) * NN);
        const unsigned ob = out_base(rank, tile, wid, lane);
#pragma unroll
        for (int m = 0; m < 4; ++m)
#pragma unroll
            for (int j = 0; j < 4; ++j) {
                u32x2 o;
                int lo = __builtin_amdgcn_cvt_pk_fp8_f32(acc[m][0][j] * inv, acc[m][1][j] * inv, 0, false);
                o[0] = (unsigned)__builtin_amdgcn_cvt_pk_fp8_f32(acc[m][2][j] * inv, acc[m][3][j] * inv, lo, true);
                int hi = __builtin_amdgcn_cvt_pk_fp8_f32(acc[m][4][j] * inv, acc[m][5][j] * inv, 0, false);
                o[1] = (unsigned)__builtin_amdgcn_cvt_pk_fp8_f32(acc[m][6][j] * inv, acc[m][7][j] * inv, hi, true);
                __builtin_amdgcn_raw_buffer_store_b64(o, ors, ob + (unsigned)((m * 16 + j) * NN), 0, AUX_OUT);
            }
    }
    if (wid == 0) STAMP(4);
    if (wid == 7) STAMP(5);
}

}

extern "C" void kernel_launch(int rank, int nranks, ncclComm_t comm,
                              hipStream_t stream,
                              void* const* d_in, const int* in_sizes, int n_in,
                              void* d_out, int out_size,
                              const int* in_dtypes, int out_dtype,
                              void* const* d_out_all,
                              void* const* d_ws_all, size_t ws_bytes) {
    (void)comm; (void)in_dtypes; (void)out_dtype; (void)d_out_all;
    if (nranks != NR || n_in < 2 || (unsigned)rank >= (unsigned)NR || ws_bytes < WS_NEED ||
        in_sizes[0] != MM * KSH || in_sizes[1] != KSH * NN || out_size != (MM / NR) * NN) {
        fprintf(stderr, "kernel_launch: unexpected problem shape\n");
        return;
    }
    Params p;
    p.x = (const char*)d_in[0];
    p.w = (const char*)d_in[1];
    for (int i = 0; i < NR; ++i) p.ws[i] = (char*)d_ws_all[(rank + i) % NR];
    p.out = (char*)d_out;
    p.rank = rank; p.pad = 0;
    hipLaunchKernelGGL(fused_gemm_relu_q8, dim3(GRID), dim3(BLOCK), 0, stream, p);
    HIP_CHECK(hipGetLastError());
}
```

```cpp
#include <hip/hip_runtime.h>
#include <rccl/rccl.h>
#include <cstdio>
#include <cstdint>

#define HIP_CHECK(expr) do { hipError_t _e = (expr); if (_e != hipSuccess) {                                          \
    fprintf(stderr, "HIP error %s at %s:%d\n", hipGetErrorString(_e), __FILE__, __LINE__);                            \
    return; } } while (0)

namespace {

typedef short bf16x8 __attribute__((ext_vector_type(8)));
typedef _Float16 f16x8 __attribute__((ext_vector_type(8)));
typedef float f32x4 __attribute__((ext_vector_type(4)));
typedef unsigned u32x4 __attribute__((ext_vector_type(4)));
typedef unsigned u32x3 __attribute__((ext_vector_type(3)));
typedef unsigned u32x2 __attribute__((ext_vector_type(2)));
typedef unsigned long long u64;
#define LAS __attribute__((address_space(3)))
#define GAS __attribute__((address_space(1)))

constexpr int NR = 8;
constexpr int MM = 4096, KSH = 512, NN = 16384, NCOL = NN / NR;
constexpr int NT = KSH / 8;
constexpr int NBLK = 24;
constexpr int PULL_PER_LINK = 18;
constexpr int PACK_DELAY = 60;
constexpr int TOUCH_AT = 6000;
constexpr int Q_JOBS = NT * NBLK;
constexpr int Q_TICKETS = Q_JOBS + PULL_PER_LINK;
constexpr int N_COMM = 128;
constexpr int N_PULL = (NR - 1) * PULL_PER_LINK;
constexpr int N_GEMM = 128;
constexpr int GRID = N_COMM + N_GEMM;
constexpr int BLOCK = 512;
constexpr int N_PACKW = N_COMM * 6;
constexpr int NGRP = NT / 8;
static_assert(N_PULL <= 2 * N_COMM, "pull waves are waves 0-1 of the comm workgroups");

__host__ __device__ constexpr int job_block(int k) { return k % 3 == 2 ? 16 + k / 3 : k / 3 * 2 + k % 3; }
__host__ __device__ constexpr unsigned chunk_pos(unsigned src, unsigned blk) { return ((src + blk) & 7u) * 4096u; }

constexpr unsigned XT_T = NR * MM * 16, WT_T = NR * NCOL * 16;
constexpr unsigned XT_OFF = 0;
constexpr unsigned WT_OFF = XT_OFF + NT * XT_T;
constexpr unsigned WS_OFF = WT_OFF + NT * WT_T;
constexpr unsigned OB_SLOT = 1024 + 8192;
constexpr unsigned OB_OFF = WS_OFF + NT * WT_T;
constexpr unsigned CTL_OFF = OB_OFF + NR * 128 * OB_SLOT;
constexpr unsigned PK_OFF = CTL_OFF;
constexpr unsigned CNT_OFF = PK_OFF + NR * 8192;
constexpr unsigned FL_OFF = CNT_OFF + NGRP * 9 * 128;
constexpr unsigned Q_OFF = FL_OFF + NBLK * NR * NT * 4;
constexpr unsigned TMAX_OFF = Q_OFF + NR * 128;
constexpr unsigned AMAX_OFF = TMAX_OFF + N_GEMM * 128;
constexpr unsigned OF_OFF = AMAX_OFF + 64 * 128;
constexpr unsigned CTR_OFF = OF_OFF + NR * 128 * 128;
constexpr unsigned TMO_OFF = CTR_OFF + GRID * 128 + 4;
constexpr size_t WS_NEED = CTR_OFF + GRID * 128 + 128;

constexpr u64 DEADLINE = 50000000ull;
constexpr int AUX_SYS = 17;

constexpr int NSTEP = 2 * NT;
constexpr int NSLOT = 5;
constexpr int LDS_SLOT = 32768;
constexpr int LDS_BYTES = NSLOT * LDS_SLOT;
constexpr int LDS_WAVE = 16384;
constexpr int LDS_MISC = 8 * LDS_WAVE;

struct Params {
    const char* x; const char* w;
    int rank, pad;
    char* ws[NR];
    char* out;
};

__device__ __forceinline__ __amdgpu_buffer_rsrc_t rsrc_of(const void* p, unsigned bytes) {
    return __builtin_amdgcn_make_buffer_rsrc(const_cast<void*>(p), 0, bytes, 0x00020000);
}
__device__ __forceinline__ unsigned ld_sys(const void* p) {
    return __hip_atomic_load((const GAS unsigned*)p, __ATOMIC_RELAXED, __HIP_MEMORY_SCOPE_SYSTEM);
}
__device__ __forceinline__ u64 ld_sys64(const void* p) {
    return __hip_atomic_load((const GAS u64*)p, __ATOMIC_RELAXED, __HIP_MEMORY_SCOPE_SYSTEM);
}
__device__ __forceinline__ void st_sys(void* p, unsigned v) {
    __hip_atomic_store((GAS unsigned*)p, v, __ATOMIC_RELAXED, __HIP_MEMORY_SCOPE_SYSTEM);
}
__device__ __forceinline__ void st_sys64(void* p, unsigned lo, unsigned hi) {
    __hip_atomic_store((GAS u64*)p, ((u64)hi << 32) | lo, __ATOMIC_RELAXED, __HIP_MEMORY_SCOPE_SYSTEM);
}
__device__ __forceinline__ unsigned ld_agent(const void* p) {
    return __hip_atomic_load((const GAS unsigned*)p, __ATOMIC_RELAXED, __HIP_MEMORY_SCOPE_AGENT);
}
__device__ __forceinline__ unsigned add_agent_lane0(void* p, int lane) {
    unsigned v = 0, z = 0;
    asm volatile("" : "+v"(z));
    if (lane == 0) v = __hip_atomic_fetch_add((GAS unsigned*)((char*)p + z), 1u, __ATOMIC_RELAXED, __HIP_MEMORY_SCOPE_AGENT);
    return v;
}
__device__ __forceinline__ u32x4 transposed(const u32x4 (&v)[8], int i) {
    u32x4 o;
#pragma unroll
    for (int d = 0; d < 4; ++d)
        o[d] = __builtin_amdgcn_perm(v[2 * d + 1][i >> 1], v[2 * d][i >> 1], (i & 1) ? 0x07060302u : 0x05040100u);
    return o;
}
constexpr int NAN_BITS = 0x7fc00000;
#define ROW_SHR_MAX(v, n) v = max(v, __builtin_amdgcn_update_dpp(0, v, 0x110 + (n), 0xf, 0xf, true))
__device__ __forceinline__ int wave_max(int v) {
    ROW_SHR_MAX(v, 1); ROW_SHR_MAX(v, 2); ROW_SHR_MAX(v, 4); ROW_SHR_MAX(v, 8);
    return max(max(__builtin_amdgcn_readlane(v, 15), __builtin_amdgcn_readlane(v, 31)),
               max(__builtin_amdgcn_readlane(v, 47), __builtin_amdgcn_readlane(v, 63)));
}
__device__ __forceinline__ unsigned sign_byte(const u32x4& mk, int u) { return (mk[u >> 2] >> (24 - 8 * (u & 3))) & 0xffu; }
__device__ __forceinline__ void stream_layout(const u32x4& mk, unsigned (&first)[16], unsigned (&last)[16]) {
    unsigned base = 0;
#pragma unroll
    for (int u = 0; u < 16; u += 2) {
        const int c = 0x00080008 - (int)(__builtin_popcount(sign_byte(mk, u)) | (__builtin_popcount(sign_byte(mk, u + 1)) << 16));
        int v = c;
        v += __builtin_amdgcn_update_dpp(0, v, 0x111, 0xf, 0xf, true);
        v += __builtin_amdgcn_update_dpp(0, v, 0x112, 0xf, 0xf, true);
        v += __builtin_amdgcn_update_dpp(0, v, 0x114, 0xf, 0xf, true);
        v += __builtin_amdgcn_update_dpp(0, v, 0x118, 0xf, 0xf, true);
        v += __builtin_amdgcn_update_dpp(0, v, 0x142, 0xa, 0xf, false);
        v += __builtin_amdgcn_update_dpp(0, v, 0x143, 0xc, 0xf, false);
        const unsigned tot = (unsigned)__builtin_amdgcn_readlane(v, 63), exc = (unsigned)(v - c);
        first[u] = base + (exc & 0xffffu); base += tot & 0xffffu; last[u] = base;
        first[u + 1] = base + (exc >> 16); base += tot >> 16; last[u + 1] = base;
    }
}
template <int SLEEP = 1>
__device__ __forceinline__ bool give_up(char* ws, unsigned call, u64 t0, unsigned& spins) {
    bool stop = __builtin_amdgcn_s_memrealtime() - t0 > DEADLINE;
    if ((++spins & 63u) == 0u) stop = stop || __builtin_amdgcn_readfirstlane(ld_agent(ws + TMO_OFF)) == call;
    if (stop) {
        __hip_atomic_store((GAS unsigned*)(ws + TMO_OFF), call, __ATOMIC_RELAXED, __HIP_MEMORY_SCOPE_AGENT);
        return true;
    }
    __builtin_amdgcn_s_sleep(SLEEP);
    return false;
}
template <int SLEEP>
__device__ __forceinline__ bool await_word(char* ws, const char* p, unsigned call) {
    const u64 t0 = __builtin_amdgcn_s_memrealtime(); unsigned sp = 0;
    while ((int)(__builtin_amdgcn_readfirstlane(ld_sys(p)) - call) < 0)
        if (give_up<SLEEP>(ws, call, t0, sp)) return false;
    return true;
}
__device__ __forceinline__ unsigned fl_word(int blk, int src, int t) { return FL_OFF + (unsigned)(((blk * 4 + (src & 3)) * NT + t) * 2 + (src >> 2)) * 4u; }
__device__ __forceinline__ int steps_known(unsigned f0, unsigned f1, unsigned call) {
    const u64 lo = __ballot((int)(f0 - call) >= 0), hi = __ballot((int)(f1 - call) >= 0);
    const u64 miss = ~(lo & hi);
    if (miss == 0) return NSTEP;
    const int c = __builtin_ctzll(miss);
    return 2 * c + (int)((lo >> c) & 1);
}
__device__ __forceinline__ unsigned out_base(int src, int tile, int wave, int lane) {
    return (unsigned)(((tile >> 3) * 256 + (wave >> 1) * 64 + (lane >> 4) * 4) * NN + src * NCOL + (tile & 7) * 256 + (wave & 1) * 128 + (lane & 15) * 8);
}
constexpr unsigned PIECE_BYTES = 896;
__device__ __forceinline__ unsigned piece_off(int lane) { return lane & 1 ? 512 + (lane >> 1) * 12 : (lane >> 1) * 16; }
#define WAIT_VM(n) asm volatile("s_waitcnt vmcnt(" #n ")" ::: "memory")
#define LGKM0_BARRIER() do { __builtin_amdgcn_s_waitcnt(0xC07F); asm volatile("" ::: "memory");                       \
        __builtin_amdgcn_s_barrier(); asm volatile("" ::: "memory"); } while (0)

__global__ void __launch_bounds__(BLOCK, 2) fused_gemm_relu_q8(const Params p) {
    __shared__ __attribute__((aligned(1024))) char smem[LDS_BYTES];
    LAS char* const lds = (LAS char*)smem;
    const int hb = blockIdx.x;
    const int lane = threadIdx.x & 63;
    const int wid = __builtin_amdgcn_readfirstlane(threadIdx.x >> 6);
    int rank = p.rank;
    char* ws = p.ws[0];
    const char* px = p.x;
    const char* pw = p.w;
    const u64 t_start = __builtin_amdgcn_s_memrealtime();
    asm volatile("" : "+s"(rank), "+s"(ws), "+s"(px), "+s"(pw));
    const unsigned ticket = add_agent_lane0(ws + CTR_OFF + (size_t)hb * 128 + wid * 4, lane);
    const int b = hb & (N_COMM - 1);

    if (hb < N_COMM) {
        const __amdgpu_buffer_rsrc_t wsr = rsrc_of(ws, CTL_OFF);
        const int cw = wid * N_COMM + b;
        if (wid < 2) {
            if (threadIdx.x == 0) *(LAS unsigned*)(lds + LDS_MISC) = 0u;
            LGKM0_BARRIER();
            if (cw < N_PULL) {
                const unsigned s = (unsigned)(rank + 1 + cw % (NR - 1)) & 7u;
                const __amdgpu_buffer_rsrc_t srs = rsrc_of(p.ws[1 + cw % (NR - 1)], CTL_OFF);
                char* const queue = ws + Q_OFF + s * 128;
                unsigned tk = add_agent_lane0(queue, lane);
                u32x4 cur[4];
                const unsigned call = __builtin_amdgcn_readfirstlane(ticket) + 1u;
                const unsigned qbase = (call - 1u) * Q_TICKETS;
                const char* const pk = ws + PK_OFF + s * 8192;
                unsigned pv = call - 1u, pg = call - 1u;
#define JOB_OF(n_, t_, j_, so_, do_) const int t_ = (int)(n_) / NBLK, j_ = job_block((int)(n_) % NBLK);               \
                const unsigned do_ = (j_ < 16 ? XT_OFF + t_ * XT_T + j_ * 32768 : WT_OFF + t_ * WT_T + (j_ - 16) * 32768) + chunk_pos(s, j_) + lane * 16; \
                const unsigned so_ = j_ < 16 ? do_ : WS_OFF + t_ * WT_T + rank * 32768 + (j_ - 16) * 4096 + lane * 16
#define SEEN(v_, l_) ((int)(__builtin_amdgcn_readlane(v_, l_) - call) >= 0)
#define WAIT_PACKED(t_, j_) do { const int sg_ = ((t_) >> 3) + ((j_) < 16 ? 0 : 8);                                   \
                if (!SEEN(pg, sg_)) {                                                                                 \
                    const u64 t0_ = __builtin_amdgcn_s_memrealtime(); unsigned sp_ = 0;                               \
                    if ((t_) < 8) { const int sl_ = (j_) < 16 ? (j_) : 24 + (t_) * 4 + ((j_) >> 1);                   \
                        if (!SEEN(pv, sl_)) {                           \
                            pv = ld_sys(pk + lane * 128); pg = ld_sys(pk + 16 * 128 + (lane & 15) * 4);               \
                            while (!SEEN(pv, sl_)) {                                                                  \
                                if (give_up(ws, call, t0_, sp_)) break;                                               \
                                if (lane == sl_) pv = ld_sys(pk + lane * 128); } }                                    \
                    } else {                                                                                          \
                        for (;;) { pg = ld_sys(pk + 16 * 128 + (lane & 15) * 4);                                      \
                            if (SEEN(pg, sg_) || give_up(ws, call, t0_, sp_)) break; } }                              \
                } } while (0)
                unsigned n = __builtin_amdgcn_readfirstlane(tk) - qbase;
                if (n < (unsigned)Q_JOBS) {
                    JOB_OF(n, t, j, so, dof);
                    WAIT_PACKED(t, j);
                    tk = add_agent_lane0(queue, lane);
#pragma unroll
                    for (int i = 0; i < 4; ++i) cur[i] = __builtin_amdgcn_raw_buffer_load_b128(srs, so + i * 1024, 0, AUX_SYS);
                    unsigned dcur = dof, fcur = fl_word(j, s, t);
                    for (;;) {
#pragma unroll
                        for (int i = 0; i < 4; ++i) __builtin_amdgcn_raw_buffer_store_b128(cur[i], wsr, dcur + i * 1024, 0, AUX_SYS);
                        asm volatile("" ::: "memory");
                        n = __builtin_amdgcn_readfirstlane(tk) - qbase;
                        char* const flag = ws + fcur;
                        if (n >= (unsigned)Q_JOBS) {
                            WAIT_VM(0);
                            if (lane == 0) st_sys(flag, call);
                            break;
                        }
                        JOB_OF(n, t2, j2, so2, dof2);
                        WAIT_PACKED(t2, j2);
                        tk = add_agent_lane0(queue, lane);
#pragma unroll
                        for (int i = 0; i < 4; ++i) cur[i] = __builtin_amdgcn_raw_buffer_load_b128(srs, so2 + i * 1024, 0, AUX_SYS);
                        dcur = dof2; fcur = fl_word(j2, s, t2);
                        WAIT_VM(5);
                        if (lane == 0) st_sys(flag, call);
                    }
                }
            }
        } else {
            const __amdgpu_buffer_rsrc_t xr = rsrc_of(px, MM * KSH * 2);
            const __amdgpu_buffer_rsrc_t wr_ = rsrc_of(pw, KSH * NN * 2);
            bool met = false;
            bool x_todo = wid >= 4;
            for (int n = (wid - 2) * N_COMM + b; n < NT * 32 || x_todo;) {
                const bool do_x = x_todo && ((b >> 4) <= (n >> 8) || n >= NT * 32);
                const int g = do_x ? b >> 4 : n >> 8;
                const int delay = (do_x ? g * 8 : n >> 5) * PACK_DELAY;
                if (!met && delay > 0) { __builtin_amdgcn_s_barrier(); met = true; }
                while (__builtin_amdgcn_s_memrealtime() - t_start < (u64)delay) __builtin_amdgcn_s_sleep(2);
                u32x4 v[8];
                if (do_x) {
                    const unsigned rb = (wid - 4) * 64 + (lane & 7), piece = lane >> 3;
#pragma unroll
                    for (int i = 0; i < 8; ++i) v[i] = __builtin_amdgcn_raw_buffer_load_b128(xr, ((b & 15) * 256 + rb + i * 8) * (KSH * 2) + g * 128 + piece * 16, 0, 0);
                    if (!met) { __builtin_amdgcn_s_barrier(); met = true; }
                    const unsigned d0 = XT_OFF + (g * 8 + piece) * XT_T + (b & 15) * 32768 + chunk_pos(rank, b & 15) + rb * 16;
#pragma unroll
                    for (int i = 0; i < 8; ++i) __builtin_amdgcn_raw_buffer_store_b128(v[i], wsr, d0 + i * 128, 0, AUX_SYS);
                    char* fl = lane < 8 ? p.ws[lane & 7] + PK_OFF + rank * 8192 + (b & 15) * 128
                                        : ws + fl_word(b & 15, rank, g * 8 + (lane & 7));
                    asm volatile("s_waitcnt vmcnt(0)" : "+v"(fl) :: "memory");
                    unsigned old = 0;
                    if (lane == 0) old = __hip_atomic_fetch_add((LAS unsigned*)(lds + LDS_MISC), 1u, __ATOMIC_RELAXED, __HIP_MEMORY_SCOPE_WORKGROUP);
                    const unsigned call = __builtin_amdgcn_readfirstlane(ticket) + 1u;
                    if (__builtin_amdgcn_readfirstlane(old) == 3u) {
                        if (lane >= (g == 0 ? 1 : 8) && lane < 16) st_sys(fl, call);
                        if (__builtin_amdgcn_readfirstlane(add_agent_lane0(ws + CNT_OFF + g * 9 * 128, lane)) + 1u == call * 16u && lane >= 1 && lane < 8)
                            st_sys(fl + (16 - (b & 15)) * 128 + g * 4, call);
                    }
                    x_todo = false;
                } else {
                    const int t = n >> 5, cb = n & 31;
                    const unsigned dest = (unsigned)cb >> 2;
#pragma unroll
                    for (int i = 0; i < 8; ++i) v[i] = __builtin_amdgcn_raw_buffer_load_b128(wr_, (unsigned)((t * 8 + i) * (NN * 2) + cb * 1024 + lane * 16), 0, 0);
                    if (!met) { __builtin_amdgcn_s_barrier(); met = true; }
                    const unsigned d0 = (dest == (unsigned)rank ? WT_OFF + t * WT_T + ((cb & 3) * 2 + (lane >> 5)) * 32768 + chunk_pos(rank, (cb & 3) * 2 + (lane >> 5))
                                                                : WS_OFF + t * WT_T + dest * 32768 + ((cb & 3) * 2 + (lane >> 5)) * 4096)
                                      + (((lane >> 4) & 1) * 128 + (lane & 15)) * 16;
#pragma unroll
                    for (int i = 0; i < 8; ++i) __builtin_amdgcn_raw_buffer_store_b128(transposed(v, i), wsr, d0 + i * 256, 0, AUX_SYS);
                    char* const dws = p.ws[(dest - rank) & 7u];
                    WAIT_VM(0);
                    const unsigned call = __builtin_amdgcn_readfirstlane(ticket) + 1u;
                    if (dest == (unsigned)rank) { if (lane < 2) st_sys(ws + fl_word(16 + (cb & 3) * 2 + lane, rank, t), call); }
                    else {
                        if (g == 0 && lane == 0) st_sys(dws + PK_OFF + rank * 8192 + (32 + t * 4 + (cb & 3)) * 128, call);
                        if (__builtin_amdgcn_readfirstlane(add_agent_lane0(ws + CNT_OFF + (g * 9 + 1 + dest) * 128, lane)) + 1u == call * 32u && lane == 0)
                            st_sys(dws + PK_OFF + rank * 8192 + 16 * 128 + (8 + g) * 4, call);
                    }
                    n += N_PACKW;
                }
            }
        }
        const unsigned call = __builtin_amdgcn_readfirstlane(ticket) + 1u;
        if (wid == 7) {
            {
                while (__builtin_amdgcn_s_memrealtime() - t_start < (u64)(TOUCH_AT + b * 16)) __builtin_amdgcn_s_sleep(8);
                unsigned tv = 0;
#pragma unroll
                for (int i = 1; i < NR; ++i) tv += ld_sys(ws + OB_OFF + (unsigned)(((rank + i) & 7) * 128 + b) * OB_SLOT + lane * 128);
                asm volatile("" :: "v"(tv));
            }
            if (b == N_COMM - 1) {
                char* amax_out = p.ws[lane & 7] + AMAX_OFF + ((lane >> 3) * NR + rank) * 128;
                asm volatile("" : "+v"(amax_out));
                const char* tm = ws + TMAX_OFF + lane * 256;
                const u64 t0 = __builtin_amdgcn_s_memrealtime(); unsigned sp = 0;
                int mx;
                for (;;) {
                    const u64 v0 = ld_sys64(tm), v1 = ld_sys64(tm + 128);
                    mx = max((int)(unsigned)v0, (int)(unsigned)v1);
                    if (__all((int)((unsigned)(v0 >> 32) - call) >= 0 && (int)((unsigned)(v1 >> 32) - call) >= 0)) break;
                    if (give_up(ws, call, t0, sp)) { mx = NAN_BITS; break; }
                }
                st_sys64(amax_out, (unsigned)wave_max(mx), call);
            }
            return;
        }
        {
            const int src = (rank + 1 + wid) & 7;
            const unsigned slot = OB_OFF + (unsigned)(src * 128 + b) * OB_SLOT;
            const char* const of = ws + OF_OFF + (src * 128 + b) * 128;
            const unsigned lbase = wid * LDS_WAVE;
#pragma unroll
            for (int e = 0; e < 4; ++e) {
                const unsigned idx = lane * 4 + e;
                u32x2 sel = {0u, 0u};
                unsigned rk = 0;
#pragma unroll
                for (int n = 0; n < 8; ++n) {
                    const bool neg = (idx >> (7 - n)) & 1u;
                    sel[n >> 2] |= (neg ? 0x0cu : rk) << (8 * (n & 3));
                    rk += neg ? 0u : 1u;
                }
                *(LAS u32x2*)(lds + lbase + 12288 + idx * 8) = sel;
            }
            char* outp = p.out;
            asm volatile("" : "+s"(outp));
            const unsigned ob = out_base(src, b >> 3, b & 7, lane);
            {
                bool ok = await_word<8>(ws, of, call);
                const u32x4 mk = __builtin_amdgcn_raw_buffer_load_b128(wsr, slot + lane * 16, 0, AUX_SYS);
                unsigned first[16], last[16];
                stream_layout(mk, first, last);
                u32x2 sel[16];
#pragma unroll
                for (int u = 0; u < 16; ++u) {
                    sel[u] = *(const LAS u32x2*)(lds + lbase + 12288 + sign_byte(mk, u) * 8);
                    first[u] += lbase;
                    asm volatile("" : "+v"(first[u]), "+v"(sel[u]));
                }
                const unsigned cnt = last[15];
                const __amdgpu_buffer_rsrc_t ors = rsrc_of(outp, (MM / NR) * NN);
#define TAKE_PIECES(i0_, i1_) do { u32x4 pc[(i1_) - (i0_)];                                                           \
                    _Pragma("unroll") for (int i = (i0_); i < (i1_); ++i)                                             \
                        if (i * 1024u < cnt) pc[i - (i0_)] = __builtin_amdgcn_raw_buffer_load_b128(wsr, slot + 1024 + i * PIECE_BYTES + piece_off(lane), 0, AUX_SYS); \
                    fill = ok && !__any(ld_sys(lane & 1 ? ws + TMO_OFF : of + 12) == call) ? 0u : 0x7f7f7f7fu;        \
                    _Pragma("unroll") for (int i = (i0_); i < (i1_); ++i)                                             \
                        if (i * 1024u < cnt) {                                                                        \
                            u32x4 q = pc[i - (i0_)];                                                                  \
                            unsigned h = ((q[0] & 0x80808080u) >> 7) | ((q[1] & 0x80808080u) >> 6) | ((q[2] & 0x80808080u) >> 5); \
                            const unsigned he = h | ((q[3] & 0x80808080u) >> 4);     \
                            h = (h << 4) | (unsigned)__builtin_amdgcn_update_dpp(0, (int)he, 0xa0, 0xf, 0xf, true);   \
                            q[0] &= 0x7f7f7f7fu; q[1] &= 0x7f7f7f7fu; q[2] &= 0x7f7f7f7fu;                            \
                            q[3] = lane & 1 ? h : q[3] & 0x7f7f7f7fu;                                                 \
                            *(LAS u32x4*)(lds + lbase + i * 1024 + lane * 16) = q;                                    \
                        } } while (0)
                unsigned fill;
                ok = await_word<1>(ws, of + 8, call) && ok;
                TAKE_PIECES(0, 8);
                unsigned d[16][3];
#pragma unroll
                for (int u = 0; u < 16; ++u)
#pragma unroll
                    for (int i = 0; i < 3; ++i) d[u][i] = *(const LAS unsigned*)(lds + (first[u] & ~3u) + i * 4);
                __builtin_amdgcn_sched_barrier(0);
#pragma unroll
                for (int u = 0; u < 16; ++u) {
                    const unsigned lo = __builtin_amdgcn_alignbyte(d[u][1], d[u][0], first[u]), hi = __builtin_amdgcn_alignbyte(d[u][2], d[u][1], first[u]);
                    u32x2 o;
                    o[0] = __builtin_amdgcn_perm(hi, lo, sel[u][0]) | fill;
                    o[1] = __builtin_amdgcn_perm(hi, lo, sel[u][1]) | fill;
                    __builtin_amdgcn_raw_buffer_store_b64(o, ors, ob + (unsigned)(((u >> 2) * 16 + (u & 3)) * NN), 0, AUX_SYS);
                }
            }
        }
        return;
    }

    const int g = b;
    const int pm = (g >> 4) * 2 + (g & 1);
    const int pn = ((g & 7) >> 1) * 2 + ((g >> 3) & 1);
    const int fr = lane & 15, fq = lane >> 4, wr = wid >> 1, wc = wid & 1;
    const int owner = pm >> 1, tile = (pm & 1) * 8 + pn;
    char* ows = p.ws[(owner - rank) & 7];
    char* outp = p.out;
    asm volatile("" : "+s"(ows), "+s"(outp));
    const unsigned call = __builtin_amdgcn_readfirstlane(ticket) + 1u;

    const int op = wid >> 2;
    const char* const pollp = ws + fl_word(op ? 16 + pn : pm, wid & 3, lane);
    const unsigned unit = op ? WT_T : XT_T;
    const char* const gbase = ws + (op ? WT_OFF + pn * 32768 : XT_OFF + pm * 32768) + lane * 16;
    const unsigned slot0 = chunk_pos(wid & 3, op ? pn : pm);
    LAS char* const ldst = lds + wid * 4096;
    const int aoff = fq * 4096 + (wr * 64 + fr) * 16;
    const int boff = 16384 + fq * 4096 + (wc * 128 + fr) * 16;

    int known = 0;
    int next = 0, nslot = 0;
    u32x2 pvn = {0u, 0u};
#define STAGE_NEXT() do { const char* g_ = gbase + ((slot0 ^ ((next & 1) << 14)) + (unsigned)(next >> 1) * unit);     \
        LAS char* l_ = ldst + nslot * LDS_SLOT;                                                                       \
        __builtin_amdgcn_global_load_lds((const GAS unsigned*)g_, (LAS unsigned*)l_, 16, 0, AUX_SYS);                 \
        __builtin_amdgcn_global_load_lds((const GAS unsigned*)g_, (LAS unsigned*)l_, 16, 1024, AUX_SYS);              \
        __builtin_amdgcn_global_load_lds((const GAS unsigned*)g_, (LAS unsigned*)l_, 16, 2048, AUX_SYS);              \
        __builtin_amdgcn_global_load_lds((const GAS unsigned*)g_, (LAS unsigned*)l_, 16, 3072, AUX_SYS);              \
        ++next; nslot = nslot == NSLOT - 1 ? 0 : nslot + 1; } while (0)
#define POLL_ISSUE() asm volatile("global_load_dwordx2 %0, %1, off sc0 sc1" : "+v"(pvn) : "v"(pollp) : "memory")
#define POLL_TAKE() do { u32x2 f_; asm volatile("s_waitcnt vmcnt(0)\n\tv_mov_b64 %0, %1" : "=v"(f_) : "v"(pvn) : "memory");    \
        known = steps_known(f_[0], f_[1], call); } while (0)
#define AWAIT_NEXT() do { if (next >= known) {                                                                        \
            const u64 t0_ = __builtin_amdgcn_s_memrealtime(); unsigned sp_ = 0;                                       \
            for (;;) { const u64 f_ = ld_sys64(pollp); known = steps_known((unsigned)f_, (unsigned)(f_ >> 32), call); \
                if (next < known) break;                                                                              \
                if (give_up<2>(ws, call, t0_, sp_)) { known = NSTEP; break; } } } } while (0)
#define MFMA_COLS(A_, B_, n0_, n1_) do { __builtin_amdgcn_sched_barrier(0);                                           \
        _Pragma("unroll") for (int n = (n0_); n < (n1_); ++n)                                                         \
            _Pragma("unroll") for (int m = 0; m < 4; ++m)                                                             \
                acc[m][n] = __builtin_amdgcn_mfma_f32_16x16x32_bf16(A_[m], B_[n], acc[m][n], 0, 0, 0);                \
        __builtin_amdgcn_sched_barrier(0); } while (0)
#define RD_A(A_, m_) A_[m_] = *(const LAS bf16x8*)(lb_ + aoff + (m_) * 256)
#define RD_B(B_, n_) B_[n_] = *(const LAS bf16x8*)(lb_ + boff + (n_) * 256)
#define KSTEP(st_, A_, B_, AN_, BN_) do {                                                                             \
        MFMA_COLS(A_, B_, 0, 1);                                                                                      \
        if (polled) POLL_TAKE();                                          \
        const int had_ = next;                                                                                        \
        const int lim_ = min((st_) + NSLOT, known);                                   \
        if (next < lim_) { STAGE_NEXT(); if (next < lim_) STAGE_NEXT(); }                                             \
        const bool full_ = (polled ? had_ + NSLOT - 2 : next) >= (st_) + NSLOT;                                       \
        MFMA_COLS(A_, B_, 1, 4);                                                                                      \
        if (!full_) {                                                              \
            if (next == (st_) + 1 && next < NSTEP) {                                                                  \
                AWAIT_NEXT(); STAGE_NEXT();                                                                           \
                if (next < known) STAGE_NEXT();                                                                       \
            }                                                                                                         \
            const int k_ = next - (st_) - 2;                                                                          \
            if (k_ <= 0) WAIT_VM(0); else if (k_ == 1) WAIT_VM(4); else WAIT_VM(8);                                   \
        }                                                                                                             \
        WAIT_VM(12);                                                                                                  \
        LGKM0_BARRIER();                                                                                              \
        const LAS char* lb_ = lds + rslot * LDS_SLOT;               \
        RD_A(AN_, 0); RD_A(AN_, 1); RD_A(AN_, 2); RD_A(AN_, 3);                                                       \
        polled = known < min(next + 6, NSTEP);                                                                        \
        if (polled) POLL_ISSUE();                                                                                     \
        MFMA_COLS(A_, B_, 4, 5);                                                                                      \
        RD_B(BN_, 0); RD_B(BN_, 1); RD_B(BN_, 2); RD_B(BN_, 3);                                                       \
        MFMA_COLS(A_, B_, 5, 6);                                                                                      \
        RD_B(BN_, 4); RD_B(BN_, 5); RD_B(BN_, 6); RD_B(BN_, 7);                                                       \
        rslot = rslot == NSLOT - 1 ? 0 : rslot + 1;                                                                   \
        MFMA_COLS(A_, B_, 6, 8); } while (0)

    f32x4 acc[4][8];
#pragma unroll
    for (int m = 0; m < 4; ++m)
#pragma unroll
        for (int n = 0; n < 8; ++n) acc[m][n] = (f32x4){0.f, 0.f, 0.f, 0.f};

    AWAIT_NEXT(); STAGE_NEXT();
    WAIT_VM(0);
    LGKM0_BARRIER();
    int rslot = 1;
    bool polled = false;
    bf16x8 a0[4], b0[8], a1[4], b1[8];
    {
        const LAS char* lb_ = lds;
#pragma unroll
        for (int m = 0; m < 4; ++m) RD_A(a0, m);
#pragma unroll
        for (int n = 0; n < 8; ++n) RD_B(b0, n);
    }
    for (int st = 0; st < NSTEP; st += 2) {
        KSTEP(st, a0, b0, a1, b1);
        KSTEP(st + 1, a1, b1, a0, b0);
    }

    int mx = 0;
#pragma unroll
    for (int m = 0; m < 4; ++m)
#pragma unroll
        for (int n = 0; n < 8; ++n)
#pragma unroll
            for (int j = 0; j < 4; ++j) mx = max(mx, __float_as_int(acc[m][n][j]));
    mx = wave_max(mx);
    LAS int* red = (LAS int*)(lds + LDS_MISC);
    if (lane == 0) red[wid] = mx;
    LGKM0_BARRIER();
    if (wid == 0) {
        int t = red[lane & 7];
        ROW_SHR_MAX(t, 1); ROW_SHR_MAX(t, 2); ROW_SHR_MAX(t, 4);
        if (lane == 7) st_sys64(ws + TMAX_OFF + g * 128, (unsigned)t, call);
    }

    const bool remote = owner != rank;
    const __amdgpu_buffer_rsrc_t sls = rsrc_of(ows + OB_OFF + (unsigned)(rank * 128 + tile * 8 + wid) * OB_SLOT, OB_SLOT);
    char* const of = ows + OF_OFF + (rank * 128 + tile * 8 + wid) * 128;
    const int efield = min(268 - (mx >> 23), 253);
    const float sc = __int_as_float(efield << 23);
    const unsigned pos0 = (unsigned)(size_t)(lds + wid * LDS_WAVE);
    const unsigned spill = (unsigned)(size_t)(lds + LDS_MISC + 64);
    unsigned first[16], last[16];
#define VAL(u_, n_) acc[(u_) >> 2][n_][(u_) & 3]
#define PACK1(n_) "v_ashrrev_i32 %[s], 31, %[v" #n_ "]\n\tv_fma_mixlo_f16 %[h], %[v" #n_ "], %[sc], 0\n\t"            \
                  "v_bfi_b32 %[a], %[s], %[sp], %[p]\n\tv_lshl_add_u32 %[p], %[s], 1, %[p]\n\t"                       \
                  "ds_write_b16 %[a], %[h] offset:2*" #n_ "\n\t"
#define PACK8(u_) do { unsigned p_ = pos0 + 2 * first[u_], s_, h_, a_;                                                \
        asm volatile(PACK1(0) PACK1(1) PACK1(2) PACK1(3) PACK1(4) PACK1(5) PACK1(6) PACK1(7)                          \
                     : [p] "+v"(p_), [s] "=&v"(s_), [h] "=&v"(h_), [a] "=&v"(a_)                                      \
                     : [v0] "v"(VAL(u_, 0)), [v1] "v"(VAL(u_, 1)), [v2] "v"(VAL(u_, 2)), [v3] "v"(VAL(u_, 3)),        \
                       [v4] "v"(VAL(u_, 4)), [v5] "v"(VAL(u_, 5)), [v6] "v"(VAL(u_, 6)), [v7] "v"(VAL(u_, 7)),        \
                       [sc] "s"(sc), [sp] "v"(spill) : "memory"); } while (0)
    u32x4 mk = {0u, 0u, 0u, 0u};
    if (remote) {
#pragma unroll
        for (int r = 0; r < 128; ++r) mk[r >> 5] = __builtin_amdgcn_alignbit(mk[r >> 5], __float_as_uint(VAL(r >> 3, r & 7)), 31);
        if (wid != 0) __builtin_amdgcn_raw_buffer_store_b128(mk, sls, lane * 16, 0, AUX_SYS);
        stream_layout(mk, first, last);
        PACK8(0); PACK8(1); PACK8(2); PACK8(3); PACK8(4);
    } else {
#pragma unroll
        for (int m = 0; m < 4; ++m) {
#pragma unroll
            for (int n = 0; n < 8; ++n)
#pragma unroll
                for (int j = 0; j < 4; ++j) acc[m][n][j] = __int_as_float(max(0, __float_as_int(acc[m][n][j])));
            asm volatile("" : "+v"(acc[m][0]), "+v"(acc[m][1]), "+v"(acc[m][2]), "+v"(acc[m][3]),
                              "+v"(acc[m][4]), "+v"(acc[m][5]), "+v"(acc[m][6]), "+v"(acc[m][7]));
        }
    }
    if (wid == 0) {
        const char* ap = lane < NR ? ws + AMAX_OFF + ((g & 7) * NR + lane) * 128 : ws + TMO_OFF - 4;
        const u64 t0 = __builtin_amdgcn_s_memrealtime(); unsigned sp = 0;
        u64 v = 0;
        for (;;) {
            if (lane <= NR) v = ld_sys64(ap);
            if ((__ballot((int)((unsigned)(v >> 32) - call) >= 0) & 0xffu) == 0xffu) break;
            if (give_up(ws, call, t0, sp)) { v = (u64)call << 32; break; }
        }
        int t = lane < NR ? (int)(unsigned)v : ((unsigned)(v >> 32) == call ? NAN_BITS : 0);
        ROW_SHR_MAX(t, 1); ROW_SHR_MAX(t, 2); ROW_SHR_MAX(t, 4); ROW_SHR_MAX(t, 8);
        if (lane == NR) red[8] = t;
    }
    LGKM0_BARRIER();
    const float inv = 448.0f / __int_as_float(red[8]);

    if (remote) {
        const float f = inv * __int_as_float((254 - efield) << 23);
        unsigned sent = 0;
        const unsigned hshift = (lane & 1) * 4, woff = piece_off(lane);
#define SEND(cnt_, upto_) do { __builtin_amdgcn_s_waitcnt(0xC07F); asm volatile("" ::: "memory");                     \
            _Pragma("nounroll") for (; sent < (upto_); sent += 1024) {                                                \
                const unsigned o_ = sent + lane * 16;                                                                 \
                const f16x8 h0 = *(const LAS f16x8*)(lds + wid * LDS_WAVE + o_ * 2), h1 = *(const LAS f16x8*)(lds + wid * LDS_WAVE + o_ * 2 + 16); \
                u32x4 q;                                                                                              \
                _Pragma("unroll") for (int d = 0; d < 2; ++d) {                                                       \
                    int w0 = __builtin_amdgcn_cvt_pk_fp8_f32(__builtin_fmaf((float)h0[4 * d], f, 0.f), __builtin_fmaf((float)h0[4 * d + 1], f, 0.f), 0, false); \
                    q[d] = (unsigned)__builtin_amdgcn_cvt_pk_fp8_f32(__builtin_fmaf((float)h0[4 * d + 2], f, 0.f), __builtin_fmaf((float)h0[4 * d + 3], f, 0.f), w0, true); \
                    int w1 = __builtin_amdgcn_cvt_pk_fp8_f32(__builtin_fmaf((float)h1[4 * d], f, 0.f), __builtin_fmaf((float)h1[4 * d + 1], f, 0.f), 0, false); \
                    q[2 + d] = (unsigned)__builtin_amdgcn_cvt_pk_fp8_f32(__builtin_fmaf((float)h1[4 * d + 2], f, 0.f), __builtin_fmaf((float)h1[4 * d + 3], f, 0.f), w1, true); \
                }                                                                                                     \
                  \
                const unsigned h_ = (unsigned)__builtin_amdgcn_update_dpp(0, (int)q[3], 0xf5, 0xf, 0xf, true) >> hshift; \
                _Pragma("unroll") for (int d = 0; d < 4; ++d) q[d] |= (h_ << (7 - d)) & 0x80808080u;                  \
                WAIT_VM(2);                                                                                           \
                if ((o_ & ~31u) < (cnt_)) {                                                                           \
                    const unsigned w_ = 1024 + (sent >> 10) * PIECE_BYTES + woff;                                     \
                    if (lane & 1) __builtin_amdgcn_raw_buffer_store_b96((u32x3){q[0], q[1], q[2]}, sls, w_, 0, AUX_SYS); \
                    else __builtin_amdgcn_raw_buffer_store_b128(q, sls, w_, 0, AUX_SYS);                              \
                }                                                                                                     \
            } } while (0)
        if (wid == 0) __builtin_amdgcn_raw_buffer_store_b128(mk, sls, lane * 16, 0, AUX_SYS);
        SEND(last[4], last[4] & ~1023u);
        PACK8(5); PACK8(6); PACK8(7); PACK8(8);
        if (sent == 1024u) WAIT_VM(2); else WAIT_VM(0);
        if (lane == 0) st_sys(of, call);
        SEND(last[8], last[8] & ~1023u);
        PACK8(9); PACK8(10); PACK8(11); PACK8(12);
        SEND(last[12], last[12] & ~1023u);
        PACK8(13); PACK8(14); PACK8(15);
        SEND(last[15], last[15]);
        WAIT_VM(0);
        if (lane == 0) st_sys64(of + 8, call, red[8] == NAN_BITS ? call : 0u);
    } else {
        const __amdgpu_buffer_rsrc_t ors = rsrc_of(outp, (MM / NR) * NN);
        const unsigned ob = out_base(rank, tile, wid, lane);
#pragma unroll
        for (int m = 0; m < 4; ++m)
#pragma unroll
            for (int j = 0; j < 4; ++j) {
                u32x2 o;
                int lo = __builtin_amdgcn_cvt_pk_fp8_f32(acc[m][0][j] * inv, acc[m][1][j] * inv, 0, false);
                o[0] = (unsigned)__builtin_amdgcn_cvt_pk_fp8_f32(acc[m][2][j] * inv, acc[m][3][j] * inv, lo, true);
                int hi = __builtin_amdgcn_cvt_pk_fp8_f32(acc[m][4][j] * inv, acc[m][5][j] * inv, 0, false);
                o[1] = (unsigned)__builtin_amdgcn_cvt_pk_fp8_f32(acc[m][6][j] * inv, acc[m][7][j] * inv, hi, true);
                __builtin_amdgcn_raw_buffer_store_b64(o, ors, ob + (unsigned)((m * 16 + j) * NN), 0, AUX_SYS);
            }
    }
}

}

extern "C" void kernel_launch(int rank, int nranks, ncclComm_t comm,
                              hipStream_t stream,
                              void* const* d_in, const int* in_sizes, int n_in,
                              void* d_out, int out_size,
                              const int* in_dtypes, int out_dtype,
                              void* const* d_out_all,
                              void* const* d_ws_all, size_t ws_bytes) {
    (void)comm; (void)in_dtypes; (void)out_dtype; (void)d_out_all;
    if (nranks != NR || n_in < 2 || (unsigned)rank >= (unsigned)NR || ws_bytes < WS_NEED ||
        in_sizes[0] != MM * KSH || in_sizes[1] != KSH * NN || out_size != (MM / NR) * NN) {
        fprintf(stderr, "kernel_launch: unexpected problem shape\n");
        return;
    }
    Params p;
    p.x = (const char*)d_in[0];
    p.w = (const char*)d_in[1];
    for (int i = 0; i < NR; ++i) p.ws[i] = (char*)d_ws_all[(rank + i) % NR];
    p.out = (char*)d_out;
    p.rank = rank; p.pad = 0;
    hipLaunchKernelGGL(fused_gemm_relu_q8, dim3(GRID), dim3(BLOCK), 0, stream, p);
    HIP_CHECK(hipGetLastError());
}
```
